```python
import numpy as np

kernel_name = "gpu_a2a_gemm_m4096_k4096_n32768_f32_silu"


ABSMAX_RTOL = 0.001

COMPUTE_DTYPE = "float32"
EPILOGUE = "silu"
IN_QUANT = None
OUT_QUANT = None
IN_DTYPES = {"x": "float32", "w_mat": "float32"}
WORLD_SIZE = 8

def setup_inputs():

    x = np.random.randn(4096, 4096).astype(np.float32)
    w_mat = (np.random.randn(4096, 32768) / np.sqrt(4096)).astype(np.float32)
    return {"x": x, "w_mat": w_mat}

def _epilogue(y):
    return (y / (1.0 + np.exp(-np.clip(y, -60.0, 60.0)))).astype(np.float32)

def reference(x, w_mat):
    y = (x @ w_mat).astype(np.float32)
    return _epilogue(y)
```

```cpp
#include <hip/hip_runtime.h>
#include <rccl/rccl.h>
#include <cstdio>

#define LDS_BYTES 116736
#define FUSED_ASM \
  "s_load_dwordx2 s[96:97], %6, 0x0\n" \
  "s_memrealtime s[98:99]\n" \
  "s_mov_b64 s[84:85], %4\n" \
  "s_mov_b32 s86, 0x8000\n" \
  "s_mov_b32 s87, 0x00020000\n" \
  "s_mov_b64 s[64:65], %0\n" \
  "s_mov_b32 s66, 0x20080000\n" \
  "s_mov_b32 s67, 0x00020000\n" \
  "s_lshl_b32 s76, %3, 26\n" \
  "s_mov_b32 s23, %8\n" \
  "s_cmp_eq_u32 s23, 0\n" \
  "s_cbranch_scc1 XFIRST%=\n" \
  "s_sleep 8\n" \
  "buffer_load_dwordx4 v[64:67], %11, s[64:67], s76 offen nt\n" \
  "s_add_u32 s76, s76, 262144\n" \
  "buffer_load_dwordx4 v[68:71], %11, s[64:67], s76 offen nt\n" \
  "s_add_u32 s76, s76, 262144\n" \
  "buffer_load_dwordx4 v[72:75], %11, s[64:67], s76 offen nt\n" \
  "s_add_u32 s76, s76, 262144\n" \
  "buffer_load_dwordx4 v[76:79], %11, s[64:67], s76 offen nt\n" \
  "s_add_u32 s76, s76, 3407872\n" \
  "buffer_load_dwordx4 v[80:83], %11, s[64:67], s76 offen nt\n" \
  "s_add_u32 s76, s76, 262144\n" \
  "buffer_load_dwordx4 v[84:87], %11, s[64:67], s76 offen nt\n" \
  "s_add_u32 s76, s76, 262144\n" \
  "buffer_load_dwordx4 v[88:91], %11, s[64:67], s76 offen nt\n" \
  "s_add_u32 s76, s76, 262144\n" \
  "buffer_load_dwordx4 v[92:95], %11, s[64:67], s76 offen nt\n" \
  "s_add_u32 s76, s76, 3407872\n" \
  "s_and_b32 s76, s76, 0x1fffffff\n" \
  "XFIRST%=:\n" \
  "buffer_load_dwordx4 v[0:3], %17, s[84:87], 0 offen offset:0 nt\n" \
  "buffer_load_dwordx4 v[4:7], %17, s[84:87], 0 offen offset:16 nt\n" \
  "buffer_load_dwordx4 v[8:11], %17, s[84:87], 0 offen offset:512 nt\n" \
  "buffer_load_dwordx4 v[12:15], %17, s[84:87], 0 offen offset:528 nt\n" \
  "buffer_load_dwordx4 v[16:19], %17, s[84:87], 0 offen offset:1024 nt\n" \
  "buffer_load_dwordx4 v[20:23], %17, s[84:87], 0 offen offset:1040 nt\n" \
  "buffer_load_dwordx4 v[24:27], %17, s[84:87], 0 offen offset:1536 nt\n" \
  "buffer_load_dwordx4 v[28:31], %17, s[84:87], 0 offen offset:1552 nt\n" \
  "s_lshl_b32 s77, %3, 19\n" \
  "s_mov_b64 s[68:69], %1\n" \
  "s_mov_b32 s70, 0x400000\n" \
  "s_mov_b32 s71, 0x00020000\n" \
  "s_mov_b64 s[88:89], %5\n" \
  "s_mov_b32 s90, 8388608\n" \
  "s_mov_b32 s91, 0x00020000\n" \
  "s_mov_b64 s[92:93], %1\n" \
  "s_add_u32 s92, s92, 8388608\n" \
  "s_addc_u32 s93, s93, 0\n" \
  "s_mov_b32 s94, 65536\n" \
  "s_mov_b32 s95, 0x00020000\n" \
  "v_lshlrev_b32 v180, 4, %19\n" \
  "v_mov_b32 v182, 1\n" \
  "v_lshlrev_b32 v179, 7, %19\n" \
  "s_waitcnt lgkmcnt(0)\n" \
  "s_add_u32 s96, s96, 1\n" \
  "s_sub_u32 s97, s98, s97\n" \
  "s_lshl_b32 s83, s96, 7\n" \
  "s_and_b32 s60, s96, 1\n" \
  "s_lshl_b32 s60, s60, 22\n" \
  "s_add_u32 s68, s68, s60\n" \
  "s_addc_u32 s69, s69, 0\n" \
  "s_add_u32 s34, s60, %7\n" \
  "v_add_u32 v172, 0x1000, %12\n" \
  "v_add_u32 v173, 0x8000, %12\n" \
  "v_add_u32 v174, 0x9000, %12\n" \
  "v_lshlrev_b32 v176, 24, %19\n" \
  "s_lshl_b32 s60, %10, 12\n" \
  "s_add_u32 s60, s60, 98304\n" \
  "v_add_u32 v181, s60, v180\n" \
  "v_mov_b32 v177, %13\n" \
  "v_mov_b32 v183, %14\n" \
  "s_add_u32 s56, s77, 0x20000\n" \
  "s_mov_b32 s55, s56\n" \
  "s_mov_b32 s26, 0\n" \
  "s_lshl_b32 s35, %3, 9\n" \
  "s_mov_b32 s59, %9\n" \
  "s_lshl_b32 s36, %3, 2\n" \
  "s_lshl_b32 s31, 15, s36\n" \
  "s_add_u32 s36, s36, 1\n" \
  "s_mov_b32 s30, 0\n" \
  "s_mov_b32 s82, 0\n" \
  "s_cmp_eq_u32 %10, 0\n" \
  "s_cselect_b32 s31, s31, -1\n" \
  "v_mov_b32 v160, 0\n" \
  "v_mov_b32 v161, 0\n" \
  "v_mov_b32 v162, 0\n" \
  "v_mov_b32 v163, 0\n" \
  "s_nop 1\n" \
  "v_mfma_f32_32x32x16_f16 a[0:15], v[160:163], v[160:163], 0\n" \
  "v_mfma_f32_32x32x16_f16 a[16:31], v[160:163], v[160:163], 0\n" \
  "v_mfma_f32_32x32x16_f16 a[32:47], v[160:163], v[160:163], 0\n" \
  "v_mfma_f32_32x32x16_f16 a[48:63], v[160:163], v[160:163], 0\n" \
  "v_mfma_f32_32x32x16_f16 a[64:79], v[160:163], v[160:163], 0\n" \
  "v_mfma_f32_32x32x16_f16 a[80:95], v[160:163], v[160:163], 0\n" \
  "v_mfma_f32_32x32x16_f16 a[96:111], v[160:163], v[160:163], 0\n" \
  "v_mfma_f32_32x32x16_f16 a[112:127], v[160:163], v[160:163], 0\n" \
  "v_mfma_f32_32x32x16_f16 a[128:143], v[160:163], v[160:163], 0\n" \
  "v_mfma_f32_32x32x16_f16 a[144:159], v[160:163], v[160:163], 0\n" \
  "v_mfma_f32_32x32x16_f16 a[160:175], v[160:163], v[160:163], 0\n" \
  "v_mfma_f32_32x32x16_f16 a[176:191], v[160:163], v[160:163], 0\n" \
  "v_mfma_f32_32x32x16_f16 a[192:207], v[160:163], v[160:163], 0\n" \
  "v_mfma_f32_32x32x16_f16 a[208:223], v[160:163], v[160:163], 0\n" \
  "v_mfma_f32_32x32x16_f16 a[224:239], v[160:163], v[160:163], 0\n" \
  "v_mfma_f32_32x32x16_f16 a[240:255], v[160:163], v[160:163], 0\n" \
  "s_cmp_eq_u32 s23, 0\n" \
  "s_cbranch_scc0 REMOTE%=\n" \
  "s_mov_b32 s60, s34\n" \
  "s_waitcnt vmcnt(6)\n" \
  "v_mul_f32 v0, 0xbfb8aa3b, v0\n" \
  "v_mul_f32 v1, 0xbfb8aa3b, v1\n" \
  "v_mul_f32 v2, 0xbfb8aa3b, v2\n" \
  "v_mul_f32 v3, 0xbfb8aa3b, v3\n" \
  "v_mul_f32 v4, 0xbfb8aa3b, v4\n" \
  "v_mul_f32 v5, 0xbfb8aa3b, v5\n" \
  "v_mul_f32 v6, 0xbfb8aa3b, v6\n" \
  "v_mul_f32 v7, 0xbfb8aa3b, v7\n" \
  "v_cvt_pk_f16_f32 v32, v0, v2\n" \
  "v_cvt_pk_f16_f32 v33, v4, v6\n" \
  "v_cvt_pk_f16_f32 v34, v1, v3\n" \
  "v_cvt_pk_f16_f32 v35, v5, v7\n" \
  "buffer_store_dwordx4 v[32:35], v180, s[88:91], s60 offen sc0 sc1\n" \
  "s_add_u32 s60, s60, 0x20000\n" \
  "s_waitcnt vmcnt(5)\n" \
  "v_mul_f32 v8, 0xbfb8aa3b, v8\n" \
  "v_mul_f32 v9, 0xbfb8aa3b, v9\n" \
  "v_mul_f32 v10, 0xbfb8aa3b, v10\n" \
  "v_mul_f32 v11, 0xbfb8aa3b, v11\n" \
  "v_mul_f32 v12, 0xbfb8aa3b, v12\n" \
  "v_mul_f32 v13, 0xbfb8aa3b, v13\n" \
  "v_mul_f32 v14, 0xbfb8aa3b, v14\n" \
  "v_mul_f32 v15, 0xbfb8aa3b, v15\n" \
  "v_cvt_pk_f16_f32 v36, v8, v10\n" \
  "v_cvt_pk_f16_f32 v37, v12, v14\n" \
  "v_cvt_pk_f16_f32 v38, v9, v11\n" \
  "v_cvt_pk_f16_f32 v39, v13, v15\n" \
  "buffer_store_dwordx4 v[36:39], v180, s[88:91], s60 offen sc0 sc1\n" \
  "s_add_u32 s60, s60, 0x20000\n" \
  "s_waitcnt vmcnt(4)\n" \
  "v_mul_f32 v16, 0xbfb8aa3b, v16\n" \
  "v_mul_f32 v17, 0xbfb8aa3b, v17\n" \
  "v_mul_f32 v18, 0xbfb8aa3b, v18\n" \
  "v_mul_f32 v19, 0xbfb8aa3b, v19\n" \
  "v_mul_f32 v20, 0xbfb8aa3b, v20\n" \
  "v_mul_f32 v21, 0xbfb8aa3b, v21\n" \
  "v_mul_f32 v22, 0xbfb8aa3b, v22\n" \
  "v_mul_f32 v23, 0xbfb8aa3b, v23\n" \
  "v_cvt_pk_f16_f32 v40, v16, v18\n" \
  "v_cvt_pk_f16_f32 v41, v20, v22\n" \
  "v_cvt_pk_f16_f32 v42, v17, v19\n" \
  "v_cvt_pk_f16_f32 v43, v21, v23\n" \
  "buffer_store_dwordx4 v[40:43], v180, s[88:91], s60 offen sc0 sc1\n" \
  "s_add_u32 s60, s60, 0x20000\n" \
  "s_waitcnt vmcnt(3)\n" \
  "v_mul_f32 v24, 0xbfb8aa3b, v24\n" \
  "v_mul_f32 v25, 0xbfb8aa3b, v25\n" \
  "v_mul_f32 v26, 0xbfb8aa3b, v26\n" \
  "v_mul_f32 v27, 0xbfb8aa3b, v27\n" \
  "v_mul_f32 v28, 0xbfb8aa3b, v28\n" \
  "v_mul_f32 v29, 0xbfb8aa3b, v29\n" \
  "v_mul_f32 v30, 0xbfb8aa3b, v30\n" \
  "v_mul_f32 v31, 0xbfb8aa3b, v31\n" \
  "v_cvt_pk_f16_f32 v44, v24, v26\n" \
  "v_cvt_pk_f16_f32 v45, v28, v30\n" \
  "v_cvt_pk_f16_f32 v46, v25, v27\n" \
  "v_cvt_pk_f16_f32 v47, v29, v31\n" \
  "buffer_store_dwordx4 v[44:47], v180, s[88:91], s60 offen sc0 sc1\n" \
  "s_mov_b32 s60, 32768\n" \
  "s_waitcnt vmcnt(0)\n" \
  "s_barrier\n" \
  "v_mov_b32 v182, 4\n" \
  "s_cmp_eq_u32 %10, 0\n" \
  "s_cselect_b32 s94, 65536, 0\n" \
  "buffer_atomic_add v182, v179, s[92:95], s60 offen sc1\n" \
  "s_mov_b32 s94, 65536\n" \
  "buffer_load_dwordx4 v[64:67], %11, s[64:67], s76 offen nt\n" \
  "s_add_u32 s76, s76, 262144\n" \
  "buffer_load_dwordx4 v[68:71], %11, s[64:67], s76 offen nt\n" \
  "s_add_u32 s76, s76, 262144\n" \
  "buffer_load_dwordx4 v[72:75], %11, s[64:67], s76 offen nt\n" \
  "s_add_u32 s76, s76, 262144\n" \
  "buffer_load_dwordx4 v[76:79], %11, s[64:67], s76 offen nt\n" \
  "s_add_u32 s76, s76, 3407872\n" \
  "buffer_load_dwordx4 v[80:83], %11, s[64:67], s76 offen nt\n" \
  "s_add_u32 s76, s76, 262144\n" \
  "buffer_load_dwordx4 v[84:87], %11, s[64:67], s76 offen nt\n" \
  "s_add_u32 s76, s76, 262144\n" \
  "buffer_load_dwordx4 v[88:91], %11, s[64:67], s76 offen nt\n" \
  "s_add_u32 s76, s76, 262144\n" \
  "buffer_load_dwordx4 v[92:95], %11, s[64:67], s76 offen nt\n" \
  "s_add_u32 s76, s76, 3407872\n" \
  "s_and_b32 s76, s76, 0x1fffffff\n" \
  "buffer_load_dwordx4 v[128:131], %11, s[64:67], s76 offen nt\n" \
  "s_add_u32 s76, s76, 262144\n" \
  "buffer_load_dwordx4 v[132:135], %11, s[64:67], s76 offen nt\n" \
  "s_add_u32 s76, s76, 262144\n" \
  "buffer_load_dwordx4 v[136:139], %11, s[64:67], s76 offen nt\n" \
  "s_add_u32 s76, s76, 262144\n" \
  "buffer_load_dwordx4 v[140:143], %11, s[64:67], s76 offen nt\n" \
  "s_add_u32 s76, s76, 3407872\n" \
  "buffer_load_dwordx4 v[144:147], %11, s[64:67], s76 offen nt\n" \
  "s_add_u32 s76, s76, 262144\n" \
  "buffer_load_dwordx4 v[148:151], %11, s[64:67], s76 offen nt\n" \
  "s_add_u32 s76, s76, 262144\n" \
  "buffer_load_dwordx4 v[152:155], %11, s[64:67], s76 offen nt\n" \
  "s_add_u32 s76, s76, 262144\n" \
  "buffer_load_dwordx4 v[156:159], %11, s[64:67], s76 offen nt\n" \
  "s_add_u32 s76, s76, 3407872\n" \
  "s_and_b32 s76, s76, 0x1fffffff\n" \
  "s_waitcnt vmcnt(8)\n" \
  "v_cvt_pk_f16_f32 v160, v64, v68\n" \
  "v_cvt_pk_f16_f32 v161, v72, v76\n" \
  "v_cvt_pk_f16_f32 v162, v65, v69\n" \
  "v_cvt_pk_f16_f32 v163, v73, v77\n" \
  "v_cvt_pk_f16_f32 v164, v66, v70\n" \
  "v_cvt_pk_f16_f32 v165, v74, v78\n" \
  "v_cvt_pk_f16_f32 v166, v67, v71\n" \
  "v_cvt_pk_f16_f32 v167, v75, v79\n" \
  "ds_write_b64 v177, v[160:161] offset:0\n" \
  "ds_write_b64 v177, v[162:163] offset:512\n" \
  "ds_write_b64 v177, v[164:165] offset:1024\n" \
  "ds_write_b64 v177, v[166:167] offset:1536\n" \
  "v_cvt_pk_f16_f32 v160, v80, v84\n" \
  "v_cvt_pk_f16_f32 v161, v88, v92\n" \
  "v_cvt_pk_f16_f32 v162, v81, v85\n" \
  "v_cvt_pk_f16_f32 v163, v89, v93\n" \
  "v_cvt_pk_f16_f32 v164, v82, v86\n" \
  "v_cvt_pk_f16_f32 v165, v90, v94\n" \
  "v_cvt_pk_f16_f32 v166, v83, v87\n" \
  "v_cvt_pk_f16_f32 v167, v91, v95\n" \
  "ds_write_b64 v177, v[160:161] offset:8192\n" \
  "ds_write_b64 v177, v[162:163] offset:8704\n" \
  "ds_write_b64 v177, v[164:165] offset:9216\n" \
  "ds_write_b64 v177, v[166:167] offset:9728\n" \
  "s_branch JOIN%=\n" \
  "REMOTE%=:\n" \
  "buffer_load_dwordx4 v[128:131], %11, s[64:67], s76 offen nt\n" \
  "s_add_u32 s76, s76, 262144\n" \
  "buffer_load_dwordx4 v[132:135], %11, s[64:67], s76 offen nt\n" \
  "s_add_u32 s76, s76, 262144\n" \
  "buffer_load_dwordx4 v[136:139], %11, s[64:67], s76 offen nt\n" \
  "s_add_u32 s76, s76, 262144\n" \
  "buffer_load_dwordx4 v[140:143], %11, s[64:67], s76 offen nt\n" \
  "s_add_u32 s76, s76, 3407872\n" \
  "buffer_load_dwordx4 v[144:147], %11, s[64:67], s76 offen nt\n" \
  "s_add_u32 s76, s76, 262144\n" \
  "buffer_load_dwordx4 v[148:151], %11, s[64:67], s76 offen nt\n" \
  "s_add_u32 s76, s76, 262144\n" \
  "buffer_load_dwordx4 v[152:155], %11, s[64:67], s76 offen nt\n" \
  "s_add_u32 s76, s76, 262144\n" \
  "buffer_load_dwordx4 v[156:159], %11, s[64:67], s76 offen nt\n" \
  "s_add_u32 s76, s76, 3407872\n" \
  "s_and_b32 s76, s76, 0x1fffffff\n" \
  "s_waitcnt vmcnt(16)\n" \
  "v_cvt_pk_f16_f32 v160, v64, v68\n" \
  "v_cvt_pk_f16_f32 v161, v72, v76\n" \
  "v_cvt_pk_f16_f32 v162, v65, v69\n" \
  "v_cvt_pk_f16_f32 v163, v73, v77\n" \
  "v_cvt_pk_f16_f32 v164, v66, v70\n" \
  "v_cvt_pk_f16_f32 v165, v74, v78\n" \
  "v_cvt_pk_f16_f32 v166, v67, v71\n" \
  "v_cvt_pk_f16_f32 v167, v75, v79\n" \
  "ds_write_b64 v177, v[160:161] offset:0\n" \
  "ds_write_b64 v177, v[162:163] offset:512\n" \
  "ds_write_b64 v177, v[164:165] offset:1024\n" \
  "ds_write_b64 v177, v[166:167] offset:1536\n" \
  "v_cvt_pk_f16_f32 v160, v80, v84\n" \
  "v_cvt_pk_f16_f32 v161, v88, v92\n" \
  "v_cvt_pk_f16_f32 v162, v81, v85\n" \
  "v_cvt_pk_f16_f32 v163, v89, v93\n" \
  "v_cvt_pk_f16_f32 v164, v82, v86\n" \
  "v_cvt_pk_f16_f32 v165, v90, v94\n" \
  "v_cvt_pk_f16_f32 v166, v83, v87\n" \
  "v_cvt_pk_f16_f32 v167, v91, v95\n" \
  "ds_write_b64 v177, v[160:161] offset:8192\n" \
  "ds_write_b64 v177, v[162:163] offset:8704\n" \
  "ds_write_b64 v177, v[164:165] offset:9216\n" \
  "ds_write_b64 v177, v[166:167] offset:9728\n" \
  "s_waitcnt vmcnt(14)\n" \
  "v_mul_f32 v0, 0xbfb8aa3b, v0\n" \
  "v_mul_f32 v1, 0xbfb8aa3b, v1\n" \
  "v_mul_f32 v2, 0xbfb8aa3b, v2\n" \
  "v_mul_f32 v3, 0xbfb8aa3b, v3\n" \
  "v_mul_f32 v4, 0xbfb8aa3b, v4\n" \
  "v_mul_f32 v5, 0xbfb8aa3b, v5\n" \
  "v_mul_f32 v6, 0xbfb8aa3b, v6\n" \
  "v_mul_f32 v7, 0xbfb8aa3b, v7\n" \
  "v_cvt_pk_f16_f32 v32, v0, v2\n" \
  "v_cvt_pk_f16_f32 v33, v4, v6\n" \
  "v_cvt_pk_f16_f32 v34, v1, v3\n" \
  "v_cvt_pk_f16_f32 v35, v5, v7\n" \
  "ds_write_b128 v181, v[32:35] offset:0\n" \
  "s_waitcnt vmcnt(12)\n" \
  "v_mul_f32 v8, 0xbfb8aa3b, v8\n" \
  "v_mul_f32 v9, 0xbfb8aa3b, v9\n" \
  "v_mul_f32 v10, 0xbfb8aa3b, v10\n" \
  "v_mul_f32 v11, 0xbfb8aa3b, v11\n" \
  "v_mul_f32 v12, 0xbfb8aa3b, v12\n" \
  "v_mul_f32 v13, 0xbfb8aa3b, v13\n" \
  "v_mul_f32 v14, 0xbfb8aa3b, v14\n" \
  "v_mul_f32 v15, 0xbfb8aa3b, v15\n" \
  "v_cvt_pk_f16_f32 v36, v8, v10\n" \
  "v_cvt_pk_f16_f32 v37, v12, v14\n" \
  "v_cvt_pk_f16_f32 v38, v9, v11\n" \
  "v_cvt_pk_f16_f32 v39, v13, v15\n" \
  "ds_write_b128 v181, v[36:39] offset:1024\n" \
  "s_waitcnt vmcnt(10)\n" \
  "v_mul_f32 v16, 0xbfb8aa3b, v16\n" \
  "v_mul_f32 v17, 0xbfb8aa3b, v17\n" \
  "v_mul_f32 v18, 0xbfb8aa3b, v18\n" \
  "v_mul_f32 v19, 0xbfb8aa3b, v19\n" \
  "v_mul_f32 v20, 0xbfb8aa3b, v20\n" \
  "v_mul_f32 v21, 0xbfb8aa3b, v21\n" \
  "v_mul_f32 v22, 0xbfb8aa3b, v22\n" \
  "v_mul_f32 v23, 0xbfb8aa3b, v23\n" \
  "v_cvt_pk_f16_f32 v40, v16, v18\n" \
  "v_cvt_pk_f16_f32 v41, v20, v22\n" \
  "v_cvt_pk_f16_f32 v42, v17, v19\n" \
  "v_cvt_pk_f16_f32 v43, v21, v23\n" \
  "ds_write_b128 v181, v[40:43] offset:2048\n" \
  "s_waitcnt vmcnt(8)\n" \
  "v_mul_f32 v24, 0xbfb8aa3b, v24\n" \
  "v_mul_f32 v25, 0xbfb8aa3b, v25\n" \
  "v_mul_f32 v26, 0xbfb8aa3b, v26\n" \
  "v_mul_f32 v27, 0xbfb8aa3b, v27\n" \
  "v_mul_f32 v28, 0xbfb8aa3b, v28\n" \
  "v_mul_f32 v29, 0xbfb8aa3b, v29\n" \
  "v_mul_f32 v30, 0xbfb8aa3b, v30\n" \
  "v_mul_f32 v31, 0xbfb8aa3b, v31\n" \
  "v_cvt_pk_f16_f32 v44, v24, v26\n" \
  "v_cvt_pk_f16_f32 v45, v28, v30\n" \
  "v_cvt_pk_f16_f32 v46, v25, v27\n" \
  "v_cvt_pk_f16_f32 v47, v29, v31\n" \
  "ds_write_b128 v181, v[44:47] offset:3072\n" \
  "JOIN%=:\n" \
  "s_mov_b64 s[84:85], %5\n" \
  "s_add_u32 s84, s84, 8388608\n" \
  "s_addc_u32 s85, s85, 0\n" \
  "s_mov_b32 s86, 0\n" \
  "s_bitcmp1_b32 s23, 0\n" \
  "s_cselect_b32 s90, 8388608, 0\n" \
  "s_cmp_eq_u32 %10, 0\n" \
  "s_cbranch_scc0 READY%=\n" \
  "s_and_b32 s79, s59, 0x1f8\n" \
  "s_lshl_b32 s79, s79, 4\n" \
  "s_add_u32 s79, s79, 32768\n" \
  "s_mov_b64 vcc, 0\n" \
  "RETRYA%=:\n" \
  "s_bitcmp1_b32 vcc_lo, 0\n" \
  "s_cbranch_scc1 READY%=\n" \
  "s_sleep 1\n" \
  "buffer_load_dword v178, off, s[92:95], s79 sc1\n" \
  "s_movk_i32 s60, 8192\n" \
  "buffer_load_dword v179, off, s[92:95], s60 sc1\n" \
  "s_memrealtime s[62:63]\n" \
  "s_waitcnt vmcnt(0) lgkmcnt(0)\n" \
  "v_cmp_le_u32 vcc, s83, v178\n" \
  "v_readfirstlane_b32 s61, v179\n" \
  "s_sub_u32 s62, s62, s98\n" \
  "s_cmp_gt_u32 s62, 50000000\n" \
  "s_cbranch_scc1 TMOA%=\n" \
  "s_cmp_eq_u32 s61, s96\n" \
  "s_cbranch_scc0 RETRYA%=\n" \
  "TMOA%=:\n" \
  "v_mov_b32 v179, s96\n" \
  "buffer_store_dword v179, v176, s[92:95], s60 offen sc1\n" \
  "s_waitcnt vmcnt(0)\n" \
  "s_mov_b64 vcc, -1\n" \
  "s_branch RETRYA%=\n" \
  "READY%=:\n" \
  "v_mov_b32 v178, 0\n" \
  "s_waitcnt lgkmcnt(0)\n" \
  "s_barrier\n" \
  "buffer_load_dwordx4 v[0:3], %12, s[68:71], s77 offen offset:0 sc0 sc1\n" \
  "buffer_load_dwordx4 v[4:7], v173, s[68:71], s77 offen offset:0 sc0 sc1\n" \
  "buffer_load_dwordx4 v[8:11], %12, s[68:71], s77 offen offset:1024 sc0 sc1\n" \
  "buffer_load_dwordx4 v[12:15], v173, s[68:71], s77 offen offset:1024 sc0 sc1\n" \
  "buffer_load_dwordx4 v[16:19], %12, s[68:71], s77 offen offset:2048 sc0 sc1\n" \
  "buffer_load_dwordx4 v[20:23], v173, s[68:71], s77 offen offset:2048 sc0 sc1\n" \
  "buffer_load_dwordx4 v[24:27], %12, s[68:71], s77 offen offset:3072 sc0 sc1\n" \
  "buffer_load_dwordx4 v[28:31], v173, s[68:71], s77 offen offset:3072 sc0 sc1\n" \
  "buffer_load_dwordx4 v[32:35], v172, s[68:71], s77 offen offset:0 sc0 sc1\n" \
  "buffer_load_dwordx4 v[36:39], v174, s[68:71], s77 offen offset:0 sc0 sc1\n" \
  "buffer_load_dwordx4 v[40:43], v172, s[68:71], s77 offen offset:1024 sc0 sc1\n" \
  "buffer_load_dwordx4 v[44:47], v174, s[68:71], s77 offen offset:1024 sc0 sc1\n" \
  "buffer_load_dwordx4 v[48:51], v172, s[68:71], s77 offen offset:2048 sc0 sc1\n" \
  "buffer_load_dwordx4 v[52:55], v174, s[68:71], s77 offen offset:2048 sc0 sc1\n" \
  "ds_read_b128 v[168:171], v181\n" \
  "ds_read_b128 v[64:67], v183 offset:0\n" \
  "ds_read_b128 v[96:99], v183 offset:8192\n" \
  "ds_read_b128 v[100:103], v183 offset:8448\n" \
  "ds_read_b128 v[68:71], v183 offset:256\n" \
  "ds_read_b128 v[72:75], v183 offset:512\n" \
  "ds_read_b128 v[104:107], v183 offset:8704\n" \
  "ds_read_b128 v[108:111], v183 offset:8960\n" \
  "ds_read_b128 v[76:79], v183 offset:768\n" \
  "ds_read_b128 v[80:83], v183 offset:1024\n" \
  "ds_read_b128 v[112:115], v183 offset:9216\n" \
  "ds_read_b128 v[116:119], v183 offset:9472\n" \
  "ds_read_b128 v[84:87], v183 offset:1280\n" \
  "ds_read_b128 v[88:91], v183 offset:1536\n" \
  "ds_read_b128 v[120:123], v183 offset:9728\n" \
  "ds_read_b128 v[124:127], v183 offset:9984\n" \
  "ds_read_b128 v[92:95], v183 offset:1792\n" \
  "s_mov_b32 s57, 0x4000\n" \
  "s_mov_b32 s58, 0\n" \
  "v_add_u32 v177, s57, %13\n" \
  "s_mov_b32 s78, 0\n" \
  "s_mov_b32 s29, 3968\n" \
  "s_mov_b32 s37, 2496\n" \
  "s_add_u32 s25, s98, 500\n" \
  "s_memtime s[72:73]\n" \
  "s_memrealtime s[62:63]\n" \
  "s_waitcnt lgkmcnt(0)\n" \
  "s_mov_b32 s73, s62\n" \
  "s_mov_b32 s53, 0\n" \
  "s_mov_b32 s28, 4000\n" \
  "s_mov_b32 s52, 0\n" \
  "s_mov_b32 s99, 3200\n" \
  "s_mov_b32 s74, 0\n" \
  "s_mov_b32 s75, s25\n" \
  "s_waitcnt vmcnt(0)\n" \
  "LOOPX%=:\n" \
  "s_waitcnt vmcnt(12) lgkmcnt(8)\n" \
  "v_mfma_f32_16x16x32_f16 a[0:3], v[0:3], v[64:67], a[0:3]\n" \
  "buffer_load_dwordx4 v[56:59], v172, s[68:71], s77 offen offset:3072 sc0 sc1\n" \
  "v_mfma_f32_16x16x32_f16 a[0:3], v[4:7], v[96:99], a[0:3]\n" \
  "buffer_load_dwordx4 v[60:63], v174, s[68:71], s77 offen offset:3072 sc0 sc1\n" \
  "v_mfma_f32_16x16x32_f16 a[4:7], v[4:7], v[100:103], a[4:7]\n" \
  "v_cvt_pk_f16_f32 v160, v128, v132\n" \
  "v_cvt_pk_f16_f32 v161, v136, v140\n" \
  "v_mfma_f32_16x16x32_f16 a[4:7], v[0:3], v[68:71], a[4:7]\n" \
  "ds_write_b64 v177, v[160:161] offset:0\n" \
  "v_mfma_f32_16x16x32_f16 a[8:11], v[0:3], v[72:75], a[8:11]\n" \
  "v_cvt_pk_f16_f32 v162, v129, v133\n" \
  "v_cvt_pk_f16_f32 v163, v137, v141\n" \
  "v_mfma_f32_16x16x32_f16 a[8:11], v[4:7], v[104:107], a[8:11]\n" \
  "ds_write_b64 v177, v[162:163] offset:512\n" \
  "v_mfma_f32_16x16x32_f16 a[12:15], v[4:7], v[108:111], a[12:15]\n" \
  "v_cvt_pk_f16_f32 v164, v130, v134\n" \
  "v_cvt_pk_f16_f32 v165, v138, v142\n" \
  "v_mfma_f32_16x16x32_f16 a[12:15], v[0:3], v[76:79], a[12:15]\n" \
  "ds_write_b64 v177, v[164:165] offset:1024\n" \
  "s_waitcnt lgkmcnt(3)\n" \
  "v_mfma_f32_16x16x32_f16 a[16:19], v[0:3], v[80:83], a[16:19]\n" \
  "v_cvt_pk_f16_f32 v166, v131, v135\n" \
  "v_cvt_pk_f16_f32 v167, v139, v143\n" \
  "v_mfma_f32_16x16x32_f16 a[16:19], v[4:7], v[112:115], a[16:19]\n" \
  "ds_write_b64 v177, v[166:167] offset:1536\n" \
  "v_mfma_f32_16x16x32_f16 a[20:23], v[4:7], v[116:119], a[20:23]\n" \
  "v_cvt_pk_f16_f32 v160, v144, v148\n" \
  "v_cvt_pk_f16_f32 v161, v152, v156\n" \
  "v_mfma_f32_16x16x32_f16 a[20:23], v[0:3], v[84:87], a[20:23]\n" \
  "ds_write_b64 v177, v[160:161] offset:8192\n" \
  "v_mfma_f32_16x16x32_f16 a[24:27], v[0:3], v[88:91], a[24:27]\n" \
  "v_cvt_pk_f16_f32 v162, v145, v149\n" \
  "v_cvt_pk_f16_f32 v163, v153, v157\n" \
  "v_mfma_f32_16x16x32_f16 a[24:27], v[4:7], v[120:123], a[24:27]\n" \
  "ds_write_b64 v177, v[162:163] offset:8704\n" \
  "v_mfma_f32_16x16x32_f16 a[28:31], v[4:7], v[124:127], a[28:31]\n" \
  "v_cvt_pk_f16_f32 v164, v146, v150\n" \
  "v_cvt_pk_f16_f32 v165, v154, v158\n" \
  "v_mfma_f32_16x16x32_f16 a[28:31], v[0:3], v[92:95], a[28:31]\n" \
  "ds_write_b64 v177, v[164:165] offset:9216\n" \
  "s_waitcnt vmcnt(12)\n" \
  "v_mfma_f32_16x16x32_f16 a[32:35], v[8:11], v[64:67], a[32:35]\n" \
  "v_cvt_pk_f16_f32 v166, v147, v151\n" \
  "v_cvt_pk_f16_f32 v167, v155, v159\n" \
  "v_mfma_f32_16x16x32_f16 a[32:35], v[12:15], v[96:99], a[32:35]\n" \
  "ds_write_b64 v177, v[166:167] offset:9728\n" \
  "v_mfma_f32_16x16x32_f16 a[36:39], v[12:15], v[100:103], a[36:39]\n" \
  "buffer_atomic_add v182, v176, s[84:87], s35 offen sc1\n" \
  "v_mfma_f32_16x16x32_f16 a[36:39], v[8:11], v[68:71], a[36:39]\n" \
  "buffer_store_dwordx4 v[168:171], v180, s[88:91], s34 offen sc0 sc1\n" \
  "v_mfma_f32_16x16x32_f16 a[40:43], v[8:11], v[72:75], a[40:43]\n" \
  "buffer_load_dword v175, %16, s[68:71], s55 offen sc0 sc1\n" \
  "v_mfma_f32_16x16x32_f16 a[40:43], v[12:15], v[104:107], a[40:43]\n" \
  "buffer_load_dwordx4 v[128:131], %11, s[64:67], s76 offen nt\n" \
  "v_mfma_f32_16x16x32_f16 a[44:47], v[12:15], v[108:111], a[44:47]\n" \
  "s_add_u32 s76, s76, 262144\n" \
  "v_mfma_f32_16x16x32_f16 a[44:47], v[8:11], v[76:79], a[44:47]\n" \
  "s_add_u32 s57, s57, 0x4000\n" \
  "v_mfma_f32_16x16x32_f16 a[48:51], v[8:11], v[80:83], a[48:51]\n" \
  "buffer_load_dwordx4 v[132:135], %11, s[64:67], s76 offen nt\n" \
  "v_mfma_f32_16x16x32_f16 a[48:51], v[12:15], v[112:115], a[48:51]\n" \
  "s_add_u32 s76, s76, 262144\n" \
  "v_mfma_f32_16x16x32_f16 a[52:55], v[12:15], v[116:119], a[52:55]\n" \
  "s_cmp_eq_u32 s57, 98304\n" \
  "s_cselect_b32 s57, 0, s57\n" \
  "v_mfma_f32_16x16x32_f16 a[52:55], v[8:11], v[84:87], a[52:55]\n" \
  "buffer_load_dwordx4 v[136:139], %11, s[64:67], s76 offen nt\n" \
  "v_mfma_f32_16x16x32_f16 a[56:59], v[8:11], v[88:91], a[56:59]\n" \
  "s_add_u32 s76, s76, 262144\n" \
  "v_mfma_f32_16x16x32_f16 a[56:59], v[12:15], v[120:123], a[56:59]\n" \
  "v_add_u32 v177, s57, %13\n" \
  "v_mfma_f32_16x16x32_f16 a[60:63], v[12:15], v[124:127], a[60:63]\n" \
  "buffer_load_dwordx4 v[140:143], %11, s[64:67], s76 offen nt\n" \
  "v_mfma_f32_16x16x32_f16 a[60:63], v[8:11], v[92:95], a[60:63]\n" \
  "s_add_u32 s76, s76, 3407872\n" \
  "s_waitcnt vmcnt(17)\n" \
  "v_mfma_f32_16x16x32_f16 a[64:67], v[16:19], v[64:67], a[64:67]\n" \
  "s_add_u32 s77, s77, 0x10000\n" \
  "s_and_b32 s77, s77, 0x3fffff\n" \
  "v_mfma_f32_16x16x32_f16 a[64:67], v[20:23], v[96:99], a[64:67]\n" \
  "buffer_load_dwordx4 v[144:147], %11, s[64:67], s76 offen nt\n" \
  "v_mfma_f32_16x16x32_f16 a[68:71], v[20:23], v[100:103], a[68:71]\n" \
  "s_add_u32 s76, s76, 262144\n" \
  "v_mfma_f32_16x16x32_f16 a[68:71], v[16:19], v[68:71], a[68:71]\n" \
  "s_add_u32 s56, s56, 0x10000\n" \
  "s_and_b32 s56, s56, 0x3fffff\n" \
  "v_mfma_f32_16x16x32_f16 a[72:75], v[16:19], v[72:75], a[72:75]\n" \
  "buffer_load_dwordx4 v[148:151], %11, s[64:67], s76 offen nt\n" \
  "v_mfma_f32_16x16x32_f16 a[72:75], v[20:23], v[104:107], a[72:75]\n" \
  "s_add_u32 s76, s76, 262144\n" \
  "v_mfma_f32_16x16x32_f16 a[76:79], v[20:23], v[108:111], a[76:79]\n" \
  "s_cmp_lg_u32 s30, 0\n" \
  "s_cbranch_scc1 SAMPLEX%=\n" \
  "SAMPLEDX%=:\n" \
  "v_mfma_f32_16x16x32_f16 a[76:79], v[16:19], v[76:79], a[76:79]\n" \
  "buffer_load_dwordx4 v[152:155], %11, s[64:67], s76 offen nt\n" \
  "v_mfma_f32_16x16x32_f16 a[80:83], v[16:19], v[80:83], a[80:83]\n" \
  "s_add_u32 s76, s76, 262144\n" \
  "v_mfma_f32_16x16x32_f16 a[80:83], v[20:23], v[112:115], a[80:83]\n" \
  "v_mfma_f32_16x16x32_f16 a[84:87], v[20:23], v[116:119], a[84:87]\n" \
  "buffer_load_dwordx4 v[156:159], %11, s[64:67], s76 offen nt\n" \
  "v_mfma_f32_16x16x32_f16 a[84:87], v[16:19], v[84:87], a[84:87]\n" \
  "s_add_u32 s76, s76, 3407872\n" \
  "s_and_b32 s76, s76, 0x1fffffff\n" \
  "v_mfma_f32_16x16x32_f16 a[88:91], v[16:19], v[88:91], a[88:91]\n" \
  "buffer_load_dwordx4 v[188:191], %11, s[64:67], s76 offen nt\n" \
  "v_mfma_f32_16x16x32_f16 a[88:91], v[20:23], v[120:123], a[88:91]\n" \
  "s_add_u32 s76, s76, 262144\n" \
  "v_mfma_f32_16x16x32_f16 a[92:95], v[20:23], v[124:127], a[92:95]\n" \
  "buffer_load_dwordx4 v[192:195], %11, s[64:67], s76 offen nt\n" \
  "v_mfma_f32_16x16x32_f16 a[92:95], v[16:19], v[92:95], a[92:95]\n" \
  "s_add_u32 s76, s76, 262144\n" \
  "s_waitcnt vmcnt(21)\n" \
  "v_mfma_f32_16x16x32_f16 a[96:99], v[24:27], v[64:67], a[96:99]\n" \
  "buffer_load_dwordx4 v[196:199], %11, s[64:67], s76 offen nt\n" \
  "v_mfma_f32_16x16x32_f16 a[96:99], v[28:31], v[96:99], a[96:99]\n" \
  "s_add_u32 s76, s76, 262144\n" \
  "v_mfma_f32_16x16x32_f16 a[100:103], v[28:31], v[100:103], a[100:103]\n" \
  "buffer_load_dwordx4 v[200:203], %11, s[64:67], s76 offen nt\n" \
  "v_mfma_f32_16x16x32_f16 a[100:103], v[24:27], v[68:71], a[100:103]\n" \
  "s_add_u32 s76, s76, 3407872\n" \
  "v_mfma_f32_16x16x32_f16 a[104:107], v[24:27], v[72:75], a[104:107]\n" \
  "buffer_load_dwordx4 v[204:207], %11, s[64:67], s76 offen nt\n" \
  "v_mfma_f32_16x16x32_f16 a[104:107], v[28:31], v[104:107], a[104:107]\n" \
  "s_add_u32 s76, s76, 262144\n" \
  "v_mfma_f32_16x16x32_f16 a[108:111], v[28:31], v[108:111], a[108:111]\n" \
  "buffer_load_dwordx4 v[208:211], %11, s[64:67], s76 offen nt\n" \
  "v_mfma_f32_16x16x32_f16 a[108:111], v[24:27], v[76:79], a[108:111]\n" \
  "s_add_u32 s76, s76, 262144\n" \
  "v_mfma_f32_16x16x32_f16 a[112:115], v[24:27], v[80:83], a[112:115]\n" \
  "buffer_load_dwordx4 v[212:215], %11, s[64:67], s76 offen nt\n" \
  "v_mfma_f32_16x16x32_f16 a[112:115], v[28:31], v[112:115], a[112:115]\n" \
  "s_add_u32 s76, s76, 262144\n" \
  "v_mfma_f32_16x16x32_f16 a[116:119], v[28:31], v[116:119], a[116:119]\n" \
  "buffer_load_dwordx4 v[216:219], %11, s[64:67], s76 offen nt\n" \
  "v_mfma_f32_16x16x32_f16 a[116:119], v[24:27], v[84:87], a[116:119]\n" \
  "s_memrealtime s[38:39]\n" \
  "s_memtime s[80:81]\n" \
  "s_add_u32 s76, s76, 3407872\n" \
  "s_and_b32 s76, s76, 0x1fffffff\n" \
  "v_mfma_f32_16x16x32_f16 a[120:123], v[24:27], v[88:91], a[120:123]\n" \
  "v_mfma_f32_16x16x32_f16 a[120:123], v[28:31], v[120:123], a[120:123]\n" \
  "v_mfma_f32_16x16x32_f16 a[124:127], v[28:31], v[124:127], a[124:127]\n" \
  "s_waitcnt lgkmcnt(0)\n" \
  "s_sub_u32 s24, s38, s25\n" \
  "s_cmp_lt_i32 s24, 0\n" \
  "s_cbranch_scc1 PACEX0%=\n" \
  "PACEDX0%=:\n" \
  "s_barrier\n" \
  "v_mfma_f32_16x16x32_f16 a[124:127], v[24:27], v[92:95], a[124:127]\n" \
  "s_add_u32 s35, s35, s26\n" \
  "s_lshr_b32 s86, s90, 7\n" \
  "s_waitcnt vmcnt(25)\n" \
  "v_mfma_f32_16x16x32_f16 a[128:131], v[32:35], v[64:67], a[128:131]\n" \
  "s_lshr_b32 s26, s90, 16\n" \
  "s_lshr_b32 s22, s90, 6\n" \
  "v_mfma_f32_16x16x32_f16 a[128:131], v[36:39], v[96:99], a[128:131]\n" \
  "s_add_u32 s34, s34, s22\n" \
  "s_lshr_b32 s22, s90, 13\n" \
  "v_mfma_f32_16x16x32_f16 a[132:135], v[36:39], v[100:103], a[132:135]\n" \
  "v_add_u32 v181, s22, v181\n" \
  "s_lshr_b32 s23, s23, 1\n" \
  "v_mfma_f32_16x16x32_f16 a[132:135], v[32:35], v[68:71], a[132:135]\n" \
  "s_bitcmp1_b32 s23, 0\n" \
  "s_cselect_b32 s90, 8388608, 0\n" \
  "v_mfma_f32_16x16x32_f16 a[136:139], v[32:35], v[72:75], a[136:139]\n" \
  "buffer_load_dwordx4 v[0:3], %12, s[68:71], s77 offen offset:0 sc0 sc1\n" \
  "v_mfma_f32_16x16x32_f16 a[136:139], v[36:39], v[104:107], a[136:139]\n" \
  "ds_read_b128 v[168:171], v181\n" \
  "v_mfma_f32_16x16x32_f16 a[140:143], v[36:39], v[108:111], a[140:143]\n" \
  "s_add_u32 s25, s25, s24\n" \
  "s_sub_i32 s74, s74, 88\n" \
  "v_mfma_f32_16x16x32_f16 a[140:143], v[32:35], v[76:79], a[140:143]\n" \
  "s_sub_u32 s60, s80, s72\n" \
  "s_sub_u32 s61, s38, s73\n" \
  "v_mfma_f32_16x16x32_f16 a[144:147], v[32:35], v[80:83], a[144:147]\n" \
  "buffer_load_dwordx4 v[4:7], v173, s[68:71], s77 offen offset:0 sc0 sc1\n" \
  "v_mfma_f32_16x16x32_f16 a[144:147], v[36:39], v[112:115], a[144:147]\n" \
  "s_mov_b32 s72, s80\n" \
  "s_mov_b32 s73, s38\n" \
  "v_mfma_f32_16x16x32_f16 a[148:151], v[36:39], v[116:119], a[148:151]\n" \
  "s_mul_i32 s62, s60, s99\n" \
  "s_mul_i32 s63, s61, s28\n" \
  "v_mfma_f32_16x16x32_f16 a[148:151], v[32:35], v[84:87], a[148:151]\n" \
  "s_lshr_b32 s63, s63, 1\n" \
  "v_mfma_f32_16x16x32_f16 a[152:155], v[32:35], v[88:91], a[152:155]\n" \
  "buffer_load_dwordx4 v[8:11], %12, s[68:71], s77 offen offset:1024 sc0 sc1\n" \
  "v_mfma_f32_16x16x32_f16 a[152:155], v[36:39], v[120:123], a[152:155]\n" \
  "s_mul_i32 s63, s63, 35\n" \
  "v_mfma_f32_16x16x32_f16 a[156:159], v[36:39], v[124:127], a[156:159]\n" \
  "s_sub_i32 s27, s61, 8\n" \
  "s_min_i32 s27, s27, 190\n" \
  "v_mfma_f32_16x16x32_f16 a[156:159], v[32:35], v[92:95], a[156:159]\n" \
  "s_lshl_b32 s27, s27, 4\n" \
  "s_waitcnt vmcnt(26)\n" \
  "v_mfma_f32_16x16x32_f16 a[160:163], v[40:43], v[64:67], a[160:163]\n" \
  "buffer_load_dwordx4 v[12:15], v173, s[68:71], s77 offen offset:1024 sc0 sc1\n" \
  "v_mfma_f32_16x16x32_f16 a[160:163], v[44:47], v[96:99], a[160:163]\n" \
  "s_cmp_gt_u32 s62, s63\n" \
  "s_cselect_b32 s27, s27, 0\n" \
  "v_mfma_f32_16x16x32_f16 a[164:167], v[44:47], v[100:103], a[164:167]\n" \
  "s_cmp_gt_u32 s62, s63\n" \
  "s_cselect_b32 s24, s24, 0\n" \
  "v_mfma_f32_16x16x32_f16 a[164:167], v[40:43], v[68:71], a[164:167]\n" \
  "s_max_i32 s74, s74, s27\n" \
  "s_sub_i32 s27, s29, 192\n" \
  "v_mfma_f32_16x16x32_f16 a[168:171], v[40:43], v[72:75], a[168:171]\n" \
  "buffer_load_dwordx4 v[16:19], %12, s[68:71], s77 offen offset:2048 sc0 sc1\n" \
  "v_mfma_f32_16x16x32_f16 a[168:171], v[44:47], v[104:107], a[168:171]\n" \
  "s_max_i32 s74, s74, s27\n" \
  "s_add_i32 s52, s52, s24\n" \
  "v_mfma_f32_16x16x32_f16 a[172:175], v[44:47], v[108:111], a[172:175]\n" \
  "s_sub_i32 s27, s74, s29\n" \
  "s_ashr_i32 s27, s27, 4\n" \
  "v_mfma_f32_16x16x32_f16 a[172:175], v[40:43], v[76:79], a[172:175]\n" \
  "s_max_i32 s27, s27, -6\n" \
  "s_add_i32 s52, s52, s27\n" \
  "v_mfma_f32_16x16x32_f16 a[176:179], v[40:43], v[80:83], a[176:179]\n" \
  "buffer_load_dwordx4 v[20:23], v173, s[68:71], s77 offen offset:2048 sc0 sc1\n" \
  "v_mfma_f32_16x16x32_f16 a[176:179], v[44:47], v[112:115], a[176:179]\n" \
  "s_max_i32 s52, s52, 0\n" \
  "s_lshr_b32 s27, s29, 4\n" \
  "v_mfma_f32_16x16x32_f16 a[180:183], v[44:47], v[116:119], a[180:183]\n" \
  "s_add_u32 s75, s75, s27\n" \
  "s_lshr_b32 s27, s74, 4\n" \
  "s_add_u32 s75, s75, 24\n" \
  "s_add_u32 s27, s27, 24\n" \
  "v_mfma_f32_16x16x32_f16 a[180:183], v[40:43], v[84:87], a[180:183]\n" \
  "s_add_u32 s25, s25, s27\n" \
  "s_add_u32 s27, s75, s52\n" \
  "v_mfma_f32_16x16x32_f16 a[184:187], v[40:43], v[88:91], a[184:187]\n" \
  "buffer_load_dwordx4 v[24:27], %12, s[68:71], s77 offen offset:3072 sc0 sc1\n" \
  "v_mfma_f32_16x16x32_f16 a[184:187], v[44:47], v[120:123], a[184:187]\n" \
  "s_sub_u32 s27, s27, s25\n" \
  "s_max_i32 s27, s27, 0\n" \
  "v_mfma_f32_16x16x32_f16 a[188:191], v[44:47], v[124:127], a[188:191]\n" \
  "s_add_u32 s25, s25, s27\n" \
  "s_sub_i32 s29, s29, 84\n" \
  "v_mfma_f32_16x16x32_f16 a[188:191], v[40:43], v[92:95], a[188:191]\n" \
  "s_max_i32 s29, s29, s37\n" \
  "s_waitcnt vmcnt(28)\n" \
  "v_mfma_f32_16x16x32_f16 a[192:195], v[48:51], v[64:67], a[192:195]\n" \
  "buffer_load_dwordx4 v[28:31], v173, s[68:71], s77 offen offset:3072 sc0 sc1\n" \
  "v_mfma_f32_16x16x32_f16 a[192:195], v[52:55], v[96:99], a[192:195]\n" \
  "s_lshr_b32 s27, s56, 17\n" \
  "s_bitcmp1_b32 s31, s27\n" \
  "s_cselect_b32 s55, s56, s77\n" \
  "v_mfma_f32_16x16x32_f16 a[196:199], v[52:55], v[100:103], a[196:199]\n" \
  "s_add_u32 s58, s58, 0x4000\n" \
  "s_cmp_eq_u32 s58, 98304\n" \
  "s_cselect_b32 s58, 0, s58\n" \
  "v_add_u32 v183, s58, %14\n" \
  "v_mfma_f32_16x16x32_f16 a[196:199], v[48:51], v[68:71], a[196:199]\n" \
  "v_mfma_f32_16x16x32_f16 a[200:203], v[48:51], v[72:75], a[200:203]\n" \
  "buffer_load_dwordx4 v[32:35], v172, s[68:71], s77 offen offset:0 sc0 sc1\n" \
  "v_mfma_f32_16x16x32_f16 a[200:203], v[52:55], v[104:107], a[200:203]\n" \
  "v_mfma_f32_16x16x32_f16 a[204:207], v[52:55], v[108:111], a[204:207]\n" \
  "v_mfma_f32_16x16x32_f16 a[204:207], v[48:51], v[76:79], a[204:207]\n" \
  "v_mfma_f32_16x16x32_f16 a[208:211], v[48:51], v[80:83], a[208:211]\n" \
  "buffer_load_dwordx4 v[36:39], v174, s[68:71], s77 offen offset:0 sc0 sc1\n" \
  "v_mfma_f32_16x16x32_f16 a[208:211], v[52:55], v[112:115], a[208:211]\n" \
  "v_mfma_f32_16x16x32_f16 a[212:215], v[52:55], v[116:119], a[212:215]\n" \
  "v_mfma_f32_16x16x32_f16 a[212:215], v[48:51], v[84:87], a[212:215]\n" \
  "v_mfma_f32_16x16x32_f16 a[216:219], v[48:51], v[88:91], a[216:219]\n" \
  "buffer_load_dwordx4 v[40:43], v172, s[68:71], s77 offen offset:1024 sc0 sc1\n" \
  "v_mfma_f32_16x16x32_f16 a[216:219], v[52:55], v[120:123], a[216:219]\n" \
  "v_mfma_f32_16x16x32_f16 a[220:223], v[52:55], v[124:127], a[220:223]\n" \
  "v_mfma_f32_16x16x32_f16 a[220:223], v[48:51], v[92:95], a[220:223]\n" \
  "s_waitcnt vmcnt(30)\n" \
  "v_mfma_f32_16x16x32_f16 a[224:227], v[56:59], v[64:67], a[224:227]\n" \
  "buffer_load_dwordx4 v[44:47], v174, s[68:71], s77 offen offset:1024 sc0 sc1\n" \
  "ds_read_b128 v[64:67], v183 offset:0\n" \
  "v_mfma_f32_16x16x32_f16 a[224:227], v[60:63], v[96:99], a[224:227]\n" \
  "ds_read_b128 v[96:99], v183 offset:8192\n" \
  "v_mfma_f32_16x16x32_f16 a[228:231], v[60:63], v[100:103], a[228:231]\n" \
  "ds_read_b128 v[100:103], v183 offset:8448\n" \
  "v_mfma_f32_16x16x32_f16 a[228:231], v[56:59], v[68:71], a[228:231]\n" \
  "ds_read_b128 v[68:71], v183 offset:256\n" \
  "v_mfma_f32_16x16x32_f16 a[232:235], v[56:59], v[72:75], a[232:235]\n" \
  "buffer_load_dwordx4 v[48:51], v172, s[68:71], s77 offen offset:2048 sc0 sc1\n" \
  "ds_read_b128 v[72:75], v183 offset:512\n" \
  "v_mfma_f32_16x16x32_f16 a[232:235], v[60:63], v[104:107], a[232:235]\n" \
  "ds_read_b128 v[104:107], v183 offset:8704\n" \
  "v_mfma_f32_16x16x32_f16 a[236:239], v[60:63], v[108:111], a[236:239]\n" \
  "ds_read_b128 v[108:111], v183 offset:8960\n" \
  "v_mfma_f32_16x16x32_f16 a[236:239], v[56:59], v[76:79], a[236:239]\n" \
  "ds_read_b128 v[76:79], v183 offset:768\n" \
  "v_mfma_f32_16x16x32_f16 a[240:243], v[56:59], v[80:83], a[240:243]\n" \
  "buffer_load_dwordx4 v[52:55], v174, s[68:71], s77 offen offset:2048 sc0 sc1\n" \
  "ds_read_b128 v[80:83], v183 offset:1024\n" \
  "v_mfma_f32_16x16x32_f16 a[240:243], v[60:63], v[112:115], a[240:243]\n" \
  "ds_read_b128 v[112:115], v183 offset:9216\n" \
  "v_mfma_f32_16x16x32_f16 a[244:247], v[60:63], v[116:119], a[244:247]\n" \
  "ds_read_b128 v[116:119], v183 offset:9472\n" \
  "v_mfma_f32_16x16x32_f16 a[244:247], v[56:59], v[84:87], a[244:247]\n" \
  "ds_read_b128 v[84:87], v183 offset:1280\n" \
  "v_mfma_f32_16x16x32_f16 a[248:251], v[56:59], v[88:91], a[248:251]\n" \
  "ds_read_b128 v[88:91], v183 offset:1536\n" \
  "v_mfma_f32_16x16x32_f16 a[248:251], v[60:63], v[120:123], a[248:251]\n" \
  "ds_read_b128 v[120:123], v183 offset:9728\n" \
  "v_mfma_f32_16x16x32_f16 a[252:255], v[60:63], v[124:127], a[252:255]\n" \
  "ds_read_b128 v[124:127], v183 offset:9984\n" \
  "v_mfma_f32_16x16x32_f16 a[252:255], v[56:59], v[92:95], a[252:255]\n" \
  "ds_read_b128 v[92:95], v183 offset:1792\n" \
  "s_waitcnt vmcnt(12) lgkmcnt(8)\n" \
  "v_mfma_f32_16x16x32_f16 a[0:3], v[0:3], v[64:67], a[0:3]\n" \
  "buffer_load_dwordx4 v[56:59], v172, s[68:71], s77 offen offset:3072 sc0 sc1\n" \
  "v_mfma_f32_16x16x32_f16 a[0:3], v[4:7], v[96:99], a[0:3]\n" \
  "buffer_load_dwordx4 v[60:63], v174, s[68:71], s77 offen offset:3072 sc0 sc1\n" \
  "v_mfma_f32_16x16x32_f16 a[4:7], v[4:7], v[100:103], a[4:7]\n" \
  "v_cvt_pk_f16_f32 v160, v128, v132\n" \
  "v_cvt_pk_f16_f32 v161, v136, v140\n" \
  "v_mfma_f32_16x16x32_f16 a[4:7], v[0:3], v[68:71], a[4:7]\n" \
  "ds_write_b64 v177, v[160:161] offset:0\n" \
  "v_mfma_f32_16x16x32_f16 a[8:11], v[0:3], v[72:75], a[8:11]\n" \
  "v_cvt_pk_f16_f32 v162, v129, v133\n" \
  "v_cvt_pk_f16_f32 v163, v137, v141\n" \
  "v_mfma_f32_16x16x32_f16 a[8:11], v[4:7], v[104:107], a[8:11]\n" \
  "ds_write_b64 v177, v[162:163] offset:512\n" \
  "v_mfma_f32_16x16x32_f16 a[12:15], v[4:7], v[108:111], a[12:15]\n" \
  "v_cvt_pk_f16_f32 v164, v130, v134\n" \
  "v_cvt_pk_f16_f32 v165, v138, v142\n" \
  "v_mfma_f32_16x16x32_f16 a[12:15], v[0:3], v[76:79], a[12:15]\n" \
  "ds_write_b64 v177, v[164:165] offset:1024\n" \
  "s_waitcnt lgkmcnt(3)\n" \
  "v_mfma_f32_16x16x32_f16 a[16:19], v[0:3], v[80:83], a[16:19]\n" \
  "v_cvt_pk_f16_f32 v166, v131, v135\n" \
  "v_cvt_pk_f16_f32 v167, v139, v143\n" \
  "v_mfma_f32_16x16x32_f16 a[16:19], v[4:7], v[112:115], a[16:19]\n" \
  "ds_write_b64 v177, v[166:167] offset:1536\n" \
  "v_mfma_f32_16x16x32_f16 a[20:23], v[4:7], v[116:119], a[20:23]\n" \
  "v_cvt_pk_f16_f32 v160, v144, v148\n" \
  "v_cvt_pk_f16_f32 v161, v152, v156\n" \
  "v_mfma_f32_16x16x32_f16 a[20:23], v[0:3], v[84:87], a[20:23]\n" \
  "ds_write_b64 v177, v[160:161] offset:8192\n" \
  "v_mfma_f32_16x16x32_f16 a[24:27], v[0:3], v[88:91], a[24:27]\n" \
  "v_cvt_pk_f16_f32 v162, v145, v149\n" \
  "v_cvt_pk_f16_f32 v163, v153, v157\n" \
  "v_mfma_f32_16x16x32_f16 a[24:27], v[4:7], v[120:123], a[24:27]\n" \
  "ds_write_b64 v177, v[162:163] offset:8704\n" \
  "v_mfma_f32_16x16x32_f16 a[28:31], v[4:7], v[124:127], a[28:31]\n" \
  "v_cvt_pk_f16_f32 v164, v146, v150\n" \
  "v_cvt_pk_f16_f32 v165, v154, v158\n" \
  "v_mfma_f32_16x16x32_f16 a[28:31], v[0:3], v[92:95], a[28:31]\n" \
  "ds_write_b64 v177, v[164:165] offset:9216\n" \
  "s_waitcnt vmcnt(12)\n" \
  "v_mfma_f32_16x16x32_f16 a[32:35], v[8:11], v[64:67], a[32:35]\n" \
  "v_cvt_pk_f16_f32 v166, v147, v151\n" \
  "v_cvt_pk_f16_f32 v167, v155, v159\n" \
  "v_mfma_f32_16x16x32_f16 a[32:35], v[12:15], v[96:99], a[32:35]\n" \
  "ds_write_b64 v177, v[166:167] offset:9728\n" \
  "v_mfma_f32_16x16x32_f16 a[36:39], v[12:15], v[100:103], a[36:39]\n" \
  "buffer_atomic_add v182, v176, s[84:87], s35 offen sc1\n" \
  "v_mfma_f32_16x16x32_f16 a[36:39], v[8:11], v[68:71], a[36:39]\n" \
  "buffer_store_dwordx4 v[168:171], v180, s[88:91], s34 offen sc0 sc1\n" \
  "v_mfma_f32_16x16x32_f16 a[40:43], v[8:11], v[72:75], a[40:43]\n" \
  "buffer_load_dword v175, %16, s[68:71], s55 offen sc0 sc1\n" \
  "v_mfma_f32_16x16x32_f16 a[40:43], v[12:15], v[104:107], a[40:43]\n" \
  "buffer_load_dwordx4 v[128:131], %11, s[64:67], s76 offen nt\n" \
  "v_mfma_f32_16x16x32_f16 a[44:47], v[12:15], v[108:111], a[44:47]\n" \
  "s_add_u32 s76, s76, 262144\n" \
  "v_mfma_f32_16x16x32_f16 a[44:47], v[8:11], v[76:79], a[44:47]\n" \
  "s_add_u32 s57, s57, 0x4000\n" \
  "v_mfma_f32_16x16x32_f16 a[48:51], v[8:11], v[80:83], a[48:51]\n" \
  "buffer_load_dwordx4 v[132:135], %11, s[64:67], s76 offen nt\n" \
  "v_mfma_f32_16x16x32_f16 a[48:51], v[12:15], v[112:115], a[48:51]\n" \
  "s_add_u32 s76, s76, 262144\n" \
  "v_mfma_f32_16x16x32_f16 a[52:55], v[12:15], v[116:119], a[52:55]\n" \
  "s_cmp_eq_u32 s57, 98304\n" \
  "s_cselect_b32 s57, 0, s57\n" \
  "v_mfma_f32_16x16x32_f16 a[52:55], v[8:11], v[84:87], a[52:55]\n" \
  "buffer_load_dwordx4 v[136:139], %11, s[64:67], s76 offen nt\n" \
  "v_mfma_f32_16x16x32_f16 a[56:59], v[8:11], v[88:91], a[56:59]\n" \
  "s_add_u32 s76, s76, 262144\n" \
  "v_mfma_f32_16x16x32_f16 a[56:59], v[12:15], v[120:123], a[56:59]\n" \
  "v_add_u32 v177, s57, %13\n" \
  "v_mfma_f32_16x16x32_f16 a[60:63], v[12:15], v[124:127], a[60:63]\n" \
  "buffer_load_dwordx4 v[140:143], %11, s[64:67], s76 offen nt\n" \
  "v_mfma_f32_16x16x32_f16 a[60:63], v[8:11], v[92:95], a[60:63]\n" \
  "s_add_u32 s76, s76, 3407872\n" \
  "s_waitcnt vmcnt(17)\n" \
  "v_mfma_f32_16x16x32_f16 a[64:67], v[16:19], v[64:67], a[64:67]\n" \
  "s_add_u32 s77, s77, 0x10000\n" \
  "s_and_b32 s77, s77, 0x3fffff\n" \
  "v_mfma_f32_16x16x32_f16 a[64:67], v[20:23], v[96:99], a[64:67]\n" \
  "buffer_load_dwordx4 v[144:147], %11, s[64:67], s76 offen nt\n" \
  "v_mfma_f32_16x16x32_f16 a[68:71], v[20:23], v[100:103], a[68:71]\n" \
  "s_add_u32 s76, s76, 262144\n" \
  "v_mfma_f32_16x16x32_f16 a[68:71], v[16:19], v[68:71], a[68:71]\n" \
  "s_add_u32 s56, s56, 0x10000\n" \
  "s_and_b32 s56, s56, 0x3fffff\n" \
  "v_mfma_f32_16x16x32_f16 a[72:75], v[16:19], v[72:75], a[72:75]\n" \
  "buffer_load_dwordx4 v[148:151], %11, s[64:67], s76 offen nt\n" \
  "v_mfma_f32_16x16x32_f16 a[72:75], v[20:23], v[104:107], a[72:75]\n" \
  "s_add_u32 s76, s76, 262144\n" \
  "v_mfma_f32_16x16x32_f16 a[76:79], v[20:23], v[108:111], a[76:79]\n" \
  "v_cmp_le_u32 vcc, s83, v178\n" \
  "v_mfma_f32_16x16x32_f16 a[76:79], v[16:19], v[76:79], a[76:79]\n" \
  "buffer_load_dwordx4 v[152:155], %11, s[64:67], s76 offen nt\n" \
  "v_mfma_f32_16x16x32_f16 a[80:83], v[16:19], v[80:83], a[80:83]\n" \
  "s_add_u32 s76, s76, 262144\n" \
  "v_mfma_f32_16x16x32_f16 a[80:83], v[20:23], v[112:115], a[80:83]\n" \
  "s_or_b32 s31, s31, vcc_lo\n" \
  "v_mfma_f32_16x16x32_f16 a[84:87], v[20:23], v[116:119], a[84:87]\n" \
  "buffer_load_dwordx4 v[156:159], %11, s[64:67], s76 offen nt\n" \
  "v_mfma_f32_16x16x32_f16 a[84:87], v[16:19], v[84:87], a[84:87]\n" \
  "s_add_u32 s76, s76, 3407872\n" \
  "s_and_b32 s76, s76, 0x1fffffff\n" \
  "v_mfma_f32_16x16x32_f16 a[88:91], v[16:19], v[88:91], a[88:91]\n" \
  "v_cvt_pk_f16_f32 v160, v188, v192\n" \
  "v_cvt_pk_f16_f32 v161, v196, v200\n" \
  "v_mfma_f32_16x16x32_f16 a[88:91], v[20:23], v[120:123], a[88:91]\n" \
  "ds_write_b64 v177, v[160:161] offset:0\n" \
  "v_mfma_f32_16x16x32_f16 a[92:95], v[20:23], v[124:127], a[92:95]\n" \
  "v_cvt_pk_f16_f32 v162, v189, v193\n" \
  "v_cvt_pk_f16_f32 v163, v197, v201\n" \
  "v_mfma_f32_16x16x32_f16 a[92:95], v[16:19], v[92:95], a[92:95]\n" \
  "ds_write_b64 v177, v[162:163] offset:512\n" \
  "s_waitcnt vmcnt(19)\n" \
  "v_mfma_f32_16x16x32_f16 a[96:99], v[24:27], v[64:67], a[96:99]\n" \
  "v_cvt_pk_f16_f32 v164, v190, v194\n" \
  "v_cvt_pk_f16_f32 v165, v198, v202\n" \
  "v_mfma_f32_16x16x32_f16 a[96:99], v[28:31], v[96:99], a[96:99]\n" \
  "ds_write_b64 v177, v[164:165] offset:1024\n" \
  "v_mfma_f32_16x16x32_f16 a[100:103], v[28:31], v[100:103], a[100:103]\n" \
  "v_cvt_pk_f16_f32 v166, v191, v195\n" \
  "v_cvt_pk_f16_f32 v167, v199, v203\n" \
  "v_mfma_f32_16x16x32_f16 a[100:103], v[24:27], v[68:71], a[100:103]\n" \
  "ds_write_b64 v177, v[166:167] offset:1536\n" \
  "v_mfma_f32_16x16x32_f16 a[104:107], v[24:27], v[72:75], a[104:107]\n" \
  "v_cvt_pk_f16_f32 v160, v204, v208\n" \
  "v_cvt_pk_f16_f32 v161, v212, v216\n" \
  "v_mfma_f32_16x16x32_f16 a[104:107], v[28:31], v[104:107], a[104:107]\n" \
  "ds_write_b64 v177, v[160:161] offset:8192\n" \
  "v_mfma_f32_16x16x32_f16 a[108:111], v[28:31], v[108:111], a[108:111]\n" \
  "v_cvt_pk_f16_f32 v162, v205, v209\n" \
  "v_cvt_pk_f16_f32 v163, v213, v217\n" \
  "v_mfma_f32_16x16x32_f16 a[108:111], v[24:27], v[76:79], a[108:111]\n" \
  "ds_write_b64 v177, v[162:163] offset:8704\n" \
  "v_mfma_f32_16x16x32_f16 a[112:115], v[24:27], v[80:83], a[112:115]\n" \
  "v_cvt_pk_f16_f32 v164, v206, v210\n" \
  "v_cvt_pk_f16_f32 v165, v214, v218\n" \
  "v_mfma_f32_16x16x32_f16 a[112:115], v[28:31], v[112:115], a[112:115]\n" \
  "ds_write_b64 v177, v[164:165] offset:9216\n" \
  "v_mfma_f32_16x16x32_f16 a[116:119], v[28:31], v[116:119], a[116:119]\n" \
  "v_cvt_pk_f16_f32 v166, v207, v211\n" \
  "v_cvt_pk_f16_f32 v167, v215, v219\n" \
  "v_mfma_f32_16x16x32_f16 a[116:119], v[24:27], v[84:87], a[116:119]\n" \
  "s_memrealtime s[38:39]\n" \
  "s_memtime s[80:81]\n" \
  "ds_write_b64 v177, v[166:167] offset:9728\n" \
  "v_mfma_f32_16x16x32_f16 a[120:123], v[24:27], v[88:91], a[120:123]\n" \
  "s_andn2_b32 s60, s82, s31\n" \
  "s_cbranch_scc1 SLOWX%=\n" \
  "DECIDEDX%=:\n" \
  "v_mfma_f32_16x16x32_f16 a[120:123], v[28:31], v[120:123], a[120:123]\n" \
  "v_mfma_f32_16x16x32_f16 a[124:127], v[28:31], v[124:127], a[124:127]\n" \
  "s_waitcnt lgkmcnt(0)\n" \
  "s_sub_u32 s24, s38, s25\n" \
  "s_cmp_lt_i32 s24, 0\n" \
  "s_cbranch_scc1 PACEX1%=\n" \
  "PACEDX1%=:\n" \
  "s_barrier\n" \
  "v_mfma_f32_16x16x32_f16 a[124:127], v[24:27], v[92:95], a[124:127]\n" \
  "s_add_u32 s57, s57, 0x4000\n" \
  "s_cmp_eq_u32 s57, 98304\n" \
  "s_cselect_b32 s57, 0, s57\n" \
  "v_add_u32 v177, s57, %13\n" \
  "s_waitcnt vmcnt(17)\n" \
  "v_mfma_f32_16x16x32_f16 a[128:131], v[32:35], v[64:67], a[128:131]\n" \
  "s_add_u32 s35, s35, s26\n" \
  "s_lshr_b32 s86, s90, 7\n" \
  "v_mfma_f32_16x16x32_f16 a[128:131], v[36:39], v[96:99], a[128:131]\n" \
  "s_lshr_b32 s26, s90, 16\n" \
  "s_lshr_b32 s22, s90, 6\n" \
  "v_mfma_f32_16x16x32_f16 a[132:135], v[36:39], v[100:103], a[132:135]\n" \
  "s_add_u32 s34, s34, s22\n" \
  "s_lshr_b32 s22, s90, 13\n" \
  "v_mfma_f32_16x16x32_f16 a[132:135], v[32:35], v[68:71], a[132:135]\n" \
  "v_add_u32 v181, s22, v181\n" \
  "s_lshr_b32 s23, s23, 1\n" \
  "v_mfma_f32_16x16x32_f16 a[136:139], v[32:35], v[72:75], a[136:139]\n" \
  "buffer_load_dwordx4 v[0:3], %12, s[68:71], s77 offen offset:0 sc0 sc1\n" \
  "v_mfma_f32_16x16x32_f16 a[136:139], v[36:39], v[104:107], a[136:139]\n" \
  "s_bitcmp1_b32 s23, 0\n" \
  "s_cselect_b32 s90, 8388608, 0\n" \
  "v_mfma_f32_16x16x32_f16 a[140:143], v[36:39], v[108:111], a[140:143]\n" \
  "ds_read_b128 v[168:171], v181\n" \
  "v_mfma_f32_16x16x32_f16 a[140:143], v[32:35], v[76:79], a[140:143]\n" \
  "s_add_u32 s25, s25, s24\n" \
  "s_sub_i32 s74, s74, 88\n" \
  "v_mfma_f32_16x16x32_f16 a[144:147], v[32:35], v[80:83], a[144:147]\n" \
  "buffer_load_dwordx4 v[4:7], v173, s[68:71], s77 offen offset:0 sc0 sc1\n" \
  "v_mfma_f32_16x16x32_f16 a[144:147], v[36:39], v[112:115], a[144:147]\n" \
  "s_sub_u32 s60, s80, s72\n" \
  "s_sub_u32 s61, s38, s73\n" \
  "v_mfma_f32_16x16x32_f16 a[148:151], v[36:39], v[116:119], a[148:151]\n" \
  "s_mov_b32 s72, s80\n" \
  "s_mov_b32 s73, s38\n" \
  "v_mfma_f32_16x16x32_f16 a[148:151], v[32:35], v[84:87], a[148:151]\n" \
  "s_mul_i32 s62, s60, s99\n" \
  "s_mul_i32 s63, s61, s28\n" \
  "v_mfma_f32_16x16x32_f16 a[152:155], v[32:35], v[88:91], a[152:155]\n" \
  "buffer_load_dwordx4 v[8:11], %12, s[68:71], s77 offen offset:1024 sc0 sc1\n" \
  "v_mfma_f32_16x16x32_f16 a[152:155], v[36:39], v[120:123], a[152:155]\n" \
  "s_lshr_b32 s63, s63, 1\n" \
  "v_mfma_f32_16x16x32_f16 a[156:159], v[36:39], v[124:127], a[156:159]\n" \
  "s_mul_i32 s63, s63, 35\n" \
  "v_mfma_f32_16x16x32_f16 a[156:159], v[32:35], v[92:95], a[156:159]\n" \
  "s_sub_i32 s27, s61, 8\n" \
  "s_min_i32 s27, s27, 190\n" \
  "s_waitcnt vmcnt(18)\n" \
  "v_mfma_f32_16x16x32_f16 a[160:163], v[40:43], v[64:67], a[160:163]\n" \
  "buffer_load_dwordx4 v[12:15], v173, s[68:71], s77 offen offset:1024 sc0 sc1\n" \
  "v_mfma_f32_16x16x32_f16 a[160:163], v[44:47], v[96:99], a[160:163]\n" \
  "s_lshl_b32 s27, s27, 4\n" \
  "v_mfma_f32_16x16x32_f16 a[164:167], v[44:47], v[100:103], a[164:167]\n" \
  "s_cmp_gt_u32 s62, s63\n" \
  "s_cselect_b32 s27, s27, 0\n" \
  "v_mfma_f32_16x16x32_f16 a[164:167], v[40:43], v[68:71], a[164:167]\n" \
  "s_cmp_gt_u32 s62, s63\n" \
  "s_cselect_b32 s24, s24, 0\n" \
  "v_mfma_f32_16x16x32_f16 a[168:171], v[40:43], v[72:75], a[168:171]\n" \
  "buffer_load_dwordx4 v[16:19], %12, s[68:71], s77 offen offset:2048 sc0 sc1\n" \
  "v_mfma_f32_16x16x32_f16 a[168:171], v[44:47], v[104:107], a[168:171]\n" \
  "s_max_i32 s74, s74, s27\n" \
  "s_sub_i32 s27, s29, 192\n" \
  "v_mfma_f32_16x16x32_f16 a[172:175], v[44:47], v[108:111], a[172:175]\n" \
  "s_max_i32 s74, s74, s27\n" \
  "s_add_i32 s52, s52, s24\n" \
  "v_mfma_f32_16x16x32_f16 a[172:175], v[40:43], v[76:79], a[172:175]\n" \
  "s_sub_i32 s27, s74, s29\n" \
  "s_ashr_i32 s27, s27, 4\n" \
  "v_mfma_f32_16x16x32_f16 a[176:179], v[40:43], v[80:83], a[176:179]\n" \
  "buffer_load_dwordx4 v[20:23], v173, s[68:71], s77 offen offset:2048 sc0 sc1\n" \
  "v_mfma_f32_16x16x32_f16 a[176:179], v[44:47], v[112:115], a[176:179]\n" \
  "s_max_i32 s27, s27, -6\n" \
  "s_add_i32 s52, s52, s27\n" \
  "v_mfma_f32_16x16x32_f16 a[180:183], v[44:47], v[116:119], a[180:183]\n" \
  "s_max_i32 s52, s52, 0\n" \
  "s_lshr_b32 s27, s29, 4\n" \
  "v_mfma_f32_16x16x32_f16 a[180:183], v[40:43], v[84:87], a[180:183]\n" \
  "s_add_u32 s75, s75, s27\n" \
  "s_lshr_b32 s27, s74, 4\n" \
  "s_add_u32 s75, s75, 24\n" \
  "s_add_u32 s27, s27, 24\n" \
  "v_mfma_f32_16x16x32_f16 a[184:187], v[40:43], v[88:91], a[184:187]\n" \
  "buffer_load_dwordx4 v[24:27], %12, s[68:71], s77 offen offset:3072 sc0 sc1\n" \
  "v_mfma_f32_16x16x32_f16 a[184:187], v[44:47], v[120:123], a[184:187]\n" \
  "s_add_u32 s25, s25, s27\n" \
  "s_add_u32 s27, s75, s52\n" \
  "v_mfma_f32_16x16x32_f16 a[188:191], v[44:47], v[124:127], a[188:191]\n" \
  "s_sub_u32 s27, s27, s25\n" \
  "s_max_i32 s27, s27, 0\n" \
  "v_mfma_f32_16x16x32_f16 a[188:191], v[40:43], v[92:95], a[188:191]\n" \
  "s_add_u32 s25, s25, s27\n" \
  "s_sub_i32 s29, s29, 84\n" \
  "s_waitcnt vmcnt(20)\n" \
  "v_mfma_f32_16x16x32_f16 a[192:195], v[48:51], v[64:67], a[192:195]\n" \
  "buffer_load_dwordx4 v[28:31], v173, s[68:71], s77 offen offset:3072 sc0 sc1\n" \
  "v_mfma_f32_16x16x32_f16 a[192:195], v[52:55], v[96:99], a[192:195]\n" \
  "s_max_i32 s29, s29, s37\n" \
  "v_mfma_f32_16x16x32_f16 a[196:199], v[52:55], v[100:103], a[196:199]\n" \
  "s_lshr_b32 s27, s56, 17\n" \
  "s_bitcmp1_b32 s31, s27\n" \
  "s_cselect_b32 s55, s56, s77\n" \
  "v_mfma_f32_16x16x32_f16 a[196:199], v[48:51], v[68:71], a[196:199]\n" \
  "s_add_u32 s58, s58, 0x4000\n" \
  "s_cmp_eq_u32 s58, 98304\n" \
  "s_cselect_b32 s58, 0, s58\n" \
  "v_add_u32 v183, s58, %14\n" \
  "v_mfma_f32_16x16x32_f16 a[200:203], v[48:51], v[72:75], a[200:203]\n" \
  "buffer_load_dwordx4 v[32:35], v172, s[68:71], s77 offen offset:0 sc0 sc1\n" \
  "v_mfma_f32_16x16x32_f16 a[200:203], v[52:55], v[104:107], a[200:203]\n" \
  "s_add_u32 s36, s36, 1\n" \
  "s_lshl_b32 s82, 1, s36\n" \
  "s_cmp_ge_u32 s78, 26\n" \
  "s_cselect_b32 s82, -1, s82\n" \
  "v_mfma_f32_16x16x32_f16 a[204:207], v[52:55], v[108:111], a[204:207]\n" \
  "s_add_u32 s62, s36, 1\n" \
  "s_mov_b32 s30, s82\n" \
  "s_bitset1_b32 s30, s62\n" \
  "s_cmp_ge_u32 s78, 6\n" \
  "s_cselect_b32 s30, -1, s30\n" \
  "s_andn2_b32 s30, s30, s31\n" \
  "s_add_u32 s78, s78, 1\n" \
  "v_mfma_f32_16x16x32_f16 a[204:207], v[48:51], v[76:79], a[204:207]\n" \
  "v_mfma_f32_16x16x32_f16 a[208:211], v[48:51], v[80:83], a[208:211]\n" \
  "buffer_load_dwordx4 v[36:39], v174, s[68:71], s77 offen offset:0 sc0 sc1\n" \
  "v_mfma_f32_16x16x32_f16 a[208:211], v[52:55], v[112:115], a[208:211]\n" \
  "v_mfma_f32_16x16x32_f16 a[212:215], v[52:55], v[116:119], a[212:215]\n" \
  "v_mfma_f32_16x16x32_f16 a[212:215], v[48:51], v[84:87], a[212:215]\n" \
  "v_mfma_f32_16x16x32_f16 a[216:219], v[48:51], v[88:91], a[216:219]\n" \
  "buffer_load_dwordx4 v[40:43], v172, s[68:71], s77 offen offset:1024 sc0 sc1\n" \
  "v_mfma_f32_16x16x32_f16 a[216:219], v[52:55], v[120:123], a[216:219]\n" \
  "v_mfma_f32_16x16x32_f16 a[220:223], v[52:55], v[124:127], a[220:223]\n" \
  "v_mfma_f32_16x16x32_f16 a[220:223], v[48:51], v[92:95], a[220:223]\n" \
  "s_cmp_lt_u32 s78, 4\n" \
  "s_waitcnt vmcnt(22)\n" \
  "v_mfma_f32_16x16x32_f16 a[224:227], v[56:59], v[64:67], a[224:227]\n" \
  "buffer_load_dwordx4 v[44:47], v174, s[68:71], s77 offen offset:1024 sc0 sc1\n" \
  "ds_read_b128 v[64:67], v183 offset:0\n" \
  "v_mfma_f32_16x16x32_f16 a[224:227], v[60:63], v[96:99], a[224:227]\n" \
  "ds_read_b128 v[96:99], v183 offset:8192\n" \
  "v_mfma_f32_16x16x32_f16 a[228:231], v[60:63], v[100:103], a[228:231]\n" \
  "ds_read_b128 v[100:103], v183 offset:8448\n" \
  "v_mfma_f32_16x16x32_f16 a[228:231], v[56:59], v[68:71], a[228:231]\n" \
  "ds_read_b128 v[68:71], v183 offset:256\n" \
  "v_mfma_f32_16x16x32_f16 a[232:235], v[56:59], v[72:75], a[232:235]\n" \
  "buffer_load_dwordx4 v[48:51], v172, s[68:71], s77 offen offset:2048 sc0 sc1\n" \
  "ds_read_b128 v[72:75], v183 offset:512\n" \
  "v_mfma_f32_16x16x32_f16 a[232:235], v[60:63], v[104:107], a[232:235]\n" \
  "ds_read_b128 v[104:107], v183 offset:8704\n" \
  "v_mfma_f32_16x16x32_f16 a[236:239], v[60:63], v[108:111], a[236:239]\n" \
  "ds_read_b128 v[108:111], v183 offset:8960\n" \
  "v_mfma_f32_16x16x32_f16 a[236:239], v[56:59], v[76:79], a[236:239]\n" \
  "ds_read_b128 v[76:79], v183 offset:768\n" \
  "v_mfma_f32_16x16x32_f16 a[240:243], v[56:59], v[80:83], a[240:243]\n" \
  "buffer_load_dwordx4 v[52:55], v174, s[68:71], s77 offen offset:2048 sc0 sc1\n" \
  "ds_read_b128 v[80:83], v183 offset:1024\n" \
  "v_mfma_f32_16x16x32_f16 a[240:243], v[60:63], v[112:115], a[240:243]\n" \
  "ds_read_b128 v[112:115], v183 offset:9216\n" \
  "v_mfma_f32_16x16x32_f16 a[244:247], v[60:63], v[116:119], a[244:247]\n" \
  "ds_read_b128 v[116:119], v183 offset:9472\n" \
  "v_mfma_f32_16x16x32_f16 a[244:247], v[56:59], v[84:87], a[244:247]\n" \
  "ds_read_b128 v[84:87], v183 offset:1280\n" \
  "v_mfma_f32_16x16x32_f16 a[248:251], v[56:59], v[88:91], a[248:251]\n" \
  "ds_read_b128 v[88:91], v183 offset:1536\n" \
  "v_mfma_f32_16x16x32_f16 a[248:251], v[60:63], v[120:123], a[248:251]\n" \
  "ds_read_b128 v[120:123], v183 offset:9728\n" \
  "v_mfma_f32_16x16x32_f16 a[252:255], v[60:63], v[124:127], a[252:255]\n" \
  "ds_read_b128 v[124:127], v183 offset:9984\n" \
  "v_mfma_f32_16x16x32_f16 a[252:255], v[56:59], v[92:95], a[252:255]\n" \
  "ds_read_b128 v[92:95], v183 offset:1792\n" \
  "s_cbranch_scc1 LOOPX%=\n" \
  "s_mov_b32 s54, 29\n" \
  "PICK%=:\n" \
  "s_cmp_lt_u32 s97, 10000\n" \
  "s_cbranch_scc1 LOOPW%=\n" \
  "LOOP%=:\n" \
  "s_waitcnt vmcnt(12) lgkmcnt(8)\n" \
  "v_mfma_f32_16x16x32_f16 a[0:3], v[0:3], v[64:67], a[0:3]\n" \
  "buffer_load_dwordx4 v[56:59], v172, s[68:71], s77 offen offset:3072 sc0 sc1\n" \
  "v_mfma_f32_16x16x32_f16 a[0:3], v[4:7], v[96:99], a[0:3]\n" \
  "buffer_load_dwordx4 v[60:63], v174, s[68:71], s77 offen offset:3072 sc0 sc1\n" \
  "s_cmp_eq_u32 s78, 28\n" \
  "s_cbranch_scc1 TPFL%=\n" \
  "TPFDL%=:\n" \
  "v_mfma_f32_16x16x32_f16 a[4:7], v[4:7], v[100:103], a[4:7]\n" \
  "s_add_u32 s77, s77, 0x10000\n" \
  "s_and_b32 s77, s77, 0x3fffff\n" \
  "v_mfma_f32_16x16x32_f16 a[4:7], v[0:3], v[68:71], a[4:7]\n" \
  "v_mfma_f32_16x16x32_f16 a[8:11], v[0:3], v[72:75], a[8:11]\n" \
  "v_mfma_f32_16x16x32_f16 a[8:11], v[4:7], v[104:107], a[8:11]\n" \
  "buffer_load_dword v175, %16, s[68:71], s55 offen sc0 sc1\n" \
  "v_mfma_f32_16x16x32_f16 a[12:15], v[4:7], v[108:111], a[12:15]\n" \
  "buffer_load_dwordx4 v[188:191], %11, s[64:67], s76 offen nt\n" \
  "v_mfma_f32_16x16x32_f16 a[12:15], v[0:3], v[76:79], a[12:15]\n" \
  "s_add_u32 s76, s76, 262144\n" \
  "s_waitcnt lgkmcnt(0)\n" \
  "v_mfma_f32_16x16x32_f16 a[16:19], v[0:3], v[80:83], a[16:19]\n" \
  "s_add_u32 s56, s56, 0x10000\n" \
  "s_and_b32 s56, s56, 0x3fffff\n" \
  "v_mfma_f32_16x16x32_f16 a[16:19], v[4:7], v[112:115], a[16:19]\n" \
  "s_cmp_lg_u32 s30, 0\n" \
  "s_cbranch_scc1 SAMPLEL%=\n" \
  "SAMPLEDL%=:\n" \
  "v_mfma_f32_16x16x32_f16 a[20:23], v[4:7], v[116:119], a[20:23]\n" \
  "buffer_load_dwordx4 v[192:195], %11, s[64:67], s76 offen nt\n" \
  "v_mfma_f32_16x16x32_f16 a[20:23], v[0:3], v[84:87], a[20:23]\n" \
  "s_add_u32 s76, s76, 262144\n" \
  "v_mfma_f32_16x16x32_f16 a[24:27], v[0:3], v[88:91], a[24:27]\n" \
  "v_mfma_f32_16x16x32_f16 a[24:27], v[4:7], v[120:123], a[24:27]\n" \
  "v_mfma_f32_16x16x32_f16 a[28:31], v[4:7], v[124:127], a[28:31]\n" \
  "buffer_load_dwordx4 v[196:199], %11, s[64:67], s76 offen nt\n" \
  "v_mfma_f32_16x16x32_f16 a[28:31], v[0:3], v[92:95], a[28:31]\n" \
  "s_add_u32 s76, s76, 262144\n" \
  "s_waitcnt vmcnt(16)\n" \
  "v_mfma_f32_16x16x32_f16 a[32:35], v[8:11], v[64:67], a[32:35]\n" \
  "v_mfma_f32_16x16x32_f16 a[32:35], v[12:15], v[96:99], a[32:35]\n" \
  "v_mfma_f32_16x16x32_f16 a[36:39], v[12:15], v[100:103], a[36:39]\n" \
  "buffer_load_dwordx4 v[200:203], %11, s[64:67], s76 offen nt\n" \
  "v_mfma_f32_16x16x32_f16 a[36:39], v[8:11], v[68:71], a[36:39]\n" \
  "s_add_u32 s76, s76, 3407872\n" \
  "v_mfma_f32_16x16x32_f16 a[40:43], v[8:11], v[72:75], a[40:43]\n" \
  "v_mfma_f32_16x16x32_f16 a[40:43], v[12:15], v[104:107], a[40:43]\n" \
  "v_mfma_f32_16x16x32_f16 a[44:47], v[12:15], v[108:111], a[44:47]\n" \
  "buffer_load_dwordx4 v[204:207], %11, s[64:67], s76 offen nt\n" \
  "v_mfma_f32_16x16x32_f16 a[44:47], v[8:11], v[76:79], a[44:47]\n" \
  "s_add_u32 s76, s76, 262144\n" \
  "v_mfma_f32_16x16x32_f16 a[48:51], v[8:11], v[80:83], a[48:51]\n" \
  "v_mfma_f32_16x16x32_f16 a[48:51], v[12:15], v[112:115], a[48:51]\n" \
  "v_mfma_f32_16x16x32_f16 a[52:55], v[12:15], v[116:119], a[52:55]\n" \
  "buffer_load_dwordx4 v[208:211], %11, s[64:67], s76 offen nt\n" \
  "v_mfma_f32_16x16x32_f16 a[52:55], v[8:11], v[84:87], a[52:55]\n" \
  "s_add_u32 s76, s76, 262144\n" \
  "v_mfma_f32_16x16x32_f16 a[56:59], v[8:11], v[88:91], a[56:59]\n" \
  "v_mfma_f32_16x16x32_f16 a[56:59], v[12:15], v[120:123], a[56:59]\n" \
  "v_mfma_f32_16x16x32_f16 a[60:63], v[12:15], v[124:127], a[60:63]\n" \
  "buffer_load_dwordx4 v[212:215], %11, s[64:67], s76 offen nt\n" \
  "v_mfma_f32_16x16x32_f16 a[60:63], v[8:11], v[92:95], a[60:63]\n" \
  "s_add_u32 s76, s76, 262144\n" \
  "s_waitcnt vmcnt(18)\n" \
  "v_mfma_f32_16x16x32_f16 a[64:67], v[16:19], v[64:67], a[64:67]\n" \
  "v_mfma_f32_16x16x32_f16 a[64:67], v[20:23], v[96:99], a[64:67]\n" \
  "v_mfma_f32_16x16x32_f16 a[68:71], v[20:23], v[100:103], a[68:71]\n" \
  "buffer_load_dwordx4 v[216:219], %11, s[64:67], s76 offen nt\n" \
  "v_mfma_f32_16x16x32_f16 a[68:71], v[16:19], v[68:71], a[68:71]\n" \
  "s_add_u32 s76, s76, 3407872\n" \
  "s_and_b32 s76, s76, 0x1fffffff\n" \
  "v_mfma_f32_16x16x32_f16 a[72:75], v[16:19], v[72:75], a[72:75]\n" \
  "v_mfma_f32_16x16x32_f16 a[72:75], v[20:23], v[104:107], a[72:75]\n" \
  "v_mfma_f32_16x16x32_f16 a[76:79], v[20:23], v[108:111], a[76:79]\n" \
  "v_mfma_f32_16x16x32_f16 a[76:79], v[16:19], v[76:79], a[76:79]\n" \
  "v_mfma_f32_16x16x32_f16 a[80:83], v[16:19], v[80:83], a[80:83]\n" \
  "v_cvt_pk_f16_f32 v160, v128, v132\n" \
  "v_cvt_pk_f16_f32 v161, v136, v140\n" \
  "v_mfma_f32_16x16x32_f16 a[80:83], v[20:23], v[112:115], a[80:83]\n" \
  "ds_write_b64 v177, v[160:161] offset:0\n" \
  "v_mfma_f32_16x16x32_f16 a[84:87], v[20:23], v[116:119], a[84:87]\n" \
  "v_cvt_pk_f16_f32 v162, v129, v133\n" \
  "v_cvt_pk_f16_f32 v163, v137, v141\n" \
  "v_mfma_f32_16x16x32_f16 a[84:87], v[16:19], v[84:87], a[84:87]\n" \
  "ds_write_b64 v177, v[162:163] offset:512\n" \
  "v_mfma_f32_16x16x32_f16 a[88:91], v[16:19], v[88:91], a[88:91]\n" \
  "v_cvt_pk_f16_f32 v164, v130, v134\n" \
  "v_cvt_pk_f16_f32 v165, v138, v142\n" \
  "v_mfma_f32_16x16x32_f16 a[88:91], v[20:23], v[120:123], a[88:91]\n" \
  "ds_write_b64 v177, v[164:165] offset:1024\n" \
  "v_mfma_f32_16x16x32_f16 a[92:95], v[20:23], v[124:127], a[92:95]\n" \
  "v_cvt_pk_f16_f32 v166, v131, v135\n" \
  "v_cvt_pk_f16_f32 v167, v139, v143\n" \
  "v_mfma_f32_16x16x32_f16 a[92:95], v[16:19], v[92:95], a[92:95]\n" \
  "ds_write_b64 v177, v[166:167] offset:1536\n" \
  "s_waitcnt vmcnt(17)\n" \
  "v_mfma_f32_16x16x32_f16 a[96:99], v[24:27], v[64:67], a[96:99]\n" \
  "v_cvt_pk_f16_f32 v160, v144, v148\n" \
  "v_cvt_pk_f16_f32 v161, v152, v156\n" \
  "v_mfma_f32_16x16x32_f16 a[96:99], v[28:31], v[96:99], a[96:99]\n" \
  "ds_write_b64 v177, v[160:161] offset:8192\n" \
  "v_mfma_f32_16x16x32_f16 a[100:103], v[28:31], v[100:103], a[100:103]\n" \
  "v_cvt_pk_f16_f32 v162, v145, v149\n" \
  "v_cvt_pk_f16_f32 v163, v153, v157\n" \
  "v_mfma_f32_16x16x32_f16 a[100:103], v[24:27], v[68:71], a[100:103]\n" \
  "ds_write_b64 v177, v[162:163] offset:8704\n" \
  "v_mfma_f32_16x16x32_f16 a[104:107], v[24:27], v[72:75], a[104:107]\n" \
  "v_cvt_pk_f16_f32 v164, v146, v150\n" \
  "v_cvt_pk_f16_f32 v165, v154, v158\n" \
  "v_mfma_f32_16x16x32_f16 a[104:107], v[28:31], v[104:107], a[104:107]\n" \
  "ds_write_b64 v177, v[164:165] offset:9216\n" \
  "v_mfma_f32_16x16x32_f16 a[108:111], v[28:31], v[108:111], a[108:111]\n" \
  "v_cvt_pk_f16_f32 v166, v147, v151\n" \
  "v_cvt_pk_f16_f32 v167, v155, v159\n" \
  "v_mfma_f32_16x16x32_f16 a[108:111], v[24:27], v[76:79], a[108:111]\n" \
  "ds_write_b64 v177, v[166:167] offset:9728\n" \
  "v_mfma_f32_16x16x32_f16 a[112:115], v[24:27], v[80:83], a[112:115]\n" \
  "s_add_u32 s57, s57, 0x4000\n" \
  "v_mfma_f32_16x16x32_f16 a[112:115], v[28:31], v[112:115], a[112:115]\n" \
  "s_cmp_eq_u32 s57, 98304\n" \
  "s_cselect_b32 s57, 0, s57\n" \
  "v_mfma_f32_16x16x32_f16 a[116:119], v[28:31], v[116:119], a[116:119]\n" \
  "v_add_u32 v177, s57, %13\n" \
  "v_mfma_f32_16x16x32_f16 a[116:119], v[24:27], v[84:87], a[116:119]\n" \
  "s_memrealtime s[38:39]\n" \
  "s_memtime s[80:81]\n" \
  "v_mfma_f32_16x16x32_f16 a[120:123], v[24:27], v[88:91], a[120:123]\n" \
  "s_add_u32 s58, s58, 0x4000\n" \
  "v_mfma_f32_16x16x32_f16 a[120:123], v[28:31], v[120:123], a[120:123]\n" \
  "s_cmp_eq_u32 s58, 98304\n" \
  "s_cselect_b32 s58, 0, s58\n" \
  "v_add_u32 v183, s58, %14\n" \
  "v_mfma_f32_16x16x32_f16 a[124:127], v[28:31], v[124:127], a[124:127]\n" \
  "s_waitcnt lgkmcnt(0)\n" \
  "s_sub_u32 s24, s38, s25\n" \
  "s_cmp_lt_i32 s24, 0\n" \
  "s_cbranch_scc1 PACEL0%=\n" \
  "PACEDL0%=:\n" \
  "s_barrier\n" \
  "v_mfma_f32_16x16x32_f16 a[124:127], v[24:27], v[92:95], a[124:127]\n" \
  "s_add_u32 s25, s25, s24\n" \
  "s_sub_i32 s74, s74, 88\n" \
  "s_waitcnt vmcnt(15)\n" \
  "v_mfma_f32_16x16x32_f16 a[128:131], v[32:35], v[64:67], a[128:131]\n" \
  "s_sub_u32 s60, s80, s72\n" \
  "s_sub_u32 s61, s38, s73\n" \
  "v_mfma_f32_16x16x32_f16 a[128:131], v[36:39], v[96:99], a[128:131]\n" \
  "s_mov_b32 s72, s80\n" \
  "s_mov_b32 s73, s38\n" \
  "v_mfma_f32_16x16x32_f16 a[132:135], v[36:39], v[100:103], a[132:135]\n" \
  "s_mul_i32 s62, s60, s99\n" \
  "s_mul_i32 s63, s61, s28\n" \
  "v_mfma_f32_16x16x32_f16 a[132:135], v[32:35], v[68:71], a[132:135]\n" \
  "s_lshr_b32 s63, s63, 1\n" \
  "v_mfma_f32_16x16x32_f16 a[136:139], v[32:35], v[72:75], a[136:139]\n" \
  "buffer_load_dwordx4 v[0:3], %12, s[68:71], s77 offen offset:0 sc0 sc1\n" \
  "v_mfma_f32_16x16x32_f16 a[136:139], v[36:39], v[104:107], a[136:139]\n" \
  "s_lshr_b32 s27, s29, 4\n" \
  "s_sub_i32 s27, s61, s27\n" \
  "v_mfma_f32_16x16x32_f16 a[140:143], v[36:39], v[108:111], a[140:143]\n" \
  "s_sub_i32 s27, s27, 14\n" \
  "s_min_i32 s27, s27, s24\n" \
  "v_mfma_f32_16x16x32_f16 a[140:143], v[32:35], v[76:79], a[140:143]\n" \
  "s_add_u32 s53, s53, 1\n" \
  "s_cmp_gt_i32 s27, 0\n" \
  "s_cselect_b32 s53, s53, 0\n" \
  "v_mfma_f32_16x16x32_f16 a[144:147], v[32:35], v[80:83], a[144:147]\n" \
  "buffer_load_dwordx4 v[4:7], v173, s[68:71], s77 offen offset:0 sc0 sc1\n" \
  "v_mfma_f32_16x16x32_f16 a[144:147], v[36:39], v[112:115], a[144:147]\n" \
  "s_cmp_ge_u32 s53, 3\n" \
  "s_cbranch_scc1 THRL0%=\n" \
  "THRDL0%=:\n" \
  "v_mfma_f32_16x16x32_f16 a[148:151], v[36:39], v[116:119], a[148:151]\n" \
  "s_mul_i32 s63, s63, 34\n" \
  "v_mfma_f32_16x16x32_f16 a[148:151], v[32:35], v[84:87], a[148:151]\n" \
  "s_sub_i32 s27, s61, 8\n" \
  "s_min_i32 s27, s27, 190\n" \
  "v_mfma_f32_16x16x32_f16 a[152:155], v[32:35], v[88:91], a[152:155]\n" \
  "buffer_load_dwordx4 v[8:11], %12, s[68:71], s77 offen offset:1024 sc0 sc1\n" \
  "v_mfma_f32_16x16x32_f16 a[152:155], v[36:39], v[120:123], a[152:155]\n" \
  "s_sub_i32 s60, s60, s28\n" \
  "s_lshl_b32 s61, s61, 4\n" \
  "v_mfma_f32_16x16x32_f16 a[156:159], v[36:39], v[124:127], a[156:159]\n" \
  "s_ashr_i32 s60, s60, 3\n" \
  "s_sub_i32 s61, s61, s99\n" \
  "v_mfma_f32_16x16x32_f16 a[156:159], v[32:35], v[92:95], a[156:159]\n" \
  "s_ashr_i32 s61, s61, 3\n" \
  "s_lshl_b32 s27, s27, 4\n" \
  "s_waitcnt vmcnt(16)\n" \
  "v_mfma_f32_16x16x32_f16 a[160:163], v[40:43], v[64:67], a[160:163]\n" \
  "buffer_load_dwordx4 v[12:15], v173, s[68:71], s77 offen offset:1024 sc0 sc1\n" \
  "v_mfma_f32_16x16x32_f16 a[160:163], v[44:47], v[96:99], a[160:163]\n" \
  "s_cmp_gt_u32 s62, s63\n" \
  "s_cselect_b64 s[60:61], 0, s[60:61]\n" \
  "v_mfma_f32_16x16x32_f16 a[164:167], v[44:47], v[100:103], a[164:167]\n" \
  "s_add_i32 s28, s28, s60\n" \
  "s_add_i32 s99, s99, s61\n" \
  "v_mfma_f32_16x16x32_f16 a[164:167], v[40:43], v[68:71], a[164:167]\n" \
  "s_cmp_gt_u32 s62, s63\n" \
  "s_cselect_b32 s27, s27, 0\n" \
  "v_mfma_f32_16x16x32_f16 a[168:171], v[40:43], v[72:75], a[168:171]\n" \
  "buffer_load_dwordx4 v[16:19], %12, s[68:71], s77 offen offset:2048 sc0 sc1\n" \
  "v_mfma_f32_16x16x32_f16 a[168:171], v[44:47], v[104:107], a[168:171]\n" \
  "s_cmp_gt_u32 s62, s63\n" \
  "s_cselect_b32 s24, s24, 0\n" \
  "v_mfma_f32_16x16x32_f16 a[172:175], v[44:47], v[108:111], a[172:175]\n" \
  "s_max_i32 s74, s74, s27\n" \
  "s_sub_i32 s27, s29, 192\n" \
  "v_mfma_f32_16x16x32_f16 a[172:175], v[40:43], v[76:79], a[172:175]\n" \
  "s_max_i32 s74, s74, s27\n" \
  "s_add_i32 s52, s52, s24\n" \
  "v_mfma_f32_16x16x32_f16 a[176:179], v[40:43], v[80:83], a[176:179]\n" \
  "buffer_load_dwordx4 v[20:23], v173, s[68:71], s77 offen offset:2048 sc0 sc1\n" \
  "v_mfma_f32_16x16x32_f16 a[176:179], v[44:47], v[112:115], a[176:179]\n" \
  "s_sub_i32 s27, s74, s29\n" \
  "s_ashr_i32 s27, s27, 4\n" \
  "v_mfma_f32_16x16x32_f16 a[180:183], v[44:47], v[116:119], a[180:183]\n" \
  "s_max_i32 s27, s27, -6\n" \
  "s_add_i32 s52, s52, s27\n" \
  "v_mfma_f32_16x16x32_f16 a[180:183], v[40:43], v[84:87], a[180:183]\n" \
  "s_max_i32 s52, s52, 0\n" \
  "s_lshr_b32 s27, s29, 4\n" \
  "v_mfma_f32_16x16x32_f16 a[184:187], v[40:43], v[88:91], a[184:187]\n" \
  "buffer_load_dwordx4 v[24:27], %12, s[68:71], s77 offen offset:3072 sc0 sc1\n" \
  "v_mfma_f32_16x16x32_f16 a[184:187], v[44:47], v[120:123], a[184:187]\n" \
  "s_add_u32 s75, s75, s27\n" \
  "s_lshr_b32 s27, s74, 4\n" \
  "v_mfma_f32_16x16x32_f16 a[188:191], v[44:47], v[124:127], a[188:191]\n" \
  "s_add_u32 s25, s25, s27\n" \
  "s_add_u32 s27, s75, s52\n" \
  "v_mfma_f32_16x16x32_f16 a[188:191], v[40:43], v[92:95], a[188:191]\n" \
  "s_sub_u32 s27, s27, s25\n" \
  "s_max_i32 s27, s27, 0\n" \
  "s_waitcnt vmcnt(18)\n" \
  "v_mfma_f32_16x16x32_f16 a[192:195], v[48:51], v[64:67], a[192:195]\n" \
  "buffer_load_dwordx4 v[28:31], v173, s[68:71], s77 offen offset:3072 sc0 sc1\n" \
  "v_mfma_f32_16x16x32_f16 a[192:195], v[52:55], v[96:99], a[192:195]\n" \
  "s_add_u32 s25, s25, s27\n" \
  "s_sub_i32 s29, s29, 84\n" \
  "v_mfma_f32_16x16x32_f16 a[196:199], v[52:55], v[100:103], a[196:199]\n" \
  "s_max_i32 s29, s29, s37\n" \
  "v_mfma_f32_16x16x32_f16 a[196:199], v[48:51], v[68:71], a[196:199]\n" \
  "s_lshr_b32 s27, s56, 17\n" \
  "s_bitcmp1_b32 s31, s27\n" \
  "s_cselect_b32 s55, s56, s77\n" \
  "v_mfma_f32_16x16x32_f16 a[200:203], v[48:51], v[72:75], a[200:203]\n" \
  "buffer_load_dwordx4 v[32:35], v172, s[68:71], s77 offen offset:0 sc0 sc1\n" \
  "v_mfma_f32_16x16x32_f16 a[200:203], v[52:55], v[104:107], a[200:203]\n" \
  "v_mfma_f32_16x16x32_f16 a[204:207], v[52:55], v[108:111], a[204:207]\n" \
  "v_mfma_f32_16x16x32_f16 a[204:207], v[48:51], v[76:79], a[204:207]\n" \
  "v_mfma_f32_16x16x32_f16 a[208:211], v[48:51], v[80:83], a[208:211]\n" \
  "buffer_load_dwordx4 v[36:39], v174, s[68:71], s77 offen offset:0 sc0 sc1\n" \
  "v_mfma_f32_16x16x32_f16 a[208:211], v[52:55], v[112:115], a[208:211]\n" \
  "v_mfma_f32_16x16x32_f16 a[212:215], v[52:55], v[116:119], a[212:215]\n" \
  "v_mfma_f32_16x16x32_f16 a[212:215], v[48:51], v[84:87], a[212:215]\n" \
  "v_mfma_f32_16x16x32_f16 a[216:219], v[48:51], v[88:91], a[216:219]\n" \
  "buffer_load_dwordx4 v[40:43], v172, s[68:71], s77 offen offset:1024 sc0 sc1\n" \
  "v_mfma_f32_16x16x32_f16 a[216:219], v[52:55], v[120:123], a[216:219]\n" \
  "v_mfma_f32_16x16x32_f16 a[220:223], v[52:55], v[124:127], a[220:223]\n" \
  "v_mfma_f32_16x16x32_f16 a[220:223], v[48:51], v[92:95], a[220:223]\n" \
  "s_waitcnt vmcnt(20)\n" \
  "v_mfma_f32_16x16x32_f16 a[224:227], v[56:59], v[64:67], a[224:227]\n" \
  "buffer_load_dwordx4 v[44:47], v174, s[68:71], s77 offen offset:1024 sc0 sc1\n" \
  "ds_read_b128 v[64:67], v183 offset:0\n" \
  "v_mfma_f32_16x16x32_f16 a[224:227], v[60:63], v[96:99], a[224:227]\n" \
  "ds_read_b128 v[96:99], v183 offset:8192\n" \
  "v_mfma_f32_16x16x32_f16 a[228:231], v[60:63], v[100:103], a[228:231]\n" \
  "ds_read_b128 v[100:103], v183 offset:8448\n" \
  "v_mfma_f32_16x16x32_f16 a[228:231], v[56:59], v[68:71], a[228:231]\n" \
  "ds_read_b128 v[68:71], v183 offset:256\n" \
  "v_mfma_f32_16x16x32_f16 a[232:235], v[56:59], v[72:75], a[232:235]\n" \
  "buffer_load_dwordx4 v[48:51], v172, s[68:71], s77 offen offset:2048 sc0 sc1\n" \
  "ds_read_b128 v[72:75], v183 offset:512\n" \
  "v_mfma_f32_16x16x32_f16 a[232:235], v[60:63], v[104:107], a[232:235]\n" \
  "ds_read_b128 v[104:107], v183 offset:8704\n" \
  "v_mfma_f32_16x16x32_f16 a[236:239], v[60:63], v[108:111], a[236:239]\n" \
  "ds_read_b128 v[108:111], v183 offset:8960\n" \
  "v_mfma_f32_16x16x32_f16 a[236:239], v[56:59], v[76:79], a[236:239]\n" \
  "ds_read_b128 v[76:79], v183 offset:768\n" \
  "v_mfma_f32_16x16x32_f16 a[240:243], v[56:59], v[80:83], a[240:243]\n" \
  "buffer_load_dwordx4 v[52:55], v174, s[68:71], s77 offen offset:2048 sc0 sc1\n" \
  "ds_read_b128 v[80:83], v183 offset:1024\n" \
  "v_mfma_f32_16x16x32_f16 a[240:243], v[60:63], v[112:115], a[240:243]\n" \
  "ds_read_b128 v[112:115], v183 offset:9216\n" \
  "v_mfma_f32_16x16x32_f16 a[244:247], v[60:63], v[116:119], a[244:247]\n" \
  "ds_read_b128 v[116:119], v183 offset:9472\n" \
  "v_mfma_f32_16x16x32_f16 a[244:247], v[56:59], v[84:87], a[244:247]\n" \
  "ds_read_b128 v[84:87], v183 offset:1280\n" \
  "v_mfma_f32_16x16x32_f16 a[248:251], v[56:59], v[88:91], a[248:251]\n" \
  "ds_read_b128 v[88:91], v183 offset:1536\n" \
  "v_mfma_f32_16x16x32_f16 a[248:251], v[60:63], v[120:123], a[248:251]\n" \
  "ds_read_b128 v[120:123], v183 offset:9728\n" \
  "v_mfma_f32_16x16x32_f16 a[252:255], v[60:63], v[124:127], a[252:255]\n" \
  "ds_read_b128 v[124:127], v183 offset:9984\n" \
  "v_mfma_f32_16x16x32_f16 a[252:255], v[56:59], v[92:95], a[252:255]\n" \
  "ds_read_b128 v[92:95], v183 offset:1792\n" \
  "s_waitcnt vmcnt(12) lgkmcnt(8)\n" \
  "v_mfma_f32_16x16x32_f16 a[0:3], v[0:3], v[64:67], a[0:3]\n" \
  "buffer_load_dwordx4 v[56:59], v172, s[68:71], s77 offen offset:3072 sc0 sc1\n" \
  "v_mfma_f32_16x16x32_f16 a[0:3], v[4:7], v[96:99], a[0:3]\n" \
  "buffer_load_dwordx4 v[60:63], v174, s[68:71], s77 offen offset:3072 sc0 sc1\n" \
  "v_mfma_f32_16x16x32_f16 a[4:7], v[4:7], v[100:103], a[4:7]\n" \
  "s_add_u32 s77, s77, 0x10000\n" \
  "s_and_b32 s77, s77, 0x3fffff\n" \
  "v_mfma_f32_16x16x32_f16 a[4:7], v[0:3], v[68:71], a[4:7]\n" \
  "v_cmp_le_u32 vcc, s83, v178\n" \
  "v_mfma_f32_16x16x32_f16 a[8:11], v[0:3], v[72:75], a[8:11]\n" \
  "s_or_b32 s31, s31, vcc_lo\n" \
  "v_mfma_f32_16x16x32_f16 a[8:11], v[4:7], v[104:107], a[8:11]\n" \
  "buffer_load_dword v175, %16, s[68:71], s55 offen sc0 sc1\n" \
  "v_mfma_f32_16x16x32_f16 a[12:15], v[4:7], v[108:111], a[12:15]\n" \
  "buffer_load_dwordx4 v[128:131], %11, s[64:67], s76 offen nt\n" \
  "v_mfma_f32_16x16x32_f16 a[12:15], v[0:3], v[76:79], a[12:15]\n" \
  "s_add_u32 s76, s76, 262144\n" \
  "s_waitcnt lgkmcnt(0)\n" \
  "v_mfma_f32_16x16x32_f16 a[16:19], v[0:3], v[80:83], a[16:19]\n" \
  "s_add_u32 s56, s56, 0x10000\n" \
  "s_and_b32 s56, s56, 0x3fffff\n" \
  "v_mfma_f32_16x16x32_f16 a[16:19], v[4:7], v[112:115], a[16:19]\n" \
  "s_andn2_b32 s60, s82, s31\n" \
  "s_cbranch_scc1 SLOWL%=\n" \
  "DECIDEDL%=:\n" \
  "v_mfma_f32_16x16x32_f16 a[20:23], v[4:7], v[116:119], a[20:23]\n" \
  "buffer_load_dwordx4 v[132:135], %11, s[64:67], s76 offen nt\n" \
  "v_mfma_f32_16x16x32_f16 a[20:23], v[0:3], v[84:87], a[20:23]\n" \
  "s_add_u32 s76, s76, 262144\n" \
  "v_mfma_f32_16x16x32_f16 a[24:27], v[0:3], v[88:91], a[24:27]\n" \
  "v_mfma_f32_16x16x32_f16 a[24:27], v[4:7], v[120:123], a[24:27]\n" \
  "v_mfma_f32_16x16x32_f16 a[28:31], v[4:7], v[124:127], a[28:31]\n" \
  "buffer_load_dwordx4 v[136:139], %11, s[64:67], s76 offen nt\n" \
  "v_mfma_f32_16x16x32_f16 a[28:31], v[0:3], v[92:95], a[28:31]\n" \
  "s_add_u32 s76, s76, 262144\n" \
  "s_waitcnt vmcnt(16)\n" \
  "v_mfma_f32_16x16x32_f16 a[32:35], v[8:11], v[64:67], a[32:35]\n" \
  "v_mfma_f32_16x16x32_f16 a[32:35], v[12:15], v[96:99], a[32:35]\n" \
  "v_mfma_f32_16x16x32_f16 a[36:39], v[12:15], v[100:103], a[36:39]\n" \
  "buffer_load_dwordx4 v[140:143], %11, s[64:67], s76 offen nt\n" \
  "v_mfma_f32_16x16x32_f16 a[36:39], v[8:11], v[68:71], a[36:39]\n" \
  "s_add_u32 s76, s76, 3407872\n" \
  "v_mfma_f32_16x16x32_f16 a[40:43], v[8:11], v[72:75], a[40:43]\n" \
  "v_mfma_f32_16x16x32_f16 a[40:43], v[12:15], v[104:107], a[40:43]\n" \
  "v_mfma_f32_16x16x32_f16 a[44:47], v[12:15], v[108:111], a[44:47]\n" \
  "buffer_load_dwordx4 v[144:147], %11, s[64:67], s76 offen nt\n" \
  "v_mfma_f32_16x16x32_f16 a[44:47], v[8:11], v[76:79], a[44:47]\n" \
  "s_add_u32 s76, s76, 262144\n" \
  "v_mfma_f32_16x16x32_f16 a[48:51], v[8:11], v[80:83], a[48:51]\n" \
  "v_mfma_f32_16x16x32_f16 a[48:51], v[12:15], v[112:115], a[48:51]\n" \
  "v_mfma_f32_16x16x32_f16 a[52:55], v[12:15], v[116:119], a[52:55]\n" \
  "buffer_load_dwordx4 v[148:151], %11, s[64:67], s76 offen nt\n" \
  "v_mfma_f32_16x16x32_f16 a[52:55], v[8:11], v[84:87], a[52:55]\n" \
  "s_add_u32 s76, s76, 262144\n" \
  "v_mfma_f32_16x16x32_f16 a[56:59], v[8:11], v[88:91], a[56:59]\n" \
  "v_mfma_f32_16x16x32_f16 a[56:59], v[12:15], v[120:123], a[56:59]\n" \
  "v_mfma_f32_16x16x32_f16 a[60:63], v[12:15], v[124:127], a[60:63]\n" \
  "buffer_load_dwordx4 v[152:155], %11, s[64:67], s76 offen nt\n" \
  "v_mfma_f32_16x16x32_f16 a[60:63], v[8:11], v[92:95], a[60:63]\n" \
  "s_add_u32 s76, s76, 262144\n" \
  "s_waitcnt vmcnt(18)\n" \
  "v_mfma_f32_16x16x32_f16 a[64:67], v[16:19], v[64:67], a[64:67]\n" \
  "v_mfma_f32_16x16x32_f16 a[64:67], v[20:23], v[96:99], a[64:67]\n" \
  "v_mfma_f32_16x16x32_f16 a[68:71], v[20:23], v[100:103], a[68:71]\n" \
  "buffer_load_dwordx4 v[156:159], %11, s[64:67], s76 offen nt\n" \
  "v_mfma_f32_16x16x32_f16 a[68:71], v[16:19], v[68:71], a[68:71]\n" \
  "s_add_u32 s76, s76, 3407872\n" \
  "s_and_b32 s76, s76, 0x1fffffff\n" \
  "v_mfma_f32_16x16x32_f16 a[72:75], v[16:19], v[72:75], a[72:75]\n" \
  "v_mfma_f32_16x16x32_f16 a[72:75], v[20:23], v[104:107], a[72:75]\n" \
  "v_mfma_f32_16x16x32_f16 a[76:79], v[20:23], v[108:111], a[76:79]\n" \
  "v_mfma_f32_16x16x32_f16 a[76:79], v[16:19], v[76:79], a[76:79]\n" \
  "v_mfma_f32_16x16x32_f16 a[80:83], v[16:19], v[80:83], a[80:83]\n" \
  "v_cvt_pk_f16_f32 v160, v188, v192\n" \
  "v_cvt_pk_f16_f32 v161, v196, v200\n" \
  "v_mfma_f32_16x16x32_f16 a[80:83], v[20:23], v[112:115], a[80:83]\n" \
  "ds_write_b64 v177, v[160:161] offset:0\n" \
  "v_mfma_f32_16x16x32_f16 a[84:87], v[20:23], v[116:119], a[84:87]\n" \
  "v_cvt_pk_f16_f32 v162, v189, v193\n" \
  "v_cvt_pk_f16_f32 v163, v197, v201\n" \
  "v_mfma_f32_16x16x32_f16 a[84:87], v[16:19], v[84:87], a[84:87]\n" \
  "ds_write_b64 v177, v[162:163] offset:512\n" \
  "v_mfma_f32_16x16x32_f16 a[88:91], v[16:19], v[88:91], a[88:91]\n" \
  "v_cvt_pk_f16_f32 v164, v190, v194\n" \
  "v_cvt_pk_f16_f32 v165, v198, v202\n" \
  "v_mfma_f32_16x16x32_f16 a[88:91], v[20:23], v[120:123], a[88:91]\n" \
  "ds_write_b64 v177, v[164:165] offset:1024\n" \
  "v_mfma_f32_16x16x32_f16 a[92:95], v[20:23], v[124:127], a[92:95]\n" \
  "v_cvt_pk_f16_f32 v166, v191, v195\n" \
  "v_cvt_pk_f16_f32 v167, v199, v203\n" \
  "v_mfma_f32_16x16x32_f16 a[92:95], v[16:19], v[92:95], a[92:95]\n" \
  "ds_write_b64 v177, v[166:167] offset:1536\n" \
  "s_waitcnt vmcnt(17)\n" \
  "v_mfma_f32_16x16x32_f16 a[96:99], v[24:27], v[64:67], a[96:99]\n" \
  "v_cvt_pk_f16_f32 v160, v204, v208\n" \
  "v_cvt_pk_f16_f32 v161, v212, v216\n" \
  "v_mfma_f32_16x16x32_f16 a[96:99], v[28:31], v[96:99], a[96:99]\n" \
  "ds_write_b64 v177, v[160:161] offset:8192\n" \
  "v_mfma_f32_16x16x32_f16 a[100:103], v[28:31], v[100:103], a[100:103]\n" \
  "v_cvt_pk_f16_f32 v162, v205, v209\n" \
  "v_cvt_pk_f16_f32 v163, v213, v217\n" \
  "v_mfma_f32_16x16x32_f16 a[100:103], v[24:27], v[68:71], a[100:103]\n" \
  "ds_write_b64 v177, v[162:163] offset:8704\n" \
  "v_mfma_f32_16x16x32_f16 a[104:107], v[24:27], v[72:75], a[104:107]\n" \
  "v_cvt_pk_f16_f32 v164, v206, v210\n" \
  "v_cvt_pk_f16_f32 v165, v214, v218\n" \
  "v_mfma_f32_16x16x32_f16 a[104:107], v[28:31], v[104:107], a[104:107]\n" \
  "ds_write_b64 v177, v[164:165] offset:9216\n" \
  "v_mfma_f32_16x16x32_f16 a[108:111], v[28:31], v[108:111], a[108:111]\n" \
  "v_cvt_pk_f16_f32 v166, v207, v211\n" \
  "v_cvt_pk_f16_f32 v167, v215, v219\n" \
  "v_mfma_f32_16x16x32_f16 a[108:111], v[24:27], v[76:79], a[108:111]\n" \
  "ds_write_b64 v177, v[166:167] offset:9728\n" \
  "v_mfma_f32_16x16x32_f16 a[112:115], v[24:27], v[80:83], a[112:115]\n" \
  "s_add_u32 s57, s57, 0x4000\n" \
  "v_mfma_f32_16x16x32_f16 a[112:115], v[28:31], v[112:115], a[112:115]\n" \
  "s_cmp_eq_u32 s57, 98304\n" \
  "s_cselect_b32 s57, 0, s57\n" \
  "v_mfma_f32_16x16x32_f16 a[116:119], v[28:31], v[116:119], a[116:119]\n" \
  "v_add_u32 v177, s57, %13\n" \
  "v_mfma_f32_16x16x32_f16 a[116:119], v[24:27], v[84:87], a[116:119]\n" \
  "s_memrealtime s[38:39]\n" \
  "s_memtime s[80:81]\n" \
  "v_mfma_f32_16x16x32_f16 a[120:123], v[24:27], v[88:91], a[120:123]\n" \
  "s_add_u32 s58, s58, 0x4000\n" \
  "v_mfma_f32_16x16x32_f16 a[120:123], v[28:31], v[120:123], a[120:123]\n" \
  "s_cmp_eq_u32 s58, 98304\n" \
  "s_cselect_b32 s58, 0, s58\n" \
  "v_add_u32 v183, s58, %14\n" \
  "s_add_u32 s36, s36, 1\n" \
  "s_lshl_b32 s82, 1, s36\n" \
  "s_cmp_ge_u32 s78, 26\n" \
  "s_cselect_b32 s82, -1, s82\n" \
  "s_add_u32 s62, s36, 1\n" \
  "s_mov_b32 s30, s82\n" \
  "s_bitset1_b32 s30, s62\n" \
  "s_cmp_ge_u32 s78, 6\n" \
  "s_cselect_b32 s30, -1, s30\n" \
  "s_andn2_b32 s30, s30, s31\n" \
  "v_mfma_f32_16x16x32_f16 a[124:127], v[28:31], v[124:127], a[124:127]\n" \
  "s_waitcnt lgkmcnt(0)\n" \
  "s_sub_u32 s24, s38, s25\n" \
  "s_cmp_lt_i32 s24, 0\n" \
  "s_cbranch_scc1 PACEL1%=\n" \
  "PACEDL1%=:\n" \
  "s_barrier\n" \
  "v_mfma_f32_16x16x32_f16 a[124:127], v[24:27], v[92:95], a[124:127]\n" \
  "s_add_u32 s25, s25, s24\n" \
  "s_sub_i32 s74, s74, 88\n" \
  "s_waitcnt vmcnt(15)\n" \
  "v_mfma_f32_16x16x32_f16 a[128:131], v[32:35], v[64:67], a[128:131]\n" \
  "s_sub_u32 s60, s80, s72\n" \
  "s_sub_u32 s61, s38, s73\n" \
  "v_mfma_f32_16x16x32_f16 a[128:131], v[36:39], v[96:99], a[128:131]\n" \
  "s_mov_b32 s72, s80\n" \
  "s_mov_b32 s73, s38\n" \
  "v_mfma_f32_16x16x32_f16 a[132:135], v[36:39], v[100:103], a[132:135]\n" \
  "s_mul_i32 s62, s60, s99\n" \
  "s_mul_i32 s63, s61, s28\n" \
  "v_mfma_f32_16x16x32_f16 a[132:135], v[32:35], v[68:71], a[132:135]\n" \
  "s_lshr_b32 s63, s63, 1\n" \
  "v_mfma_f32_16x16x32_f16 a[136:139], v[32:35], v[72:75], a[136:139]\n" \
  "buffer_load_dwordx4 v[0:3], %12, s[68:71], s77 offen offset:0 sc0 sc1\n" \
  "v_mfma_f32_16x16x32_f16 a[136:139], v[36:39], v[104:107], a[136:139]\n" \
  "s_lshr_b32 s27, s29, 4\n" \
  "s_sub_i32 s27, s61, s27\n" \
  "v_mfma_f32_16x16x32_f16 a[140:143], v[36:39], v[108:111], a[140:143]\n" \
  "s_sub_i32 s27, s27, 14\n" \
  "s_min_i32 s27, s27, s24\n" \
  "v_mfma_f32_16x16x32_f16 a[140:143], v[32:35], v[76:79], a[140:143]\n" \
  "s_add_u32 s53, s53, 1\n" \
  "s_cmp_gt_i32 s27, 0\n" \
  "s_cselect_b32 s53, s53, 0\n" \
  "v_mfma_f32_16x16x32_f16 a[144:147], v[32:35], v[80:83], a[144:147]\n" \
  "buffer_load_dwordx4 v[4:7], v173, s[68:71], s77 offen offset:0 sc0 sc1\n" \
  "v_mfma_f32_16x16x32_f16 a[144:147], v[36:39], v[112:115], a[144:147]\n" \
  "s_cmp_ge_u32 s53, 3\n" \
  "s_cbranch_scc1 THRL1%=\n" \
  "THRDL1%=:\n" \
  "v_mfma_f32_16x16x32_f16 a[148:151], v[36:39], v[116:119], a[148:151]\n" \
  "s_mul_i32 s63, s63, 34\n" \
  "v_mfma_f32_16x16x32_f16 a[148:151], v[32:35], v[84:87], a[148:151]\n" \
  "s_sub_i32 s27, s61, 8\n" \
  "s_min_i32 s27, s27, 190\n" \
  "v_mfma_f32_16x16x32_f16 a[152:155], v[32:35], v[88:91], a[152:155]\n" \
  "buffer_load_dwordx4 v[8:11], %12, s[68:71], s77 offen offset:1024 sc0 sc1\n" \
  "v_mfma_f32_16x16x32_f16 a[152:155], v[36:39], v[120:123], a[152:155]\n" \
  "s_sub_i32 s60, s60, s28\n" \
  "s_lshl_b32 s61, s61, 4\n" \
  "v_mfma_f32_16x16x32_f16 a[156:159], v[36:39], v[124:127], a[156:159]\n" \
  "s_ashr_i32 s60, s60, 3\n" \
  "s_sub_i32 s61, s61, s99\n" \
  "v_mfma_f32_16x16x32_f16 a[156:159], v[32:35], v[92:95], a[156:159]\n" \
  "s_ashr_i32 s61, s61, 3\n" \
  "s_lshl_b32 s27, s27, 4\n" \
  "s_waitcnt vmcnt(16)\n" \
  "v_mfma_f32_16x16x32_f16 a[160:163], v[40:43], v[64:67], a[160:163]\n" \
  "buffer_load_dwordx4 v[12:15], v173, s[68:71], s77 offen offset:1024 sc0 sc1\n" \
  "v_mfma_f32_16x16x32_f16 a[160:163], v[44:47], v[96:99], a[160:163]\n" \
  "s_cmp_gt_u32 s62, s63\n" \
  "s_cselect_b64 s[60:61], 0, s[60:61]\n" \
  "v_mfma_f32_16x16x32_f16 a[164:167], v[44:47], v[100:103], a[164:167]\n" \
  "s_add_i32 s28, s28, s60\n" \
  "s_add_i32 s99, s99, s61\n" \
  "v_mfma_f32_16x16x32_f16 a[164:167], v[40:43], v[68:71], a[164:167]\n" \
  "s_cmp_gt_u32 s62, s63\n" \
  "s_cselect_b32 s27, s27, 0\n" \
  "v_mfma_f32_16x16x32_f16 a[168:171], v[40:43], v[72:75], a[168:171]\n" \
  "buffer_load_dwordx4 v[16:19], %12, s[68:71], s77 offen offset:2048 sc0 sc1\n" \
  "v_mfma_f32_16x16x32_f16 a[168:171], v[44:47], v[104:107], a[168:171]\n" \
  "s_cmp_gt_u32 s62, s63\n" \
  "s_cselect_b32 s24, s24, 0\n" \
  "v_mfma_f32_16x16x32_f16 a[172:175], v[44:47], v[108:111], a[172:175]\n" \
  "s_max_i32 s74, s74, s27\n" \
  "s_sub_i32 s27, s29, 192\n" \
  "v_mfma_f32_16x16x32_f16 a[172:175], v[40:43], v[76:79], a[172:175]\n" \
  "s_max_i32 s74, s74, s27\n" \
  "s_add_i32 s52, s52, s24\n" \
  "v_mfma_f32_16x16x32_f16 a[176:179], v[40:43], v[80:83], a[176:179]\n" \
  "buffer_load_dwordx4 v[20:23], v173, s[68:71], s77 offen offset:2048 sc0 sc1\n" \
  "v_mfma_f32_16x16x32_f16 a[176:179], v[44:47], v[112:115], a[176:179]\n" \
  "s_sub_i32 s27, s74, s29\n" \
  "s_ashr_i32 s27, s27, 4\n" \
  "v_mfma_f32_16x16x32_f16 a[180:183], v[44:47], v[116:119], a[180:183]\n" \
  "s_max_i32 s27, s27, -6\n" \
  "s_add_i32 s52, s52, s27\n" \
  "v_mfma_f32_16x16x32_f16 a[180:183], v[40:43], v[84:87], a[180:183]\n" \
  "s_max_i32 s52, s52, 0\n" \
  "s_lshr_b32 s27, s29, 4\n" \
  "v_mfma_f32_16x16x32_f16 a[184:187], v[40:43], v[88:91], a[184:187]\n" \
  "buffer_load_dwordx4 v[24:27], %12, s[68:71], s77 offen offset:3072 sc0 sc1\n" \
  "v_mfma_f32_16x16x32_f16 a[184:187], v[44:47], v[120:123], a[184:187]\n" \
  "s_add_u32 s75, s75, s27\n" \
  "s_lshr_b32 s27, s74, 4\n" \
  "v_mfma_f32_16x16x32_f16 a[188:191], v[44:47], v[124:127], a[188:191]\n" \
  "s_add_u32 s25, s25, s27\n" \
  "s_add_u32 s27, s75, s52\n" \
  "v_mfma_f32_16x16x32_f16 a[188:191], v[40:43], v[92:95], a[188:191]\n" \
  "s_sub_u32 s27, s27, s25\n" \
  "s_max_i32 s27, s27, 0\n" \
  "s_waitcnt vmcnt(18)\n" \
  "v_mfma_f32_16x16x32_f16 a[192:195], v[48:51], v[64:67], a[192:195]\n" \
  "buffer_load_dwordx4 v[28:31], v173, s[68:71], s77 offen offset:3072 sc0 sc1\n" \
  "v_mfma_f32_16x16x32_f16 a[192:195], v[52:55], v[96:99], a[192:195]\n" \
  "s_add_u32 s25, s25, s27\n" \
  "s_sub_i32 s29, s29, 84\n" \
  "v_mfma_f32_16x16x32_f16 a[196:199], v[52:55], v[100:103], a[196:199]\n" \
  "s_max_i32 s29, s29, s37\n" \
  "v_mfma_f32_16x16x32_f16 a[196:199], v[48:51], v[68:71], a[196:199]\n" \
  "s_lshr_b32 s27, s56, 17\n" \
  "s_bitcmp1_b32 s31, s27\n" \
  "s_cselect_b32 s55, s56, s77\n" \
  "v_mfma_f32_16x16x32_f16 a[200:203], v[48:51], v[72:75], a[200:203]\n" \
  "buffer_load_dwordx4 v[32:35], v172, s[68:71], s77 offen offset:0 sc0 sc1\n" \
  "v_mfma_f32_16x16x32_f16 a[200:203], v[52:55], v[104:107], a[200:203]\n" \
  "s_add_u32 s78, s78, 1\n" \
  "v_mfma_f32_16x16x32_f16 a[204:207], v[52:55], v[108:111], a[204:207]\n" \
  "v_mfma_f32_16x16x32_f16 a[204:207], v[48:51], v[76:79], a[204:207]\n" \
  "v_mfma_f32_16x16x32_f16 a[208:211], v[48:51], v[80:83], a[208:211]\n" \
  "buffer_load_dwordx4 v[36:39], v174, s[68:71], s77 offen offset:0 sc0 sc1\n" \
  "v_mfma_f32_16x16x32_f16 a[208:211], v[52:55], v[112:115], a[208:211]\n" \
  "v_mfma_f32_16x16x32_f16 a[212:215], v[52:55], v[116:119], a[212:215]\n" \
  "v_mfma_f32_16x16x32_f16 a[212:215], v[48:51], v[84:87], a[212:215]\n" \
  "v_mfma_f32_16x16x32_f16 a[216:219], v[48:51], v[88:91], a[216:219]\n" \
  "buffer_load_dwordx4 v[40:43], v172, s[68:71], s77 offen offset:1024 sc0 sc1\n" \
  "v_mfma_f32_16x16x32_f16 a[216:219], v[52:55], v[120:123], a[216:219]\n" \
  "v_mfma_f32_16x16x32_f16 a[220:223], v[52:55], v[124:127], a[220:223]\n" \
  "v_mfma_f32_16x16x32_f16 a[220:223], v[48:51], v[92:95], a[220:223]\n" \
  "s_cmp_lt_u32 s78, s54\n" \
  "s_waitcnt vmcnt(20)\n" \
  "v_mfma_f32_16x16x32_f16 a[224:227], v[56:59], v[64:67], a[224:227]\n" \
  "buffer_load_dwordx4 v[44:47], v174, s[68:71], s77 offen offset:1024 sc0 sc1\n" \
  "ds_read_b128 v[64:67], v183 offset:0\n" \
  "v_mfma_f32_16x16x32_f16 a[224:227], v[60:63], v[96:99], a[224:227]\n" \
  "ds_read_b128 v[96:99], v183 offset:8192\n" \
  "v_mfma_f32_16x16x32_f16 a[228:231], v[60:63], v[100:103], a[228:231]\n" \
  "ds_read_b128 v[100:103], v183 offset:8448\n" \
  "v_mfma_f32_16x16x32_f16 a[228:231], v[56:59], v[68:71], a[228:231]\n" \
  "ds_read_b128 v[68:71], v183 offset:256\n" \
  "v_mfma_f32_16x16x32_f16 a[232:235], v[56:59], v[72:75], a[232:235]\n" \
  "buffer_load_dwordx4 v[48:51], v172, s[68:71], s77 offen offset:2048 sc0 sc1\n" \
  "ds_read_b128 v[72:75], v183 offset:512\n" \
  "v_mfma_f32_16x16x32_f16 a[232:235], v[60:63], v[104:107], a[232:235]\n" \
  "ds_read_b128 v[104:107], v183 offset:8704\n" \
  "v_mfma_f32_16x16x32_f16 a[236:239], v[60:63], v[108:111], a[236:239]\n" \
  "ds_read_b128 v[108:111], v183 offset:8960\n" \
  "v_mfma_f32_16x16x32_f16 a[236:239], v[56:59], v[76:79], a[236:239]\n" \
  "ds_read_b128 v[76:79], v183 offset:768\n" \
  "v_mfma_f32_16x16x32_f16 a[240:243], v[56:59], v[80:83], a[240:243]\n" \
  "buffer_load_dwordx4 v[52:55], v174, s[68:71], s77 offen offset:2048 sc0 sc1\n" \
  "ds_read_b128 v[80:83], v183 offset:1024\n" \
  "v_mfma_f32_16x16x32_f16 a[240:243], v[60:63], v[112:115], a[240:243]\n" \
  "ds_read_b128 v[112:115], v183 offset:9216\n" \
  "v_mfma_f32_16x16x32_f16 a[244:247], v[60:63], v[116:119], a[244:247]\n" \
  "ds_read_b128 v[116:119], v183 offset:9472\n" \
  "v_mfma_f32_16x16x32_f16 a[244:247], v[56:59], v[84:87], a[244:247]\n" \
  "ds_read_b128 v[84:87], v183 offset:1280\n" \
  "v_mfma_f32_16x16x32_f16 a[248:251], v[56:59], v[88:91], a[248:251]\n" \
  "ds_read_b128 v[88:91], v183 offset:1536\n" \
  "v_mfma_f32_16x16x32_f16 a[248:251], v[60:63], v[120:123], a[248:251]\n" \
  "ds_read_b128 v[120:123], v183 offset:9728\n" \
  "v_mfma_f32_16x16x32_f16 a[252:255], v[60:63], v[124:127], a[252:255]\n" \
  "ds_read_b128 v[124:127], v183 offset:9984\n" \
  "v_mfma_f32_16x16x32_f16 a[252:255], v[56:59], v[92:95], a[252:255]\n" \
  "ds_read_b128 v[92:95], v183 offset:1792\n" \
  "s_cbranch_scc1 LOOP%=\n" \
  "s_branch NEXT%=\n" \
  "PACEX0%=:\n" \
  "s_sleep 1\n" \
  "s_memrealtime s[62:63]\n" \
  "s_waitcnt lgkmcnt(0)\n" \
  "s_sub_u32 s24, s62, s25\n" \
  "s_cmp_lt_i32 s24, 0\n" \
  "s_cbranch_scc1 PACEX0%=\n" \
  "s_mov_b32 s24, 0\n" \
  "s_branch PACEDX0%=\n" \
  "PACEX1%=:\n" \
  "s_sleep 1\n" \
  "s_memrealtime s[62:63]\n" \
  "s_waitcnt lgkmcnt(0)\n" \
  "s_sub_u32 s24, s62, s25\n" \
  "s_cmp_lt_i32 s24, 0\n" \
  "s_cbranch_scc1 PACEX1%=\n" \
  "s_mov_b32 s24, 0\n" \
  "s_branch PACEDX1%=\n" \
  "SAMPLEX%=:\n" \
  "s_mov_b32 exec_lo, s30\n" \
  "s_mov_b32 exec_hi, 0\n" \
  "buffer_load_dword v178, %18, s[92:95], 0 offen sc0 sc1\n" \
  "s_mov_b64 exec, -1\n" \
  "s_branch SAMPLEDX%=\n" \
  "SLOWX%=:\n" \
  "s_sleep 1\n" \
  "buffer_load_dword v178, %18, s[92:95], 0 offen sc0 sc1\n" \
  "s_movk_i32 s60, 8192\n" \
  "buffer_load_dword v179, off, s[92:95], s60 sc1\n" \
  "s_memrealtime s[62:63]\n" \
  "s_waitcnt vmcnt(0) lgkmcnt(0)\n" \
  "v_cmp_le_u32 vcc, s83, v178\n" \
  "v_readfirstlane_b32 s61, v179\n" \
  "s_sub_u32 s62, s62, s98\n" \
  "s_cmp_gt_u32 s62, 50000000\n" \
  "s_cbranch_scc1 TMOX%=\n" \
  "s_cmp_eq_u32 s61, s96\n" \
  "s_cbranch_scc0 RETRYX%=\n" \
  "TMOX%=:\n" \
  "v_mov_b32 v179, s96\n" \
  "buffer_store_dword v179, v176, s[92:95], s60 offen sc1\n" \
  "s_waitcnt vmcnt(0)\n" \
  "s_mov_b64 vcc, -1\n" \
  "s_branch RETRYX%=\n" \
  "RETRYX%=:\n" \
  "s_or_b32 s31, s31, vcc_lo\n" \
  "s_andn2_b32 s60, s82, s31\n" \
  "s_cbranch_scc1 SLOWX%=\n" \
  "s_branch DECIDEDX%=\n" \
  "PACEL0%=:\n" \
  "s_sleep 1\n" \
  "s_memrealtime s[62:63]\n" \
  "s_waitcnt lgkmcnt(0)\n" \
  "s_sub_u32 s24, s62, s25\n" \
  "s_cmp_lt_i32 s24, 0\n" \
  "s_cbranch_scc1 PACEL0%=\n" \
  "s_mov_b32 s24, 0\n" \
  "s_branch PACEDL0%=\n" \
  "THRL0%=:\n" \
  "s_sub_u32 s27, s25, s75\n" \
  "s_max_i32 s52, s52, s27\n" \
  "s_mov_b32 s24, 0\n" \
  "s_max_i32 s74, s74, 2880\n" \
  "s_cmp_eq_u32 s53, 3\n" \
  "s_cselect_b32 s27, 48, 0\n" \
  "s_add_u32 s37, s37, s27\n" \
  "s_branch THRDL0%=\n" \
  "PACEL1%=:\n" \
  "s_sleep 1\n" \
  "s_memrealtime s[62:63]\n" \
  "s_waitcnt lgkmcnt(0)\n" \
  "s_sub_u32 s24, s62, s25\n" \
  "s_cmp_lt_i32 s24, 0\n" \
  "s_cbranch_scc1 PACEL1%=\n" \
  "s_mov_b32 s24, 0\n" \
  "s_branch PACEDL1%=\n" \
  "THRL1%=:\n" \
  "s_sub_u32 s27, s25, s75\n" \
  "s_max_i32 s52, s52, s27\n" \
  "s_mov_b32 s24, 0\n" \
  "s_max_i32 s74, s74, 2880\n" \
  "s_cmp_eq_u32 s53, 3\n" \
  "s_cselect_b32 s27, 48, 0\n" \
  "s_add_u32 s37, s37, s27\n" \
  "s_branch THRDL1%=\n" \
  "TPFL%=:\n" \
  "s_getpc_b64 s[84:85]\n" \
  "TPBL%=:\n" \
  "s_add_u32 s84, s84, TAIL%=-TPBL%=\n" \
  "s_addc_u32 s85, s85, 0\n" \
  "s_mov_b32 s86, CPFE%=-TAIL%=\n" \
  "buffer_load_dword v175, %16, s[84:87], 0 offen\n" \
  "s_add_u32 s60, s56, 131072\n" \
  "buffer_load_dword v175, %16, s[68:71], s60 offen sc0 sc1\n" \
  "s_add_u32 s60, s56, 196608\n" \
  "buffer_load_dword v175, %16, s[68:71], s60 offen sc0 sc1\n" \
  "s_add_u32 s60, s56, 262144\n" \
  "buffer_load_dword v175, %16, s[68:71], s60 offen sc0 sc1\n" \
  "s_add_u32 s60, s56, 327680\n" \
  "buffer_load_dword v175, %16, s[68:71], s60 offen sc0 sc1\n" \
  "s_branch TPFDL%=\n" \
  "SAMPLEL%=:\n" \
  "s_mov_b32 exec_lo, s30\n" \
  "s_mov_b32 exec_hi, 0\n" \
  "buffer_load_dword v178, %18, s[92:95], 0 offen sc0 sc1\n" \
  "s_mov_b64 exec, -1\n" \
  "s_branch SAMPLEDL%=\n" \
  "SLOWL%=:\n" \
  "s_sleep 1\n" \
  "buffer_load_dword v178, %18, s[92:95], 0 offen sc0 sc1\n" \
  "s_movk_i32 s60, 8192\n" \
  "buffer_load_dword v179, off, s[92:95], s60 sc1\n" \
  "s_memrealtime s[62:63]\n" \
  "s_waitcnt vmcnt(0) lgkmcnt(0)\n" \
  "v_cmp_le_u32 vcc, s83, v178\n" \
  "v_readfirstlane_b32 s61, v179\n" \
  "s_sub_u32 s62, s62, s98\n" \
  "s_cmp_gt_u32 s62, 50000000\n" \
  "s_cbranch_scc1 TMOL%=\n" \
  "s_cmp_eq_u32 s61, s96\n" \
  "s_cbranch_scc0 RETRYL%=\n" \
  "TMOL%=:\n" \
  "v_mov_b32 v179, s96\n" \
  "buffer_store_dword v179, v176, s[92:95], s60 offen sc1\n" \
  "s_waitcnt vmcnt(0)\n" \
  "s_mov_b64 vcc, -1\n" \
  "s_branch RETRYL%=\n" \
  "RETRYL%=:\n" \
  "s_or_b32 s31, s31, vcc_lo\n" \
  "s_andn2_b32 s60, s82, s31\n" \
  "s_cbranch_scc1 SLOWL%=\n" \
  "s_branch DECIDEDL%=\n" \
  "PACEW0%=:\n" \
  "s_sleep 1\n" \
  "s_memrealtime s[62:63]\n" \
  "s_waitcnt lgkmcnt(0)\n" \
  "s_sub_u32 s24, s62, s25\n" \
  "s_cmp_lt_i32 s24, 0\n" \
  "s_cbranch_scc1 PACEW0%=\n" \
  "s_mov_b32 s24, 0\n" \
  "s_branch PACEDW0%=\n" \
  "THRW0%=:\n" \
  "s_sub_u32 s27, s25, s75\n" \
  "s_max_i32 s52, s52, s27\n" \
  "s_mov_b32 s24, 0\n" \
  "s_max_i32 s74, s74, 2880\n" \
  "s_cmp_eq_u32 s53, 3\n" \
  "s_cselect_b32 s27, 48, 0\n" \
  "s_add_u32 s37, s37, s27\n" \
  "s_branch THRDW0%=\n" \
  "PACEW1%=:\n" \
  "s_sleep 1\n" \
  "s_memrealtime s[62:63]\n" \
  "s_waitcnt lgkmcnt(0)\n" \
  "s_sub_u32 s24, s62, s25\n" \
  "s_cmp_lt_i32 s24, 0\n" \
  "s_cbranch_scc1 PACEW1%=\n" \
  "s_mov_b32 s24, 0\n" \
  "s_branch PACEDW1%=\n" \
  "THRW1%=:\n" \
  "s_sub_u32 s27, s25, s75\n" \
  "s_max_i32 s52, s52, s27\n" \
  "s_mov_b32 s24, 0\n" \
  "s_max_i32 s74, s74, 2880\n" \
  "s_cmp_eq_u32 s53, 3\n" \
  "s_cselect_b32 s27, 48, 0\n" \
  "s_add_u32 s37, s37, s27\n" \
  "s_branch THRDW1%=\n" \
  "TPFW%=:\n" \
  "s_getpc_b64 s[84:85]\n" \
  "TPBW%=:\n" \
  "s_add_u32 s84, s84, TAIL%=-TPBW%=\n" \
  "s_addc_u32 s85, s85, 0\n" \
  "s_mov_b32 s86, CPFE%=-TAIL%=\n" \
  "buffer_load_dword v175, %16, s[84:87], 0 offen\n" \
  "s_add_u32 s60, s56, 131072\n" \
  "buffer_load_dword v175, %16, s[68:71], s60 offen sc0 sc1\n" \
  "s_add_u32 s60, s56, 196608\n" \
  "buffer_load_dword v175, %16, s[68:71], s60 offen sc0 sc1\n" \
  "s_add_u32 s60, s56, 262144\n" \
  "buffer_load_dword v175, %16, s[68:71], s60 offen sc0 sc1\n" \
  "s_add_u32 s60, s56, 327680\n" \
  "buffer_load_dword v175, %16, s[68:71], s60 offen sc0 sc1\n" \
  "s_branch TPFDW%=\n" \
  "SAMPLEW%=:\n" \
  "s_mov_b32 exec_lo, s30\n" \
  "s_mov_b32 exec_hi, 0\n" \
  "buffer_load_dword v178, %18, s[92:95], 0 offen sc0 sc1\n" \
  "s_mov_b64 exec, -1\n" \
  "s_branch SAMPLEDW%=\n" \
  "SLOWW%=:\n" \
  "s_sleep 1\n" \
  "buffer_load_dword v178, %18, s[92:95], 0 offen sc0 sc1\n" \
  "s_movk_i32 s60, 8192\n" \
  "buffer_load_dword v179, off, s[92:95], s60 sc1\n" \
  "s_memrealtime s[62:63]\n" \
  "s_waitcnt vmcnt(0) lgkmcnt(0)\n" \
  "v_cmp_le_u32 vcc, s83, v178\n" \
  "v_readfirstlane_b32 s61, v179\n" \
  "s_sub_u32 s62, s62, s98\n" \
  "s_cmp_gt_u32 s62, 50000000\n" \
  "s_cbranch_scc1 TMOW%=\n" \
  "s_cmp_eq_u32 s61, s96\n" \
  "s_cbranch_scc0 RETRYW%=\n" \
  "TMOW%=:\n" \
  "v_mov_b32 v179, s96\n" \
  "buffer_store_dword v179, v176, s[92:95], s60 offen sc1\n" \
  "s_waitcnt vmcnt(0)\n" \
  "s_mov_b64 vcc, -1\n" \
  "s_branch RETRYW%=\n" \
  "RETRYW%=:\n" \
  "s_or_b32 s31, s31, vcc_lo\n" \
  "s_andn2_b32 s60, s82, s31\n" \
  "s_cbranch_scc1 SLOWW%=\n" \
  "s_branch DECIDEDW%=\n" \
  "LOOPW%=:\n" \
  "s_waitcnt vmcnt(12) lgkmcnt(8)\n" \
  "v_mfma_f32_16x16x32_f16 a[0:3], v[0:3], v[64:67], a[0:3]\n" \
  "buffer_load_dwordx4 v[56:59], v172, s[68:71], s77 offen offset:3072 sc0 sc1\n" \
  "v_mfma_f32_16x16x32_f16 a[0:3], v[4:7], v[96:99], a[0:3]\n" \
  "buffer_load_dwordx4 v[60:63], v174, s[68:71], s77 offen offset:3072 sc0 sc1\n" \
  "s_cmp_eq_u32 s78, 28\n" \
  "s_cbranch_scc1 TPFW%=\n" \
  "TPFDW%=:\n" \
  "v_mfma_f32_16x16x32_f16 a[4:7], v[4:7], v[100:103], a[4:7]\n" \
  "s_add_u32 s77, s77, 0x10000\n" \
  "s_and_b32 s77, s77, 0x3fffff\n" \
  "v_mfma_f32_16x16x32_f16 a[4:7], v[0:3], v[68:71], a[4:7]\n" \
  "v_mfma_f32_16x16x32_f16 a[8:11], v[0:3], v[72:75], a[8:11]\n" \
  "v_mfma_f32_16x16x32_f16 a[8:11], v[4:7], v[104:107], a[8:11]\n" \
  "buffer_load_dword v175, %16, s[68:71], s55 offen sc0 sc1\n" \
  "v_mfma_f32_16x16x32_f16 a[12:15], v[4:7], v[108:111], a[12:15]\n" \
  "buffer_load_dwordx4 v[188:191], %11, s[64:67], s76 offen\n" \
  "v_mfma_f32_16x16x32_f16 a[12:15], v[0:3], v[76:79], a[12:15]\n" \
  "s_add_u32 s76, s76, 262144\n" \
  "s_waitcnt lgkmcnt(0)\n" \
  "v_mfma_f32_16x16x32_f16 a[16:19], v[0:3], v[80:83], a[16:19]\n" \
  "s_add_u32 s56, s56, 0x10000\n" \
  "s_and_b32 s56, s56, 0x3fffff\n" \
  "v_mfma_f32_16x16x32_f16 a[16:19], v[4:7], v[112:115], a[16:19]\n" \
  "s_cmp_lg_u32 s30, 0\n" \
  "s_cbranch_scc1 SAMPLEW%=\n" \
  "SAMPLEDW%=:\n" \
  "v_mfma_f32_16x16x32_f16 a[20:23], v[4:7], v[116:119], a[20:23]\n" \
  "buffer_load_dwordx4 v[192:195], %11, s[64:67], s76 offen nt\n" \
  "v_mfma_f32_16x16x32_f16 a[20:23], v[0:3], v[84:87], a[20:23]\n" \
  "s_add_u32 s76, s76, 262144\n" \
  "v_mfma_f32_16x16x32_f16 a[24:27], v[0:3], v[88:91], a[24:27]\n" \
  "v_mfma_f32_16x16x32_f16 a[24:27], v[4:7], v[120:123], a[24:27]\n" \
  "v_mfma_f32_16x16x32_f16 a[28:31], v[4:7], v[124:127], a[28:31]\n" \
  "buffer_load_dwordx4 v[196:199], %11, s[64:67], s76 offen\n" \
  "v_mfma_f32_16x16x32_f16 a[28:31], v[0:3], v[92:95], a[28:31]\n" \
  "s_add_u32 s76, s76, 262144\n" \
  "s_waitcnt vmcnt(16)\n" \
  "v_mfma_f32_16x16x32_f16 a[32:35], v[8:11], v[64:67], a[32:35]\n" \
  "v_mfma_f32_16x16x32_f16 a[32:35], v[12:15], v[96:99], a[32:35]\n" \
  "v_mfma_f32_16x16x32_f16 a[36:39], v[12:15], v[100:103], a[36:39]\n" \
  "buffer_load_dwordx4 v[200:203], %11, s[64:67], s76 offen nt\n" \
  "v_mfma_f32_16x16x32_f16 a[36:39], v[8:11], v[68:71], a[36:39]\n" \
  "s_add_u32 s76, s76, 3407872\n" \
  "v_mfma_f32_16x16x32_f16 a[40:43], v[8:11], v[72:75], a[40:43]\n" \
  "v_mfma_f32_16x16x32_f16 a[40:43], v[12:15], v[104:107], a[40:43]\n" \
  "v_mfma_f32_16x16x32_f16 a[44:47], v[12:15], v[108:111], a[44:47]\n" \
  "buffer_load_dwordx4 v[204:207], %11, s[64:67], s76 offen\n" \
  "v_mfma_f32_16x16x32_f16 a[44:47], v[8:11], v[76:79], a[44:47]\n" \
  "s_add_u32 s76, s76, 262144\n" \
  "v_mfma_f32_16x16x32_f16 a[48:51], v[8:11], v[80:83], a[48:51]\n" \
  "v_mfma_f32_16x16x32_f16 a[48:51], v[12:15], v[112:115], a[48:51]\n" \
  "v_mfma_f32_16x16x32_f16 a[52:55], v[12:15], v[116:119], a[52:55]\n" \
  "buffer_load_dwordx4 v[208:211], %11, s[64:67], s76 offen nt\n" \
  "v_mfma_f32_16x16x32_f16 a[52:55], v[8:11], v[84:87], a[52:55]\n" \
  "s_add_u32 s76, s76, 262144\n" \
  "v_mfma_f32_16x16x32_f16 a[56:59], v[8:11], v[88:91], a[56:59]\n" \
  "v_mfma_f32_16x16x32_f16 a[56:59], v[12:15], v[120:123], a[56:59]\n" \
  "v_mfma_f32_16x16x32_f16 a[60:63], v[12:15], v[124:127], a[60:63]\n" \
  "buffer_load_dwordx4 v[212:215], %11, s[64:67], s76 offen\n" \
  "v_mfma_f32_16x16x32_f16 a[60:63], v[8:11], v[92:95], a[60:63]\n" \
  "s_add_u32 s76, s76, 262144\n" \
  "s_waitcnt vmcnt(18)\n" \
  "v_mfma_f32_16x16x32_f16 a[64:67], v[16:19], v[64:67], a[64:67]\n" \
  "v_mfma_f32_16x16x32_f16 a[64:67], v[20:23], v[96:99], a[64:67]\n" \
  "v_mfma_f32_16x16x32_f16 a[68:71], v[20:23], v[100:103], a[68:71]\n" \
  "buffer_load_dwordx4 v[216:219], %11, s[64:67], s76 offen nt\n" \
  "v_mfma_f32_16x16x32_f16 a[68:71], v[16:19], v[68:71], a[68:71]\n" \
  "s_add_u32 s76, s76, 3407872\n" \
  "s_and_b32 s76, s76, 0x1fffffff\n" \
  "v_mfma_f32_16x16x32_f16 a[72:75], v[16:19], v[72:75], a[72:75]\n" \
  "v_mfma_f32_16x16x32_f16 a[72:75], v[20:23], v[104:107], a[72:75]\n" \
  "v_mfma_f32_16x16x32_f16 a[76:79], v[20:23], v[108:111], a[76:79]\n" \
  "v_mfma_f32_16x16x32_f16 a[76:79], v[16:19], v[76:79], a[76:79]\n" \
  "v_mfma_f32_16x16x32_f16 a[80:83], v[16:19], v[80:83], a[80:83]\n" \
  "v_cvt_pk_f16_f32 v160, v128, v132\n" \
  "v_cvt_pk_f16_f32 v161, v136, v140\n" \
  "v_mfma_f32_16x16x32_f16 a[80:83], v[20:23], v[112:115], a[80:83]\n" \
  "ds_write_b64 v177, v[160:161] offset:0\n" \
  "v_mfma_f32_16x16x32_f16 a[84:87], v[20:23], v[116:119], a[84:87]\n" \
  "v_cvt_pk_f16_f32 v162, v129, v133\n" \
  "v_cvt_pk_f16_f32 v163, v137, v141\n" \
  "v_mfma_f32_16x16x32_f16 a[84:87], v[16:19], v[84:87], a[84:87]\n" \
  "ds_write_b64 v177, v[162:163] offset:512\n" \
  "v_mfma_f32_16x16x32_f16 a[88:91], v[16:19], v[88:91], a[88:91]\n" \
  "v_cvt_pk_f16_f32 v164, v130, v134\n" \
  "v_cvt_pk_f16_f32 v165, v138, v142\n" \
  "v_mfma_f32_16x16x32_f16 a[88:91], v[20:23], v[120:123], a[88:91]\n" \
  "ds_write_b64 v177, v[164:165] offset:1024\n" \
  "v_mfma_f32_16x16x32_f16 a[92:95], v[20:23], v[124:127], a[92:95]\n" \
  "v_cvt_pk_f16_f32 v166, v131, v135\n" \
  "v_cvt_pk_f16_f32 v167, v139, v143\n" \
  "v_mfma_f32_16x16x32_f16 a[92:95], v[16:19], v[92:95], a[92:95]\n" \
  "ds_write_b64 v177, v[166:167] offset:1536\n" \
  "s_waitcnt vmcnt(17)\n" \
  "v_mfma_f32_16x16x32_f16 a[96:99], v[24:27], v[64:67], a[96:99]\n" \
  "v_cvt_pk_f16_f32 v160, v144, v148\n" \
  "v_cvt_pk_f16_f32 v161, v152, v156\n" \
  "v_mfma_f32_16x16x32_f16 a[96:99], v[28:31], v[96:99], a[96:99]\n" \
  "ds_write_b64 v177, v[160:161] offset:8192\n" \
  "v_mfma_f32_16x16x32_f16 a[100:103], v[28:31], v[100:103], a[100:103]\n" \
  "v_cvt_pk_f16_f32 v162, v145, v149\n" \
  "v_cvt_pk_f16_f32 v163, v153, v157\n" \
  "v_mfma_f32_16x16x32_f16 a[100:103], v[24:27], v[68:71], a[100:103]\n" \
  "ds_write_b64 v177, v[162:163] offset:8704\n" \
  "v_mfma_f32_16x16x32_f16 a[104:107], v[24:27], v[72:75], a[104:107]\n" \
  "v_cvt_pk_f16_f32 v164, v146, v150\n" \
  "v_cvt_pk_f16_f32 v165, v154, v158\n" \
  "v_mfma_f32_16x16x32_f16 a[104:107], v[28:31], v[104:107], a[104:107]\n" \
  "ds_write_b64 v177, v[164:165] offset:9216\n" \
  "v_mfma_f32_16x16x32_f16 a[108:111], v[28:31], v[108:111], a[108:111]\n" \
  "v_cvt_pk_f16_f32 v166, v147, v151\n" \
  "v_cvt_pk_f16_f32 v167, v155, v159\n" \
  "v_mfma_f32_16x16x32_f16 a[108:111], v[24:27], v[76:79], a[108:111]\n" \
  "ds_write_b64 v177, v[166:167] offset:9728\n" \
  "v_mfma_f32_16x16x32_f16 a[112:115], v[24:27], v[80:83], a[112:115]\n" \
  "s_add_u32 s57, s57, 0x4000\n" \
  "v_mfma_f32_16x16x32_f16 a[112:115], v[28:31], v[112:115], a[112:115]\n" \
  "s_cmp_eq_u32 s57, 98304\n" \
  "s_cselect_b32 s57, 0, s57\n" \
  "v_mfma_f32_16x16x32_f16 a[116:119], v[28:31], v[116:119], a[116:119]\n" \
  "v_add_u32 v177, s57, %13\n" \
  "v_mfma_f32_16x16x32_f16 a[116:119], v[24:27], v[84:87], a[116:119]\n" \
  "s_memrealtime s[38:39]\n" \
  "s_memtime s[80:81]\n" \
  "v_mfma_f32_16x16x32_f16 a[120:123], v[24:27], v[88:91], a[120:123]\n" \
  "s_add_u32 s58, s58, 0x4000\n" \
  "v_mfma_f32_16x16x32_f16 a[120:123], v[28:31], v[120:123], a[120:123]\n" \
  "s_cmp_eq_u32 s58, 98304\n" \
  "s_cselect_b32 s58, 0, s58\n" \
  "v_add_u32 v183, s58, %14\n" \
  "v_mfma_f32_16x16x32_f16 a[124:127], v[28:31], v[124:127], a[124:127]\n" \
  "s_waitcnt lgkmcnt(0)\n" \
  "s_sub_u32 s24, s38, s25\n" \
  "s_cmp_lt_i32 s24, 0\n" \
  "s_cbranch_scc1 PACEW0%=\n" \
  "PACEDW0%=:\n" \
  "s_barrier\n" \
  "v_mfma_f32_16x16x32_f16 a[124:127], v[24:27], v[92:95], a[124:127]\n" \
  "s_add_u32 s25, s25, s24\n" \
  "s_sub_i32 s74, s74, 88\n" \
  "s_waitcnt vmcnt(15)\n" \
  "v_mfma_f32_16x16x32_f16 a[128:131], v[32:35], v[64:67], a[128:131]\n" \
  "s_sub_u32 s60, s80, s72\n" \
  "s_sub_u32 s61, s38, s73\n" \
  "v_mfma_f32_16x16x32_f16 a[128:131], v[36:39], v[96:99], a[128:131]\n" \
  "s_mov_b32 s72, s80\n" \
  "s_mov_b32 s73, s38\n" \
  "v_mfma_f32_16x16x32_f16 a[132:135], v[36:39], v[100:103], a[132:135]\n" \
  "s_mul_i32 s62, s60, s99\n" \
  "s_mul_i32 s63, s61, s28\n" \
  "v_mfma_f32_16x16x32_f16 a[132:135], v[32:35], v[68:71], a[132:135]\n" \
  "s_lshr_b32 s63, s63, 1\n" \
  "v_mfma_f32_16x16x32_f16 a[136:139], v[32:35], v[72:75], a[136:139]\n" \
  "buffer_load_dwordx4 v[0:3], %12, s[68:71], s77 offen offset:0 sc0 sc1\n" \
  "v_mfma_f32_16x16x32_f16 a[136:139], v[36:39], v[104:107], a[136:139]\n" \
  "s_lshr_b32 s27, s29, 4\n" \
  "s_sub_i32 s27, s61, s27\n" \
  "v_mfma_f32_16x16x32_f16 a[140:143], v[36:39], v[108:111], a[140:143]\n" \
  "s_sub_i32 s27, s27, 14\n" \
  "s_min_i32 s27, s27, s24\n" \
  "v_mfma_f32_16x16x32_f16 a[140:143], v[32:35], v[76:79], a[140:143]\n" \
  "s_add_u32 s53, s53, 1\n" \
  "s_cmp_gt_i32 s27, 0\n" \
  "s_cselect_b32 s53, s53, 0\n" \
  "v_mfma_f32_16x16x32_f16 a[144:147], v[32:35], v[80:83], a[144:147]\n" \
  "buffer_load_dwordx4 v[4:7], v173, s[68:71], s77 offen offset:0 sc0 sc1\n" \
  "v_mfma_f32_16x16x32_f16 a[144:147], v[36:39], v[112:115], a[144:147]\n" \
  "s_cmp_ge_u32 s53, 3\n" \
  "s_cbranch_scc1 THRW0%=\n" \
  "THRDW0%=:\n" \
  "v_mfma_f32_16x16x32_f16 a[148:151], v[36:39], v[116:119], a[148:151]\n" \
  "s_mul_i32 s63, s63, 34\n" \
  "v_mfma_f32_16x16x32_f16 a[148:151], v[32:35], v[84:87], a[148:151]\n" \
  "s_sub_i32 s27, s61, 8\n" \
  "s_min_i32 s27, s27, 190\n" \
  "v_mfma_f32_16x16x32_f16 a[152:155], v[32:35], v[88:91], a[152:155]\n" \
  "buffer_load_dwordx4 v[8:11], %12, s[68:71], s77 offen offset:1024 sc0 sc1\n" \
  "v_mfma_f32_16x16x32_f16 a[152:155], v[36:39], v[120:123], a[152:155]\n" \
  "s_sub_i32 s60, s60, s28\n" \
  "s_lshl_b32 s61, s61, 4\n" \
  "v_mfma_f32_16x16x32_f16 a[156:159], v[36:39], v[124:127], a[156:159]\n" \
  "s_ashr_i32 s60, s60, 3\n" \
  "s_sub_i32 s61, s61, s99\n" \
  "v_mfma_f32_16x16x32_f16 a[156:159], v[32:35], v[92:95], a[156:159]\n" \
  "s_ashr_i32 s61, s61, 3\n" \
  "s_lshl_b32 s27, s27, 4\n" \
  "s_waitcnt vmcnt(16)\n" \
  "v_mfma_f32_16x16x32_f16 a[160:163], v[40:43], v[64:67], a[160:163]\n" \
  "buffer_load_dwordx4 v[12:15], v173, s[68:71], s77 offen offset:1024 sc0 sc1\n" \
  "v_mfma_f32_16x16x32_f16 a[160:163], v[44:47], v[96:99], a[160:163]\n" \
  "s_cmp_gt_u32 s62, s63\n" \
  "s_cselect_b64 s[60:61], 0, s[60:61]\n" \
  "v_mfma_f32_16x16x32_f16 a[164:167], v[44:47], v[100:103], a[164:167]\n" \
  "s_add_i32 s28, s28, s60\n" \
  "s_add_i32 s99, s99, s61\n" \
  "v_mfma_f32_16x16x32_f16 a[164:167], v[40:43], v[68:71], a[164:167]\n" \
  "s_cmp_gt_u32 s62, s63\n" \
  "s_cselect_b32 s27, s27, 0\n" \
  "v_mfma_f32_16x16x32_f16 a[168:171], v[40:43], v[72:75], a[168:171]\n" \
  "buffer_load_dwordx4 v[16:19], %12, s[68:71], s77 offen offset:2048 sc0 sc1\n" \
  "v_mfma_f32_16x16x32_f16 a[168:171], v[44:47], v[104:107], a[168:171]\n" \
  "s_cmp_gt_u32 s62, s63\n" \
  "s_cselect_b32 s24, s24, 0\n" \
  "v_mfma_f32_16x16x32_f16 a[172:175], v[44:47], v[108:111], a[172:175]\n" \
  "s_max_i32 s74, s74, s27\n" \
  "s_sub_i32 s27, s29, 192\n" \
  "v_mfma_f32_16x16x32_f16 a[172:175], v[40:43], v[76:79], a[172:175]\n" \
  "s_max_i32 s74, s74, s27\n" \
  "s_add_i32 s52, s52, s24\n" \
  "v_mfma_f32_16x16x32_f16 a[176:179], v[40:43], v[80:83], a[176:179]\n" \
  "buffer_load_dwordx4 v[20:23], v173, s[68:71], s77 offen offset:2048 sc0 sc1\n" \
  "v_mfma_f32_16x16x32_f16 a[176:179], v[44:47], v[112:115], a[176:179]\n" \
  "s_sub_i32 s27, s74, s29\n" \
  "s_ashr_i32 s27, s27, 4\n" \
  "v_mfma_f32_16x16x32_f16 a[180:183], v[44:47], v[116:119], a[180:183]\n" \
  "s_max_i32 s27, s27, -6\n" \
  "s_add_i32 s52, s52, s27\n" \
  "v_mfma_f32_16x16x32_f16 a[180:183], v[40:43], v[84:87], a[180:183]\n" \
  "s_max_i32 s52, s52, 0\n" \
  "s_lshr_b32 s27, s29, 4\n" \
  "v_mfma_f32_16x16x32_f16 a[184:187], v[40:43], v[88:91], a[184:187]\n" \
  "buffer_load_dwordx4 v[24:27], %12, s[68:71], s77 offen offset:3072 sc0 sc1\n" \
  "v_mfma_f32_16x16x32_f16 a[184:187], v[44:47], v[120:123], a[184:187]\n" \
  "s_add_u32 s75, s75, s27\n" \
  "s_lshr_b32 s27, s74, 4\n" \
  "v_mfma_f32_16x16x32_f16 a[188:191], v[44:47], v[124:127], a[188:191]\n" \
  "s_add_u32 s25, s25, s27\n" \
  "s_add_u32 s27, s75, s52\n" \
  "v_mfma_f32_16x16x32_f16 a[188:191], v[40:43], v[92:95], a[188:191]\n" \
  "s_sub_u32 s27, s27, s25\n" \
  "s_max_i32 s27, s27, 0\n" \
  "s_waitcnt vmcnt(18)\n" \
  "v_mfma_f32_16x16x32_f16 a[192:195], v[48:51], v[64:67], a[192:195]\n" \
  "buffer_load_dwordx4 v[28:31], v173, s[68:71], s77 offen offset:3072 sc0 sc1\n" \
  "v_mfma_f32_16x16x32_f16 a[192:195], v[52:55], v[96:99], a[192:195]\n" \
  "s_add_u32 s25, s25, s27\n" \
  "s_sub_i32 s29, s29, 84\n" \
  "v_mfma_f32_16x16x32_f16 a[196:199], v[52:55], v[100:103], a[196:199]\n" \
  "s_max_i32 s29, s29, s37\n" \
  "v_mfma_f32_16x16x32_f16 a[196:199], v[48:51], v[68:71], a[196:199]\n" \
  "s_lshr_b32 s27, s56, 17\n" \
  "s_bitcmp1_b32 s31, s27\n" \
  "s_cselect_b32 s55, s56, s77\n" \
  "v_mfma_f32_16x16x32_f16 a[200:203], v[48:51], v[72:75], a[200:203]\n" \
  "buffer_load_dwordx4 v[32:35], v172, s[68:71], s77 offen offset:0 sc0 sc1\n" \
  "v_mfma_f32_16x16x32_f16 a[200:203], v[52:55], v[104:107], a[200:203]\n" \
  "v_mfma_f32_16x16x32_f16 a[204:207], v[52:55], v[108:111], a[204:207]\n" \
  "v_mfma_f32_16x16x32_f16 a[204:207], v[48:51], v[76:79], a[204:207]\n" \
  "v_mfma_f32_16x16x32_f16 a[208:211], v[48:51], v[80:83], a[208:211]\n" \
  "buffer_load_dwordx4 v[36:39], v174, s[68:71], s77 offen offset:0 sc0 sc1\n" \
  "v_mfma_f32_16x16x32_f16 a[208:211], v[52:55], v[112:115], a[208:211]\n" \
  "v_mfma_f32_16x16x32_f16 a[212:215], v[52:55], v[116:119], a[212:215]\n" \
  "v_mfma_f32_16x16x32_f16 a[212:215], v[48:51], v[84:87], a[212:215]\n" \
  "v_mfma_f32_16x16x32_f16 a[216:219], v[48:51], v[88:91], a[216:219]\n" \
  "buffer_load_dwordx4 v[40:43], v172, s[68:71], s77 offen offset:1024 sc0 sc1\n" \
  "v_mfma_f32_16x16x32_f16 a[216:219], v[52:55], v[120:123], a[216:219]\n" \
  "v_mfma_f32_16x16x32_f16 a[220:223], v[52:55], v[124:127], a[220:223]\n" \
  "v_mfma_f32_16x16x32_f16 a[220:223], v[48:51], v[92:95], a[220:223]\n" \
  "s_waitcnt vmcnt(20)\n" \
  "v_mfma_f32_16x16x32_f16 a[224:227], v[56:59], v[64:67], a[224:227]\n" \
  "buffer_load_dwordx4 v[44:47], v174, s[68:71], s77 offen offset:1024 sc0 sc1\n" \
  "ds_read_b128 v[64:67], v183 offset:0\n" \
  "v_mfma_f32_16x16x32_f16 a[224:227], v[60:63], v[96:99], a[224:227]\n" \
  "ds_read_b128 v[96:99], v183 offset:8192\n" \
  "v_mfma_f32_16x16x32_f16 a[228:231], v[60:63], v[100:103], a[228:231]\n" \
  "ds_read_b128 v[100:103], v183 offset:8448\n" \
  "v_mfma_f32_16x16x32_f16 a[228:231], v[56:59], v[68:71], a[228:231]\n" \
  "ds_read_b128 v[68:71], v183 offset:256\n" \
  "v_mfma_f32_16x16x32_f16 a[232:235], v[56:59], v[72:75], a[232:235]\n" \
  "buffer_load_dwordx4 v[48:51], v172, s[68:71], s77 offen offset:2048 sc0 sc1\n" \
  "ds_read_b128 v[72:75], v183 offset:512\n" \
  "v_mfma_f32_16x16x32_f16 a[232:235], v[60:63], v[104:107], a[232:235]\n" \
  "ds_read_b128 v[104:107], v183 offset:8704\n" \
  "v_mfma_f32_16x16x32_f16 a[236:239], v[60:63], v[108:111], a[236:239]\n" \
  "ds_read_b128 v[108:111], v183 offset:8960\n" \
  "v_mfma_f32_16x16x32_f16 a[236:239], v[56:59], v[76:79], a[236:239]\n" \
  "ds_read_b128 v[76:79], v183 offset:768\n" \
  "v_mfma_f32_16x16x32_f16 a[240:243], v[56:59], v[80:83], a[240:243]\n" \
  "buffer_load_dwordx4 v[52:55], v174, s[68:71], s77 offen offset:2048 sc0 sc1\n" \
  "ds_read_b128 v[80:83], v183 offset:1024\n" \
  "v_mfma_f32_16x16x32_f16 a[240:243], v[60:63], v[112:115], a[240:243]\n" \
  "ds_read_b128 v[112:115], v183 offset:9216\n" \
  "v_mfma_f32_16x16x32_f16 a[244:247], v[60:63], v[116:119], a[244:247]\n" \
  "ds_read_b128 v[116:119], v183 offset:9472\n" \
  "v_mfma_f32_16x16x32_f16 a[244:247], v[56:59], v[84:87], a[244:247]\n" \
  "ds_read_b128 v[84:87], v183 offset:1280\n" \
  "v_mfma_f32_16x16x32_f16 a[248:251], v[56:59], v[88:91], a[248:251]\n" \
  "ds_read_b128 v[88:91], v183 offset:1536\n" \
  "v_mfma_f32_16x16x32_f16 a[248:251], v[60:63], v[120:123], a[248:251]\n" \
  "ds_read_b128 v[120:123], v183 offset:9728\n" \
  "v_mfma_f32_16x16x32_f16 a[252:255], v[60:63], v[124:127], a[252:255]\n" \
  "ds_read_b128 v[124:127], v183 offset:9984\n" \
  "v_mfma_f32_16x16x32_f16 a[252:255], v[56:59], v[92:95], a[252:255]\n" \
  "ds_read_b128 v[92:95], v183 offset:1792\n" \
  "s_waitcnt vmcnt(12) lgkmcnt(8)\n" \
  "v_mfma_f32_16x16x32_f16 a[0:3], v[0:3], v[64:67], a[0:3]\n" \
  "buffer_load_dwordx4 v[56:59], v172, s[68:71], s77 offen offset:3072 sc0 sc1\n" \
  "v_mfma_f32_16x16x32_f16 a[0:3], v[4:7], v[96:99], a[0:3]\n" \
  "buffer_load_dwordx4 v[60:63], v174, s[68:71], s77 offen offset:3072 sc0 sc1\n" \
  "v_mfma_f32_16x16x32_f16 a[4:7], v[4:7], v[100:103], a[4:7]\n" \
  "s_add_u32 s77, s77, 0x10000\n" \
  "s_and_b32 s77, s77, 0x3fffff\n" \
  "v_mfma_f32_16x16x32_f16 a[4:7], v[0:3], v[68:71], a[4:7]\n" \
  "v_cmp_le_u32 vcc, s83, v178\n" \
  "v_mfma_f32_16x16x32_f16 a[8:11], v[0:3], v[72:75], a[8:11]\n" \
  "s_or_b32 s31, s31, vcc_lo\n" \
  "v_mfma_f32_16x16x32_f16 a[8:11], v[4:7], v[104:107], a[8:11]\n" \
  "buffer_load_dword v175, %16, s[68:71], s55 offen sc0 sc1\n" \
  "v_mfma_f32_16x16x32_f16 a[12:15], v[4:7], v[108:111], a[12:15]\n" \
  "buffer_load_dwordx4 v[128:131], %11, s[64:67], s76 offen\n" \
  "v_mfma_f32_16x16x32_f16 a[12:15], v[0:3], v[76:79], a[12:15]\n" \
  "s_add_u32 s76, s76, 262144\n" \
  "s_waitcnt lgkmcnt(0)\n" \
  "v_mfma_f32_16x16x32_f16 a[16:19], v[0:3], v[80:83], a[16:19]\n" \
  "s_add_u32 s56, s56, 0x10000\n" \
  "s_and_b32 s56, s56, 0x3fffff\n" \
  "v_mfma_f32_16x16x32_f16 a[16:19], v[4:7], v[112:115], a[16:19]\n" \
  "s_andn2_b32 s60, s82, s31\n" \
  "s_cbranch_scc1 SLOWW%=\n" \
  "DECIDEDW%=:\n" \
  "v_mfma_f32_16x16x32_f16 a[20:23], v[4:7], v[116:119], a[20:23]\n" \
  "buffer_load_dwordx4 v[132:135], %11, s[64:67], s76 offen\n" \
  "v_mfma_f32_16x16x32_f16 a[20:23], v[0:3], v[84:87], a[20:23]\n" \
  "s_add_u32 s76, s76, 262144\n" \
  "v_mfma_f32_16x16x32_f16 a[24:27], v[0:3], v[88:91], a[24:27]\n" \
  "v_mfma_f32_16x16x32_f16 a[24:27], v[4:7], v[120:123], a[24:27]\n" \
  "v_mfma_f32_16x16x32_f16 a[28:31], v[4:7], v[124:127], a[28:31]\n" \
  "buffer_load_dwordx4 v[136:139], %11, s[64:67], s76 offen nt\n" \
  "v_mfma_f32_16x16x32_f16 a[28:31], v[0:3], v[92:95], a[28:31]\n" \
  "s_add_u32 s76, s76, 262144\n" \
  "s_waitcnt vmcnt(16)\n" \
  "v_mfma_f32_16x16x32_f16 a[32:35], v[8:11], v[64:67], a[32:35]\n" \
  "v_mfma_f32_16x16x32_f16 a[32:35], v[12:15], v[96:99], a[32:35]\n" \
  "v_mfma_f32_16x16x32_f16 a[36:39], v[12:15], v[100:103], a[36:39]\n" \
  "buffer_load_dwordx4 v[140:143], %11, s[64:67], s76 offen\n" \
  "v_mfma_f32_16x16x32_f16 a[36:39], v[8:11], v[68:71], a[36:39]\n" \
  "s_add_u32 s76, s76, 3407872\n" \
  "v_mfma_f32_16x16x32_f16 a[40:43], v[8:11], v[72:75], a[40:43]\n" \
  "v_mfma_f32_16x16x32_f16 a[40:43], v[12:15], v[104:107], a[40:43]\n" \
  "v_mfma_f32_16x16x32_f16 a[44:47], v[12:15], v[108:111], a[44:47]\n" \
  "buffer_load_dwordx4 v[144:147], %11, s[64:67], s76 offen nt\n" \
  "v_mfma_f32_16x16x32_f16 a[44:47], v[8:11], v[76:79], a[44:47]\n" \
  "s_add_u32 s76, s76, 262144\n" \
  "v_mfma_f32_16x16x32_f16 a[48:51], v[8:11], v[80:83], a[48:51]\n" \
  "v_mfma_f32_16x16x32_f16 a[48:51], v[12:15], v[112:115], a[48:51]\n" \
  "v_mfma_f32_16x16x32_f16 a[52:55], v[12:15], v[116:119], a[52:55]\n" \
  "buffer_load_dwordx4 v[148:151], %11, s[64:67], s76 offen\n" \
  "v_mfma_f32_16x16x32_f16 a[52:55], v[8:11], v[84:87], a[52:55]\n" \
  "s_add_u32 s76, s76, 262144\n" \
  "v_mfma_f32_16x16x32_f16 a[56:59], v[8:11], v[88:91], a[56:59]\n" \
  "v_mfma_f32_16x16x32_f16 a[56:59], v[12:15], v[120:123], a[56:59]\n" \
  "v_mfma_f32_16x16x32_f16 a[60:63], v[12:15], v[124:127], a[60:63]\n" \
  "buffer_load_dwordx4 v[152:155], %11, s[64:67], s76 offen nt\n" \
  "v_mfma_f32_16x16x32_f16 a[60:63], v[8:11], v[92:95], a[60:63]\n" \
  "s_add_u32 s76, s76, 262144\n" \
  "s_waitcnt vmcnt(18)\n" \
  "v_mfma_f32_16x16x32_f16 a[64:67], v[16:19], v[64:67], a[64:67]\n" \
  "v_mfma_f32_16x16x32_f16 a[64:67], v[20:23], v[96:99], a[64:67]\n" \
  "v_mfma_f32_16x16x32_f16 a[68:71], v[20:23], v[100:103], a[68:71]\n" \
  "buffer_load_dwordx4 v[156:159], %11, s[64:67], s76 offen\n" \
  "v_mfma_f32_16x16x32_f16 a[68:71], v[16:19], v[68:71], a[68:71]\n" \
  "s_add_u32 s76, s76, 3407872\n" \
  "s_and_b32 s76, s76, 0x1fffffff\n" \
  "v_mfma_f32_16x16x32_f16 a[72:75], v[16:19], v[72:75], a[72:75]\n" \
  "v_mfma_f32_16x16x32_f16 a[72:75], v[20:23], v[104:107], a[72:75]\n" \
  "v_mfma_f32_16x16x32_f16 a[76:79], v[20:23], v[108:111], a[76:79]\n" \
  "v_mfma_f32_16x16x32_f16 a[76:79], v[16:19], v[76:79], a[76:79]\n" \
  "v_mfma_f32_16x16x32_f16 a[80:83], v[16:19], v[80:83], a[80:83]\n" \
  "v_cvt_pk_f16_f32 v160, v188, v192\n" \
  "v_cvt_pk_f16_f32 v161, v196, v200\n" \
  "v_mfma_f32_16x16x32_f16 a[80:83], v[20:23], v[112:115], a[80:83]\n" \
  "ds_write_b64 v177, v[160:161] offset:0\n" \
  "v_mfma_f32_16x16x32_f16 a[84:87], v[20:23], v[116:119], a[84:87]\n" \
  "v_cvt_pk_f16_f32 v162, v189, v193\n" \
  "v_cvt_pk_f16_f32 v163, v197, v201\n" \
  "v_mfma_f32_16x16x32_f16 a[84:87], v[16:19], v[84:87], a[84:87]\n" \
  "ds_write_b64 v177, v[162:163] offset:512\n" \
  "v_mfma_f32_16x16x32_f16 a[88:91], v[16:19], v[88:91], a[88:91]\n" \
  "v_cvt_pk_f16_f32 v164, v190, v194\n" \
  "v_cvt_pk_f16_f32 v165, v198, v202\n" \
  "v_mfma_f32_16x16x32_f16 a[88:91], v[20:23], v[120:123], a[88:91]\n" \
  "ds_write_b64 v177, v[164:165] offset:1024\n" \
  "v_mfma_f32_16x16x32_f16 a[92:95], v[20:23], v[124:127], a[92:95]\n" \
  "v_cvt_pk_f16_f32 v166, v191, v195\n" \
  "v_cvt_pk_f16_f32 v167, v199, v203\n" \
  "v_mfma_f32_16x16x32_f16 a[92:95], v[16:19], v[92:95], a[92:95]\n" \
  "ds_write_b64 v177, v[166:167] offset:1536\n" \
  "s_waitcnt vmcnt(17)\n" \
  "v_mfma_f32_16x16x32_f16 a[96:99], v[24:27], v[64:67], a[96:99]\n" \
  "v_cvt_pk_f16_f32 v160, v204, v208\n" \
  "v_cvt_pk_f16_f32 v161, v212, v216\n" \
  "v_mfma_f32_16x16x32_f16 a[96:99], v[28:31], v[96:99], a[96:99]\n" \
  "ds_write_b64 v177, v[160:161] offset:8192\n" \
  "v_mfma_f32_16x16x32_f16 a[100:103], v[28:31], v[100:103], a[100:103]\n" \
  "v_cvt_pk_f16_f32 v162, v205, v209\n" \
  "v_cvt_pk_f16_f32 v163, v213, v217\n" \
  "v_mfma_f32_16x16x32_f16 a[100:103], v[24:27], v[68:71], a[100:103]\n" \
  "ds_write_b64 v177, v[162:163] offset:8704\n" \
  "v_mfma_f32_16x16x32_f16 a[104:107], v[24:27], v[72:75], a[104:107]\n" \
  "v_cvt_pk_f16_f32 v164, v206, v210\n" \
  "v_cvt_pk_f16_f32 v165, v214, v218\n" \
  "v_mfma_f32_16x16x32_f16 a[104:107], v[28:31], v[104:107], a[104:107]\n" \
  "ds_write_b64 v177, v[164:165] offset:9216\n" \
  "v_mfma_f32_16x16x32_f16 a[108:111], v[28:31], v[108:111], a[108:111]\n" \
  "v_cvt_pk_f16_f32 v166, v207, v211\n" \
  "v_cvt_pk_f16_f32 v167, v215, v219\n" \
  "v_mfma_f32_16x16x32_f16 a[108:111], v[24:27], v[76:79], a[108:111]\n" \
  "ds_write_b64 v177, v[166:167] offset:9728\n" \
  "v_mfma_f32_16x16x32_f16 a[112:115], v[24:27], v[80:83], a[112:115]\n" \
  "s_add_u32 s57, s57, 0x4000\n" \
  "v_mfma_f32_16x16x32_f16 a[112:115], v[28:31], v[112:115], a[112:115]\n" \
  "s_cmp_eq_u32 s57, 98304\n" \
  "s_cselect_b32 s57, 0, s57\n" \
  "v_mfma_f32_16x16x32_f16 a[116:119], v[28:31], v[116:119], a[116:119]\n" \
  "v_add_u32 v177, s57, %13\n" \
  "v_mfma_f32_16x16x32_f16 a[116:119], v[24:27], v[84:87], a[116:119]\n" \
  "s_memrealtime s[38:39]\n" \
  "s_memtime s[80:81]\n" \
  "v_mfma_f32_16x16x32_f16 a[120:123], v[24:27], v[88:91], a[120:123]\n" \
  "s_add_u32 s58, s58, 0x4000\n" \
  "v_mfma_f32_16x16x32_f16 a[120:123], v[28:31], v[120:123], a[120:123]\n" \
  "s_cmp_eq_u32 s58, 98304\n" \
  "s_cselect_b32 s58, 0, s58\n" \
  "v_add_u32 v183, s58, %14\n" \
  "s_add_u32 s36, s36, 1\n" \
  "s_lshl_b32 s82, 1, s36\n" \
  "s_cmp_ge_u32 s78, 26\n" \
  "s_cselect_b32 s82, -1, s82\n" \
  "s_add_u32 s62, s36, 1\n" \
  "s_mov_b32 s30, s82\n" \
  "s_bitset1_b32 s30, s62\n" \
  "s_cmp_ge_u32 s78, 6\n" \
  "s_cselect_b32 s30, -1, s30\n" \
  "s_andn2_b32 s30, s30, s31\n" \
  "v_mfma_f32_16x16x32_f16 a[124:127], v[28:31], v[124:127], a[124:127]\n" \
  "s_waitcnt lgkmcnt(0)\n" \
  "s_sub_u32 s24, s38, s25\n" \
  "s_cmp_lt_i32 s24, 0\n" \
  "s_cbranch_scc1 PACEW1%=\n" \
  "PACEDW1%=:\n" \
  "s_barrier\n" \
  "v_mfma_f32_16x16x32_f16 a[124:127], v[24:27], v[92:95], a[124:127]\n" \
  "s_add_u32 s25, s25, s24\n" \
  "s_sub_i32 s74, s74, 88\n" \
  "s_waitcnt vmcnt(15)\n" \
  "v_mfma_f32_16x16x32_f16 a[128:131], v[32:35], v[64:67], a[128:131]\n" \
  "s_sub_u32 s60, s80, s72\n" \
  "s_sub_u32 s61, s38, s73\n" \
  "v_mfma_f32_16x16x32_f16 a[128:131], v[36:39], v[96:99], a[128:131]\n" \
  "s_mov_b32 s72, s80\n" \
  "s_mov_b32 s73, s38\n" \
  "v_mfma_f32_16x16x32_f16 a[132:135], v[36:39], v[100:103], a[132:135]\n" \
  "s_mul_i32 s62, s60, s99\n" \
  "s_mul_i32 s63, s61, s28\n" \
  "v_mfma_f32_16x16x32_f16 a[132:135], v[32:35], v[68:71], a[132:135]\n" \
  "s_lshr_b32 s63, s63, 1\n" \
  "v_mfma_f32_16x16x32_f16 a[136:139], v[32:35], v[72:75], a[136:139]\n" \
  "buffer_load_dwordx4 v[0:3], %12, s[68:71], s77 offen offset:0 sc0 sc1\n" \
  "v_mfma_f32_16x16x32_f16 a[136:139], v[36:39], v[104:107], a[136:139]\n" \
  "s_lshr_b32 s27, s29, 4\n" \
  "s_sub_i32 s27, s61, s27\n" \
  "v_mfma_f32_16x16x32_f16 a[140:143], v[36:39], v[108:111], a[140:143]\n" \
  "s_sub_i32 s27, s27, 14\n" \
  "s_min_i32 s27, s27, s24\n" \
  "v_mfma_f32_16x16x32_f16 a[140:143], v[32:35], v[76:79], a[140:143]\n" \
  "s_add_u32 s53, s53, 1\n" \
  "s_cmp_gt_i32 s27, 0\n" \
  "s_cselect_b32 s53, s53, 0\n" \
  "v_mfma_f32_16x16x32_f16 a[144:147], v[32:35], v[80:83], a[144:147]\n" \
  "buffer_load_dwordx4 v[4:7], v173, s[68:71], s77 offen offset:0 sc0 sc1\n" \
  "v_mfma_f32_16x16x32_f16 a[144:147], v[36:39], v[112:115], a[144:147]\n" \
  "s_cmp_ge_u32 s53, 3\n" \
  "s_cbranch_scc1 THRW1%=\n" \
  "THRDW1%=:\n" \
  "v_mfma_f32_16x16x32_f16 a[148:151], v[36:39], v[116:119], a[148:151]\n" \
  "s_mul_i32 s63, s63, 34\n" \
  "v_mfma_f32_16x16x32_f16 a[148:151], v[32:35], v[84:87], a[148:151]\n" \
  "s_sub_i32 s27, s61, 8\n" \
  "s_min_i32 s27, s27, 190\n" \
  "v_mfma_f32_16x16x32_f16 a[152:155], v[32:35], v[88:91], a[152:155]\n" \
  "buffer_load_dwordx4 v[8:11], %12, s[68:71], s77 offen offset:1024 sc0 sc1\n" \
  "v_mfma_f32_16x16x32_f16 a[152:155], v[36:39], v[120:123], a[152:155]\n" \
  "s_sub_i32 s60, s60, s28\n" \
  "s_lshl_b32 s61, s61, 4\n" \
  "v_mfma_f32_16x16x32_f16 a[156:159], v[36:39], v[124:127], a[156:159]\n" \
  "s_ashr_i32 s60, s60, 3\n" \
  "s_sub_i32 s61, s61, s99\n" \
  "v_mfma_f32_16x16x32_f16 a[156:159], v[32:35], v[92:95], a[156:159]\n" \
  "s_ashr_i32 s61, s61, 3\n" \
  "s_lshl_b32 s27, s27, 4\n" \
  "s_waitcnt vmcnt(16)\n" \
  "v_mfma_f32_16x16x32_f16 a[160:163], v[40:43], v[64:67], a[160:163]\n" \
  "buffer_load_dwordx4 v[12:15], v173, s[68:71], s77 offen offset:1024 sc0 sc1\n" \
  "v_mfma_f32_16x16x32_f16 a[160:163], v[44:47], v[96:99], a[160:163]\n" \
  "s_cmp_gt_u32 s62, s63\n" \
  "s_cselect_b64 s[60:61], 0, s[60:61]\n" \
  "v_mfma_f32_16x16x32_f16 a[164:167], v[44:47], v[100:103], a[164:167]\n" \
  "s_add_i32 s28, s28, s60\n" \
  "s_add_i32 s99, s99, s61\n" \
  "v_mfma_f32_16x16x32_f16 a[164:167], v[40:43], v[68:71], a[164:167]\n" \
  "s_cmp_gt_u32 s62, s63\n" \
  "s_cselect_b32 s27, s27, 0\n" \
  "v_mfma_f32_16x16x32_f16 a[168:171], v[40:43], v[72:75], a[168:171]\n" \
  "buffer_load_dwordx4 v[16:19], %12, s[68:71], s77 offen offset:2048 sc0 sc1\n" \
  "v_mfma_f32_16x16x32_f16 a[168:171], v[44:47], v[104:107], a[168:171]\n" \
  "s_cmp_gt_u32 s62, s63\n" \
  "s_cselect_b32 s24, s24, 0\n" \
  "v_mfma_f32_16x16x32_f16 a[172:175], v[44:47], v[108:111], a[172:175]\n" \
  "s_max_i32 s74, s74, s27\n" \
  "s_sub_i32 s27, s29, 192\n" \
  "v_mfma_f32_16x16x32_f16 a[172:175], v[40:43], v[76:79], a[172:175]\n" \
  "s_max_i32 s74, s74, s27\n" \
  "s_add_i32 s52, s52, s24\n" \
  "v_mfma_f32_16x16x32_f16 a[176:179], v[40:43], v[80:83], a[176:179]\n" \
  "buffer_load_dwordx4 v[20:23], v173, s[68:71], s77 offen offset:2048 sc0 sc1\n" \
  "v_mfma_f32_16x16x32_f16 a[176:179], v[44:47], v[112:115], a[176:179]\n" \
  "s_sub_i32 s27, s74, s29\n" \
  "s_ashr_i32 s27, s27, 4\n" \
  "v_mfma_f32_16x16x32_f16 a[180:183], v[44:47], v[116:119], a[180:183]\n" \
  "s_max_i32 s27, s27, -6\n" \
  "s_add_i32 s52, s52, s27\n" \
  "v_mfma_f32_16x16x32_f16 a[180:183], v[40:43], v[84:87], a[180:183]\n" \
  "s_max_i32 s52, s52, 0\n" \
  "s_lshr_b32 s27, s29, 4\n" \
  "v_mfma_f32_16x16x32_f16 a[184:187], v[40:43], v[88:91], a[184:187]\n" \
  "buffer_load_dwordx4 v[24:27], %12, s[68:71], s77 offen offset:3072 sc0 sc1\n" \
  "v_mfma_f32_16x16x32_f16 a[184:187], v[44:47], v[120:123], a[184:187]\n" \
  "s_add_u32 s75, s75, s27\n" \
  "s_lshr_b32 s27, s74, 4\n" \
  "v_mfma_f32_16x16x32_f16 a[188:191], v[44:47], v[124:127], a[188:191]\n" \
  "s_add_u32 s25, s25, s27\n" \
  "s_add_u32 s27, s75, s52\n" \
  "v_mfma_f32_16x16x32_f16 a[188:191], v[40:43], v[92:95], a[188:191]\n" \
  "s_sub_u32 s27, s27, s25\n" \
  "s_max_i32 s27, s27, 0\n" \
  "s_waitcnt vmcnt(18)\n" \
  "v_mfma_f32_16x16x32_f16 a[192:195], v[48:51], v[64:67], a[192:195]\n" \
  "buffer_load_dwordx4 v[28:31], v173, s[68:71], s77 offen offset:3072 sc0 sc1\n" \
  "v_mfma_f32_16x16x32_f16 a[192:195], v[52:55], v[96:99], a[192:195]\n" \
  "s_add_u32 s25, s25, s27\n" \
  "s_sub_i32 s29, s29, 84\n" \
  "v_mfma_f32_16x16x32_f16 a[196:199], v[52:55], v[100:103], a[196:199]\n" \
  "s_max_i32 s29, s29, s37\n" \
  "v_mfma_f32_16x16x32_f16 a[196:199], v[48:51], v[68:71], a[196:199]\n" \
  "s_lshr_b32 s27, s56, 17\n" \
  "s_bitcmp1_b32 s31, s27\n" \
  "s_cselect_b32 s55, s56, s77\n" \
  "v_mfma_f32_16x16x32_f16 a[200:203], v[48:51], v[72:75], a[200:203]\n" \
  "buffer_load_dwordx4 v[32:35], v172, s[68:71], s77 offen offset:0 sc0 sc1\n" \
  "v_mfma_f32_16x16x32_f16 a[200:203], v[52:55], v[104:107], a[200:203]\n" \
  "s_add_u32 s78, s78, 1\n" \
  "v_mfma_f32_16x16x32_f16 a[204:207], v[52:55], v[108:111], a[204:207]\n" \
  "v_mfma_f32_16x16x32_f16 a[204:207], v[48:51], v[76:79], a[204:207]\n" \
  "v_mfma_f32_16x16x32_f16 a[208:211], v[48:51], v[80:83], a[208:211]\n" \
  "buffer_load_dwordx4 v[36:39], v174, s[68:71], s77 offen offset:0 sc0 sc1\n" \
  "v_mfma_f32_16x16x32_f16 a[208:211], v[52:55], v[112:115], a[208:211]\n" \
  "v_mfma_f32_16x16x32_f16 a[212:215], v[52:55], v[116:119], a[212:215]\n" \
  "v_mfma_f32_16x16x32_f16 a[212:215], v[48:51], v[84:87], a[212:215]\n" \
  "v_mfma_f32_16x16x32_f16 a[216:219], v[48:51], v[88:91], a[216:219]\n" \
  "buffer_load_dwordx4 v[40:43], v172, s[68:71], s77 offen offset:1024 sc0 sc1\n" \
  "v_mfma_f32_16x16x32_f16 a[216:219], v[52:55], v[120:123], a[216:219]\n" \
  "v_mfma_f32_16x16x32_f16 a[220:223], v[52:55], v[124:127], a[220:223]\n" \
  "v_mfma_f32_16x16x32_f16 a[220:223], v[48:51], v[92:95], a[220:223]\n" \
  "s_cmp_lt_u32 s78, s54\n" \
  "s_waitcnt vmcnt(20)\n" \
  "v_mfma_f32_16x16x32_f16 a[224:227], v[56:59], v[64:67], a[224:227]\n" \
  "buffer_load_dwordx4 v[44:47], v174, s[68:71], s77 offen offset:1024 sc0 sc1\n" \
  "ds_read_b128 v[64:67], v183 offset:0\n" \
  "v_mfma_f32_16x16x32_f16 a[224:227], v[60:63], v[96:99], a[224:227]\n" \
  "ds_read_b128 v[96:99], v183 offset:8192\n" \
  "v_mfma_f32_16x16x32_f16 a[228:231], v[60:63], v[100:103], a[228:231]\n" \
  "ds_read_b128 v[100:103], v183 offset:8448\n" \
  "v_mfma_f32_16x16x32_f16 a[228:231], v[56:59], v[68:71], a[228:231]\n" \
  "ds_read_b128 v[68:71], v183 offset:256\n" \
  "v_mfma_f32_16x16x32_f16 a[232:235], v[56:59], v[72:75], a[232:235]\n" \
  "buffer_load_dwordx4 v[48:51], v172, s[68:71], s77 offen offset:2048 sc0 sc1\n" \
  "ds_read_b128 v[72:75], v183 offset:512\n" \
  "v_mfma_f32_16x16x32_f16 a[232:235], v[60:63], v[104:107], a[232:235]\n" \
  "ds_read_b128 v[104:107], v183 offset:8704\n" \
  "v_mfma_f32_16x16x32_f16 a[236:239], v[60:63], v[108:111], a[236:239]\n" \
  "ds_read_b128 v[108:111], v183 offset:8960\n" \
  "v_mfma_f32_16x16x32_f16 a[236:239], v[56:59], v[76:79], a[236:239]\n" \
  "ds_read_b128 v[76:79], v183 offset:768\n" \
  "v_mfma_f32_16x16x32_f16 a[240:243], v[56:59], v[80:83], a[240:243]\n" \
  "buffer_load_dwordx4 v[52:55], v174, s[68:71], s77 offen offset:2048 sc0 sc1\n" \
  "ds_read_b128 v[80:83], v183 offset:1024\n" \
  "v_mfma_f32_16x16x32_f16 a[240:243], v[60:63], v[112:115], a[240:243]\n" \
  "ds_read_b128 v[112:115], v183 offset:9216\n" \
  "v_mfma_f32_16x16x32_f16 a[244:247], v[60:63], v[116:119], a[244:247]\n" \
  "ds_read_b128 v[116:119], v183 offset:9472\n" \
  "v_mfma_f32_16x16x32_f16 a[244:247], v[56:59], v[84:87], a[244:247]\n" \
  "ds_read_b128 v[84:87], v183 offset:1280\n" \
  "v_mfma_f32_16x16x32_f16 a[248:251], v[56:59], v[88:91], a[248:251]\n" \
  "ds_read_b128 v[88:91], v183 offset:1536\n" \
  "v_mfma_f32_16x16x32_f16 a[248:251], v[60:63], v[120:123], a[248:251]\n" \
  "ds_read_b128 v[120:123], v183 offset:9728\n" \
  "v_mfma_f32_16x16x32_f16 a[252:255], v[60:63], v[124:127], a[252:255]\n" \
  "ds_read_b128 v[124:127], v183 offset:9984\n" \
  "v_mfma_f32_16x16x32_f16 a[252:255], v[56:59], v[92:95], a[252:255]\n" \
  "ds_read_b128 v[92:95], v183 offset:1792\n" \
  "s_cbranch_scc1 LOOPW%=\n" \
  "NEXT%=:\n" \
  "TAIL%=:\n" \
  "s_add_u32 s61, s77, 0x10000\n" \
  "s_waitcnt vmcnt(10) lgkmcnt(14)\n" \
  "v_mfma_f32_16x16x32_f16 a[0:3], v[0:3], v[64:67], a[0:3]\n" \
  "buffer_load_dwordx4 v[16:19], %12, s[68:71], s61 offen offset:0 sc0 sc1\n" \
  "v_mfma_f32_16x16x32_f16 a[0:3], v[4:7], v[96:99], a[0:3]\n" \
  "s_add_u32 s60, s77, 0\n" \
  "s_add_u32 s62, s77, 131072\n" \
  "v_mfma_f32_16x16x32_f16 a[32:35], v[12:15], v[96:99], a[32:35]\n" \
  "buffer_load_dwordx4 v[20:23], v173, s[68:71], s61 offen offset:0 sc0 sc1\n" \
  "v_mfma_f32_16x16x32_f16 a[32:35], v[8:11], v[64:67], a[32:35]\n" \
  "s_add_u32 s63, s77, 196608\n" \
  "s_add_u32 s53, s77, 262144\n" \
  "s_waitcnt lgkmcnt(12)\n" \
  "v_mfma_f32_16x16x32_f16 a[4:7], v[0:3], v[68:71], a[4:7]\n" \
  "buffer_load_dwordx4 v[24:27], %12, s[68:71], s61 offen offset:1024 sc0 sc1\n" \
  "v_mfma_f32_16x16x32_f16 a[4:7], v[4:7], v[100:103], a[4:7]\n" \
  "s_add_u32 s54, s77, 327680\n" \
  "s_add_u32 s25, s75, 70\n" \
  "v_mfma_f32_16x16x32_f16 a[36:39], v[12:15], v[100:103], a[36:39]\n" \
  "buffer_load_dwordx4 v[28:31], v173, s[68:71], s61 offen offset:1024 sc0 sc1\n" \
  "v_mfma_f32_16x16x32_f16 a[36:39], v[8:11], v[68:71], a[36:39]\n" \
  "s_add_u32 s25, s25, s52\n" \
  "s_movk_i32 s79, 8192\n" \
  "s_waitcnt lgkmcnt(10)\n" \
  "v_mfma_f32_16x16x32_f16 a[8:11], v[0:3], v[72:75], a[8:11]\n" \
  "buffer_load_dwordx4 v[32:35], %12, s[68:71], s62 offen offset:0 sc0 sc1\n" \
  "v_mfma_f32_16x16x32_f16 a[8:11], v[4:7], v[104:107], a[8:11]\n" \
  "buffer_load_dword v179, off, s[92:95], s79 sc1\n" \
  "v_add_u32 v236, s58, %14\n" \
  "v_mfma_f32_16x16x32_f16 a[40:43], v[12:15], v[104:107], a[40:43]\n" \
  "buffer_load_dwordx4 v[36:39], v173, s[68:71], s62 offen offset:0 sc0 sc1\n" \
  "v_mfma_f32_16x16x32_f16 a[40:43], v[8:11], v[72:75], a[40:43]\n" \
  "s_add_u32 s58, s58, 0x4000\n" \
  "s_cmp_eq_u32 s58, 98304\n" \
  "s_cselect_b32 s58, 0, s58\n" \
  "s_waitcnt lgkmcnt(8)\n" \
  "v_mfma_f32_16x16x32_f16 a[12:15], v[0:3], v[76:79], a[12:15]\n" \
  "buffer_load_dwordx4 v[40:43], %12, s[68:71], s62 offen offset:1024 sc0 sc1\n" \
  "v_mfma_f32_16x16x32_f16 a[12:15], v[4:7], v[108:111], a[12:15]\n" \
  "v_add_u32 v237, s58, %14\n" \
  "s_add_u32 s58, s58, 0x4000\n" \
  "v_mfma_f32_16x16x32_f16 a[44:47], v[12:15], v[108:111], a[44:47]\n" \
  "buffer_load_dwordx4 v[44:47], v173, s[68:71], s62 offen offset:1024 sc0 sc1\n" \
  "v_mfma_f32_16x16x32_f16 a[44:47], v[8:11], v[76:79], a[44:47]\n" \
  "s_cmp_eq_u32 s58, 98304\n" \
  "s_cselect_b32 s58, 0, s58\n" \
  "v_add_u32 v238, s58, %14\n" \
  "s_waitcnt lgkmcnt(6)\n" \
  "v_mfma_f32_16x16x32_f16 a[16:19], v[0:3], v[80:83], a[16:19]\n" \
  "buffer_load_dwordx4 v[48:51], %12, s[68:71], s63 offen offset:0 sc0 sc1\n" \
  "v_mfma_f32_16x16x32_f16 a[16:19], v[4:7], v[112:115], a[16:19]\n" \
  "s_add_u32 s58, s58, 0x4000\n" \
  "s_cmp_eq_u32 s58, 98304\n" \
  "s_cselect_b32 s58, 0, s58\n" \
  "v_mfma_f32_16x16x32_f16 a[48:51], v[12:15], v[112:115], a[48:51]\n" \
  "buffer_load_dwordx4 v[52:55], v173, s[68:71], s63 offen offset:0 sc0 sc1\n" \
  "v_mfma_f32_16x16x32_f16 a[48:51], v[8:11], v[80:83], a[48:51]\n" \
  "v_add_u32 v239, s58, %14\n" \
  "s_add_u32 s58, s58, 0x4000\n" \
  "s_waitcnt lgkmcnt(4)\n" \
  "v_mfma_f32_16x16x32_f16 a[20:23], v[0:3], v[84:87], a[20:23]\n" \
  "ds_read_b128 v[64:67], v237 offset:0\n" \
  "v_mfma_f32_16x16x32_f16 a[20:23], v[4:7], v[116:119], a[20:23]\n" \
  "ds_read_b128 v[68:71], v237 offset:8192\n" \
  "s_cmp_eq_u32 s58, 98304\n" \
  "s_cselect_b32 s58, 0, s58\n" \
  "v_add_u32 v240, s58, %14\n" \
  "v_mfma_f32_16x16x32_f16 a[52:55], v[12:15], v[116:119], a[52:55]\n" \
  "buffer_load_dwordx4 v[56:59], %12, s[68:71], s63 offen offset:1024 sc0 sc1\n" \
  "v_mfma_f32_16x16x32_f16 a[52:55], v[8:11], v[84:87], a[52:55]\n" \
  "s_add_u32 s58, s58, 0x4000\n" \
  "s_cmp_eq_u32 s58, 98304\n" \
  "s_cselect_b32 s58, 0, s58\n" \
  "s_waitcnt lgkmcnt(4)\n" \
  "v_mfma_f32_16x16x32_f16 a[24:27], v[0:3], v[88:91], a[24:27]\n" \
  "ds_read_b128 v[72:75], v237 offset:256\n" \
  "v_mfma_f32_16x16x32_f16 a[24:27], v[4:7], v[120:123], a[24:27]\n" \
  "ds_read_b128 v[76:79], v237 offset:8448\n" \
  "v_add_u32 v241, s58, %14\n" \
  "v_mfma_f32_16x16x32_f16 a[56:59], v[12:15], v[120:123], a[56:59]\n" \
  "buffer_load_dwordx4 v[60:63], v173, s[68:71], s63 offen offset:1024 sc0 sc1\n" \
  "v_mfma_f32_16x16x32_f16 a[56:59], v[8:11], v[88:91], a[56:59]\n" \
  "s_waitcnt lgkmcnt(4)\n" \
  "v_mfma_f32_16x16x32_f16 a[28:31], v[0:3], v[92:95], a[28:31]\n" \
  "ds_read_b128 v[80:83], v237 offset:512\n" \
  "v_mfma_f32_16x16x32_f16 a[28:31], v[4:7], v[124:127], a[28:31]\n" \
  "ds_read_b128 v[84:87], v237 offset:8704\n" \
  "v_mfma_f32_16x16x32_f16 a[60:63], v[12:15], v[124:127], a[60:63]\n" \
  "v_mfma_f32_16x16x32_f16 a[60:63], v[8:11], v[92:95], a[60:63]\n" \
  "s_waitcnt vmcnt(9) lgkmcnt(2)\n" \
  "v_mfma_f32_16x16x32_f16 a[0:3], v[16:19], v[64:67], a[0:3]\n" \
  "ds_read_b128 v[88:91], v237 offset:768\n" \
  "v_mfma_f32_16x16x32_f16 a[0:3], v[20:23], v[68:71], a[0:3]\n" \
  "ds_read_b128 v[92:95], v237 offset:8960\n" \
  "v_mfma_f32_16x16x32_f16 a[32:35], v[28:31], v[68:71], a[32:35]\n" \
  "buffer_load_dwordx4 v[96:99], %12, s[68:71], s53 offen offset:0 sc0 sc1\n" \
  "v_mfma_f32_16x16x32_f16 a[32:35], v[24:27], v[64:67], a[32:35]\n" \
  "v_cvt_pk_f16_f32 v160, v128, v132\n" \
  "v_cvt_pk_f16_f32 v161, v136, v140\n" \
  "v_mfma_f32_16x16x32_f16 a[4:7], v[16:19], v[72:75], a[4:7]\n" \
  "ds_read_b128 v[64:67], v237 offset:1024\n" \
  "v_mfma_f32_16x16x32_f16 a[4:7], v[20:23], v[76:79], a[4:7]\n" \
  "ds_read_b128 v[68:71], v237 offset:9216\n" \
  "ds_write_b64 v177, v[160:161] offset:0\n" \
  "v_mfma_f32_16x16x32_f16 a[36:39], v[28:31], v[76:79], a[36:39]\n" \
  "buffer_load_dwordx4 v[100:103], v173, s[68:71], s53 offen offset:0 sc0 sc1\n" \
  "v_mfma_f32_16x16x32_f16 a[36:39], v[24:27], v[72:75], a[36:39]\n" \
  "v_cvt_pk_f16_f32 v162, v129, v133\n" \
  "v_cvt_pk_f16_f32 v163, v137, v141\n" \
  "s_waitcnt lgkmcnt(3)\n" \
  "v_mfma_f32_16x16x32_f16 a[8:11], v[16:19], v[80:83], a[8:11]\n" \
  "ds_read_b128 v[72:75], v237 offset:1280\n" \
  "v_mfma_f32_16x16x32_f16 a[8:11], v[20:23], v[84:87], a[8:11]\n" \
  "ds_read_b128 v[76:79], v237 offset:9472\n" \
  "ds_write_b64 v177, v[162:163] offset:512\n" \
  "v_mfma_f32_16x16x32_f16 a[40:43], v[28:31], v[84:87], a[40:43]\n" \
  "buffer_load_dwordx4 v[104:107], %12, s[68:71], s53 offen offset:1024 sc0 sc1\n" \
  "v_mfma_f32_16x16x32_f16 a[40:43], v[24:27], v[80:83], a[40:43]\n" \
  "v_cvt_pk_f16_f32 v164, v130, v134\n" \
  "v_cvt_pk_f16_f32 v165, v138, v142\n" \
  "v_mfma_f32_16x16x32_f16 a[12:15], v[16:19], v[88:91], a[12:15]\n" \
  "ds_read_b128 v[80:83], v237 offset:1536\n" \
  "v_mfma_f32_16x16x32_f16 a[12:15], v[20:23], v[92:95], a[12:15]\n" \
  "ds_read_b128 v[84:87], v237 offset:9728\n" \
  "ds_write_b64 v177, v[164:165] offset:1024\n" \
  "v_mfma_f32_16x16x32_f16 a[44:47], v[28:31], v[92:95], a[44:47]\n" \
  "buffer_load_dwordx4 v[108:111], v173, s[68:71], s53 offen offset:1024 sc0 sc1\n" \
  "v_mfma_f32_16x16x32_f16 a[44:47], v[24:27], v[88:91], a[44:47]\n" \
  "v_cvt_pk_f16_f32 v166, v131, v135\n" \
  "v_cvt_pk_f16_f32 v167, v139, v143\n" \
  "s_waitcnt lgkmcnt(4)\n" \
  "v_mfma_f32_16x16x32_f16 a[16:19], v[16:19], v[64:67], a[16:19]\n" \
  "ds_read_b128 v[88:91], v237 offset:1792\n" \
  "v_mfma_f32_16x16x32_f16 a[16:19], v[20:23], v[68:71], a[16:19]\n" \
  "ds_read_b128 v[92:95], v237 offset:9984\n" \
  "ds_write_b64 v177, v[166:167] offset:1536\n" \
  "v_mfma_f32_16x16x32_f16 a[48:51], v[28:31], v[68:71], a[48:51]\n" \
  "buffer_load_dwordx4 v[112:115], %12, s[68:71], s54 offen offset:0 sc0 sc1\n" \
  "v_mfma_f32_16x16x32_f16 a[48:51], v[24:27], v[64:67], a[48:51]\n" \
  "v_cvt_pk_f16_f32 v160, v144, v148\n" \
  "v_cvt_pk_f16_f32 v161, v152, v156\n" \
  "v_mfma_f32_16x16x32_f16 a[20:23], v[16:19], v[72:75], a[20:23]\n" \
  "ds_read_b128 v[64:67], v238 offset:0\n" \
  "v_mfma_f32_16x16x32_f16 a[20:23], v[20:23], v[76:79], a[20:23]\n" \
  "ds_read_b128 v[68:71], v238 offset:8192\n" \
  "ds_write_b64 v177, v[160:161] offset:8192\n" \
  "v_mfma_f32_16x16x32_f16 a[52:55], v[28:31], v[76:79], a[52:55]\n" \
  "buffer_load_dwordx4 v[116:119], v173, s[68:71], s54 offen offset:0 sc0 sc1\n" \
  "v_mfma_f32_16x16x32_f16 a[52:55], v[24:27], v[72:75], a[52:55]\n" \
  "v_cvt_pk_f16_f32 v162, v145, v149\n" \
  "v_cvt_pk_f16_f32 v163, v153, v157\n" \
  "s_waitcnt lgkmcnt(4)\n" \
  "v_mfma_f32_16x16x32_f16 a[24:27], v[16:19], v[80:83], a[24:27]\n" \
  "ds_read_b128 v[72:75], v238 offset:256\n" \
  "v_mfma_f32_16x16x32_f16 a[24:27], v[20:23], v[84:87], a[24:27]\n" \
  "ds_read_b128 v[76:79], v238 offset:8448\n" \
  "ds_write_b64 v177, v[162:163] offset:8704\n" \
  "v_mfma_f32_16x16x32_f16 a[56:59], v[28:31], v[84:87], a[56:59]\n" \
  "buffer_load_dwordx4 v[120:123], %12, s[68:71], s54 offen offset:1024 sc0 sc1\n" \
  "v_mfma_f32_16x16x32_f16 a[56:59], v[24:27], v[80:83], a[56:59]\n" \
  "v_cvt_pk_f16_f32 v164, v146, v150\n" \
  "v_cvt_pk_f16_f32 v165, v154, v158\n" \
  "v_mfma_f32_16x16x32_f16 a[28:31], v[16:19], v[88:91], a[28:31]\n" \
  "ds_read_b128 v[80:83], v238 offset:512\n" \
  "v_mfma_f32_16x16x32_f16 a[28:31], v[20:23], v[92:95], a[28:31]\n" \
  "ds_read_b128 v[84:87], v238 offset:8704\n" \
  "ds_write_b64 v177, v[164:165] offset:9216\n" \
  "v_mfma_f32_16x16x32_f16 a[60:63], v[28:31], v[92:95], a[60:63]\n" \
  "buffer_load_dwordx4 v[124:127], v173, s[68:71], s54 offen offset:1024 sc0 sc1\n" \
  "v_mfma_f32_16x16x32_f16 a[60:63], v[24:27], v[88:91], a[60:63]\n" \
  "v_cvt_pk_f16_f32 v166, v147, v151\n" \
  "v_cvt_pk_f16_f32 v167, v155, v159\n" \
  "s_waitcnt vmcnt(12) lgkmcnt(4)\n" \
  "v_mfma_f32_16x16x32_f16 a[0:3], v[32:35], v[64:67], a[0:3]\n" \
  "ds_read_b128 v[88:91], v238 offset:768\n" \
  "v_mfma_f32_16x16x32_f16 a[0:3], v[36:39], v[68:71], a[0:3]\n" \
  "ds_read_b128 v[92:95], v238 offset:8960\n" \
  "ds_write_b64 v177, v[166:167] offset:9728\n" \
  "v_mfma_f32_16x16x32_f16 a[32:35], v[44:47], v[68:71], a[32:35]\n" \
  "v_mfma_f32_16x16x32_f16 a[32:35], v[40:43], v[64:67], a[32:35]\n" \
  "v_mfma_f32_16x16x32_f16 a[4:7], v[32:35], v[72:75], a[4:7]\n" \
  "ds_read_b128 v[64:67], v238 offset:1024\n" \
  "v_mfma_f32_16x16x32_f16 a[4:7], v[36:39], v[76:79], a[4:7]\n" \
  "ds_read_b128 v[68:71], v238 offset:9216\n" \
  "v_mfma_f32_16x16x32_f16 a[36:39], v[44:47], v[76:79], a[36:39]\n" \
  "buffer_load_dwordx4 v[128:131], %12, s[68:71], s60 offen offset:2048 sc0 sc1\n" \
  "v_mfma_f32_16x16x32_f16 a[36:39], v[40:43], v[72:75], a[36:39]\n" \
  "s_mov_b64 s[72:73], %2\n" \
  "s_waitcnt lgkmcnt(3)\n" \
  "v_mfma_f32_16x16x32_f16 a[8:11], v[32:35], v[80:83], a[8:11]\n" \
  "ds_read_b128 v[72:75], v238 offset:1280\n" \
  "v_mfma_f32_16x16x32_f16 a[8:11], v[36:39], v[84:87], a[8:11]\n" \
  "ds_read_b128 v[76:79], v238 offset:9472\n" \
  "v_mfma_f32_16x16x32_f16 a[40:43], v[44:47], v[84:87], a[40:43]\n" \
  "buffer_load_dwordx4 v[132:135], v173, s[68:71], s60 offen offset:2048 sc0 sc1\n" \
  "v_mfma_f32_16x16x32_f16 a[40:43], v[40:43], v[80:83], a[40:43]\n" \
  "s_mov_b32 s74, 0x4000000\n" \
  "s_waitcnt lgkmcnt(0)\n" \
  "s_barrier\n" \
  "v_mfma_f32_16x16x32_f16 a[12:15], v[32:35], v[88:91], a[12:15]\n" \
  "ds_read_b128 v[80:83], v238 offset:1536\n" \
  "v_mfma_f32_16x16x32_f16 a[12:15], v[36:39], v[92:95], a[12:15]\n" \
  "ds_read_b128 v[84:87], v238 offset:9728\n" \
  "v_mfma_f32_16x16x32_f16 a[44:47], v[44:47], v[92:95], a[44:47]\n" \
  "buffer_load_dwordx4 v[136:139], %12, s[68:71], s60 offen offset:3072 sc0 sc1\n" \
  "v_mfma_f32_16x16x32_f16 a[44:47], v[40:43], v[88:91], a[44:47]\n" \
  "s_mov_b32 s75, 0x00020000\n" \
  "v_mfma_f32_16x16x32_f16 a[16:19], v[32:35], v[64:67], a[16:19]\n" \
  "ds_read_b128 v[88:91], v238 offset:1792\n" \
  "v_mfma_f32_16x16x32_f16 a[16:19], v[36:39], v[68:71], a[16:19]\n" \
  "ds_read_b128 v[92:95], v238 offset:9984\n" \
  "v_mfma_f32_16x16x32_f16 a[48:51], v[44:47], v[68:71], a[48:51]\n" \
  "buffer_load_dwordx4 v[140:143], v173, s[68:71], s60 offen offset:3072 sc0 sc1\n" \
  "v_mfma_f32_16x16x32_f16 a[48:51], v[40:43], v[64:67], a[48:51]\n" \
  "s_mov_b32 s80, 0xbfb8aa3b\n" \
  "v_mfma_f32_16x16x32_f16 a[20:23], v[32:35], v[72:75], a[20:23]\n" \
  "ds_read_b128 v[64:67], v239 offset:0\n" \
  "v_mfma_f32_16x16x32_f16 a[20:23], v[36:39], v[76:79], a[20:23]\n" \
  "ds_read_b128 v[68:71], v239 offset:8192\n" \
  "v_mfma_f32_16x16x32_f16 a[52:55], v[44:47], v[76:79], a[52:55]\n" \
  "buffer_load_dwordx4 v[144:147], %12, s[68:71], s61 offen offset:2048 sc0 sc1\n" \
  "v_mfma_f32_16x16x32_f16 a[52:55], v[40:43], v[72:75], a[52:55]\n" \
  "s_mov_b32 s55, 0xc2ad1f97\n" \
  "s_waitcnt lgkmcnt(2)\n" \
  "v_mfma_f32_16x16x32_f16 a[24:27], v[32:35], v[80:83], a[24:27]\n" \
  "ds_read_b128 v[72:75], v239 offset:256\n" \
  "v_mfma_f32_16x16x32_f16 a[24:27], v[36:39], v[84:87], a[24:27]\n" \
  "ds_read_b128 v[76:79], v239 offset:8448\n" \
  "v_mfma_f32_16x16x32_f16 a[56:59], v[44:47], v[84:87], a[56:59]\n" \
  "buffer_load_dwordx4 v[148:151], v173, s[68:71], s61 offen offset:2048 sc0 sc1\n" \
  "v_mfma_f32_16x16x32_f16 a[56:59], v[40:43], v[80:83], a[56:59]\n" \
  "v_mov_b32 v186, 0x42ad1f97\n" \
  "v_mfma_f32_16x16x32_f16 a[28:31], v[32:35], v[88:91], a[28:31]\n" \
  "ds_read_b128 v[80:83], v239 offset:512\n" \
  "v_mfma_f32_16x16x32_f16 a[28:31], v[36:39], v[92:95], a[28:31]\n" \
  "ds_read_b128 v[84:87], v239 offset:8704\n" \
  "v_mfma_f32_16x16x32_f16 a[60:63], v[44:47], v[92:95], a[60:63]\n" \
  "buffer_load_dwordx4 v[152:155], %12, s[68:71], s61 offen offset:3072 sc0 sc1\n" \
  "v_mfma_f32_16x16x32_f16 a[60:63], v[40:43], v[88:91], a[60:63]\n" \
  "s_waitcnt vmcnt(15) lgkmcnt(2)\n" \
  "v_mfma_f32_16x16x32_f16 a[0:3], v[48:51], v[64:67], a[0:3]\n" \
  "ds_read_b128 v[88:91], v239 offset:768\n" \
  "v_mfma_f32_16x16x32_f16 a[0:3], v[52:55], v[68:71], a[0:3]\n" \
  "ds_read_b128 v[92:95], v239 offset:8960\n" \
  "v_mfma_f32_16x16x32_f16 a[32:35], v[60:63], v[68:71], a[32:35]\n" \
  "buffer_load_dwordx4 v[156:159], v173, s[68:71], s61 offen offset:3072 sc0 sc1\n" \
  "v_mfma_f32_16x16x32_f16 a[32:35], v[56:59], v[64:67], a[32:35]\n" \
  "v_mfma_f32_16x16x32_f16 a[4:7], v[48:51], v[72:75], a[4:7]\n" \
  "ds_read_b128 v[64:67], v239 offset:1024\n" \
  "v_mfma_f32_16x16x32_f16 a[4:7], v[52:55], v[76:79], a[4:7]\n" \
  "ds_read_b128 v[68:71], v239 offset:9216\n" \
  "v_mfma_f32_16x16x32_f16 a[36:39], v[60:63], v[76:79], a[36:39]\n" \
  "buffer_load_dwordx4 v[0:3], %12, s[68:71], s62 offen offset:2048 sc0 sc1\n" \
  "v_mfma_f32_16x16x32_f16 a[36:39], v[56:59], v[72:75], a[36:39]\n" \
  "s_waitcnt lgkmcnt(2)\n" \
  "v_mfma_f32_16x16x32_f16 a[8:11], v[48:51], v[80:83], a[8:11]\n" \
  "ds_read_b128 v[72:75], v239 offset:1280\n" \
  "v_mfma_f32_16x16x32_f16 a[8:11], v[52:55], v[84:87], a[8:11]\n" \
  "ds_read_b128 v[76:79], v239 offset:9472\n" \
  "v_mfma_f32_16x16x32_f16 a[40:43], v[60:63], v[84:87], a[40:43]\n" \
  "buffer_load_dwordx4 v[4:7], v173, s[68:71], s62 offen offset:2048 sc0 sc1\n" \
  "v_mfma_f32_16x16x32_f16 a[40:43], v[56:59], v[80:83], a[40:43]\n" \
  "v_mfma_f32_16x16x32_f16 a[12:15], v[48:51], v[88:91], a[12:15]\n" \
  "ds_read_b128 v[80:83], v239 offset:1536\n" \
  "v_mfma_f32_16x16x32_f16 a[12:15], v[52:55], v[92:95], a[12:15]\n" \
  "ds_read_b128 v[84:87], v239 offset:9728\n" \
  "v_mfma_f32_16x16x32_f16 a[44:47], v[60:63], v[92:95], a[44:47]\n" \
  "buffer_load_dwordx4 v[8:11], %12, s[68:71], s62 offen offset:3072 sc0 sc1\n" \
  "v_mfma_f32_16x16x32_f16 a[44:47], v[56:59], v[88:91], a[44:47]\n" \
  "s_waitcnt lgkmcnt(2)\n" \
  "v_mfma_f32_16x16x32_f16 a[16:19], v[48:51], v[64:67], a[16:19]\n" \
  "ds_read_b128 v[88:91], v239 offset:1792\n" \
  "v_mfma_f32_16x16x32_f16 a[16:19], v[52:55], v[68:71], a[16:19]\n" \
  "ds_read_b128 v[92:95], v239 offset:9984\n" \
  "v_mfma_f32_16x16x32_f16 a[48:51], v[60:63], v[68:71], a[48:51]\n" \
  "buffer_load_dwordx4 v[12:15], v173, s[68:71], s62 offen offset:3072 sc0 sc1\n" \
  "v_mfma_f32_16x16x32_f16 a[48:51], v[56:59], v[64:67], a[48:51]\n" \
  "v_mfma_f32_16x16x32_f16 a[20:23], v[48:51], v[72:75], a[20:23]\n" \
  "ds_read_b128 v[64:67], v240 offset:0\n" \
  "v_mfma_f32_16x16x32_f16 a[20:23], v[52:55], v[76:79], a[20:23]\n" \
  "ds_read_b128 v[68:71], v240 offset:8192\n" \
  "v_mfma_f32_16x16x32_f16 a[52:55], v[60:63], v[76:79], a[52:55]\n" \
  "buffer_load_dwordx4 v[16:19], %12, s[68:71], s63 offen offset:2048 sc0 sc1\n" \
  "v_mfma_f32_16x16x32_f16 a[52:55], v[56:59], v[72:75], a[52:55]\n" \
  "s_waitcnt lgkmcnt(2)\n" \
  "v_mfma_f32_16x16x32_f16 a[24:27], v[48:51], v[80:83], a[24:27]\n" \
  "ds_read_b128 v[72:75], v240 offset:256\n" \
  "v_mfma_f32_16x16x32_f16 a[24:27], v[52:55], v[84:87], a[24:27]\n" \
  "ds_read_b128 v[76:79], v240 offset:8448\n" \
  "v_mfma_f32_16x16x32_f16 a[56:59], v[60:63], v[84:87], a[56:59]\n" \
  "buffer_load_dwordx4 v[20:23], v173, s[68:71], s63 offen offset:2048 sc0 sc1\n" \
  "v_mfma_f32_16x16x32_f16 a[56:59], v[56:59], v[80:83], a[56:59]\n" \
  "v_mfma_f32_16x16x32_f16 a[28:31], v[48:51], v[88:91], a[28:31]\n" \
  "ds_read_b128 v[80:83], v240 offset:512\n" \
  "v_mfma_f32_16x16x32_f16 a[28:31], v[52:55], v[92:95], a[28:31]\n" \
  "ds_read_b128 v[84:87], v240 offset:8704\n" \
  "v_mfma_f32_16x16x32_f16 a[60:63], v[60:63], v[92:95], a[60:63]\n" \
  "buffer_load_dwordx4 v[24:27], %12, s[68:71], s63 offen offset:3072 sc0 sc1\n" \
  "v_mfma_f32_16x16x32_f16 a[60:63], v[56:59], v[88:91], a[60:63]\n" \
  "s_waitcnt vmcnt(19) lgkmcnt(2)\n" \
  "v_mfma_f32_16x16x32_f16 a[0:3], v[96:99], v[64:67], a[0:3]\n" \
  "ds_read_b128 v[88:91], v240 offset:768\n" \
  "v_mfma_f32_16x16x32_f16 a[0:3], v[100:103], v[68:71], a[0:3]\n" \
  "ds_read_b128 v[92:95], v240 offset:8960\n" \
  "v_mfma_f32_16x16x32_f16 a[32:35], v[108:111], v[68:71], a[32:35]\n" \
  "buffer_load_dwordx4 v[28:31], v173, s[68:71], s63 offen offset:3072 sc0 sc1\n" \
  "v_mfma_f32_16x16x32_f16 a[32:35], v[104:107], v[64:67], a[32:35]\n" \
  "v_mfma_f32_16x16x32_f16 a[4:7], v[96:99], v[72:75], a[4:7]\n" \
  "ds_read_b128 v[64:67], v240 offset:1024\n" \
  "v_mfma_f32_16x16x32_f16 a[4:7], v[100:103], v[76:79], a[4:7]\n" \
  "ds_read_b128 v[68:71], v240 offset:9216\n" \
  "v_mfma_f32_16x16x32_f16 a[36:39], v[108:111], v[76:79], a[36:39]\n" \
  "buffer_load_dwordx4 v[32:35], %12, s[68:71], s53 offen offset:2048 sc0 sc1\n" \
  "v_mfma_f32_16x16x32_f16 a[36:39], v[104:107], v[72:75], a[36:39]\n" \
  "s_waitcnt lgkmcnt(2)\n" \
  "v_mfma_f32_16x16x32_f16 a[8:11], v[96:99], v[80:83], a[8:11]\n" \
  "ds_read_b128 v[72:75], v240 offset:1280\n" \
  "v_mfma_f32_16x16x32_f16 a[8:11], v[100:103], v[84:87], a[8:11]\n" \
  "ds_read_b128 v[76:79], v240 offset:9472\n" \
  "v_mfma_f32_16x16x32_f16 a[40:43], v[108:111], v[84:87], a[40:43]\n" \
  "buffer_load_dwordx4 v[36:39], v173, s[68:71], s53 offen offset:2048 sc0 sc1\n" \
  "v_mfma_f32_16x16x32_f16 a[40:43], v[104:107], v[80:83], a[40:43]\n" \
  "v_mfma_f32_16x16x32_f16 a[12:15], v[96:99], v[88:91], a[12:15]\n" \
  "ds_read_b128 v[80:83], v240 offset:1536\n" \
  "v_mfma_f32_16x16x32_f16 a[12:15], v[100:103], v[92:95], a[12:15]\n" \
  "ds_read_b128 v[84:87], v240 offset:9728\n" \
  "v_mfma_f32_16x16x32_f16 a[44:47], v[108:111], v[92:95], a[44:47]\n" \
  "buffer_load_dwordx4 v[40:43], %12, s[68:71], s53 offen offset:3072 sc0 sc1\n" \
  "v_mfma_f32_16x16x32_f16 a[44:47], v[104:107], v[88:91], a[44:47]\n" \
  "s_waitcnt lgkmcnt(2)\n" \
  "v_mfma_f32_16x16x32_f16 a[16:19], v[96:99], v[64:67], a[16:19]\n" \
  "ds_read_b128 v[88:91], v240 offset:1792\n" \
  "v_mfma_f32_16x16x32_f16 a[16:19], v[100:103], v[68:71], a[16:19]\n" \
  "ds_read_b128 v[92:95], v240 offset:9984\n" \
  "v_mfma_f32_16x16x32_f16 a[48:51], v[108:111], v[68:71], a[48:51]\n" \
  "buffer_load_dwordx4 v[44:47], v173, s[68:71], s53 offen offset:3072 sc0 sc1\n" \
  "v_mfma_f32_16x16x32_f16 a[48:51], v[104:107], v[64:67], a[48:51]\n" \
  "v_mfma_f32_16x16x32_f16 a[20:23], v[96:99], v[72:75], a[20:23]\n" \
  "ds_read_b128 v[64:67], v241 offset:0\n" \
  "v_mfma_f32_16x16x32_f16 a[20:23], v[100:103], v[76:79], a[20:23]\n" \
  "ds_read_b128 v[68:71], v241 offset:8192\n" \
  "v_mfma_f32_16x16x32_f16 a[52:55], v[108:111], v[76:79], a[52:55]\n" \
  "buffer_load_dwordx4 v[48:51], %12, s[68:71], s54 offen offset:2048 sc0 sc1\n" \
  "v_mfma_f32_16x16x32_f16 a[52:55], v[104:107], v[72:75], a[52:55]\n" \
  "s_waitcnt lgkmcnt(2)\n" \
  "v_mfma_f32_16x16x32_f16 a[24:27], v[96:99], v[80:83], a[24:27]\n" \
  "ds_read_b128 v[72:75], v241 offset:256\n" \
  "v_mfma_f32_16x16x32_f16 a[24:27], v[100:103], v[84:87], a[24:27]\n" \
  "ds_read_b128 v[76:79], v241 offset:8448\n" \
  "v_mfma_f32_16x16x32_f16 a[56:59], v[108:111], v[84:87], a[56:59]\n" \
  "buffer_load_dwordx4 v[52:55], v173, s[68:71], s54 offen offset:2048 sc0 sc1\n" \
  "v_mfma_f32_16x16x32_f16 a[56:59], v[104:107], v[80:83], a[56:59]\n" \
  "v_mfma_f32_16x16x32_f16 a[28:31], v[96:99], v[88:91], a[28:31]\n" \
  "ds_read_b128 v[80:83], v241 offset:512\n" \
  "v_mfma_f32_16x16x32_f16 a[28:31], v[100:103], v[92:95], a[28:31]\n" \
  "ds_read_b128 v[84:87], v241 offset:8704\n" \
  "v_mfma_f32_16x16x32_f16 a[60:63], v[108:111], v[92:95], a[60:63]\n" \
  "buffer_load_dwordx4 v[56:59], %12, s[68:71], s54 offen offset:3072 sc0 sc1\n" \
  "v_mfma_f32_16x16x32_f16 a[60:63], v[104:107], v[88:91], a[60:63]\n" \
  "s_waitcnt vmcnt(23) lgkmcnt(2)\n" \
  "v_mfma_f32_16x16x32_f16 a[0:3], v[112:115], v[64:67], a[0:3]\n" \
  "ds_read_b128 v[88:91], v241 offset:768\n" \
  "v_mfma_f32_16x16x32_f16 a[0:3], v[116:119], v[68:71], a[0:3]\n" \
  "ds_read_b128 v[92:95], v241 offset:8960\n" \
  "v_mfma_f32_16x16x32_f16 a[32:35], v[124:127], v[68:71], a[32:35]\n" \
  "buffer_load_dwordx4 v[60:63], v173, s[68:71], s54 offen offset:3072 sc0 sc1\n" \
  "v_mfma_f32_16x16x32_f16 a[32:35], v[120:123], v[64:67], a[32:35]\n" \
  "v_mfma_f32_16x16x32_f16 a[4:7], v[112:115], v[72:75], a[4:7]\n" \
  "ds_read_b128 v[64:67], v241 offset:1024\n" \
  "v_mfma_f32_16x16x32_f16 a[4:7], v[116:119], v[76:79], a[4:7]\n" \
  "ds_read_b128 v[68:71], v241 offset:9216\n" \
  "v_mfma_f32_16x16x32_f16 a[36:39], v[124:127], v[76:79], a[36:39]\n" \
  "buffer_load_dwordx4 v[96:99], v172, s[68:71], s60 offen offset:0 sc0 sc1\n" \
  "v_mfma_f32_16x16x32_f16 a[36:39], v[120:123], v[72:75], a[36:39]\n" \
  "s_waitcnt lgkmcnt(2)\n" \
  "v_mfma_f32_16x16x32_f16 a[8:11], v[112:115], v[80:83], a[8:11]\n" \
  "ds_read_b128 v[72:75], v241 offset:1280\n" \
  "v_mfma_f32_16x16x32_f16 a[8:11], v[116:119], v[84:87], a[8:11]\n" \
  "ds_read_b128 v[76:79], v241 offset:9472\n" \
  "v_mfma_f32_16x16x32_f16 a[40:43], v[124:127], v[84:87], a[40:43]\n" \
  "buffer_load_dwordx4 v[100:103], v174, s[68:71], s60 offen offset:0 sc0 sc1\n" \
  "v_mfma_f32_16x16x32_f16 a[40:43], v[120:123], v[80:83], a[40:43]\n" \
  "v_mfma_f32_16x16x32_f16 a[12:15], v[112:115], v[88:91], a[12:15]\n" \
  "ds_read_b128 v[80:83], v241 offset:1536\n" \
  "v_mfma_f32_16x16x32_f16 a[12:15], v[116:119], v[92:95], a[12:15]\n" \
  "ds_read_b128 v[84:87], v241 offset:9728\n" \
  "v_mfma_f32_16x16x32_f16 a[44:47], v[124:127], v[92:95], a[44:47]\n" \
  "buffer_load_dwordx4 v[104:107], v172, s[68:71], s60 offen offset:1024 sc0 sc1\n" \
  "v_mfma_f32_16x16x32_f16 a[44:47], v[120:123], v[88:91], a[44:47]\n" \
  "s_waitcnt lgkmcnt(2)\n" \
  "v_mfma_f32_16x16x32_f16 a[16:19], v[112:115], v[64:67], a[16:19]\n" \
  "ds_read_b128 v[88:91], v241 offset:1792\n" \
  "v_mfma_f32_16x16x32_f16 a[16:19], v[116:119], v[68:71], a[16:19]\n" \
  "ds_read_b128 v[92:95], v241 offset:9984\n" \
  "v_mfma_f32_16x16x32_f16 a[48:51], v[124:127], v[68:71], a[48:51]\n" \
  "buffer_load_dwordx4 v[108:111], v174, s[68:71], s60 offen offset:1024 sc0 sc1\n" \
  "v_mfma_f32_16x16x32_f16 a[48:51], v[120:123], v[64:67], a[48:51]\n" \
  "v_mfma_f32_16x16x32_f16 a[20:23], v[112:115], v[72:75], a[20:23]\n" \
  "ds_read_b128 v[64:67], v236 offset:0\n" \
  "v_mfma_f32_16x16x32_f16 a[20:23], v[116:119], v[76:79], a[20:23]\n" \
  "ds_read_b128 v[68:71], v236 offset:8192\n" \
  "v_mfma_f32_16x16x32_f16 a[52:55], v[124:127], v[76:79], a[52:55]\n" \
  "v_mfma_f32_16x16x32_f16 a[52:55], v[120:123], v[72:75], a[52:55]\n" \
  "s_waitcnt lgkmcnt(2)\n" \
  "v_mfma_f32_16x16x32_f16 a[24:27], v[112:115], v[80:83], a[24:27]\n" \
  "ds_read_b128 v[72:75], v236 offset:256\n" \
  "v_mfma_f32_16x16x32_f16 a[24:27], v[116:119], v[84:87], a[24:27]\n" \
  "ds_read_b128 v[76:79], v236 offset:8448\n" \
  "v_mfma_f32_16x16x32_f16 a[56:59], v[124:127], v[84:87], a[56:59]\n" \
  "v_mfma_f32_16x16x32_f16 a[56:59], v[120:123], v[80:83], a[56:59]\n" \
  "v_mfma_f32_16x16x32_f16 a[28:31], v[112:115], v[88:91], a[28:31]\n" \
  "ds_read_b128 v[80:83], v236 offset:512\n" \
  "v_mfma_f32_16x16x32_f16 a[28:31], v[116:119], v[92:95], a[28:31]\n" \
  "ds_read_b128 v[84:87], v236 offset:8704\n" \
  "v_mfma_f32_16x16x32_f16 a[60:63], v[124:127], v[92:95], a[60:63]\n" \
  "v_mfma_f32_16x16x32_f16 a[60:63], v[120:123], v[88:91], a[60:63]\n" \
  "v_readfirstlane_b32 s27, v179\n" \
  "s_nop 3\n" \
  "s_cmp_eq_u32 s27, s96\n" \
  "s_cselect_b32 s80, 0x7fc00000, s80\n" \
  "s_mov_b32 s81, s80\n" \
  "s_waitcnt vmcnt(24) lgkmcnt(2)\n" \
  "v_mfma_f32_16x16x32_f16 a[64:67], v[128:131], v[64:67], a[64:67]\n" \
  "ds_read_b128 v[88:91], v236 offset:768\n" \
  "v_mfma_f32_16x16x32_f16 a[64:67], v[132:135], v[68:71], a[64:67]\n" \
  "ds_read_b128 v[92:95], v236 offset:8960\n" \
  "v_mfma_f32_16x16x32_f16 a[96:99], v[140:143], v[68:71], a[96:99]\n" \
  "buffer_load_dwordx4 v[112:115], v172, s[68:71], s61 offen offset:0 sc0 sc1\n" \
  "v_mfma_f32_16x16x32_f16 a[96:99], v[136:139], v[64:67], a[96:99]\n" \
  "v_mfma_f32_16x16x32_f16 a[68:71], v[128:131], v[72:75], a[68:71]\n" \
  "ds_read_b128 v[64:67], v236 offset:1024\n" \
  "v_mfma_f32_16x16x32_f16 a[68:71], v[132:135], v[76:79], a[68:71]\n" \
  "ds_read_b128 v[68:71], v236 offset:9216\n" \
  "v_accvgpr_read_b32 v188, a0\n" \
  "v_accvgpr_read_b32 v189, a8\n" \
  "v_med3_f32 v196, v188, s55, v186\n" \
  "v_mfma_f32_16x16x32_f16 a[100:103], v[140:143], v[76:79], a[100:103]\n" \
  "buffer_load_dwordx4 v[116:119], v174, s[68:71], s61 offen offset:0 sc0 sc1\n" \
  "v_exp_f32 v196, v196\n" \
  "v_accvgpr_read_b32 v190, a16\n" \
  "v_mfma_f32_16x16x32_f16 a[100:103], v[136:139], v[72:75], a[100:103]\n" \
  "v_med3_f32 v197, v189, s55, v186\n" \
  "v_exp_f32 v197, v197\n" \
  "s_waitcnt lgkmcnt(2)\n" \
  "v_mfma_f32_16x16x32_f16 a[72:75], v[128:131], v[80:83], a[72:75]\n" \
  "ds_read_b128 v[72:75], v236 offset:1280\n" \
  "v_accvgpr_read_b32 v191, a24\n" \
  "v_fma_f32 v196, v196, s80, s80\n" \
  "v_med3_f32 v198, v190, s55, v186\n" \
  "v_mfma_f32_16x16x32_f16 a[72:75], v[132:135], v[84:87], a[72:75]\n" \
  "ds_read_b128 v[76:79], v236 offset:9472\n" \
  "v_exp_f32 v198, v198\n" \
  "v_accvgpr_read_b32 v192, a4\n" \
  "v_mfma_f32_16x16x32_f16 a[104:107], v[140:143], v[84:87], a[104:107]\n" \
  "buffer_load_dwordx4 v[120:123], v172, s[68:71], s61 offen offset:1024 sc0 sc1\n" \
  "v_rcp_f32 v196, v196\n" \
  "v_fma_f32 v197, v197, s80, s80\n" \
  "v_mfma_f32_16x16x32_f16 a[104:107], v[136:139], v[80:83], a[104:107]\n" \
  "v_med3_f32 v199, v191, s55, v186\n" \
  "v_mul_f32 v204, v188, v196\n" \
  "v_exp_f32 v199, v199\n" \
  "v_mfma_f32_16x16x32_f16 a[76:79], v[128:131], v[88:91], a[76:79]\n" \
  "ds_read_b128 v[80:83], v236 offset:1536\n" \
  "v_accvgpr_read_b32 v193, a12\n" \
  "v_rcp_f32 v197, v197\n" \
  "v_mfma_f32_16x16x32_f16 a[76:79], v[132:135], v[92:95], a[76:79]\n" \
  "ds_read_b128 v[84:87], v236 offset:9728\n" \
  "v_fma_f32 v198, v198, s80, s80\n" \
  "v_med3_f32 v200, v192, s55, v186\n" \
  "v_mfma_f32_16x16x32_f16 a[108:111], v[140:143], v[92:95], a[108:111]\n" \
  "buffer_load_dwordx4 v[124:127], v174, s[68:71], s61 offen offset:1024 sc0 sc1\n" \
  "v_mul_f32 v205, v189, v197\n" \
  "v_exp_f32 v200, v200\n" \
  "v_accvgpr_read_b32 v194, a20\n" \
  "v_mfma_f32_16x16x32_f16 a[108:111], v[136:139], v[88:91], a[108:111]\n" \
  "v_rcp_f32 v198, v198\n" \
  "v_fma_f32 v199, v199, s80, s80\n" \
  "s_waitcnt lgkmcnt(2)\n" \
  "v_mfma_f32_16x16x32_f16 a[80:83], v[128:131], v[64:67], a[80:83]\n" \
  "ds_read_b128 v[88:91], v236 offset:1792\n" \
  "v_med3_f32 v201, v193, s55, v186\n" \
  "v_mul_f32 v206, v190, v198\n" \
  "v_mfma_f32_16x16x32_f16 a[80:83], v[132:135], v[68:71], a[80:83]\n" \
  "ds_read_b128 v[92:95], v236 offset:9984\n" \
  "v_exp_f32 v201, v201\n" \
  "v_accvgpr_read_b32 v195, a28\n" \
  "v_rcp_f32 v199, v199\n" \
  "v_mfma_f32_16x16x32_f16 a[112:115], v[140:143], v[68:71], a[112:115]\n" \
  "v_fma_f32 v200, v200, s80, s80\n" \
  "v_med3_f32 v202, v194, s55, v186\n" \
  "v_mfma_f32_16x16x32_f16 a[112:115], v[136:139], v[64:67], a[112:115]\n" \
  "v_mul_f32 v207, v191, v199\n" \
  "v_exp_f32 v202, v202\n" \
  "v_mfma_f32_16x16x32_f16 a[84:87], v[128:131], v[72:75], a[84:87]\n" \
  "ds_read_b128 v[64:67], v237 offset:0\n" \
  "v_rcp_f32 v200, v200\n" \
  "v_fma_f32 v201, v201, s80, s80\n" \
  "v_mfma_f32_16x16x32_f16 a[84:87], v[132:135], v[76:79], a[84:87]\n" \
  "ds_read_b128 v[68:71], v237 offset:8192\n" \
  "v_med3_f32 v203, v195, s55, v186\n" \
  "v_mul_f32 v208, v192, v200\n" \
  "v_exp_f32 v203, v203\n" \
  "v_mfma_f32_16x16x32_f16 a[116:119], v[140:143], v[76:79], a[116:119]\n" \
  "v_rcp_f32 v201, v201\n" \
  "v_fma_f32 v202, v202, s80, s80\n" \
  "v_mfma_f32_16x16x32_f16 a[116:119], v[136:139], v[72:75], a[116:119]\n" \
  "v_mul_f32 v209, v193, v201\n" \
  "v_rcp_f32 v202, v202\n" \
  "s_waitcnt lgkmcnt(2)\n" \
  "v_mfma_f32_16x16x32_f16 a[88:91], v[128:131], v[80:83], a[88:91]\n" \
  "ds_read_b128 v[72:75], v237 offset:256\n" \
  "v_fma_f32 v203, v203, s80, s80\n" \
  "s_nop 0\n" \
  "v_mul_f32 v210, v194, v202\n" \
  "v_mfma_f32_16x16x32_f16 a[88:91], v[132:135], v[84:87], a[88:91]\n" \
  "ds_read_b128 v[76:79], v237 offset:8448\n" \
  "v_rcp_f32 v203, v203\n" \
  "s_nop 0\n" \
  "v_mfma_f32_16x16x32_f16 a[120:123], v[140:143], v[84:87], a[120:123]\n" \
  "v_mul_f32 v211, v195, v203\n" \
  "s_nop 0\n" \
  "v_mfma_f32_16x16x32_f16 a[120:123], v[136:139], v[80:83], a[120:123]\n" \
  "s_mov_b32 s79, 0\n" \
  "s_add_u32 s25, s25, 28\n" \
  "buffer_store_dwordx4 v[204:207], %15, s[72:75], s79 offen nt\n" \
  "buffer_store_dwordx4 v[208:211], %15, s[72:75], s79 offen offset:256 nt\n" \
  "v_mfma_f32_16x16x32_f16 a[92:95], v[128:131], v[88:91], a[92:95]\n" \
  "ds_read_b128 v[80:83], v237 offset:512\n" \
  "v_accvgpr_read_b32 v212, a1\n" \
  "v_accvgpr_read_b32 v213, a9\n" \
  "v_mfma_f32_16x16x32_f16 a[92:95], v[132:135], v[92:95], a[92:95]\n" \
  "ds_read_b128 v[84:87], v237 offset:8704\n" \
  "v_med3_f32 v220, v212, s55, v186\n" \
  "v_exp_f32 v220, v220\n" \
  "v_mfma_f32_16x16x32_f16 a[124:127], v[140:143], v[92:95], a[124:127]\n" \
  "v_accvgpr_read_b32 v214, a17\n" \
  "v_med3_f32 v221, v213, s55, v186\n" \
  "v_exp_f32 v221, v221\n" \
  "v_mfma_f32_16x16x32_f16 a[124:127], v[136:139], v[88:91], a[124:127]\n" \
  "v_accvgpr_read_b32 v215, a25\n" \
  "v_fma_f32 v220, v220, s80, s80\n" \
  "s_waitcnt vmcnt(26) lgkmcnt(2)\n" \
  "v_mfma_f32_16x16x32_f16 a[64:67], v[144:147], v[64:67], a[64:67]\n" \
  "ds_read_b128 v[88:91], v237 offset:768\n" \
  "v_med3_f32 v222, v214, s55, v186\n" \
  "v_exp_f32 v222, v222\n" \
  "v_mfma_f32_16x16x32_f16 a[64:67], v[148:151], v[68:71], a[64:67]\n" \
  "ds_read_b128 v[92:95], v237 offset:8960\n" \
  "v_accvgpr_read_b32 v216, a5\n" \
  "v_rcp_f32 v220, v220\n" \
  "v_mfma_f32_16x16x32_f16 a[96:99], v[156:159], v[68:71], a[96:99]\n" \
  "buffer_load_dwordx4 v[128:131], v172, s[68:71], s62 offen offset:0 sc0 sc1\n" \
  "v_fma_f32 v221, v221, s80, s80\n" \
  "v_med3_f32 v223, v215, s55, v186\n" \
  "v_mul_f32 v228, v212, v220\n" \
  "v_mfma_f32_16x16x32_f16 a[96:99], v[152:155], v[64:67], a[96:99]\n" \
  "v_exp_f32 v223, v223\n" \
  "v_accvgpr_read_b32 v217, a13\n" \
  "v_mfma_f32_16x16x32_f16 a[68:71], v[144:147], v[72:75], a[68:71]\n" \
  "ds_read_b128 v[64:67], v237 offset:1024\n" \
  "v_rcp_f32 v221, v221\n" \
  "v_fma_f32 v222, v222, s80, s80\n" \
  "v_mfma_f32_16x16x32_f16 a[68:71], v[148:151], v[76:79], a[68:71]\n" \
  "ds_read_b128 v[68:71], v237 offset:9216\n" \
  "v_med3_f32 v224, v216, s55, v186\n" \
  "v_mul_f32 v229, v213, v221\n" \
  "v_exp_f32 v224, v224\n" \
  "v_mfma_f32_16x16x32_f16 a[100:103], v[156:159], v[76:79], a[100:103]\n" \
  "buffer_load_dwordx4 v[132:135], v174, s[68:71], s62 offen offset:0 sc0 sc1\n" \
  "v_accvgpr_read_b32 v218, a21\n" \
  "v_rcp_f32 v222, v222\n" \
  "v_mfma_f32_16x16x32_f16 a[100:103], v[152:155], v[72:75], a[100:103]\n" \
  "v_fma_f32 v223, v223, s80, s80\n" \
  "v_med3_f32 v225, v217, s55, v186\n" \
  "s_waitcnt lgkmcnt(2)\n" \
  "v_mfma_f32_16x16x32_f16 a[72:75], v[144:147], v[80:83], a[72:75]\n" \
  "ds_read_b128 v[72:75], v237 offset:1280\n" \
  "v_mul_f32 v230, v214, v222\n" \
  "v_exp_f32 v225, v225\n" \
  "v_accvgpr_read_b32 v219, a29\n" \
  "v_mfma_f32_16x16x32_f16 a[72:75], v[148:151], v[84:87], a[72:75]\n" \
  "ds_read_b128 v[76:79], v237 offset:9472\n" \
  "v_rcp_f32 v223, v223\n" \
  "v_fma_f32 v224, v224, s80, s80\n" \
  "v_mfma_f32_16x16x32_f16 a[104:107], v[156:159], v[84:87], a[104:107]\n" \
  "buffer_load_dwordx4 v[136:139], v172, s[68:71], s62 offen offset:1024 sc0 sc1\n" \
  "v_med3_f32 v226, v218, s55, v186\n" \
  "v_mul_f32 v231, v215, v223\n" \
  "v_mfma_f32_16x16x32_f16 a[104:107], v[152:155], v[80:83], a[104:107]\n" \
  "v_exp_f32 v226, v226\n" \
  "v_rcp_f32 v224, v224\n" \
  "v_fma_f32 v225, v225, s80, s80\n" \
  "v_mfma_f32_16x16x32_f16 a[76:79], v[144:147], v[88:91], a[76:79]\n" \
  "ds_read_b128 v[80:83], v237 offset:1536\n" \
  "v_med3_f32 v227, v219, s55, v186\n" \
  "v_mul_f32 v232, v216, v224\n" \
  "v_mfma_f32_16x16x32_f16 a[76:79], v[148:151], v[92:95], a[76:79]\n" \
  "ds_read_b128 v[84:87], v237 offset:9728\n" \
  "v_exp_f32 v227, v227\n" \
  "v_rcp_f32 v225, v225\n" \
  "v_mfma_f32_16x16x32_f16 a[108:111], v[156:159], v[92:95], a[108:111]\n" \
  "buffer_load_dwordx4 v[140:143], v174, s[68:71], s62 offen offset:1024 sc0 sc1\n" \
  "v_fma_f32 v226, v226, s80, s80\n" \
  "v_mul_f32 v233, v217, v225\n" \
  "v_rcp_f32 v226, v226\n" \
  "v_mfma_f32_16x16x32_f16 a[108:111], v[152:155], v[88:91], a[108:111]\n" \
  "v_fma_f32 v227, v227, s80, s80\n" \
  "s_nop 0\n" \
  "s_waitcnt lgkmcnt(2)\n" \
  "v_mfma_f32_16x16x32_f16 a[80:83], v[144:147], v[64:67], a[80:83]\n" \
  "ds_read_b128 v[88:91], v237 offset:1792\n" \
  "v_mul_f32 v234, v218, v226\n" \
  "v_rcp_f32 v227, v227\n" \
  "v_mfma_f32_16x16x32_f16 a[80:83], v[148:151], v[68:71], a[80:83]\n" \
  "ds_read_b128 v[92:95], v237 offset:9984\n" \
  "s_nop 0\n" \
  "v_mul_f32 v235, v219, v227\n" \
  "v_mfma_f32_16x16x32_f16 a[112:115], v[156:159], v[68:71], a[112:115]\n" \
  "s_nop 0\n" \
  "s_mov_b32 s79, 131072\n" \
  "s_add_u32 s25, s25, 28\n" \
  "buffer_store_dwordx4 v[228:231], %15, s[72:75], s79 offen nt\n" \
  "v_mfma_f32_16x16x32_f16 a[112:115], v[152:155], v[64:67], a[112:115]\n" \
  "buffer_store_dwordx4 v[232:235], %15, s[72:75], s79 offen offset:256 nt\n" \
  "v_accvgpr_read_b32 v188, a2\n" \
  "v_mfma_f32_16x16x32_f16 a[84:87], v[144:147], v[72:75], a[84:87]\n" \
  "ds_read_b128 v[64:67], v238 offset:0\n" \
  "v_accvgpr_read_b32 v189, a10\n" \
  "v_med3_f32 v196, v188, s55, v186\n" \
  "v_mfma_f32_16x16x32_f16 a[84:87], v[148:151], v[76:79], a[84:87]\n" \
  "ds_read_b128 v[68:71], v238 offset:8192\n" \
  "v_exp_f32 v196, v196\n" \
  "v_accvgpr_read_b32 v190, a18\n" \
  "v_med3_f32 v197, v189, s55, v186\n" \
  "v_mfma_f32_16x16x32_f16 a[116:119], v[156:159], v[76:79], a[116:119]\n" \
  "v_exp_f32 v197, v197\n" \
  "v_accvgpr_read_b32 v191, a26\n" \
  "v_mfma_f32_16x16x32_f16 a[116:119], v[152:155], v[72:75], a[116:119]\n" \
  "v_fma_f32 v196, v196, s80, s80\n" \
  "v_med3_f32 v198, v190, s55, v186\n" \
  "s_waitcnt lgkmcnt(2)\n" \
  "v_mfma_f32_16x16x32_f16 a[88:91], v[144:147], v[80:83], a[88:91]\n" \
  "ds_read_b128 v[72:75], v238 offset:256\n" \
  "v_exp_f32 v198, v198\n" \
  "v_accvgpr_read_b32 v192, a6\n" \
  "v_rcp_f32 v196, v196\n" \
  "v_mfma_f32_16x16x32_f16 a[88:91], v[148:151], v[84:87], a[88:91]\n" \
  "ds_read_b128 v[76:79], v238 offset:8448\n" \
  "v_fma_f32 v197, v197, s80, s80\n" \
  "v_med3_f32 v199, v191, s55, v186\n" \
  "v_mfma_f32_16x16x32_f16 a[120:123], v[156:159], v[84:87], a[120:123]\n" \
  "v_mul_f32 v204, v188, v196\n" \
  "v_exp_f32 v199, v199\n" \
  "v_mfma_f32_16x16x32_f16 a[120:123], v[152:155], v[80:83], a[120:123]\n" \
  "v_accvgpr_read_b32 v193, a14\n" \
  "v_rcp_f32 v197, v197\n" \
  "v_fma_f32 v198, v198, s80, s80\n" \
  "v_mfma_f32_16x16x32_f16 a[92:95], v[144:147], v[88:91], a[92:95]\n" \
  "ds_read_b128 v[80:83], v238 offset:512\n" \
  "v_med3_f32 v200, v192, s55, v186\n" \
  "v_mul_f32 v205, v189, v197\n" \
  "v_mfma_f32_16x16x32_f16 a[92:95], v[148:151], v[92:95], a[92:95]\n" \
  "ds_read_b128 v[84:87], v238 offset:8704\n" \
  "v_exp_f32 v200, v200\n" \
  "v_accvgpr_read_b32 v194, a22\n" \
  "v_mfma_f32_16x16x32_f16 a[124:127], v[156:159], v[92:95], a[124:127]\n" \
  "v_rcp_f32 v198, v198\n" \
  "v_fma_f32 v199, v199, s80, s80\n" \
  "v_mfma_f32_16x16x32_f16 a[124:127], v[152:155], v[88:91], a[124:127]\n" \
  "v_med3_f32 v201, v193, s55, v186\n" \
  "v_mul_f32 v206, v190, v198\n" \
  "v_exp_f32 v201, v201\n" \
  "s_waitcnt vmcnt(28) lgkmcnt(2)\n" \
  "v_mfma_f32_16x16x32_f16 a[64:67], v[0:3], v[64:67], a[64:67]\n" \
  "ds_read_b128 v[88:91], v238 offset:768\n" \
  "v_accvgpr_read_b32 v195, a30\n" \
  "v_rcp_f32 v199, v199\n" \
  "v_mfma_f32_16x16x32_f16 a[64:67], v[4:7], v[68:71], a[64:67]\n" \
  "ds_read_b128 v[92:95], v238 offset:8960\n" \
  "v_fma_f32 v200, v200, s80, s80\n" \
  "v_med3_f32 v202, v194, s55, v186\n" \
  "v_mfma_f32_16x16x32_f16 a[96:99], v[12:15], v[68:71], a[96:99]\n" \
  "buffer_load_dwordx4 v[144:147], v172, s[68:71], s63 offen offset:0 sc0 sc1\n" \
  "v_mul_f32 v207, v191, v199\n" \
  "v_exp_f32 v202, v202\n" \
  "v_rcp_f32 v200, v200\n" \
  "v_mfma_f32_16x16x32_f16 a[96:99], v[8:11], v[64:67], a[96:99]\n" \
  "v_fma_f32 v201, v201, s80, s80\n" \
  "v_med3_f32 v203, v195, s55, v186\n" \
  "v_mfma_f32_16x16x32_f16 a[68:71], v[0:3], v[72:75], a[68:71]\n" \
  "ds_read_b128 v[64:67], v238 offset:1024\n" \
  "v_mul_f32 v208, v192, v200\n" \
  "v_exp_f32 v203, v203\n" \
  "v_mfma_f32_16x16x32_f16 a[68:71], v[4:7], v[76:79], a[68:71]\n" \
  "ds_read_b128 v[68:71], v238 offset:9216\n" \
  "v_rcp_f32 v201, v201\n" \
  "v_fma_f32 v202, v202, s80, s80\n" \
  "v_mul_f32 v209, v193, v201\n" \
  "v_mfma_f32_16x16x32_f16 a[100:103], v[12:15], v[76:79], a[100:103]\n" \
  "buffer_load_dwordx4 v[148:151], v174, s[68:71], s63 offen offset:0 sc0 sc1\n" \
  "v_rcp_f32 v202, v202\n" \
  "v_fma_f32 v203, v203, s80, s80\n" \
  "v_mfma_f32_16x16x32_f16 a[100:103], v[8:11], v[72:75], a[100:103]\n" \
  "s_nop 0\n" \
  "v_mul_f32 v210, v194, v202\n" \
  "s_waitcnt lgkmcnt(2)\n" \
  "v_mfma_f32_16x16x32_f16 a[72:75], v[0:3], v[80:83], a[72:75]\n" \
  "ds_read_b128 v[72:75], v238 offset:1280\n" \
  "v_rcp_f32 v203, v203\n" \
  "s_nop 0\n" \
  "v_mul_f32 v211, v195, v203\n" \
  "v_mfma_f32_16x16x32_f16 a[72:75], v[4:7], v[84:87], a[72:75]\n" \
  "ds_read_b128 v[76:79], v238 offset:9472\n" \
  "s_nop 0\n" \
  "s_mov_b32 s79, 262144\n" \
  "v_mfma_f32_16x16x32_f16 a[104:107], v[12:15], v[84:87], a[104:107]\n" \
  "buffer_load_dwordx4 v[152:155], v172, s[68:71], s63 offen offset:1024 sc0 sc1\n" \
  "s_add_u32 s25, s25, 28\n" \
  "buffer_store_dwordx4 v[204:207], %15, s[72:75], s79 offen nt\n" \
  "buffer_store_dwordx4 v[208:211], %15, s[72:75], s79 offen offset:256 nt\n" \
  "v_mfma_f32_16x16x32_f16 a[104:107], v[8:11], v[80:83], a[104:107]\n" \
  "v_accvgpr_read_b32 v212, a3\n" \
  "v_accvgpr_read_b32 v213, a11\n" \
  "v_med3_f32 v220, v212, s55, v186\n" \
  "v_mfma_f32_16x16x32_f16 a[76:79], v[0:3], v[88:91], a[76:79]\n" \
  "ds_read_b128 v[80:83], v238 offset:1536\n" \
  "v_exp_f32 v220, v220\n" \
  "v_accvgpr_read_b32 v214, a19\n" \
  "v_mfma_f32_16x16x32_f16 a[76:79], v[4:7], v[92:95], a[76:79]\n" \
  "ds_read_b128 v[84:87], v238 offset:9728\n" \
  "v_med3_f32 v221, v213, s55, v186\n" \
  "v_exp_f32 v221, v221\n" \
  "v_mfma_f32_16x16x32_f16 a[108:111], v[12:15], v[92:95], a[108:111]\n" \
  "buffer_load_dwordx4 v[156:159], v174, s[68:71], s63 offen offset:1024 sc0 sc1\n" \
  "v_accvgpr_read_b32 v215, a27\n" \
  "v_fma_f32 v220, v220, s80, s80\n" \
  "v_mfma_f32_16x16x32_f16 a[108:111], v[8:11], v[88:91], a[108:111]\n" \
  "v_med3_f32 v222, v214, s55, v186\n" \
  "v_exp_f32 v222, v222\n" \
  "v_accvgpr_read_b32 v216, a7\n" \
  "s_waitcnt lgkmcnt(2)\n" \
  "v_mfma_f32_16x16x32_f16 a[80:83], v[0:3], v[64:67], a[80:83]\n" \
  "ds_read_b128 v[88:91], v238 offset:1792\n" \
  "v_rcp_f32 v220, v220\n" \
  "v_fma_f32 v221, v221, s80, s80\n" \
  "v_mfma_f32_16x16x32_f16 a[80:83], v[4:7], v[68:71], a[80:83]\n" \
  "ds_read_b128 v[92:95], v238 offset:9984\n" \
  "v_med3_f32 v223, v215, s55, v186\n" \
  "v_mul_f32 v228, v212, v220\n" \
  "v_mfma_f32_16x16x32_f16 a[112:115], v[12:15], v[68:71], a[112:115]\n" \
  "v_exp_f32 v223, v223\n" \
  "v_accvgpr_read_b32 v217, a15\n" \
  "v_rcp_f32 v221, v221\n" \
  "v_mfma_f32_16x16x32_f16 a[112:115], v[8:11], v[64:67], a[112:115]\n" \
  "v_fma_f32 v222, v222, s80, s80\n" \
  "v_med3_f32 v224, v216, s55, v186\n" \
  "v_mfma_f32_16x16x32_f16 a[84:87], v[0:3], v[72:75], a[84:87]\n" \
  "ds_read_b128 v[64:67], v239 offset:0\n" \
  "v_mul_f32 v229, v213, v221\n" \
  "v_exp_f32 v224, v224\n" \
  "v_mfma_f32_16x16x32_f16 a[84:87], v[4:7], v[76:79], a[84:87]\n" \
  "ds_read_b128 v[68:71], v239 offset:8192\n" \
  "v_accvgpr_read_b32 v218, a23\n" \
  "v_rcp_f32 v222, v222\n" \
  "v_fma_f32 v223, v223, s80, s80\n" \
  "v_mfma_f32_16x16x32_f16 a[116:119], v[12:15], v[76:79], a[116:119]\n" \
  "v_med3_f32 v225, v217, s55, v186\n" \
  "v_mul_f32 v230, v214, v222\n" \
  "v_mfma_f32_16x16x32_f16 a[116:119], v[8:11], v[72:75], a[116:119]\n" \
  "v_exp_f32 v225, v225\n" \
  "v_accvgpr_read_b32 v219, a31\n" \
  "s_waitcnt lgkmcnt(2)\n" \
  "v_mfma_f32_16x16x32_f16 a[88:91], v[0:3], v[80:83], a[88:91]\n" \
  "ds_read_b128 v[72:75], v239 offset:256\n" \
  "v_rcp_f32 v223, v223\n" \
  "v_fma_f32 v224, v224, s80, s80\n" \
  "v_med3_f32 v226, v218, s55, v186\n" \
  "v_mfma_f32_16x16x32_f16 a[88:91], v[4:7], v[84:87], a[88:91]\n" \
  "ds_read_b128 v[76:79], v239 offset:8448\n" \
  "v_mul_f32 v231, v215, v223\n" \
  "v_exp_f32 v226, v226\n" \
  "v_mfma_f32_16x16x32_f16 a[120:123], v[12:15], v[84:87], a[120:123]\n" \
  "v_rcp_f32 v224, v224\n" \
  "v_fma_f32 v225, v225, s80, s80\n" \
  "v_mfma_f32_16x16x32_f16 a[120:123], v[8:11], v[80:83], a[120:123]\n" \
  "v_med3_f32 v227, v219, s55, v186\n" \
  "v_mul_f32 v232, v216, v224\n" \
  "v_mfma_f32_16x16x32_f16 a[92:95], v[0:3], v[88:91], a[92:95]\n" \
  "ds_read_b128 v[80:83], v239 offset:512\n" \
  "v_exp_f32 v227, v227\n" \
  "v_rcp_f32 v225, v225\n" \
  "v_fma_f32 v226, v226, s80, s80\n" \
  "v_mfma_f32_16x16x32_f16 a[92:95], v[4:7], v[92:95], a[92:95]\n" \
  "ds_read_b128 v[84:87], v239 offset:8704\n" \
  "v_mul_f32 v233, v217, v225\n" \
  "v_rcp_f32 v226, v226\n" \
  "v_mfma_f32_16x16x32_f16 a[124:127], v[12:15], v[92:95], a[124:127]\n" \
  "v_fma_f32 v227, v227, s80, s80\n" \
  "s_nop 0\n" \
  "v_mfma_f32_16x16x32_f16 a[124:127], v[8:11], v[88:91], a[124:127]\n" \
  "v_mul_f32 v234, v218, v226\n" \
  "v_rcp_f32 v227, v227\n" \
  "s_nop 0\n" \
  "s_waitcnt vmcnt(30) lgkmcnt(2)\n" \
  "v_mfma_f32_16x16x32_f16 a[64:67], v[16:19], v[64:67], a[64:67]\n" \
  "ds_read_b128 v[88:91], v239 offset:768\n" \
  "v_mul_f32 v235, v219, v227\n" \
  "s_nop 0\n" \
  "v_mfma_f32_16x16x32_f16 a[64:67], v[20:23], v[68:71], a[64:67]\n" \
  "ds_read_b128 v[92:95], v239 offset:8960\n" \
  "s_mov_b32 s79, 393216\n" \
  "s_add_u32 s25, s25, 28\n" \
  "buffer_store_dwordx4 v[228:231], %15, s[72:75], s79 offen nt\n" \
  "v_mfma_f32_16x16x32_f16 a[96:99], v[28:31], v[68:71], a[96:99]\n" \
  "buffer_load_dwordx4 v[0:3], v172, s[68:71], s53 offen offset:0 sc0 sc1\n" \
  "buffer_store_dwordx4 v[232:235], %15, s[72:75], s79 offen offset:256 nt\n" \
  "v_accvgpr_read_b32 v188, a32\n" \
  "v_accvgpr_read_b32 v189, a40\n" \
  "v_mfma_f32_16x16x32_f16 a[96:99], v[24:27], v[64:67], a[96:99]\n" \
  "v_med3_f32 v196, v188, s55, v186\n" \
  "v_exp_f32 v196, v196\n" \
  "v_mfma_f32_16x16x32_f16 a[68:71], v[16:19], v[72:75], a[68:71]\n" \
  "ds_read_b128 v[64:67], v239 offset:1024\n" \
  "v_accvgpr_read_b32 v190, a48\n" \
  "v_med3_f32 v197, v189, s55, v186\n" \
  "v_mfma_f32_16x16x32_f16 a[68:71], v[20:23], v[76:79], a[68:71]\n" \
  "ds_read_b128 v[68:71], v239 offset:9216\n" \
  "v_exp_f32 v197, v197\n" \
  "v_accvgpr_read_b32 v191, a56\n" \
  "v_fma_f32 v196, v196, s80, s80\n" \
  "v_mfma_f32_16x16x32_f16 a[100:103], v[28:31], v[76:79], a[100:103]\n" \
  "buffer_load_dwordx4 v[4:7], v174, s[68:71], s53 offen offset:0 sc0 sc1\n" \
  "v_med3_f32 v198, v190, s55, v186\n" \
  "v_exp_f32 v198, v198\n" \
  "v_mfma_f32_16x16x32_f16 a[100:103], v[24:27], v[72:75], a[100:103]\n" \
  "v_accvgpr_read_b32 v192, a36\n" \
  "v_rcp_f32 v196, v196\n" \
  "s_waitcnt lgkmcnt(2)\n" \
  "v_mfma_f32_16x16x32_f16 a[72:75], v[16:19], v[80:83], a[72:75]\n" \
  "ds_read_b128 v[72:75], v239 offset:1280\n" \
  "v_fma_f32 v197, v197, s80, s80\n" \
  "v_med3_f32 v199, v191, s55, v186\n" \
  "v_mul_f32 v204, v188, v196\n" \
  "v_mfma_f32_16x16x32_f16 a[72:75], v[20:23], v[84:87], a[72:75]\n" \
  "ds_read_b128 v[76:79], v239 offset:9472\n" \
  "v_exp_f32 v199, v199\n" \
  "v_accvgpr_read_b32 v193, a44\n" \
  "v_mfma_f32_16x16x32_f16 a[104:107], v[28:31], v[84:87], a[104:107]\n" \
  "buffer_load_dwordx4 v[8:11], v172, s[68:71], s53 offen offset:1024 sc0 sc1\n" \
  "v_rcp_f32 v197, v197\n" \
  "v_fma_f32 v198, v198, s80, s80\n" \
  "v_mfma_f32_16x16x32_f16 a[104:107], v[24:27], v[80:83], a[104:107]\n" \
  "v_med3_f32 v200, v192, s55, v186\n" \
  "v_mul_f32 v205, v189, v197\n" \
  "v_mfma_f32_16x16x32_f16 a[76:79], v[16:19], v[88:91], a[76:79]\n" \
  "ds_read_b128 v[80:83], v239 offset:1536\n" \
  "v_exp_f32 v200, v200\n" \
  "v_accvgpr_read_b32 v194, a52\n" \
  "v_rcp_f32 v198, v198\n" \
  "v_mfma_f32_16x16x32_f16 a[76:79], v[20:23], v[92:95], a[76:79]\n" \
  "ds_read_b128 v[84:87], v239 offset:9728\n" \
  "v_fma_f32 v199, v199, s80, s80\n" \
  "v_med3_f32 v201, v193, s55, v186\n" \
  "v_mfma_f32_16x16x32_f16 a[108:111], v[28:31], v[92:95], a[108:111]\n" \
  "buffer_load_dwordx4 v[12:15], v174, s[68:71], s53 offen offset:1024 sc0 sc1\n" \
  "v_mul_f32 v206, v190, v198\n" \
  "v_exp_f32 v201, v201\n" \
  "v_mfma_f32_16x16x32_f16 a[108:111], v[24:27], v[88:91], a[108:111]\n" \
  "v_accvgpr_read_b32 v195, a60\n" \
  "v_rcp_f32 v199, v199\n" \
  "v_fma_f32 v200, v200, s80, s80\n" \
  "s_waitcnt lgkmcnt(2)\n" \
  "v_mfma_f32_16x16x32_f16 a[80:83], v[16:19], v[64:67], a[80:83]\n" \
  "ds_read_b128 v[88:91], v239 offset:1792\n" \
  "v_med3_f32 v202, v194, s55, v186\n" \
  "v_mul_f32 v207, v191, v199\n" \
  "v_mfma_f32_16x16x32_f16 a[80:83], v[20:23], v[68:71], a[80:83]\n" \
  "ds_read_b128 v[92:95], v239 offset:9984\n" \
  "v_exp_f32 v202, v202\n" \
  "v_rcp_f32 v200, v200\n" \
  "v_mfma_f32_16x16x32_f16 a[112:115], v[28:31], v[68:71], a[112:115]\n" \
  "v_fma_f32 v201, v201, s80, s80\n" \
  "v_med3_f32 v203, v195, s55, v186\n" \
  "v_mul_f32 v208, v192, v200\n" \
  "v_mfma_f32_16x16x32_f16 a[112:115], v[24:27], v[64:67], a[112:115]\n" \
  "v_exp_f32 v203, v203\n" \
  "v_rcp_f32 v201, v201\n" \
  "v_mfma_f32_16x16x32_f16 a[84:87], v[16:19], v[72:75], a[84:87]\n" \
  "ds_read_b128 v[64:67], v240 offset:0\n" \
  "v_fma_f32 v202, v202, s80, s80\n" \
  "v_mul_f32 v209, v193, v201\n" \
  "v_mfma_f32_16x16x32_f16 a[84:87], v[20:23], v[76:79], a[84:87]\n" \
  "ds_read_b128 v[68:71], v240 offset:8192\n" \
  "v_rcp_f32 v202, v202\n" \
  "v_fma_f32 v203, v203, s80, s80\n" \
  "s_nop 0\n" \
  "v_mfma_f32_16x16x32_f16 a[116:119], v[28:31], v[76:79], a[116:119]\n" \
  "v_mul_f32 v210, v194, v202\n" \
  "v_rcp_f32 v203, v203\n" \
  "v_mfma_f32_16x16x32_f16 a[116:119], v[24:27], v[72:75], a[116:119]\n" \
  "s_nop 0\n" \
  "v_mul_f32 v211, v195, v203\n" \
  "s_waitcnt lgkmcnt(2)\n" \
  "v_mfma_f32_16x16x32_f16 a[88:91], v[16:19], v[80:83], a[88:91]\n" \
  "ds_read_b128 v[72:75], v240 offset:256\n" \
  "s_nop 0\n" \
  "s_mov_b32 s79, 2097152\n" \
  "v_mfma_f32_16x16x32_f16 a[88:91], v[20:23], v[84:87], a[88:91]\n" \
  "ds_read_b128 v[76:79], v240 offset:8448\n" \
  "s_add_u32 s25, s25, 28\n" \
  "buffer_store_dwordx4 v[204:207], %15, s[72:75], s79 offen nt\n" \
  "buffer_store_dwordx4 v[208:211], %15, s[72:75], s79 offen offset:256 nt\n" \
  "v_accvgpr_read_b32 v212, a33\n" \
  "v_mfma_f32_16x16x32_f16 a[120:123], v[28:31], v[84:87], a[120:123]\n" \
  "v_accvgpr_read_b32 v213, a41\n" \
  "v_med3_f32 v220, v212, s55, v186\n" \
  "v_mfma_f32_16x16x32_f16 a[120:123], v[24:27], v[80:83], a[120:123]\n" \
  "v_exp_f32 v220, v220\n" \
  "v_accvgpr_read_b32 v214, a49\n" \
  "v_mfma_f32_16x16x32_f16 a[92:95], v[16:19], v[88:91], a[92:95]\n" \
  "ds_read_b128 v[80:83], v240 offset:512\n" \
  "v_med3_f32 v221, v213, s55, v186\n" \
  "v_exp_f32 v221, v221\n" \
  "v_accvgpr_read_b32 v215, a57\n" \
  "v_mfma_f32_16x16x32_f16 a[92:95], v[20:23], v[92:95], a[92:95]\n" \
  "ds_read_b128 v[84:87], v240 offset:8704\n" \
  "v_fma_f32 v220, v220, s80, s80\n" \
  "v_med3_f32 v222, v214, s55, v186\n" \
  "v_mfma_f32_16x16x32_f16 a[124:127], v[28:31], v[92:95], a[124:127]\n" \
  "v_exp_f32 v222, v222\n" \
  "v_accvgpr_read_b32 v216, a37\n" \
  "v_mfma_f32_16x16x32_f16 a[124:127], v[24:27], v[88:91], a[124:127]\n" \
  "v_rcp_f32 v220, v220\n" \
  "v_fma_f32 v221, v221, s80, s80\n" \
  "v_med3_f32 v223, v215, s55, v186\n" \
  "s_waitcnt vmcnt(34) lgkmcnt(2)\n" \
  "v_mfma_f32_16x16x32_f16 a[64:67], v[32:35], v[64:67], a[64:67]\n" \
  "ds_read_b128 v[88:91], v240 offset:768\n" \
  "v_mul_f32 v228, v212, v220\n" \
  "v_exp_f32 v223, v223\n" \
  "v_mfma_f32_16x16x32_f16 a[64:67], v[36:39], v[68:71], a[64:67]\n" \
  "ds_read_b128 v[92:95], v240 offset:8960\n" \
  "v_accvgpr_read_b32 v217, a45\n" \
  "v_rcp_f32 v221, v221\n" \
  "v_mfma_f32_16x16x32_f16 a[96:99], v[44:47], v[68:71], a[96:99]\n" \
  "buffer_load_dwordx4 v[16:19], v172, s[68:71], s54 offen offset:0 sc0 sc1\n" \
  "v_fma_f32 v222, v222, s80, s80\n" \
  "v_med3_f32 v224, v216, s55, v186\n" \
  "v_mul_f32 v229, v213, v221\n" \
  "v_mfma_f32_16x16x32_f16 a[96:99], v[40:43], v[64:67], a[96:99]\n" \
  "v_exp_f32 v224, v224\n" \
  "v_accvgpr_read_b32 v218, a53\n" \
  "v_mfma_f32_16x16x32_f16 a[68:71], v[32:35], v[72:75], a[68:71]\n" \
  "ds_read_b128 v[64:67], v240 offset:1024\n" \
  "v_rcp_f32 v222, v222\n" \
  "v_fma_f32 v223, v223, s80, s80\n" \
  "v_mfma_f32_16x16x32_f16 a[68:71], v[36:39], v[76:79], a[68:71]\n" \
  "ds_read_b128 v[68:71], v240 offset:9216\n" \
  "v_med3_f32 v225, v217, s55, v186\n" \
  "v_mul_f32 v230, v214, v222\n" \
  "v_exp_f32 v225, v225\n" \
  "v_mfma_f32_16x16x32_f16 a[100:103], v[44:47], v[76:79], a[100:103]\n" \
  "buffer_load_dwordx4 v[20:23], v174, s[68:71], s54 offen offset:0 sc0 sc1\n" \
  "v_accvgpr_read_b32 v219, a61\n" \
  "v_rcp_f32 v223, v223\n" \
  "v_mfma_f32_16x16x32_f16 a[100:103], v[40:43], v[72:75], a[100:103]\n" \
  "v_fma_f32 v224, v224, s80, s80\n" \
  "v_med3_f32 v226, v218, s55, v186\n" \
  "s_waitcnt lgkmcnt(2)\n" \
  "v_mfma_f32_16x16x32_f16 a[72:75], v[32:35], v[80:83], a[72:75]\n" \
  "ds_read_b128 v[72:75], v240 offset:1280\n" \
  "v_mul_f32 v231, v215, v223\n" \
  "v_exp_f32 v226, v226\n" \
  "v_mfma_f32_16x16x32_f16 a[72:75], v[36:39], v[84:87], a[72:75]\n" \
  "ds_read_b128 v[76:79], v240 offset:9472\n" \
  "v_rcp_f32 v224, v224\n" \
  "v_fma_f32 v225, v225, s80, s80\n" \
  "v_med3_f32 v227, v219, s55, v186\n" \
  "v_mfma_f32_16x16x32_f16 a[104:107], v[44:47], v[84:87], a[104:107]\n" \
  "buffer_load_dwordx4 v[24:27], v172, s[68:71], s54 offen offset:1024 sc0 sc1\n" \
  "v_mul_f32 v232, v216, v224\n" \
  "v_exp_f32 v227, v227\n" \
  "v_mfma_f32_16x16x32_f16 a[104:107], v[40:43], v[80:83], a[104:107]\n" \
  "v_rcp_f32 v225, v225\n" \
  "v_fma_f32 v226, v226, s80, s80\n" \
  "v_mfma_f32_16x16x32_f16 a[76:79], v[32:35], v[88:91], a[76:79]\n" \
  "ds_read_b128 v[80:83], v240 offset:1536\n" \
  "v_mul_f32 v233, v217, v225\n" \
  "v_rcp_f32 v226, v226\n" \
  "v_fma_f32 v227, v227, s80, s80\n" \
  "v_mfma_f32_16x16x32_f16 a[76:79], v[36:39], v[92:95], a[76:79]\n" \
  "ds_read_b128 v[84:87], v240 offset:9728\n" \
  "s_nop 0\n" \
  "v_mul_f32 v234, v218, v226\n" \
  "v_mfma_f32_16x16x32_f16 a[108:111], v[44:47], v[92:95], a[108:111]\n" \
  "buffer_load_dwordx4 v[28:31], v174, s[68:71], s54 offen offset:1024 sc0 sc1\n" \
  "v_rcp_f32 v227, v227\n" \
  "s_nop 0\n" \
  "v_mfma_f32_16x16x32_f16 a[108:111], v[40:43], v[88:91], a[108:111]\n" \
  "v_mul_f32 v235, v219, v227\n" \
  "s_nop 0\n" \
  "s_mov_b32 s79, 2228224\n" \
  "s_waitcnt lgkmcnt(2)\n" \
  "v_mfma_f32_16x16x32_f16 a[80:83], v[32:35], v[64:67], a[80:83]\n" \
  "ds_read_b128 v[88:91], v240 offset:1792\n" \
  "s_add_u32 s25, s25, 28\n" \
  "buffer_store_dwordx4 v[228:231], %15, s[72:75], s79 offen nt\n" \
  "buffer_store_dwordx4 v[232:235], %15, s[72:75], s79 offen offset:256 nt\n" \
  "v_mfma_f32_16x16x32_f16 a[80:83], v[36:39], v[68:71], a[80:83]\n" \
  "ds_read_b128 v[92:95], v240 offset:9984\n" \
  "v_accvgpr_read_b32 v188, a34\n" \
  "v_accvgpr_read_b32 v189, a42\n" \
  "v_mfma_f32_16x16x32_f16 a[112:115], v[44:47], v[68:71], a[112:115]\n" \
  "v_med3_f32 v196, v188, s55, v186\n" \
  "v_exp_f32 v196, v196\n" \
  "v_accvgpr_read_b32 v190, a50\n" \
  "v_mfma_f32_16x16x32_f16 a[112:115], v[40:43], v[64:67], a[112:115]\n" \
  "v_med3_f32 v197, v189, s55, v186\n" \
  "v_exp_f32 v197, v197\n" \
  "v_mfma_f32_16x16x32_f16 a[84:87], v[32:35], v[72:75], a[84:87]\n" \
  "ds_read_b128 v[64:67], v241 offset:0\n" \
  "v_accvgpr_read_b32 v191, a58\n" \
  "v_fma_f32 v196, v196, s80, s80\n" \
  "v_mfma_f32_16x16x32_f16 a[84:87], v[36:39], v[76:79], a[84:87]\n" \
  "ds_read_b128 v[68:71], v241 offset:8192\n" \
  "v_med3_f32 v198, v190, s55, v186\n" \
  "v_exp_f32 v198, v198\n" \
  "v_mfma_f32_16x16x32_f16 a[116:119], v[44:47], v[76:79], a[116:119]\n" \
  "v_accvgpr_read_b32 v192, a38\n" \
  "v_rcp_f32 v196, v196\n" \
  "v_fma_f32 v197, v197, s80, s80\n" \
  "v_mfma_f32_16x16x32_f16 a[116:119], v[40:43], v[72:75], a[116:119]\n" \
  "v_med3_f32 v199, v191, s55, v186\n" \
  "v_mul_f32 v204, v188, v196\n" \
  "s_waitcnt lgkmcnt(2)\n" \
  "v_mfma_f32_16x16x32_f16 a[88:91], v[32:35], v[80:83], a[88:91]\n" \
  "ds_read_b128 v[72:75], v241 offset:256\n" \
  "v_exp_f32 v199, v199\n" \
  "v_accvgpr_read_b32 v193, a46\n" \
  "v_mfma_f32_16x16x32_f16 a[88:91], v[36:39], v[84:87], a[88:91]\n" \
  "ds_read_b128 v[76:79], v241 offset:8448\n" \
  "v_rcp_f32 v197, v197\n" \
  "v_fma_f32 v198, v198, s80, s80\n" \
  "v_med3_f32 v200, v192, s55, v186\n" \
  "v_mfma_f32_16x16x32_f16 a[120:123], v[44:47], v[84:87], a[120:123]\n" \
  "v_mul_f32 v205, v189, v197\n" \
  "v_exp_f32 v200, v200\n" \
  "v_mfma_f32_16x16x32_f16 a[120:123], v[40:43], v[80:83], a[120:123]\n" \
  "v_accvgpr_read_b32 v194, a54\n" \
  "v_rcp_f32 v198, v198\n" \
  "v_mfma_f32_16x16x32_f16 a[92:95], v[32:35], v[88:91], a[92:95]\n" \
  "ds_read_b128 v[80:83], v241 offset:512\n" \
  "v_fma_f32 v199, v199, s80, s80\n" \
  "v_med3_f32 v201, v193, s55, v186\n" \
  "v_mul_f32 v206, v190, v198\n" \
  "v_mfma_f32_16x16x32_f16 a[92:95], v[36:39], v[92:95], a[92:95]\n" \
  "ds_read_b128 v[84:87], v241 offset:8704\n" \
  "v_exp_f32 v201, v201\n" \
  "v_accvgpr_read_b32 v195, a62\n" \
  "v_mfma_f32_16x16x32_f16 a[124:127], v[44:47], v[92:95], a[124:127]\n" \
  "v_rcp_f32 v199, v199\n" \
  "v_fma_f32 v200, v200, s80, s80\n" \
  "v_mfma_f32_16x16x32_f16 a[124:127], v[40:43], v[88:91], a[124:127]\n" \
  "v_med3_f32 v202, v194, s55, v186\n" \
  "v_mul_f32 v207, v191, v199\n" \
  "v_exp_f32 v202, v202\n" \
  "s_waitcnt vmcnt(36) lgkmcnt(2)\n" \
  "v_mfma_f32_16x16x32_f16 a[64:67], v[48:51], v[64:67], a[64:67]\n" \
  "ds_read_b128 v[88:91], v241 offset:768\n" \
  "v_rcp_f32 v200, v200\n" \
  "v_fma_f32 v201, v201, s80, s80\n" \
  "v_mfma_f32_16x16x32_f16 a[64:67], v[52:55], v[68:71], a[64:67]\n" \
  "ds_read_b128 v[92:95], v241 offset:8960\n" \
  "v_med3_f32 v203, v195, s55, v186\n" \
  "v_mul_f32 v208, v192, v200\n" \
  "v_mfma_f32_16x16x32_f16 a[96:99], v[60:63], v[68:71], a[96:99]\n" \
  "buffer_load_dwordx4 v[32:35], v172, s[68:71], s60 offen offset:2048 sc0 sc1\n" \
  "v_exp_f32 v203, v203\n" \
  "v_rcp_f32 v201, v201\n" \
  "v_fma_f32 v202, v202, s80, s80\n" \
  "v_mfma_f32_16x16x32_f16 a[96:99], v[56:59], v[64:67], a[96:99]\n" \
  "v_mul_f32 v209, v193, v201\n" \
  "v_rcp_f32 v202, v202\n" \
  "v_mfma_f32_16x16x32_f16 a[68:71], v[48:51], v[72:75], a[68:71]\n" \
  "ds_read_b128 v[64:67], v241 offset:1024\n" \
  "v_fma_f32 v203, v203, s80, s80\n" \
  "s_nop 0\n" \
  "v_mfma_f32_16x16x32_f16 a[68:71], v[52:55], v[76:79], a[68:71]\n" \
  "ds_read_b128 v[68:71], v241 offset:9216\n" \
  "v_mul_f32 v210, v194, v202\n" \
  "v_rcp_f32 v203, v203\n" \
  "v_mfma_f32_16x16x32_f16 a[100:103], v[60:63], v[76:79], a[100:103]\n" \
  "buffer_load_dwordx4 v[36:39], v174, s[68:71], s60 offen offset:2048 sc0 sc1\n" \
  "s_nop 0\n" \
  "v_mul_f32 v211, v195, v203\n" \
  "s_nop 0\n" \
  "v_mfma_f32_16x16x32_f16 a[100:103], v[56:59], v[72:75], a[100:103]\n" \
  "s_mov_b32 s79, 2359296\n" \
  "s_add_u32 s25, s25, 28\n" \
  "buffer_store_dwordx4 v[204:207], %15, s[72:75], s79 offen nt\n" \
  "s_waitcnt lgkmcnt(2)\n" \
  "v_mfma_f32_16x16x32_f16 a[72:75], v[48:51], v[80:83], a[72:75]\n" \
  "ds_read_b128 v[72:75], v241 offset:1280\n" \
  "buffer_store_dwordx4 v[208:211], %15, s[72:75], s79 offen offset:256 nt\n" \
  "v_accvgpr_read_b32 v212, a35\n" \
  "v_mfma_f32_16x16x32_f16 a[72:75], v[52:55], v[84:87], a[72:75]\n" \
  "ds_read_b128 v[76:79], v241 offset:9472\n" \
  "v_accvgpr_read_b32 v213, a43\n" \
  "v_med3_f32 v220, v212, s55, v186\n" \
  "v_exp_f32 v220, v220\n" \
  "v_mfma_f32_16x16x32_f16 a[104:107], v[60:63], v[84:87], a[104:107]\n" \
  "buffer_load_dwordx4 v[40:43], v172, s[68:71], s60 offen offset:3072 sc0 sc1\n" \
  "v_accvgpr_read_b32 v214, a51\n" \
  "v_med3_f32 v221, v213, s55, v186\n" \
  "v_mfma_f32_16x16x32_f16 a[104:107], v[56:59], v[80:83], a[104:107]\n" \
  "v_exp_f32 v221, v221\n" \
  "v_accvgpr_read_b32 v215, a59\n" \
  "v_mfma_f32_16x16x32_f16 a[76:79], v[48:51], v[88:91], a[76:79]\n" \
  "ds_read_b128 v[80:83], v241 offset:1536\n" \
  "v_fma_f32 v220, v220, s80, s80\n" \
  "v_med3_f32 v222, v214, s55, v186\n" \
  "v_exp_f32 v222, v222\n" \
  "v_mfma_f32_16x16x32_f16 a[76:79], v[52:55], v[92:95], a[76:79]\n" \
  "ds_read_b128 v[84:87], v241 offset:9728\n" \
  "v_accvgpr_read_b32 v216, a39\n" \
  "v_rcp_f32 v220, v220\n" \
  "v_mfma_f32_16x16x32_f16 a[108:111], v[60:63], v[92:95], a[108:111]\n" \
  "buffer_load_dwordx4 v[44:47], v174, s[68:71], s60 offen offset:3072 sc0 sc1\n" \
  "v_fma_f32 v221, v221, s80, s80\n" \
  "v_med3_f32 v223, v215, s55, v186\n" \
  "v_mfma_f32_16x16x32_f16 a[108:111], v[56:59], v[88:91], a[108:111]\n" \
  "v_mul_f32 v228, v212, v220\n" \
  "v_exp_f32 v223, v223\n" \
  "v_accvgpr_read_b32 v217, a47\n" \
  "s_waitcnt lgkmcnt(2)\n" \
  "v_mfma_f32_16x16x32_f16 a[80:83], v[48:51], v[64:67], a[80:83]\n" \
  "ds_read_b128 v[88:91], v241 offset:1792\n" \
  "v_rcp_f32 v221, v221\n" \
  "v_fma_f32 v222, v222, s80, s80\n" \
  "v_mfma_f32_16x16x32_f16 a[80:83], v[52:55], v[68:71], a[80:83]\n" \
  "ds_read_b128 v[92:95], v241 offset:9984\n" \
  "v_med3_f32 v224, v216, s55, v186\n" \
  "v_mul_f32 v229, v213, v221\n" \
  "v_mfma_f32_16x16x32_f16 a[112:115], v[60:63], v[68:71], a[112:115]\n" \
  "v_exp_f32 v224, v224\n" \
  "v_accvgpr_read_b32 v218, a55\n" \
  "v_mfma_f32_16x16x32_f16 a[112:115], v[56:59], v[64:67], a[112:115]\n" \
  "v_rcp_f32 v222, v222\n" \
  "v_fma_f32 v223, v223, s80, s80\n" \
  "v_med3_f32 v225, v217, s55, v186\n" \
  "v_mfma_f32_16x16x32_f16 a[84:87], v[48:51], v[72:75], a[84:87]\n" \
  "ds_read_b128 v[64:67], v236 offset:0\n" \
  "v_mul_f32 v230, v214, v222\n" \
  "v_exp_f32 v225, v225\n" \
  "v_mfma_f32_16x16x32_f16 a[84:87], v[52:55], v[76:79], a[84:87]\n" \
  "ds_read_b128 v[68:71], v236 offset:8192\n" \
  "v_accvgpr_read_b32 v219, a63\n" \
  "v_rcp_f32 v223, v223\n" \
  "v_mfma_f32_16x16x32_f16 a[116:119], v[60:63], v[76:79], a[116:119]\n" \
  "v_fma_f32 v224, v224, s80, s80\n" \
  "v_med3_f32 v226, v218, s55, v186\n" \
  "v_mul_f32 v231, v215, v223\n" \
  "v_mfma_f32_16x16x32_f16 a[116:119], v[56:59], v[72:75], a[116:119]\n" \
  "v_exp_f32 v226, v226\n" \
  "v_rcp_f32 v224, v224\n" \
  "s_waitcnt lgkmcnt(2)\n" \
  "v_mfma_f32_16x16x32_f16 a[88:91], v[48:51], v[80:83], a[88:91]\n" \
  "ds_read_b128 v[72:75], v236 offset:256\n" \
  "v_fma_f32 v225, v225, s80, s80\n" \
  "v_med3_f32 v227, v219, s55, v186\n" \
  "v_mfma_f32_16x16x32_f16 a[88:91], v[52:55], v[84:87], a[88:91]\n" \
  "ds_read_b128 v[76:79], v236 offset:8448\n" \
  "v_mul_f32 v232, v216, v224\n" \
  "v_exp_f32 v227, v227\n" \
  "v_rcp_f32 v225, v225\n" \
  "v_mfma_f32_16x16x32_f16 a[120:123], v[60:63], v[84:87], a[120:123]\n" \
  "v_fma_f32 v226, v226, s80, s80\n" \
  "v_mul_f32 v233, v217, v225\n" \
  "v_mfma_f32_16x16x32_f16 a[120:123], v[56:59], v[80:83], a[120:123]\n" \
  "v_rcp_f32 v226, v226\n" \
  "v_fma_f32 v227, v227, s80, s80\n" \
  "v_mfma_f32_16x16x32_f16 a[92:95], v[48:51], v[88:91], a[92:95]\n" \
  "ds_read_b128 v[80:83], v236 offset:512\n" \
  "s_nop 0\n" \
  "v_mul_f32 v234, v218, v226\n" \
  "v_rcp_f32 v227, v227\n" \
  "v_mfma_f32_16x16x32_f16 a[92:95], v[52:55], v[92:95], a[92:95]\n" \
  "ds_read_b128 v[84:87], v236 offset:8704\n" \
  "s_nop 0\n" \
  "v_mul_f32 v235, v219, v227\n" \
  "v_mfma_f32_16x16x32_f16 a[124:127], v[60:63], v[92:95], a[124:127]\n" \
  "s_nop 0\n" \
  "s_mov_b32 s79, 2490368\n" \
  "v_mfma_f32_16x16x32_f16 a[124:127], v[56:59], v[88:91], a[124:127]\n" \
  "s_add_u32 s25, s25, 28\n" \
  "buffer_store_dwordx4 v[228:231], %15, s[72:75], s79 offen nt\n" \
  "buffer_store_dwordx4 v[232:235], %15, s[72:75], s79 offen offset:256 nt\n" \
  "s_waitcnt vmcnt(40) lgkmcnt(2)\n" \
  "v_mfma_f32_16x16x32_f16 a[128:131], v[96:99], v[64:67], a[128:131]\n" \
  "ds_read_b128 v[88:91], v236 offset:768\n" \
  "v_mfma_f32_16x16x32_f16 a[128:131], v[100:103], v[68:71], a[128:131]\n" \
  "ds_read_b128 v[92:95], v236 offset:8960\n" \
  "v_mfma_f32_16x16x32_f16 a[160:163], v[108:111], v[68:71], a[160:163]\n" \
  "buffer_load_dwordx4 v[48:51], v172, s[68:71], s61 offen offset:2048 sc0 sc1\n" \
  "v_mfma_f32_16x16x32_f16 a[160:163], v[104:107], v[64:67], a[160:163]\n" \
  "v_mfma_f32_16x16x32_f16 a[132:135], v[96:99], v[72:75], a[132:135]\n" \
  "ds_read_b128 v[64:67], v236 offset:1024\n" \
  "v_mfma_f32_16x16x32_f16 a[132:135], v[100:103], v[76:79], a[132:135]\n" \
  "ds_read_b128 v[68:71], v236 offset:9216\n" \
  "v_accvgpr_read_b32 v188, a64\n" \
  "v_accvgpr_read_b32 v189, a72\n" \
  "v_med3_f32 v196, v188, s55, v186\n" \
  "v_mfma_f32_16x16x32_f16 a[164:167], v[108:111], v[76:79], a[164:167]\n" \
  "buffer_load_dwordx4 v[52:55], v174, s[68:71], s61 offen offset:2048 sc0 sc1\n" \
  "v_exp_f32 v196, v196\n" \
  "v_accvgpr_read_b32 v190, a80\n" \
  "v_mfma_f32_16x16x32_f16 a[164:167], v[104:107], v[72:75], a[164:167]\n" \
  "v_med3_f32 v197, v189, s55, v186\n" \
  "v_exp_f32 v197, v197\n" \
  "s_waitcnt lgkmcnt(2)\n" \
  "v_mfma_f32_16x16x32_f16 a[136:139], v[96:99], v[80:83], a[136:139]\n" \
  "ds_read_b128 v[72:75], v236 offset:1280\n" \
  "v_accvgpr_read_b32 v191, a88\n" \
  "v_fma_f32 v196, v196, s80, s80\n" \
  "v_med3_f32 v198, v190, s55, v186\n" \
  "v_mfma_f32_16x16x32_f16 a[136:139], v[100:103], v[84:87], a[136:139]\n" \
  "ds_read_b128 v[76:79], v236 offset:9472\n" \
  "v_exp_f32 v198, v198\n" \
  "v_accvgpr_read_b32 v192, a68\n" \
  "v_mfma_f32_16x16x32_f16 a[168:171], v[108:111], v[84:87], a[168:171]\n" \
  "buffer_load_dwordx4 v[56:59], v172, s[68:71], s61 offen offset:3072 sc0 sc1\n" \
  "v_rcp_f32 v196, v196\n" \
  "v_fma_f32 v197, v197, s80, s80\n" \
  "v_mfma_f32_16x16x32_f16 a[168:171], v[104:107], v[80:83], a[168:171]\n" \
  "v_med3_f32 v199, v191, s55, v186\n" \
  "v_mul_f32 v204, v188, v196\n" \
  "v_exp_f32 v199, v199\n" \
  "v_mfma_f32_16x16x32_f16 a[140:143], v[96:99], v[88:91], a[140:143]\n" \
  "ds_read_b128 v[80:83], v236 offset:1536\n" \
  "v_accvgpr_read_b32 v193, a76\n" \
  "v_rcp_f32 v197, v197\n" \
  "v_mfma_f32_16x16x32_f16 a[140:143], v[100:103], v[92:95], a[140:143]\n" \
  "ds_read_b128 v[84:87], v236 offset:9728\n" \
  "v_fma_f32 v198, v198, s80, s80\n" \
  "v_med3_f32 v200, v192, s55, v186\n" \
  "v_mfma_f32_16x16x32_f16 a[172:175], v[108:111], v[92:95], a[172:175]\n" \
  "buffer_load_dwordx4 v[60:63], v174, s[68:71], s61 offen offset:3072 sc0 sc1\n" \
  "v_mul_f32 v205, v189, v197\n" \
  "v_exp_f32 v200, v200\n" \
  "v_accvgpr_read_b32 v194, a84\n" \
  "v_mfma_f32_16x16x32_f16 a[172:175], v[104:107], v[88:91], a[172:175]\n" \
  "v_rcp_f32 v198, v198\n" \
  "v_fma_f32 v199, v199, s80, s80\n" \
  "s_waitcnt lgkmcnt(2)\n" \
  "v_mfma_f32_16x16x32_f16 a[144:147], v[96:99], v[64:67], a[144:147]\n" \
  "ds_read_b128 v[88:91], v236 offset:1792\n" \
  "v_med3_f32 v201, v193, s55, v186\n" \
  "v_mul_f32 v206, v190, v198\n" \
  "v_mfma_f32_16x16x32_f16 a[144:147], v[100:103], v[68:71], a[144:147]\n" \
  "ds_read_b128 v[92:95], v236 offset:9984\n" \
  "v_exp_f32 v201, v201\n" \
  "v_accvgpr_read_b32 v195, a92\n" \
  "v_rcp_f32 v199, v199\n" \
  "v_mfma_f32_16x16x32_f16 a[176:179], v[108:111], v[68:71], a[176:179]\n" \
  "v_fma_f32 v200, v200, s80, s80\n" \
  "v_med3_f32 v202, v194, s55, v186\n" \
  "v_mfma_f32_16x16x32_f16 a[176:179], v[104:107], v[64:67], a[176:179]\n" \
  "v_mul_f32 v207, v191, v199\n" \
  "v_exp_f32 v202, v202\n" \
  "v_mfma_f32_16x16x32_f16 a[148:151], v[96:99], v[72:75], a[148:151]\n" \
  "ds_read_b128 v[64:67], v237 offset:0\n" \
  "v_rcp_f32 v200, v200\n" \
  "v_fma_f32 v201, v201, s80, s80\n" \
  "v_mfma_f32_16x16x32_f16 a[148:151], v[100:103], v[76:79], a[148:151]\n" \
  "ds_read_b128 v[68:71], v237 offset:8192\n" \
  "v_med3_f32 v203, v195, s55, v186\n" \
  "v_mul_f32 v208, v192, v200\n" \
  "v_exp_f32 v203, v203\n" \
  "v_mfma_f32_16x16x32_f16 a[180:183], v[108:111], v[76:79], a[180:183]\n" \
  "v_rcp_f32 v201, v201\n" \
  "v_fma_f32 v202, v202, s80, s80\n" \
  "v_mfma_f32_16x16x32_f16 a[180:183], v[104:107], v[72:75], a[180:183]\n" \
  "v_mul_f32 v209, v193, v201\n" \
  "v_rcp_f32 v202, v202\n" \
  "s_waitcnt lgkmcnt(2)\n" \
  "v_mfma_f32_16x16x32_f16 a[152:155], v[96:99], v[80:83], a[152:155]\n" \
  "ds_read_b128 v[72:75], v237 offset:256\n" \
  "v_fma_f32 v203, v203, s80, s80\n" \
  "s_nop 0\n" \
  "v_mul_f32 v210, v194, v202\n" \
  "v_mfma_f32_16x16x32_f16 a[152:155], v[100:103], v[84:87], a[152:155]\n" \
  "ds_read_b128 v[76:79], v237 offset:8448\n" \
  "v_rcp_f32 v203, v203\n" \
  "s_nop 0\n" \
  "v_mfma_f32_16x16x32_f16 a[184:187], v[108:111], v[84:87], a[184:187]\n" \
  "v_mul_f32 v211, v195, v203\n" \
  "s_nop 0\n" \
  "v_mfma_f32_16x16x32_f16 a[184:187], v[104:107], v[80:83], a[184:187]\n" \
  "s_mov_b32 s79, 4194304\n" \
  "s_add_u32 s25, s25, 28\n" \
  "buffer_store_dwordx4 v[204:207], %15, s[72:75], s79 offen nt\n" \
  "buffer_store_dwordx4 v[208:211], %15, s[72:75], s79 offen offset:256 nt\n" \
  "v_mfma_f32_16x16x32_f16 a[156:159], v[96:99], v[88:91], a[156:159]\n" \
  "ds_read_b128 v[80:83], v237 offset:512\n" \
  "v_accvgpr_read_b32 v212, a65\n" \
  "v_accvgpr_read_b32 v213, a73\n" \
  "v_mfma_f32_16x16x32_f16 a[156:159], v[100:103], v[92:95], a[156:159]\n" \
  "ds_read_b128 v[84:87], v237 offset:8704\n" \
  "v_med3_f32 v220, v212, s55, v186\n" \
  "v_exp_f32 v220, v220\n" \
  "v_mfma_f32_16x16x32_f16 a[188:191], v[108:111], v[92:95], a[188:191]\n" \
  "v_accvgpr_read_b32 v214, a81\n" \
  "v_med3_f32 v221, v213, s55, v186\n" \
  "v_exp_f32 v221, v221\n" \
  "v_mfma_f32_16x16x32_f16 a[188:191], v[104:107], v[88:91], a[188:191]\n" \
  "v_accvgpr_read_b32 v215, a89\n" \
  "v_fma_f32 v220, v220, s80, s80\n" \
  "s_waitcnt vmcnt(42) lgkmcnt(2)\n" \
  "v_mfma_f32_16x16x32_f16 a[128:131], v[112:115], v[64:67], a[128:131]\n" \
  "ds_read_b128 v[88:91], v237 offset:768\n" \
  "v_med3_f32 v222, v214, s55, v186\n" \
  "v_exp_f32 v222, v222\n" \
  "v_mfma_f32_16x16x32_f16 a[128:131], v[116:119], v[68:71], a[128:131]\n" \
  "ds_read_b128 v[92:95], v237 offset:8960\n" \
  "v_accvgpr_read_b32 v216, a69\n" \
  "v_rcp_f32 v220, v220\n" \
  "v_mfma_f32_16x16x32_f16 a[160:163], v[124:127], v[68:71], a[160:163]\n" \
  "buffer_load_dwordx4 v[96:99], v172, s[68:71], s62 offen offset:2048 sc0 sc1\n" \
  "v_fma_f32 v221, v221, s80, s80\n" \
  "v_med3_f32 v223, v215, s55, v186\n" \
  "v_mul_f32 v228, v212, v220\n" \
  "v_mfma_f32_16x16x32_f16 a[160:163], v[120:123], v[64:67], a[160:163]\n" \
  "v_exp_f32 v223, v223\n" \
  "v_accvgpr_read_b32 v217, a77\n" \
  "v_mfma_f32_16x16x32_f16 a[132:135], v[112:115], v[72:75], a[132:135]\n" \
  "ds_read_b128 v[64:67], v237 offset:1024\n" \
  "v_rcp_f32 v221, v221\n" \
  "v_fma_f32 v222, v222, s80, s80\n" \
  "v_mfma_f32_16x16x32_f16 a[132:135], v[116:119], v[76:79], a[132:135]\n" \
  "ds_read_b128 v[68:71], v237 offset:9216\n" \
  "v_med3_f32 v224, v216, s55, v186\n" \
  "v_mul_f32 v229, v213, v221\n" \
  "v_exp_f32 v224, v224\n" \
  "v_mfma_f32_16x16x32_f16 a[164:167], v[124:127], v[76:79], a[164:167]\n" \
  "buffer_load_dwordx4 v[100:103], v174, s[68:71], s62 offen offset:2048 sc0 sc1\n" \
  "v_accvgpr_read_b32 v218, a85\n" \
  "v_rcp_f32 v222, v222\n" \
  "v_mfma_f32_16x16x32_f16 a[164:167], v[120:123], v[72:75], a[164:167]\n" \
  "v_fma_f32 v223, v223, s80, s80\n" \
  "v_med3_f32 v225, v217, s55, v186\n" \
  "s_waitcnt lgkmcnt(2)\n" \
  "v_mfma_f32_16x16x32_f16 a[136:139], v[112:115], v[80:83], a[136:139]\n" \
  "ds_read_b128 v[72:75], v237 offset:1280\n" \
  "v_mul_f32 v230, v214, v222\n" \
  "v_exp_f32 v225, v225\n" \
  "v_accvgpr_read_b32 v219, a93\n" \
  "v_mfma_f32_16x16x32_f16 a[136:139], v[116:119], v[84:87], a[136:139]\n" \
  "ds_read_b128 v[76:79], v237 offset:9472\n" \
  "v_rcp_f32 v223, v223\n" \
  "v_fma_f32 v224, v224, s80, s80\n" \
  "v_mfma_f32_16x16x32_f16 a[168:171], v[124:127], v[84:87], a[168:171]\n" \
  "buffer_load_dwordx4 v[104:107], v172, s[68:71], s62 offen offset:3072 sc0 sc1\n" \
  "v_med3_f32 v226, v218, s55, v186\n" \
  "v_mul_f32 v231, v215, v223\n" \
  "v_mfma_f32_16x16x32_f16 a[168:171], v[120:123], v[80:83], a[168:171]\n" \
  "v_exp_f32 v226, v226\n" \
  "v_rcp_f32 v224, v224\n" \
  "v_fma_f32 v225, v225, s80, s80\n" \
  "v_mfma_f32_16x16x32_f16 a[140:143], v[112:115], v[88:91], a[140:143]\n" \
  "ds_read_b128 v[80:83], v237 offset:1536\n" \
  "v_med3_f32 v227, v219, s55, v186\n" \
  "v_mul_f32 v232, v216, v224\n" \
  "v_mfma_f32_16x16x32_f16 a[140:143], v[116:119], v[92:95], a[140:143]\n" \
  "ds_read_b128 v[84:87], v237 offset:9728\n" \
  "v_exp_f32 v227, v227\n" \
  "v_rcp_f32 v225, v225\n" \
  "v_mfma_f32_16x16x32_f16 a[172:175], v[124:127], v[92:95], a[172:175]\n" \
  "buffer_load_dwordx4 v[108:111], v174, s[68:71], s62 offen offset:3072 sc0 sc1\n" \
  "v_fma_f32 v226, v226, s80, s80\n" \
  "v_mul_f32 v233, v217, v225\n" \
  "v_rcp_f32 v226, v226\n" \
  "v_mfma_f32_16x16x32_f16 a[172:175], v[120:123], v[88:91], a[172:175]\n" \
  "v_fma_f32 v227, v227, s80, s80\n" \
  "s_nop 0\n" \
  "s_waitcnt lgkmcnt(2)\n" \
  "v_mfma_f32_16x16x32_f16 a[144:147], v[112:115], v[64:67], a[144:147]\n" \
  "ds_read_b128 v[88:91], v237 offset:1792\n" \
  "v_mul_f32 v234, v218, v226\n" \
  "v_rcp_f32 v227, v227\n" \
  "v_mfma_f32_16x16x32_f16 a[144:147], v[116:119], v[68:71], a[144:147]\n" \
  "ds_read_b128 v[92:95], v237 offset:9984\n" \
  "s_nop 0\n" \
  "v_mul_f32 v235, v219, v227\n" \
  "v_mfma_f32_16x16x32_f16 a[176:179], v[124:127], v[68:71], a[176:179]\n" \
  "s_nop 0\n" \
  "s_mov_b32 s79, 4325376\n" \
  "s_add_u32 s25, s25, 28\n" \
  "buffer_store_dwordx4 v[228:231], %15, s[72:75], s79 offen nt\n" \
  "v_mfma_f32_16x16x32_f16 a[176:179], v[120:123], v[64:67], a[176:179]\n" \
  "buffer_store_dwordx4 v[232:235], %15, s[72:75], s79 offen offset:256 nt\n" \
  "v_accvgpr_read_b32 v188, a66\n" \
  "v_mfma_f32_16x16x32_f16 a[148:151], v[112:115], v[72:75], a[148:151]\n" \
  "ds_read_b128 v[64:67], v238 offset:0\n" \
  "v_accvgpr_read_b32 v189, a74\n" \
  "v_med3_f32 v196, v188, s55, v186\n" \
  "v_mfma_f32_16x16x32_f16 a[148:151], v[116:119], v[76:79], a[148:151]\n" \
  "ds_read_b128 v[68:71], v238 offset:8192\n" \
  "v_exp_f32 v196, v196\n" \
  "v_accvgpr_read_b32 v190, a82\n" \
  "v_med3_f32 v197, v189, s55, v186\n" \
  "v_mfma_f32_16x16x32_f16 a[180:183], v[124:127], v[76:79], a[180:183]\n" \
  "v_exp_f32 v197, v197\n" \
  "v_accvgpr_read_b32 v191, a90\n" \
  "v_mfma_f32_16x16x32_f16 a[180:183], v[120:123], v[72:75], a[180:183]\n" \
  "v_fma_f32 v196, v196, s80, s80\n" \
  "v_med3_f32 v198, v190, s55, v186\n" \
  "s_waitcnt lgkmcnt(2)\n" \
  "v_mfma_f32_16x16x32_f16 a[152:155], v[112:115], v[80:83], a[152:155]\n" \
  "ds_read_b128 v[72:75], v238 offset:256\n" \
  "v_exp_f32 v198, v198\n" \
  "v_accvgpr_read_b32 v192, a70\n" \
  "v_rcp_f32 v196, v196\n" \
  "v_mfma_f32_16x16x32_f16 a[152:155], v[116:119], v[84:87], a[152:155]\n" \
  "ds_read_b128 v[76:79], v238 offset:8448\n" \
  "v_fma_f32 v197, v197, s80, s80\n" \
  "v_med3_f32 v199, v191, s55, v186\n" \
  "v_mfma_f32_16x16x32_f16 a[184:187], v[124:127], v[84:87], a[184:187]\n" \
  "v_mul_f32 v204, v188, v196\n" \
  "v_exp_f32 v199, v199\n" \
  "v_mfma_f32_16x16x32_f16 a[184:187], v[120:123], v[80:83], a[184:187]\n" \
  "v_accvgpr_read_b32 v193, a78\n" \
  "v_rcp_f32 v197, v197\n" \
  "v_fma_f32 v198, v198, s80, s80\n" \
  "v_mfma_f32_16x16x32_f16 a[156:159], v[112:115], v[88:91], a[156:159]\n" \
  "ds_read_b128 v[80:83], v238 offset:512\n" \
  "v_med3_f32 v200, v192, s55, v186\n" \
  "v_mul_f32 v205, v189, v197\n" \
  "v_mfma_f32_16x16x32_f16 a[156:159], v[116:119], v[92:95], a[156:159]\n" \
  "ds_read_b128 v[84:87], v238 offset:8704\n" \
  "v_exp_f32 v200, v200\n" \
  "v_accvgpr_read_b32 v194, a86\n" \
  "v_mfma_f32_16x16x32_f16 a[188:191], v[124:127], v[92:95], a[188:191]\n" \
  "v_rcp_f32 v198, v198\n" \
  "v_fma_f32 v199, v199, s80, s80\n" \
  "v_mfma_f32_16x16x32_f16 a[188:191], v[120:123], v[88:91], a[188:191]\n" \
  "v_med3_f32 v201, v193, s55, v186\n" \
  "v_mul_f32 v206, v190, v198\n" \
  "v_exp_f32 v201, v201\n" \
  "s_waitcnt vmcnt(42) lgkmcnt(2)\n" \
  "v_mfma_f32_16x16x32_f16 a[128:131], v[128:131], v[64:67], a[128:131]\n" \
  "ds_read_b128 v[88:91], v238 offset:768\n" \
  "v_accvgpr_read_b32 v195, a94\n" \
  "v_rcp_f32 v199, v199\n" \
  "v_mfma_f32_16x16x32_f16 a[128:131], v[132:135], v[68:71], a[128:131]\n" \
  "ds_read_b128 v[92:95], v238 offset:8960\n" \
  "v_fma_f32 v200, v200, s80, s80\n" \
  "v_med3_f32 v202, v194, s55, v186\n" \
  "v_mfma_f32_16x16x32_f16 a[160:163], v[140:143], v[68:71], a[160:163]\n" \
  "buffer_load_dwordx4 v[112:115], v172, s[68:71], s63 offen offset:2048 sc0 sc1\n" \
  "v_mul_f32 v207, v191, v199\n" \
  "v_exp_f32 v202, v202\n" \
  "v_rcp_f32 v200, v200\n" \
  "v_mfma_f32_16x16x32_f16 a[160:163], v[136:139], v[64:67], a[160:163]\n" \
  "v_fma_f32 v201, v201, s80, s80\n" \
  "v_med3_f32 v203, v195, s55, v186\n" \
  "v_mfma_f32_16x16x32_f16 a[132:135], v[128:131], v[72:75], a[132:135]\n" \
  "ds_read_b128 v[64:67], v238 offset:1024\n" \
  "v_mul_f32 v208, v192, v200\n" \
  "v_exp_f32 v203, v203\n" \
  "v_mfma_f32_16x16x32_f16 a[132:135], v[132:135], v[76:79], a[132:135]\n" \
  "ds_read_b128 v[68:71], v238 offset:9216\n" \
  "v_rcp_f32 v201, v201\n" \
  "v_fma_f32 v202, v202, s80, s80\n" \
  "v_mul_f32 v209, v193, v201\n" \
  "v_mfma_f32_16x16x32_f16 a[164:167], v[140:143], v[76:79], a[164:167]\n" \
  "buffer_load_dwordx4 v[116:119], v174, s[68:71], s63 offen offset:2048 sc0 sc1\n" \
  "v_rcp_f32 v202, v202\n" \
  "v_fma_f32 v203, v203, s80, s80\n" \
  "v_mfma_f32_16x16x32_f16 a[164:167], v[136:139], v[72:75], a[164:167]\n" \
  "s_nop 0\n" \
  "v_mul_f32 v210, v194, v202\n" \
  "s_waitcnt lgkmcnt(2)\n" \
  "v_mfma_f32_16x16x32_f16 a[136:139], v[128:131], v[80:83], a[136:139]\n" \
  "ds_read_b128 v[72:75], v238 offset:1280\n" \
  "v_rcp_f32 v203, v203\n" \
  "s_nop 0\n" \
  "v_mul_f32 v211, v195, v203\n" \
  "v_mfma_f32_16x16x32_f16 a[136:139], v[132:135], v[84:87], a[136:139]\n" \
  "ds_read_b128 v[76:79], v238 offset:9472\n" \
  "s_nop 0\n" \
  "s_mov_b32 s79, 4456448\n" \
  "v_mfma_f32_16x16x32_f16 a[168:171], v[140:143], v[84:87], a[168:171]\n" \
  "buffer_load_dwordx4 v[120:123], v172, s[68:71], s63 offen offset:3072 sc0 sc1\n" \
  "s_add_u32 s25, s25, 28\n" \
  "buffer_store_dwordx4 v[204:207], %15, s[72:75], s79 offen nt\n" \
  "buffer_store_dwordx4 v[208:211], %15, s[72:75], s79 offen offset:256 nt\n" \
  "v_mfma_f32_16x16x32_f16 a[168:171], v[136:139], v[80:83], a[168:171]\n" \
  "v_accvgpr_read_b32 v212, a67\n" \
  "v_accvgpr_read_b32 v213, a75\n" \
  "v_med3_f32 v220, v212, s55, v186\n" \
  "v_mfma_f32_16x16x32_f16 a[140:143], v[128:131], v[88:91], a[140:143]\n" \
  "ds_read_b128 v[80:83], v238 offset:1536\n" \
  "v_exp_f32 v220, v220\n" \
  "v_accvgpr_read_b32 v214, a83\n" \
  "v_mfma_f32_16x16x32_f16 a[140:143], v[132:135], v[92:95], a[140:143]\n" \
  "ds_read_b128 v[84:87], v238 offset:9728\n" \
  "v_med3_f32 v221, v213, s55, v186\n" \
  "v_exp_f32 v221, v221\n" \
  "v_mfma_f32_16x16x32_f16 a[172:175], v[140:143], v[92:95], a[172:175]\n" \
  "buffer_load_dwordx4 v[124:127], v174, s[68:71], s63 offen offset:3072 sc0 sc1\n" \
  "v_accvgpr_read_b32 v215, a91\n" \
  "v_fma_f32 v220, v220, s80, s80\n" \
  "v_mfma_f32_16x16x32_f16 a[172:175], v[136:139], v[88:91], a[172:175]\n" \
  "v_med3_f32 v222, v214, s55, v186\n" \
  "v_exp_f32 v222, v222\n" \
  "v_accvgpr_read_b32 v216, a71\n" \
  "s_waitcnt lgkmcnt(2)\n" \
  "v_mfma_f32_16x16x32_f16 a[144:147], v[128:131], v[64:67], a[144:147]\n" \
  "ds_read_b128 v[88:91], v238 offset:1792\n" \
  "v_rcp_f32 v220, v220\n" \
  "v_fma_f32 v221, v221, s80, s80\n" \
  "v_mfma_f32_16x16x32_f16 a[144:147], v[132:135], v[68:71], a[144:147]\n" \
  "ds_read_b128 v[92:95], v238 offset:9984\n" \
  "v_med3_f32 v223, v215, s55, v186\n" \
  "v_mul_f32 v228, v212, v220\n" \
  "v_mfma_f32_16x16x32_f16 a[176:179], v[140:143], v[68:71], a[176:179]\n" \
  "v_exp_f32 v223, v223\n" \
  "v_accvgpr_read_b32 v217, a79\n" \
  "v_rcp_f32 v221, v221\n" \
  "v_mfma_f32_16x16x32_f16 a[176:179], v[136:139], v[64:67], a[176:179]\n" \
  "v_fma_f32 v222, v222, s80, s80\n" \
  "v_med3_f32 v224, v216, s55, v186\n" \
  "v_mfma_f32_16x16x32_f16 a[148:151], v[128:131], v[72:75], a[148:151]\n" \
  "ds_read_b128 v[64:67], v239 offset:0\n" \
  "v_mul_f32 v229, v213, v221\n" \
  "v_exp_f32 v224, v224\n" \
  "v_mfma_f32_16x16x32_f16 a[148:151], v[132:135], v[76:79], a[148:151]\n" \
  "ds_read_b128 v[68:71], v239 offset:8192\n" \
  "v_accvgpr_read_b32 v218, a87\n" \
  "v_rcp_f32 v222, v222\n" \
  "v_fma_f32 v223, v223, s80, s80\n" \
  "v_mfma_f32_16x16x32_f16 a[180:183], v[140:143], v[76:79], a[180:183]\n" \
  "v_med3_f32 v225, v217, s55, v186\n" \
  "v_mul_f32 v230, v214, v222\n" \
  "v_mfma_f32_16x16x32_f16 a[180:183], v[136:139], v[72:75], a[180:183]\n" \
  "v_exp_f32 v225, v225\n" \
  "v_accvgpr_read_b32 v219, a95\n" \
  "s_waitcnt lgkmcnt(2)\n" \
  "v_mfma_f32_16x16x32_f16 a[152:155], v[128:131], v[80:83], a[152:155]\n" \
  "ds_read_b128 v[72:75], v239 offset:256\n" \
  "v_rcp_f32 v223, v223\n" \
  "v_fma_f32 v224, v224, s80, s80\n" \
  "v_med3_f32 v226, v218, s55, v186\n" \
  "v_mfma_f32_16x16x32_f16 a[152:155], v[132:135], v[84:87], a[152:155]\n" \
  "ds_read_b128 v[76:79], v239 offset:8448\n" \
  "v_mul_f32 v231, v215, v223\n" \
  "v_exp_f32 v226, v226\n" \
  "v_mfma_f32_16x16x32_f16 a[184:187], v[140:143], v[84:87], a[184:187]\n" \
  "v_rcp_f32 v224, v224\n" \
  "v_fma_f32 v225, v225, s80, s80\n" \
  "v_mfma_f32_16x16x32_f16 a[184:187], v[136:139], v[80:83], a[184:187]\n" \
  "v_med3_f32 v227, v219, s55, v186\n" \
  "v_mul_f32 v232, v216, v224\n" \
  "v_mfma_f32_16x16x32_f16 a[156:159], v[128:131], v[88:91], a[156:159]\n" \
  "ds_read_b128 v[80:83], v239 offset:512\n" \
  "v_exp_f32 v227, v227\n" \
  "v_rcp_f32 v225, v225\n" \
  "v_fma_f32 v226, v226, s80, s80\n" \
  "v_mfma_f32_16x16x32_f16 a[156:159], v[132:135], v[92:95], a[156:159]\n" \
  "ds_read_b128 v[84:87], v239 offset:8704\n" \
  "v_mul_f32 v233, v217, v225\n" \
  "v_rcp_f32 v226, v226\n" \
  "v_mfma_f32_16x16x32_f16 a[188:191], v[140:143], v[92:95], a[188:191]\n" \
  "v_fma_f32 v227, v227, s80, s80\n" \
  "s_nop 0\n" \
  "v_mfma_f32_16x16x32_f16 a[188:191], v[136:139], v[88:91], a[188:191]\n" \
  "v_mul_f32 v234, v218, v226\n" \
  "v_rcp_f32 v227, v227\n" \
  "s_nop 0\n" \
  "s_waitcnt vmcnt(40) lgkmcnt(2)\n" \
  "v_mfma_f32_16x16x32_f16 a[128:131], v[144:147], v[64:67], a[128:131]\n" \
  "ds_read_b128 v[88:91], v239 offset:768\n" \
  "v_mul_f32 v235, v219, v227\n" \
  "s_nop 0\n" \
  "v_mfma_f32_16x16x32_f16 a[128:131], v[148:151], v[68:71], a[128:131]\n" \
  "ds_read_b128 v[92:95], v239 offset:8960\n" \
  "s_mov_b32 s79, 4587520\n" \
  "s_add_u32 s25, s25, 28\n" \
  "buffer_store_dwordx4 v[228:231], %15, s[72:75], s79 offen nt\n" \
  "v_mfma_f32_16x16x32_f16 a[160:163], v[156:159], v[68:71], a[160:163]\n" \
  "buffer_load_dwordx4 v[128:131], v172, s[68:71], s53 offen offset:2048 sc0 sc1\n" \
  "buffer_store_dwordx4 v[232:235], %15, s[72:75], s79 offen offset:256 nt\n" \
  "v_accvgpr_read_b32 v188, a96\n" \
  "v_accvgpr_read_b32 v189, a104\n" \
  "v_mfma_f32_16x16x32_f16 a[160:163], v[152:155], v[64:67], a[160:163]\n" \
  "v_med3_f32 v196, v188, s55, v186\n" \
  "v_exp_f32 v196, v196\n" \
  "v_mfma_f32_16x16x32_f16 a[132:135], v[144:147], v[72:75], a[132:135]\n" \
  "ds_read_b128 v[64:67], v239 offset:1024\n" \
  "v_accvgpr_read_b32 v190, a112\n" \
  "v_med3_f32 v197, v189, s55, v186\n" \
  "v_mfma_f32_16x16x32_f16 a[132:135], v[148:151], v[76:79], a[132:135]\n" \
  "ds_read_b128 v[68:71], v239 offset:9216\n" \
  "v_exp_f32 v197, v197\n" \
  "v_accvgpr_read_b32 v191, a120\n" \
  "v_fma_f32 v196, v196, s80, s80\n" \
  "v_mfma_f32_16x16x32_f16 a[164:167], v[156:159], v[76:79], a[164:167]\n" \
  "buffer_load_dwordx4 v[132:135], v174, s[68:71], s53 offen offset:2048 sc0 sc1\n" \
  "v_med3_f32 v198, v190, s55, v186\n" \
  "v_exp_f32 v198, v198\n" \
  "v_mfma_f32_16x16x32_f16 a[164:167], v[152:155], v[72:75], a[164:167]\n" \
  "v_accvgpr_read_b32 v192, a100\n" \
  "v_rcp_f32 v196, v196\n" \
  "s_waitcnt lgkmcnt(2)\n" \
  "v_mfma_f32_16x16x32_f16 a[136:139], v[144:147], v[80:83], a[136:139]\n" \
  "ds_read_b128 v[72:75], v239 offset:1280\n" \
  "v_fma_f32 v197, v197, s80, s80\n" \
  "v_med3_f32 v199, v191, s55, v186\n" \
  "v_mul_f32 v204, v188, v196\n" \
  "v_mfma_f32_16x16x32_f16 a[136:139], v[148:151], v[84:87], a[136:139]\n" \
  "ds_read_b128 v[76:79], v239 offset:9472\n" \
  "v_exp_f32 v199, v199\n" \
  "v_accvgpr_read_b32 v193, a108\n" \
  "v_mfma_f32_16x16x32_f16 a[168:171], v[156:159], v[84:87], a[168:171]\n" \
  "buffer_load_dwordx4 v[136:139], v172, s[68:71], s53 offen offset:3072 sc0 sc1\n" \
  "v_rcp_f32 v197, v197\n" \
  "v_fma_f32 v198, v198, s80, s80\n" \
  "v_mfma_f32_16x16x32_f16 a[168:171], v[152:155], v[80:83], a[168:171]\n" \
  "v_med3_f32 v200, v192, s55, v186\n" \
  "v_mul_f32 v205, v189, v197\n" \
  "v_mfma_f32_16x16x32_f16 a[140:143], v[144:147], v[88:91], a[140:143]\n" \
  "ds_read_b128 v[80:83], v239 offset:1536\n" \
  "v_exp_f32 v200, v200\n" \
  "v_accvgpr_read_b32 v194, a116\n" \
  "v_rcp_f32 v198, v198\n" \
  "v_mfma_f32_16x16x32_f16 a[140:143], v[148:151], v[92:95], a[140:143]\n" \
  "ds_read_b128 v[84:87], v239 offset:9728\n" \
  "v_fma_f32 v199, v199, s80, s80\n" \
  "v_med3_f32 v201, v193, s55, v186\n" \
  "v_mfma_f32_16x16x32_f16 a[172:175], v[156:159], v[92:95], a[172:175]\n" \
  "buffer_load_dwordx4 v[140:143], v174, s[68:71], s53 offen offset:3072 sc0 sc1\n" \
  "v_mul_f32 v206, v190, v198\n" \
  "v_exp_f32 v201, v201\n" \
  "v_mfma_f32_16x16x32_f16 a[172:175], v[152:155], v[88:91], a[172:175]\n" \
  "v_accvgpr_read_b32 v195, a124\n" \
  "v_rcp_f32 v199, v199\n" \
  "v_fma_f32 v200, v200, s80, s80\n" \
  "s_waitcnt lgkmcnt(2)\n" \
  "v_mfma_f32_16x16x32_f16 a[144:147], v[144:147], v[64:67], a[144:147]\n" \
  "ds_read_b128 v[88:91], v239 offset:1792\n" \
  "v_med3_f32 v202, v194, s55, v186\n" \
  "v_mul_f32 v207, v191, v199\n" \
  "v_mfma_f32_16x16x32_f16 a[144:147], v[148:151], v[68:71], a[144:147]\n" \
  "ds_read_b128 v[92:95], v239 offset:9984\n" \
  "v_exp_f32 v202, v202\n" \
  "v_rcp_f32 v200, v200\n" \
  "v_mfma_f32_16x16x32_f16 a[176:179], v[156:159], v[68:71], a[176:179]\n" \
  "v_fma_f32 v201, v201, s80, s80\n" \
  "v_med3_f32 v203, v195, s55, v186\n" \
  "v_mul_f32 v208, v192, v200\n" \
  "v_mfma_f32_16x16x32_f16 a[176:179], v[152:155], v[64:67], a[176:179]\n" \
  "v_exp_f32 v203, v203\n" \
  "v_rcp_f32 v201, v201\n" \
  "v_mfma_f32_16x16x32_f16 a[148:151], v[144:147], v[72:75], a[148:151]\n" \
  "ds_read_b128 v[64:67], v240 offset:0\n" \
  "v_fma_f32 v202, v202, s80, s80\n" \
  "v_mul_f32 v209, v193, v201\n" \
  "v_mfma_f32_16x16x32_f16 a[148:151], v[148:151], v[76:79], a[148:151]\n" \
  "ds_read_b128 v[68:71], v240 offset:8192\n" \
  "v_rcp_f32 v202, v202\n" \
  "v_fma_f32 v203, v203, s80, s80\n" \
  "s_nop 0\n" \
  "v_mfma_f32_16x16x32_f16 a[180:183], v[156:159], v[76:79], a[180:183]\n" \
  "v_mul_f32 v210, v194, v202\n" \
  "v_rcp_f32 v203, v203\n" \
  "v_mfma_f32_16x16x32_f16 a[180:183], v[152:155], v[72:75], a[180:183]\n" \
  "s_nop 0\n" \
  "v_mul_f32 v211, v195, v203\n" \
  "s_waitcnt lgkmcnt(2)\n" \
  "v_mfma_f32_16x16x32_f16 a[152:155], v[144:147], v[80:83], a[152:155]\n" \
  "ds_read_b128 v[72:75], v240 offset:256\n" \
  "s_nop 0\n" \
  "s_mov_b32 s79, 6291456\n" \
  "v_mfma_f32_16x16x32_f16 a[152:155], v[148:151], v[84:87], a[152:155]\n" \
  "ds_read_b128 v[76:79], v240 offset:8448\n" \
  "s_add_u32 s25, s25, 28\n" \
  "buffer_store_dwordx4 v[204:207], %15, s[72:75], s79 offen nt\n" \
  "buffer_store_dwordx4 v[208:211], %15, s[72:75], s79 offen offset:256 nt\n" \
  "v_accvgpr_read_b32 v212, a97\n" \
  "v_mfma_f32_16x16x32_f16 a[184:187], v[156:159], v[84:87], a[184:187]\n" \
  "v_accvgpr_read_b32 v213, a105\n" \
  "v_med3_f32 v220, v212, s55, v186\n" \
  "v_mfma_f32_16x16x32_f16 a[184:187], v[152:155], v[80:83], a[184:187]\n" \
  "v_exp_f32 v220, v220\n" \
  "v_accvgpr_read_b32 v214, a113\n" \
  "v_mfma_f32_16x16x32_f16 a[156:159], v[144:147], v[88:91], a[156:159]\n" \
  "ds_read_b128 v[80:83], v240 offset:512\n" \
  "v_med3_f32 v221, v213, s55, v186\n" \
  "v_exp_f32 v221, v221\n" \
  "v_accvgpr_read_b32 v215, a121\n" \
  "v_mfma_f32_16x16x32_f16 a[156:159], v[148:151], v[92:95], a[156:159]\n" \
  "ds_read_b128 v[84:87], v240 offset:8704\n" \
  "v_fma_f32 v220, v220, s80, s80\n" \
  "v_med3_f32 v222, v214, s55, v186\n" \
  "v_mfma_f32_16x16x32_f16 a[188:191], v[156:159], v[92:95], a[188:191]\n" \
  "v_exp_f32 v222, v222\n" \
  "v_accvgpr_read_b32 v216, a101\n" \
  "v_mfma_f32_16x16x32_f16 a[188:191], v[152:155], v[88:91], a[188:191]\n" \
  "v_rcp_f32 v220, v220\n" \
  "v_fma_f32 v221, v221, s80, s80\n" \
  "v_med3_f32 v223, v215, s55, v186\n" \
  "s_waitcnt vmcnt(42) lgkmcnt(2)\n" \
  "v_mfma_f32_16x16x32_f16 a[128:131], v[0:3], v[64:67], a[128:131]\n" \
  "ds_read_b128 v[88:91], v240 offset:768\n" \
  "v_mul_f32 v228, v212, v220\n" \
  "v_exp_f32 v223, v223\n" \
  "v_mfma_f32_16x16x32_f16 a[128:131], v[4:7], v[68:71], a[128:131]\n" \
  "ds_read_b128 v[92:95], v240 offset:8960\n" \
  "v_accvgpr_read_b32 v217, a109\n" \
  "v_rcp_f32 v221, v221\n" \
  "v_mfma_f32_16x16x32_f16 a[160:163], v[12:15], v[68:71], a[160:163]\n" \
  "buffer_load_dwordx4 v[144:147], v172, s[68:71], s54 offen offset:2048 sc0 sc1\n" \
  "v_fma_f32 v222, v222, s80, s80\n" \
  "v_med3_f32 v224, v216, s55, v186\n" \
  "v_mul_f32 v229, v213, v221\n" \
  "v_mfma_f32_16x16x32_f16 a[160:163], v[8:11], v[64:67], a[160:163]\n" \
  "v_exp_f32 v224, v224\n" \
  "v_accvgpr_read_b32 v218, a117\n" \
  "v_mfma_f32_16x16x32_f16 a[132:135], v[0:3], v[72:75], a[132:135]\n" \
  "ds_read_b128 v[64:67], v240 offset:1024\n" \
  "v_rcp_f32 v222, v222\n" \
  "v_fma_f32 v223, v223, s80, s80\n" \
  "v_mfma_f32_16x16x32_f16 a[132:135], v[4:7], v[76:79], a[132:135]\n" \
  "ds_read_b128 v[68:71], v240 offset:9216\n" \
  "v_med3_f32 v225, v217, s55, v186\n" \
  "v_mul_f32 v230, v214, v222\n" \
  "v_exp_f32 v225, v225\n" \
  "v_mfma_f32_16x16x32_f16 a[164:167], v[12:15], v[76:79], a[164:167]\n" \
  "buffer_load_dwordx4 v[148:151], v174, s[68:71], s54 offen offset:2048 sc0 sc1\n" \
  "v_accvgpr_read_b32 v219, a125\n" \
  "v_rcp_f32 v223, v223\n" \
  "v_mfma_f32_16x16x32_f16 a[164:167], v[8:11], v[72:75], a[164:167]\n" \
  "v_fma_f32 v224, v224, s80, s80\n" \
  "v_med3_f32 v226, v218, s55, v186\n" \
  "s_waitcnt lgkmcnt(2)\n" \
  "v_mfma_f32_16x16x32_f16 a[136:139], v[0:3], v[80:83], a[136:139]\n" \
  "ds_read_b128 v[72:75], v240 offset:1280\n" \
  "v_mul_f32 v231, v215, v223\n" \
  "v_exp_f32 v226, v226\n" \
  "v_mfma_f32_16x16x32_f16 a[136:139], v[4:7], v[84:87], a[136:139]\n" \
  "ds_read_b128 v[76:79], v240 offset:9472\n" \
  "v_rcp_f32 v224, v224\n" \
  "v_fma_f32 v225, v225, s80, s80\n" \
  "v_med3_f32 v227, v219, s55, v186\n" \
  "v_mfma_f32_16x16x32_f16 a[168:171], v[12:15], v[84:87], a[168:171]\n" \
  "buffer_load_dwordx4 v[152:155], v172, s[68:71], s54 offen offset:3072 sc0 sc1\n" \
  "v_mul_f32 v232, v216, v224\n" \
  "v_exp_f32 v227, v227\n" \
  "v_mfma_f32_16x16x32_f16 a[168:171], v[8:11], v[80:83], a[168:171]\n" \
  "v_rcp_f32 v225, v225\n" \
  "v_fma_f32 v226, v226, s80, s80\n" \
  "v_mfma_f32_16x16x32_f16 a[140:143], v[0:3], v[88:91], a[140:143]\n" \
  "ds_read_b128 v[80:83], v240 offset:1536\n" \
  "v_mul_f32 v233, v217, v225\n" \
  "v_rcp_f32 v226, v226\n" \
  "v_fma_f32 v227, v227, s80, s80\n" \
  "v_mfma_f32_16x16x32_f16 a[140:143], v[4:7], v[92:95], a[140:143]\n" \
  "ds_read_b128 v[84:87], v240 offset:9728\n" \
  "s_nop 0\n" \
  "v_mul_f32 v234, v218, v226\n" \
  "v_mfma_f32_16x16x32_f16 a[172:175], v[12:15], v[92:95], a[172:175]\n" \
  "buffer_load_dwordx4 v[156:159], v174, s[68:71], s54 offen offset:3072 sc0 sc1\n" \
  "v_rcp_f32 v227, v227\n" \
  "s_nop 0\n" \
  "v_mfma_f32_16x16x32_f16 a[172:175], v[8:11], v[88:91], a[172:175]\n" \
  "v_mul_f32 v235, v219, v227\n" \
  "s_nop 0\n" \
  "s_mov_b32 s79, 6422528\n" \
  "s_waitcnt lgkmcnt(2)\n" \
  "v_mfma_f32_16x16x32_f16 a[144:147], v[0:3], v[64:67], a[144:147]\n" \
  "ds_read_b128 v[88:91], v240 offset:1792\n" \
  "s_add_u32 s25, s25, 28\n" \
  "buffer_store_dwordx4 v[228:231], %15, s[72:75], s79 offen nt\n" \
  "buffer_store_dwordx4 v[232:235], %15, s[72:75], s79 offen offset:256 nt\n" \
  "v_mfma_f32_16x16x32_f16 a[144:147], v[4:7], v[68:71], a[144:147]\n" \
  "ds_read_b128 v[92:95], v240 offset:9984\n" \
  "v_accvgpr_read_b32 v188, a98\n" \
  "v_accvgpr_read_b32 v189, a106\n" \
  "v_mfma_f32_16x16x32_f16 a[176:179], v[12:15], v[68:71], a[176:179]\n" \
  "v_med3_f32 v196, v188, s55, v186\n" \
  "v_exp_f32 v196, v196\n" \
  "v_accvgpr_read_b32 v190, a114\n" \
  "v_mfma_f32_16x16x32_f16 a[176:179], v[8:11], v[64:67], a[176:179]\n" \
  "v_med3_f32 v197, v189, s55, v186\n" \
  "v_exp_f32 v197, v197\n" \
  "v_mfma_f32_16x16x32_f16 a[148:151], v[0:3], v[72:75], a[148:151]\n" \
  "ds_read_b128 v[64:67], v241 offset:0\n" \
  "v_accvgpr_read_b32 v191, a122\n" \
  "v_fma_f32 v196, v196, s80, s80\n" \
  "v_mfma_f32_16x16x32_f16 a[148:151], v[4:7], v[76:79], a[148:151]\n" \
  "ds_read_b128 v[68:71], v241 offset:8192\n" \
  "v_med3_f32 v198, v190, s55, v186\n" \
  "v_exp_f32 v198, v198\n" \
  "v_mfma_f32_16x16x32_f16 a[180:183], v[12:15], v[76:79], a[180:183]\n" \
  "v_accvgpr_read_b32 v192, a102\n" \
  "v_rcp_f32 v196, v196\n" \
  "v_fma_f32 v197, v197, s80, s80\n" \
  "v_mfma_f32_16x16x32_f16 a[180:183], v[8:11], v[72:75], a[180:183]\n" \
  "v_med3_f32 v199, v191, s55, v186\n" \
  "v_mul_f32 v204, v188, v196\n" \
  "s_waitcnt lgkmcnt(2)\n" \
  "v_mfma_f32_16x16x32_f16 a[152:155], v[0:3], v[80:83], a[152:155]\n" \
  "ds_read_b128 v[72:75], v241 offset:256\n" \
  "v_exp_f32 v199, v199\n" \
  "v_accvgpr_read_b32 v193, a110\n" \
  "v_mfma_f32_16x16x32_f16 a[152:155], v[4:7], v[84:87], a[152:155]\n" \
  "ds_read_b128 v[76:79], v241 offset:8448\n" \
  "v_rcp_f32 v197, v197\n" \
  "v_fma_f32 v198, v198, s80, s80\n" \
  "v_med3_f32 v200, v192, s55, v186\n" \
  "v_mfma_f32_16x16x32_f16 a[184:187], v[12:15], v[84:87], a[184:187]\n" \
  "v_mul_f32 v205, v189, v197\n" \
  "v_exp_f32 v200, v200\n" \
  "v_mfma_f32_16x16x32_f16 a[184:187], v[8:11], v[80:83], a[184:187]\n" \
  "v_accvgpr_read_b32 v194, a118\n" \
  "v_rcp_f32 v198, v198\n" \
  "v_mfma_f32_16x16x32_f16 a[156:159], v[0:3], v[88:91], a[156:159]\n" \
  "ds_read_b128 v[80:83], v241 offset:512\n" \
  "v_fma_f32 v199, v199, s80, s80\n" \
  "v_med3_f32 v201, v193, s55, v186\n" \
  "v_mul_f32 v206, v190, v198\n" \
  "v_mfma_f32_16x16x32_f16 a[156:159], v[4:7], v[92:95], a[156:159]\n" \
  "ds_read_b128 v[84:87], v241 offset:8704\n" \
  "v_exp_f32 v201, v201\n" \
  "v_accvgpr_read_b32 v195, a126\n" \
  "v_mfma_f32_16x16x32_f16 a[188:191], v[12:15], v[92:95], a[188:191]\n" \
  "v_rcp_f32 v199, v199\n" \
  "v_fma_f32 v200, v200, s80, s80\n" \
  "v_mfma_f32_16x16x32_f16 a[188:191], v[8:11], v[88:91], a[188:191]\n" \
  "v_med3_f32 v202, v194, s55, v186\n" \
  "v_mul_f32 v207, v191, v199\n" \
  "v_exp_f32 v202, v202\n" \
  "s_waitcnt vmcnt(42) lgkmcnt(2)\n" \
  "v_mfma_f32_16x16x32_f16 a[128:131], v[16:19], v[64:67], a[128:131]\n" \
  "ds_read_b128 v[88:91], v241 offset:768\n" \
  "v_rcp_f32 v200, v200\n" \
  "v_fma_f32 v201, v201, s80, s80\n" \
  "v_mfma_f32_16x16x32_f16 a[128:131], v[20:23], v[68:71], a[128:131]\n" \
  "ds_read_b128 v[92:95], v241 offset:8960\n" \
  "v_med3_f32 v203, v195, s55, v186\n" \
  "v_mul_f32 v208, v192, v200\n" \
  "v_mfma_f32_16x16x32_f16 a[160:163], v[28:31], v[68:71], a[160:163]\n" \
  "v_exp_f32 v203, v203\n" \
  "v_rcp_f32 v201, v201\n" \
  "v_fma_f32 v202, v202, s80, s80\n" \
  "v_mfma_f32_16x16x32_f16 a[160:163], v[24:27], v[64:67], a[160:163]\n" \
  "v_mul_f32 v209, v193, v201\n" \
  "v_rcp_f32 v202, v202\n" \
  "v_mfma_f32_16x16x32_f16 a[132:135], v[16:19], v[72:75], a[132:135]\n" \
  "ds_read_b128 v[64:67], v241 offset:1024\n" \
  "v_fma_f32 v203, v203, s80, s80\n" \
  "s_nop 0\n" \
  "v_mfma_f32_16x16x32_f16 a[132:135], v[20:23], v[76:79], a[132:135]\n" \
  "ds_read_b128 v[68:71], v241 offset:9216\n" \
  "v_mul_f32 v210, v194, v202\n" \
  "v_rcp_f32 v203, v203\n" \
  "v_mfma_f32_16x16x32_f16 a[164:167], v[28:31], v[76:79], a[164:167]\n" \
  "s_nop 0\n" \
  "v_mul_f32 v211, v195, v203\n" \
  "s_nop 0\n" \
  "v_mfma_f32_16x16x32_f16 a[164:167], v[24:27], v[72:75], a[164:167]\n" \
  "s_mov_b32 s79, 6553600\n" \
  "s_add_u32 s25, s25, 28\n" \
  "buffer_store_dwordx4 v[204:207], %15, s[72:75], s79 offen nt\n" \
  "s_waitcnt lgkmcnt(2)\n" \
  "v_mfma_f32_16x16x32_f16 a[136:139], v[16:19], v[80:83], a[136:139]\n" \
  "ds_read_b128 v[72:75], v241 offset:1280\n" \
  "buffer_store_dwordx4 v[208:211], %15, s[72:75], s79 offen offset:256 nt\n" \
  "v_accvgpr_read_b32 v212, a99\n" \
  "v_mfma_f32_16x16x32_f16 a[136:139], v[20:23], v[84:87], a[136:139]\n" \
  "ds_read_b128 v[76:79], v241 offset:9472\n" \
  "v_accvgpr_read_b32 v213, a107\n" \
  "v_med3_f32 v220, v212, s55, v186\n" \
  "v_exp_f32 v220, v220\n" \
  "v_mfma_f32_16x16x32_f16 a[168:171], v[28:31], v[84:87], a[168:171]\n" \
  "v_accvgpr_read_b32 v214, a115\n" \
  "v_med3_f32 v221, v213, s55, v186\n" \
  "v_mfma_f32_16x16x32_f16 a[168:171], v[24:27], v[80:83], a[168:171]\n" \
  "v_exp_f32 v221, v221\n" \
  "v_accvgpr_read_b32 v215, a123\n" \
  "v_mfma_f32_16x16x32_f16 a[140:143], v[16:19], v[88:91], a[140:143]\n" \
  "ds_read_b128 v[80:83], v241 offset:1536\n" \
  "v_fma_f32 v220, v220, s80, s80\n" \
  "v_med3_f32 v222, v214, s55, v186\n" \
  "v_exp_f32 v222, v222\n" \
  "v_mfma_f32_16x16x32_f16 a[140:143], v[20:23], v[92:95], a[140:143]\n" \
  "ds_read_b128 v[84:87], v241 offset:9728\n" \
  "v_accvgpr_read_b32 v216, a103\n" \
  "v_rcp_f32 v220, v220\n" \
  "v_mfma_f32_16x16x32_f16 a[172:175], v[28:31], v[92:95], a[172:175]\n" \
  "v_fma_f32 v221, v221, s80, s80\n" \
  "v_med3_f32 v223, v215, s55, v186\n" \
  "v_mfma_f32_16x16x32_f16 a[172:175], v[24:27], v[88:91], a[172:175]\n" \
  "v_mul_f32 v228, v212, v220\n" \
  "v_exp_f32 v223, v223\n" \
  "v_accvgpr_read_b32 v217, a111\n" \
  "s_waitcnt lgkmcnt(2)\n" \
  "v_mfma_f32_16x16x32_f16 a[144:147], v[16:19], v[64:67], a[144:147]\n" \
  "ds_read_b128 v[88:91], v241 offset:1792\n" \
  "v_rcp_f32 v221, v221\n" \
  "v_fma_f32 v222, v222, s80, s80\n" \
  "v_mfma_f32_16x16x32_f16 a[144:147], v[20:23], v[68:71], a[144:147]\n" \
  "ds_read_b128 v[92:95], v241 offset:9984\n" \
  "v_med3_f32 v224, v216, s55, v186\n" \
  "v_mul_f32 v229, v213, v221\n" \
  "v_mfma_f32_16x16x32_f16 a[176:179], v[28:31], v[68:71], a[176:179]\n" \
  "v_exp_f32 v224, v224\n" \
  "v_accvgpr_read_b32 v218, a119\n" \
  "v_mfma_f32_16x16x32_f16 a[176:179], v[24:27], v[64:67], a[176:179]\n" \
  "v_rcp_f32 v222, v222\n" \
  "v_fma_f32 v223, v223, s80, s80\n" \
  "v_med3_f32 v225, v217, s55, v186\n" \
  "v_mfma_f32_16x16x32_f16 a[148:151], v[16:19], v[72:75], a[148:151]\n" \
  "ds_read_b128 v[64:67], v236 offset:0\n" \
  "v_mul_f32 v230, v214, v222\n" \
  "v_exp_f32 v225, v225\n" \
  "v_mfma_f32_16x16x32_f16 a[148:151], v[20:23], v[76:79], a[148:151]\n" \
  "ds_read_b128 v[68:71], v236 offset:8192\n" \
  "v_accvgpr_read_b32 v219, a127\n" \
  "v_rcp_f32 v223, v223\n" \
  "v_mfma_f32_16x16x32_f16 a[180:183], v[28:31], v[76:79], a[180:183]\n" \
  "v_fma_f32 v224, v224, s80, s80\n" \
  "v_med3_f32 v226, v218, s55, v186\n" \
  "v_mul_f32 v231, v215, v223\n" \
  "v_mfma_f32_16x16x32_f16 a[180:183], v[24:27], v[72:75], a[180:183]\n" \
  "v_exp_f32 v226, v226\n" \
  "v_rcp_f32 v224, v224\n" \
  "s_waitcnt lgkmcnt(2)\n" \
  "v_mfma_f32_16x16x32_f16 a[152:155], v[16:19], v[80:83], a[152:155]\n" \
  "ds_read_b128 v[72:75], v236 offset:256\n" \
  "v_fma_f32 v225, v225, s80, s80\n" \
  "v_med3_f32 v227, v219, s55, v186\n" \
  "v_mfma_f32_16x16x32_f16 a[152:155], v[20:23], v[84:87], a[152:155]\n" \
  "ds_read_b128 v[76:79], v236 offset:8448\n" \
  "v_mul_f32 v232, v216, v224\n" \
  "v_exp_f32 v227, v227\n" \
  "v_rcp_f32 v225, v225\n" \
  "v_mfma_f32_16x16x32_f16 a[184:187], v[28:31], v[84:87], a[184:187]\n" \
  "v_fma_f32 v226, v226, s80, s80\n" \
  "v_mul_f32 v233, v217, v225\n" \
  "v_mfma_f32_16x16x32_f16 a[184:187], v[24:27], v[80:83], a[184:187]\n" \
  "v_rcp_f32 v226, v226\n" \
  "v_fma_f32 v227, v227, s80, s80\n" \
  "v_mfma_f32_16x16x32_f16 a[156:159], v[16:19], v[88:91], a[156:159]\n" \
  "ds_read_b128 v[80:83], v236 offset:512\n" \
  "s_nop 0\n" \
  "v_mul_f32 v234, v218, v226\n" \
  "v_rcp_f32 v227, v227\n" \
  "v_mfma_f32_16x16x32_f16 a[156:159], v[20:23], v[92:95], a[156:159]\n" \
  "ds_read_b128 v[84:87], v236 offset:8704\n" \
  "s_nop 0\n" \
  "v_mul_f32 v235, v219, v227\n" \
  "v_mfma_f32_16x16x32_f16 a[188:191], v[28:31], v[92:95], a[188:191]\n" \
  "s_nop 0\n" \
  "s_mov_b32 s79, 6684672\n" \
  "v_mfma_f32_16x16x32_f16 a[188:191], v[24:27], v[88:91], a[188:191]\n" \
  "s_add_u32 s25, s25, 28\n" \
  "buffer_store_dwordx4 v[228:231], %15, s[72:75], s79 offen nt\n" \
  "buffer_store_dwordx4 v[232:235], %15, s[72:75], s79 offen offset:256 nt\n" \
  "s_waitcnt vmcnt(38) lgkmcnt(2)\n" \
  "v_mfma_f32_16x16x32_f16 a[192:195], v[32:35], v[64:67], a[192:195]\n" \
  "ds_read_b128 v[88:91], v236 offset:768\n" \
  "v_mfma_f32_16x16x32_f16 a[192:195], v[36:39], v[68:71], a[192:195]\n" \
  "ds_read_b128 v[92:95], v236 offset:8960\n" \
  "v_mfma_f32_16x16x32_f16 a[224:227], v[44:47], v[68:71], a[224:227]\n" \
  "v_mfma_f32_16x16x32_f16 a[224:227], v[40:43], v[64:67], a[224:227]\n" \
  "v_mfma_f32_16x16x32_f16 a[196:199], v[32:35], v[72:75], a[196:199]\n" \
  "ds_read_b128 v[64:67], v236 offset:1024\n" \
  "v_mfma_f32_16x16x32_f16 a[196:199], v[36:39], v[76:79], a[196:199]\n" \
  "ds_read_b128 v[68:71], v236 offset:9216\n" \
  "v_accvgpr_read_b32 v188, a128\n" \
  "v_accvgpr_read_b32 v189, a136\n" \
  "v_med3_f32 v196, v188, s55, v186\n" \
  "v_mfma_f32_16x16x32_f16 a[228:231], v[44:47], v[76:79], a[228:231]\n" \
  "v_exp_f32 v196, v196\n" \
  "v_accvgpr_read_b32 v190, a144\n" \
  "v_mfma_f32_16x16x32_f16 a[228:231], v[40:43], v[72:75], a[228:231]\n" \
  "v_med3_f32 v197, v189, s55, v186\n" \
  "v_exp_f32 v197, v197\n" \
  "s_waitcnt lgkmcnt(2)\n" \
  "v_mfma_f32_16x16x32_f16 a[200:203], v[32:35], v[80:83], a[200:203]\n" \
  "ds_read_b128 v[72:75], v236 offset:1280\n" \
  "v_accvgpr_read_b32 v191, a152\n" \
  "v_fma_f32 v196, v196, s80, s80\n" \
  "v_med3_f32 v198, v190, s55, v186\n" \
  "v_mfma_f32_16x16x32_f16 a[200:203], v[36:39], v[84:87], a[200:203]\n" \
  "ds_read_b128 v[76:79], v236 offset:9472\n" \
  "v_exp_f32 v198, v198\n" \
  "v_accvgpr_read_b32 v192, a132\n" \
  "v_mfma_f32_16x16x32_f16 a[232:235], v[44:47], v[84:87], a[232:235]\n" \
  "v_rcp_f32 v196, v196\n" \
  "v_fma_f32 v197, v197, s80, s80\n" \
  "v_mfma_f32_16x16x32_f16 a[232:235], v[40:43], v[80:83], a[232:235]\n" \
  "v_med3_f32 v199, v191, s55, v186\n" \
  "v_mul_f32 v204, v188, v196\n" \
  "v_exp_f32 v199, v199\n" \
  "v_mfma_f32_16x16x32_f16 a[204:207], v[32:35], v[88:91], a[204:207]\n" \
  "ds_read_b128 v[80:83], v236 offset:1536\n" \
  "v_accvgpr_read_b32 v193, a140\n" \
  "v_rcp_f32 v197, v197\n" \
  "v_mfma_f32_16x16x32_f16 a[204:207], v[36:39], v[92:95], a[204:207]\n" \
  "ds_read_b128 v[84:87], v236 offset:9728\n" \
  "v_fma_f32 v198, v198, s80, s80\n" \
  "v_med3_f32 v200, v192, s55, v186\n" \
  "v_mfma_f32_16x16x32_f16 a[236:239], v[44:47], v[92:95], a[236:239]\n" \
  "v_mul_f32 v205, v189, v197\n" \
  "v_exp_f32 v200, v200\n" \
  "v_accvgpr_read_b32 v194, a148\n" \
  "v_mfma_f32_16x16x32_f16 a[236:239], v[40:43], v[88:91], a[236:239]\n" \
  "v_rcp_f32 v198, v198\n" \
  "v_fma_f32 v199, v199, s80, s80\n" \
  "s_waitcnt lgkmcnt(2)\n" \
  "v_mfma_f32_16x16x32_f16 a[208:211], v[32:35], v[64:67], a[208:211]\n" \
  "ds_read_b128 v[88:91], v236 offset:1792\n" \
  "v_med3_f32 v201, v193, s55, v186\n" \
  "v_mul_f32 v206, v190, v198\n" \
  "v_mfma_f32_16x16x32_f16 a[208:211], v[36:39], v[68:71], a[208:211]\n" \
  "ds_read_b128 v[92:95], v236 offset:9984\n" \
  "v_exp_f32 v201, v201\n" \
  "v_accvgpr_read_b32 v195, a156\n" \
  "v_rcp_f32 v199, v199\n" \
  "v_mfma_f32_16x16x32_f16 a[240:243], v[44:47], v[68:71], a[240:243]\n" \
  "v_fma_f32 v200, v200, s80, s80\n" \
  "v_med3_f32 v202, v194, s55, v186\n" \
  "v_mfma_f32_16x16x32_f16 a[240:243], v[40:43], v[64:67], a[240:243]\n" \
  "v_mul_f32 v207, v191, v199\n" \
  "v_exp_f32 v202, v202\n" \
  "v_mfma_f32_16x16x32_f16 a[212:215], v[32:35], v[72:75], a[212:215]\n" \
  "ds_read_b128 v[64:67], v237 offset:0\n" \
  "v_rcp_f32 v200, v200\n" \
  "v_fma_f32 v201, v201, s80, s80\n" \
  "v_mfma_f32_16x16x32_f16 a[212:215], v[36:39], v[76:79], a[212:215]\n" \
  "ds_read_b128 v[68:71], v237 offset:8192\n" \
  "v_med3_f32 v203, v195, s55, v186\n" \
  "v_mul_f32 v208, v192, v200\n" \
  "v_exp_f32 v203, v203\n" \
  "v_mfma_f32_16x16x32_f16 a[244:247], v[44:47], v[76:79], a[244:247]\n" \
  "v_rcp_f32 v201, v201\n" \
  "v_fma_f32 v202, v202, s80, s80\n" \
  "v_mfma_f32_16x16x32_f16 a[244:247], v[40:43], v[72:75], a[244:247]\n" \
  "v_mul_f32 v209, v193, v201\n" \
  "v_rcp_f32 v202, v202\n" \
  "s_waitcnt lgkmcnt(2)\n" \
  "v_mfma_f32_16x16x32_f16 a[216:219], v[32:35], v[80:83], a[216:219]\n" \
  "ds_read_b128 v[72:75], v237 offset:256\n" \
  "v_fma_f32 v203, v203, s80, s80\n" \
  "s_nop 0\n" \
  "v_mul_f32 v210, v194, v202\n" \
  "v_mfma_f32_16x16x32_f16 a[216:219], v[36:39], v[84:87], a[216:219]\n" \
  "ds_read_b128 v[76:79], v237 offset:8448\n" \
  "v_rcp_f32 v203, v203\n" \
  "s_nop 0\n" \
  "v_mfma_f32_16x16x32_f16 a[248:251], v[44:47], v[84:87], a[248:251]\n" \
  "v_mul_f32 v211, v195, v203\n" \
  "s_nop 0\n" \
  "v_mfma_f32_16x16x32_f16 a[248:251], v[40:43], v[80:83], a[248:251]\n" \
  "s_mov_b32 s79, 8388608\n" \
  "s_add_u32 s25, s25, 28\n" \
  "buffer_store_dwordx4 v[204:207], %15, s[72:75], s79 offen nt\n" \
  "buffer_store_dwordx4 v[208:211], %15, s[72:75], s79 offen offset:256 nt\n" \
  "v_mfma_f32_16x16x32_f16 a[220:223], v[32:35], v[88:91], a[220:223]\n" \
  "ds_read_b128 v[80:83], v237 offset:512\n" \
  "v_accvgpr_read_b32 v212, a129\n" \
  "v_accvgpr_read_b32 v213, a137\n" \
  "v_mfma_f32_16x16x32_f16 a[220:223], v[36:39], v[92:95], a[220:223]\n" \
  "ds_read_b128 v[84:87], v237 offset:8704\n" \
  "v_med3_f32 v220, v212, s55, v186\n" \
  "v_exp_f32 v220, v220\n" \
  "v_mfma_f32_16x16x32_f16 a[252:255], v[44:47], v[92:95], a[252:255]\n" \
  "v_accvgpr_read_b32 v214, a145\n" \
  "v_med3_f32 v221, v213, s55, v186\n" \
  "v_exp_f32 v221, v221\n" \
  "v_mfma_f32_16x16x32_f16 a[252:255], v[40:43], v[88:91], a[252:255]\n" \
  "v_accvgpr_read_b32 v215, a153\n" \
  "v_fma_f32 v220, v220, s80, s80\n" \
  "s_waitcnt vmcnt(34) lgkmcnt(2)\n" \
  "v_mfma_f32_16x16x32_f16 a[192:195], v[48:51], v[64:67], a[192:195]\n" \
  "ds_read_b128 v[88:91], v237 offset:768\n" \
  "v_med3_f32 v222, v214, s55, v186\n" \
  "v_exp_f32 v222, v222\n" \
  "v_mfma_f32_16x16x32_f16 a[192:195], v[52:55], v[68:71], a[192:195]\n" \
  "ds_read_b128 v[92:95], v237 offset:8960\n" \
  "v_accvgpr_read_b32 v216, a133\n" \
  "v_rcp_f32 v220, v220\n" \
  "v_mfma_f32_16x16x32_f16 a[224:227], v[60:63], v[68:71], a[224:227]\n" \
  "v_fma_f32 v221, v221, s80, s80\n" \
  "v_med3_f32 v223, v215, s55, v186\n" \
  "v_mul_f32 v228, v212, v220\n" \
  "v_mfma_f32_16x16x32_f16 a[224:227], v[56:59], v[64:67], a[224:227]\n" \
  "v_exp_f32 v223, v223\n" \
  "v_accvgpr_read_b32 v217, a141\n" \
  "v_mfma_f32_16x16x32_f16 a[196:199], v[48:51], v[72:75], a[196:199]\n" \
  "ds_read_b128 v[64:67], v237 offset:1024\n" \
  "v_rcp_f32 v221, v221\n" \
  "v_fma_f32 v222, v222, s80, s80\n" \
  "v_mfma_f32_16x16x32_f16 a[196:199], v[52:55], v[76:79], a[196:199]\n" \
  "ds_read_b128 v[68:71], v237 offset:9216\n" \
  "v_med3_f32 v224, v216, s55, v186\n" \
  "v_mul_f32 v229, v213, v221\n" \
  "v_exp_f32 v224, v224\n" \
  "v_mfma_f32_16x16x32_f16 a[228:231], v[60:63], v[76:79], a[228:231]\n" \
  "v_accvgpr_read_b32 v218, a149\n" \
  "v_rcp_f32 v222, v222\n" \
  "v_mfma_f32_16x16x32_f16 a[228:231], v[56:59], v[72:75], a[228:231]\n" \
  "v_fma_f32 v223, v223, s80, s80\n" \
  "v_med3_f32 v225, v217, s55, v186\n" \
  "s_waitcnt lgkmcnt(2)\n" \
  "v_mfma_f32_16x16x32_f16 a[200:203], v[48:51], v[80:83], a[200:203]\n" \
  "ds_read_b128 v[72:75], v237 offset:1280\n" \
  "v_mul_f32 v230, v214, v222\n" \
  "v_exp_f32 v225, v225\n" \
  "v_accvgpr_read_b32 v219, a157\n" \
  "v_mfma_f32_16x16x32_f16 a[200:203], v[52:55], v[84:87], a[200:203]\n" \
  "ds_read_b128 v[76:79], v237 offset:9472\n" \
  "v_rcp_f32 v223, v223\n" \
  "v_fma_f32 v224, v224, s80, s80\n" \
  "v_mfma_f32_16x16x32_f16 a[232:235], v[60:63], v[84:87], a[232:235]\n" \
  "v_med3_f32 v226, v218, s55, v186\n" \
  "v_mul_f32 v231, v215, v223\n" \
  "v_mfma_f32_16x16x32_f16 a[232:235], v[56:59], v[80:83], a[232:235]\n" \
  "v_exp_f32 v226, v226\n" \
  "v_rcp_f32 v224, v224\n" \
  "v_fma_f32 v225, v225, s80, s80\n" \
  "v_mfma_f32_16x16x32_f16 a[204:207], v[48:51], v[88:91], a[204:207]\n" \
  "ds_read_b128 v[80:83], v237 offset:1536\n" \
  "v_med3_f32 v227, v219, s55, v186\n" \
  "v_mul_f32 v232, v216, v224\n" \
  "v_mfma_f32_16x16x32_f16 a[204:207], v[52:55], v[92:95], a[204:207]\n" \
  "ds_read_b128 v[84:87], v237 offset:9728\n" \
  "v_exp_f32 v227, v227\n" \
  "v_rcp_f32 v225, v225\n" \
  "v_mfma_f32_16x16x32_f16 a[236:239], v[60:63], v[92:95], a[236:239]\n" \
  "v_fma_f32 v226, v226, s80, s80\n" \
  "v_mul_f32 v233, v217, v225\n" \
  "v_rcp_f32 v226, v226\n" \
  "v_mfma_f32_16x16x32_f16 a[236:239], v[56:59], v[88:91], a[236:239]\n" \
  "v_fma_f32 v227, v227, s80, s80\n" \
  "s_nop 0\n" \
  "s_waitcnt lgkmcnt(2)\n" \
  "v_mfma_f32_16x16x32_f16 a[208:211], v[48:51], v[64:67], a[208:211]\n" \
  "ds_read_b128 v[88:91], v237 offset:1792\n" \
  "v_mul_f32 v234, v218, v226\n" \
  "v_rcp_f32 v227, v227\n" \
  "v_mfma_f32_16x16x32_f16 a[208:211], v[52:55], v[68:71], a[208:211]\n" \
  "ds_read_b128 v[92:95], v237 offset:9984\n" \
  "s_nop 0\n" \
  "v_mul_f32 v235, v219, v227\n" \
  "v_mfma_f32_16x16x32_f16 a[240:243], v[60:63], v[68:71], a[240:243]\n" \
  "s_nop 0\n" \
  "s_mov_b32 s79, 8519680\n" \
  "s_add_u32 s25, s25, 28\n" \
  "buffer_store_dwordx4 v[228:231], %15, s[72:75], s79 offen nt\n" \
  "v_mfma_f32_16x16x32_f16 a[240:243], v[56:59], v[64:67], a[240:243]\n" \
  "buffer_store_dwordx4 v[232:235], %15, s[72:75], s79 offen offset:256 nt\n" \
  "v_accvgpr_read_b32 v188, a130\n" \
  "v_mfma_f32_16x16x32_f16 a[212:215], v[48:51], v[72:75], a[212:215]\n" \
  "ds_read_b128 v[64:67], v238 offset:0\n" \
  "v_accvgpr_read_b32 v189, a138\n" \
  "v_med3_f32 v196, v188, s55, v186\n" \
  "v_mfma_f32_16x16x32_f16 a[212:215], v[52:55], v[76:79], a[212:215]\n" \
  "ds_read_b128 v[68:71], v238 offset:8192\n" \
  "v_exp_f32 v196, v196\n" \
  "v_accvgpr_read_b32 v190, a146\n" \
  "v_med3_f32 v197, v189, s55, v186\n" \
  "v_mfma_f32_16x16x32_f16 a[244:247], v[60:63], v[76:79], a[244:247]\n" \
  "v_exp_f32 v197, v197\n" \
  "v_accvgpr_read_b32 v191, a154\n" \
  "v_mfma_f32_16x16x32_f16 a[244:247], v[56:59], v[72:75], a[244:247]\n" \
  "v_fma_f32 v196, v196, s80, s80\n" \
  "v_med3_f32 v198, v190, s55, v186\n" \
  "s_waitcnt lgkmcnt(2)\n" \
  "v_mfma_f32_16x16x32_f16 a[216:219], v[48:51], v[80:83], a[216:219]\n" \
  "ds_read_b128 v[72:75], v238 offset:256\n" \
  "v_exp_f32 v198, v198\n" \
  "v_accvgpr_read_b32 v192, a134\n" \
  "v_rcp_f32 v196, v196\n" \
  "v_mfma_f32_16x16x32_f16 a[216:219], v[52:55], v[84:87], a[216:219]\n" \
  "ds_read_b128 v[76:79], v238 offset:8448\n" \
  "v_fma_f32 v197, v197, s80, s80\n" \
  "v_med3_f32 v199, v191, s55, v186\n" \
  "v_mfma_f32_16x16x32_f16 a[248:251], v[60:63], v[84:87], a[248:251]\n" \
  "v_mul_f32 v204, v188, v196\n" \
  "v_exp_f32 v199, v199\n" \
  "v_mfma_f32_16x16x32_f16 a[248:251], v[56:59], v[80:83], a[248:251]\n" \
  "v_accvgpr_read_b32 v193, a142\n" \
  "v_rcp_f32 v197, v197\n" \
  "v_fma_f32 v198, v198, s80, s80\n" \
  "v_mfma_f32_16x16x32_f16 a[220:223], v[48:51], v[88:91], a[220:223]\n" \
  "ds_read_b128 v[80:83], v238 offset:512\n" \
  "v_med3_f32 v200, v192, s55, v186\n" \
  "v_mul_f32 v205, v189, v197\n" \
  "v_mfma_f32_16x16x32_f16 a[220:223], v[52:55], v[92:95], a[220:223]\n" \
  "ds_read_b128 v[84:87], v238 offset:8704\n" \
  "v_exp_f32 v200, v200\n" \
  "v_accvgpr_read_b32 v194, a150\n" \
  "v_mfma_f32_16x16x32_f16 a[252:255], v[60:63], v[92:95], a[252:255]\n" \
  "v_rcp_f32 v198, v198\n" \
  "v_fma_f32 v199, v199, s80, s80\n" \
  "v_mfma_f32_16x16x32_f16 a[252:255], v[56:59], v[88:91], a[252:255]\n" \
  "v_med3_f32 v201, v193, s55, v186\n" \
  "v_mul_f32 v206, v190, v198\n" \
  "v_exp_f32 v201, v201\n" \
  "s_waitcnt vmcnt(30) lgkmcnt(2)\n" \
  "v_mfma_f32_16x16x32_f16 a[192:195], v[96:99], v[64:67], a[192:195]\n" \
  "ds_read_b128 v[88:91], v238 offset:768\n" \
  "v_accvgpr_read_b32 v195, a158\n" \
  "v_rcp_f32 v199, v199\n" \
  "v_mfma_f32_16x16x32_f16 a[192:195], v[100:103], v[68:71], a[192:195]\n" \
  "ds_read_b128 v[92:95], v238 offset:8960\n" \
  "v_fma_f32 v200, v200, s80, s80\n" \
  "v_med3_f32 v202, v194, s55, v186\n" \
  "v_mfma_f32_16x16x32_f16 a[224:227], v[108:111], v[68:71], a[224:227]\n" \
  "v_mul_f32 v207, v191, v199\n" \
  "v_exp_f32 v202, v202\n" \
  "v_rcp_f32 v200, v200\n" \
  "v_mfma_f32_16x16x32_f16 a[224:227], v[104:107], v[64:67], a[224:227]\n" \
  "v_fma_f32 v201, v201, s80, s80\n" \
  "v_med3_f32 v203, v195, s55, v186\n" \
  "v_mfma_f32_16x16x32_f16 a[196:199], v[96:99], v[72:75], a[196:199]\n" \
  "ds_read_b128 v[64:67], v238 offset:1024\n" \
  "v_mul_f32 v208, v192, v200\n" \
  "v_exp_f32 v203, v203\n" \
  "v_mfma_f32_16x16x32_f16 a[196:199], v[100:103], v[76:79], a[196:199]\n" \
  "ds_read_b128 v[68:71], v238 offset:9216\n" \
  "v_rcp_f32 v201, v201\n" \
  "v_fma_f32 v202, v202, s80, s80\n" \
  "v_mul_f32 v209, v193, v201\n" \
  "v_mfma_f32_16x16x32_f16 a[228:231], v[108:111], v[76:79], a[228:231]\n" \
  "v_rcp_f32 v202, v202\n" \
  "v_fma_f32 v203, v203, s80, s80\n" \
  "v_mfma_f32_16x16x32_f16 a[228:231], v[104:107], v[72:75], a[228:231]\n" \
  "s_nop 0\n" \
  "v_mul_f32 v210, v194, v202\n" \
  "s_waitcnt lgkmcnt(2)\n" \
  "v_mfma_f32_16x16x32_f16 a[200:203], v[96:99], v[80:83], a[200:203]\n" \
  "ds_read_b128 v[72:75], v238 offset:1280\n" \
  "v_rcp_f32 v203, v203\n" \
  "s_nop 0\n" \
  "v_mul_f32 v211, v195, v203\n" \
  "v_mfma_f32_16x16x32_f16 a[200:203], v[100:103], v[84:87], a[200:203]\n" \
  "ds_read_b128 v[76:79], v238 offset:9472\n" \
  "s_nop 0\n" \
  "s_mov_b32 s79, 8650752\n" \
  "v_mfma_f32_16x16x32_f16 a[232:235], v[108:111], v[84:87], a[232:235]\n" \
  "s_add_u32 s25, s25, 28\n" \
  "buffer_store_dwordx4 v[204:207], %15, s[72:75], s79 offen nt\n" \
  "buffer_store_dwordx4 v[208:211], %15, s[72:75], s79 offen offset:256 nt\n" \
  "v_mfma_f32_16x16x32_f16 a[232:235], v[104:107], v[80:83], a[232:235]\n" \
  "v_accvgpr_read_b32 v212, a131\n" \
  "v_accvgpr_read_b32 v213, a139\n" \
  "v_med3_f32 v220, v212, s55, v186\n" \
  "v_mfma_f32_16x16x32_f16 a[204:207], v[96:99], v[88:91], a[204:207]\n" \
  "ds_read_b128 v[80:83], v238 offset:1536\n" \
  "v_exp_f32 v220, v220\n" \
  "v_accvgpr_read_b32 v214, a147\n" \
  "v_mfma_f32_16x16x32_f16 a[204:207], v[100:103], v[92:95], a[204:207]\n" \
  "ds_read_b128 v[84:87], v238 offset:9728\n" \
  "v_med3_f32 v221, v213, s55, v186\n" \
  "v_exp_f32 v221, v221\n" \
  "v_mfma_f32_16x16x32_f16 a[236:239], v[108:111], v[92:95], a[236:239]\n" \
  "v_accvgpr_read_b32 v215, a155\n" \
  "v_fma_f32 v220, v220, s80, s80\n" \
  "v_mfma_f32_16x16x32_f16 a[236:239], v[104:107], v[88:91], a[236:239]\n" \
  "v_med3_f32 v222, v214, s55, v186\n" \
  "v_exp_f32 v222, v222\n" \
  "v_accvgpr_read_b32 v216, a135\n" \
  "s_waitcnt lgkmcnt(2)\n" \
  "v_mfma_f32_16x16x32_f16 a[208:211], v[96:99], v[64:67], a[208:211]\n" \
  "ds_read_b128 v[88:91], v238 offset:1792\n" \
  "v_rcp_f32 v220, v220\n" \
  "v_fma_f32 v221, v221, s80, s80\n" \
  "v_mfma_f32_16x16x32_f16 a[208:211], v[100:103], v[68:71], a[208:211]\n" \
  "ds_read_b128 v[92:95], v238 offset:9984\n" \
  "v_med3_f32 v223, v215, s55, v186\n" \
  "v_mul_f32 v228, v212, v220\n" \
  "v_mfma_f32_16x16x32_f16 a[240:243], v[108:111], v[68:71], a[240:243]\n" \
  "v_exp_f32 v223, v223\n" \
  "v_accvgpr_read_b32 v217, a143\n" \
  "v_rcp_f32 v221, v221\n" \
  "v_mfma_f32_16x16x32_f16 a[240:243], v[104:107], v[64:67], a[240:243]\n" \
  "v_fma_f32 v222, v222, s80, s80\n" \
  "v_med3_f32 v224, v216, s55, v186\n" \
  "v_mfma_f32_16x16x32_f16 a[212:215], v[96:99], v[72:75], a[212:215]\n" \
  "ds_read_b128 v[64:67], v239 offset:0\n" \
  "v_mul_f32 v229, v213, v221\n" \
  "v_exp_f32 v224, v224\n" \
  "v_mfma_f32_16x16x32_f16 a[212:215], v[100:103], v[76:79], a[212:215]\n" \
  "ds_read_b128 v[68:71], v239 offset:8192\n" \
  "v_accvgpr_read_b32 v218, a151\n" \
  "v_rcp_f32 v222, v222\n" \
  "v_fma_f32 v223, v223, s80, s80\n" \
  "v_mfma_f32_16x16x32_f16 a[244:247], v[108:111], v[76:79], a[244:247]\n" \
  "v_med3_f32 v225, v217, s55, v186\n" \
  "v_mul_f32 v230, v214, v222\n" \
  "v_mfma_f32_16x16x32_f16 a[244:247], v[104:107], v[72:75], a[244:247]\n" \
  "v_exp_f32 v225, v225\n" \
  "v_accvgpr_read_b32 v219, a159\n" \
  "s_waitcnt lgkmcnt(2)\n" \
  "v_mfma_f32_16x16x32_f16 a[216:219], v[96:99], v[80:83], a[216:219]\n" \
  "ds_read_b128 v[72:75], v239 offset:256\n" \
  "v_rcp_f32 v223, v223\n" \
  "v_fma_f32 v224, v224, s80, s80\n" \
  "v_med3_f32 v226, v218, s55, v186\n" \
  "v_mfma_f32_16x16x32_f16 a[216:219], v[100:103], v[84:87], a[216:219]\n" \
  "ds_read_b128 v[76:79], v239 offset:8448\n" \
  "v_mul_f32 v231, v215, v223\n" \
  "v_exp_f32 v226, v226\n" \
  "v_mfma_f32_16x16x32_f16 a[248:251], v[108:111], v[84:87], a[248:251]\n" \
  "v_rcp_f32 v224, v224\n" \
  "v_fma_f32 v225, v225, s80, s80\n" \
  "v_mfma_f32_16x16x32_f16 a[248:251], v[104:107], v[80:83], a[248:251]\n" \
  "v_med3_f32 v227, v219, s55, v186\n" \
  "v_mul_f32 v232, v216, v224\n" \
  "v_mfma_f32_16x16x32_f16 a[220:223], v[96:99], v[88:91], a[220:223]\n" \
  "ds_read_b128 v[80:83], v239 offset:512\n" \
  "v_exp_f32 v227, v227\n" \
  "v_rcp_f32 v225, v225\n" \
  "v_fma_f32 v226, v226, s80, s80\n" \
  "v_mfma_f32_16x16x32_f16 a[220:223], v[100:103], v[92:95], a[220:223]\n" \
  "ds_read_b128 v[84:87], v239 offset:8704\n" \
  "v_mul_f32 v233, v217, v225\n" \
  "v_rcp_f32 v226, v226\n" \
  "v_mfma_f32_16x16x32_f16 a[252:255], v[108:111], v[92:95], a[252:255]\n" \
  "v_fma_f32 v227, v227, s80, s80\n" \
  "s_nop 0\n" \
  "v_mfma_f32_16x16x32_f16 a[252:255], v[104:107], v[88:91], a[252:255]\n" \
  "v_mul_f32 v234, v218, v226\n" \
  "v_rcp_f32 v227, v227\n" \
  "s_nop 0\n" \
  "s_waitcnt vmcnt(24) lgkmcnt(2)\n" \
  "v_mfma_f32_16x16x32_f16 a[192:195], v[112:115], v[64:67], a[192:195]\n" \
  "ds_read_b128 v[88:91], v239 offset:768\n" \
  "v_mul_f32 v235, v219, v227\n" \
  "s_nop 0\n" \
  "v_mfma_f32_16x16x32_f16 a[192:195], v[116:119], v[68:71], a[192:195]\n" \
  "ds_read_b128 v[92:95], v239 offset:8960\n" \
  "s_mov_b32 s79, 8781824\n" \
  "s_add_u32 s25, s25, 28\n" \
  "buffer_store_dwordx4 v[228:231], %15, s[72:75], s79 offen nt\n" \
  "v_mfma_f32_16x16x32_f16 a[224:227], v[124:127], v[68:71], a[224:227]\n" \
  "buffer_store_dwordx4 v[232:235], %15, s[72:75], s79 offen offset:256 nt\n" \
  "v_accvgpr_read_b32 v188, a160\n" \
  "v_accvgpr_read_b32 v189, a168\n" \
  "v_mfma_f32_16x16x32_f16 a[224:227], v[120:123], v[64:67], a[224:227]\n" \
  "v_med3_f32 v196, v188, s55, v186\n" \
  "v_exp_f32 v196, v196\n" \
  "v_mfma_f32_16x16x32_f16 a[196:199], v[112:115], v[72:75], a[196:199]\n" \
  "ds_read_b128 v[64:67], v239 offset:1024\n" \
  "v_accvgpr_read_b32 v190, a176\n" \
  "v_med3_f32 v197, v189, s55, v186\n" \
  "v_mfma_f32_16x16x32_f16 a[196:199], v[116:119], v[76:79], a[196:199]\n" \
  "ds_read_b128 v[68:71], v239 offset:9216\n" \
  "v_exp_f32 v197, v197\n" \
  "v_accvgpr_read_b32 v191, a184\n" \
  "v_fma_f32 v196, v196, s80, s80\n" \
  "v_mfma_f32_16x16x32_f16 a[228:231], v[124:127], v[76:79], a[228:231]\n" \
  "v_med3_f32 v198, v190, s55, v186\n" \
  "v_exp_f32 v198, v198\n" \
  "v_mfma_f32_16x16x32_f16 a[228:231], v[120:123], v[72:75], a[228:231]\n" \
  "v_accvgpr_read_b32 v192, a164\n" \
  "v_rcp_f32 v196, v196\n" \
  "s_waitcnt lgkmcnt(2)\n" \
  "v_mfma_f32_16x16x32_f16 a[200:203], v[112:115], v[80:83], a[200:203]\n" \
  "ds_read_b128 v[72:75], v239 offset:1280\n" \
  "v_fma_f32 v197, v197, s80, s80\n" \
  "v_med3_f32 v199, v191, s55, v186\n" \
  "v_mul_f32 v204, v188, v196\n" \
  "v_mfma_f32_16x16x32_f16 a[200:203], v[116:119], v[84:87], a[200:203]\n" \
  "ds_read_b128 v[76:79], v239 offset:9472\n" \
  "v_exp_f32 v199, v199\n" \
  "v_accvgpr_read_b32 v193, a172\n" \
  "v_mfma_f32_16x16x32_f16 a[232:235], v[124:127], v[84:87], a[232:235]\n" \
  "v_rcp_f32 v197, v197\n" \
  "v_fma_f32 v198, v198, s80, s80\n" \
  "v_mfma_f32_16x16x32_f16 a[232:235], v[120:123], v[80:83], a[232:235]\n" \
  "v_med3_f32 v200, v192, s55, v186\n" \
  "v_mul_f32 v205, v189, v197\n" \
  "v_mfma_f32_16x16x32_f16 a[204:207], v[112:115], v[88:91], a[204:207]\n" \
  "ds_read_b128 v[80:83], v239 offset:1536\n" \
  "v_exp_f32 v200, v200\n" \
  "v_accvgpr_read_b32 v194, a180\n" \
  "v_rcp_f32 v198, v198\n" \
  "v_mfma_f32_16x16x32_f16 a[204:207], v[116:119], v[92:95], a[204:207]\n" \
  "ds_read_b128 v[84:87], v239 offset:9728\n" \
  "v_fma_f32 v199, v199, s80, s80\n" \
  "v_med3_f32 v201, v193, s55, v186\n" \
  "v_mfma_f32_16x16x32_f16 a[236:239], v[124:127], v[92:95], a[236:239]\n" \
  "v_mul_f32 v206, v190, v198\n" \
  "v_exp_f32 v201, v201\n" \
  "v_mfma_f32_16x16x32_f16 a[236:239], v[120:123], v[88:91], a[236:239]\n" \
  "v_accvgpr_read_b32 v195, a188\n" \
  "v_rcp_f32 v199, v199\n" \
  "v_fma_f32 v200, v200, s80, s80\n" \
  "s_waitcnt lgkmcnt(2)\n" \
  "v_mfma_f32_16x16x32_f16 a[208:211], v[112:115], v[64:67], a[208:211]\n" \
  "ds_read_b128 v[88:91], v239 offset:1792\n" \
  "v_med3_f32 v202, v194, s55, v186\n" \
  "v_mul_f32 v207, v191, v199\n" \
  "v_mfma_f32_16x16x32_f16 a[208:211], v[116:119], v[68:71], a[208:211]\n" \
  "ds_read_b128 v[92:95], v239 offset:9984\n" \
  "v_exp_f32 v202, v202\n" \
  "v_rcp_f32 v200, v200\n" \
  "v_mfma_f32_16x16x32_f16 a[240:243], v[124:127], v[68:71], a[240:243]\n" \
  "v_fma_f32 v201, v201, s80, s80\n" \
  "v_med3_f32 v203, v195, s55, v186\n" \
  "v_mul_f32 v208, v192, v200\n" \
  "v_mfma_f32_16x16x32_f16 a[240:243], v[120:123], v[64:67], a[240:243]\n" \
  "v_exp_f32 v203, v203\n" \
  "v_rcp_f32 v201, v201\n" \
  "v_mfma_f32_16x16x32_f16 a[212:215], v[112:115], v[72:75], a[212:215]\n" \
  "ds_read_b128 v[64:67], v240 offset:0\n" \
  "v_fma_f32 v202, v202, s80, s80\n" \
  "v_mul_f32 v209, v193, v201\n" \
  "v_mfma_f32_16x16x32_f16 a[212:215], v[116:119], v[76:79], a[212:215]\n" \
  "ds_read_b128 v[68:71], v240 offset:8192\n" \
  "v_rcp_f32 v202, v202\n" \
  "v_fma_f32 v203, v203, s80, s80\n" \
  "s_nop 0\n" \
  "v_mfma_f32_16x16x32_f16 a[244:247], v[124:127], v[76:79], a[244:247]\n" \
  "v_mul_f32 v210, v194, v202\n" \
  "v_rcp_f32 v203, v203\n" \
  "v_mfma_f32_16x16x32_f16 a[244:247], v[120:123], v[72:75], a[244:247]\n" \
  "s_nop 0\n" \
  "v_mul_f32 v211, v195, v203\n" \
  "s_waitcnt lgkmcnt(2)\n" \
  "v_mfma_f32_16x16x32_f16 a[216:219], v[112:115], v[80:83], a[216:219]\n" \
  "ds_read_b128 v[72:75], v240 offset:256\n" \
  "s_nop 0\n" \
  "s_mov_b32 s79, 10485760\n" \
  "v_mfma_f32_16x16x32_f16 a[216:219], v[116:119], v[84:87], a[216:219]\n" \
  "ds_read_b128 v[76:79], v240 offset:8448\n" \
  "s_add_u32 s25, s25, 28\n" \
  "buffer_store_dwordx4 v[204:207], %15, s[72:75], s79 offen nt\n" \
  "buffer_store_dwordx4 v[208:211], %15, s[72:75], s79 offen offset:256 nt\n" \
  "v_accvgpr_read_b32 v212, a161\n" \
  "v_mfma_f32_16x16x32_f16 a[248:251], v[124:127], v[84:87], a[248:251]\n" \
  "v_accvgpr_read_b32 v213, a169\n" \
  "v_med3_f32 v220, v212, s55, v186\n" \
  "v_mfma_f32_16x16x32_f16 a[248:251], v[120:123], v[80:83], a[248:251]\n" \
  "v_exp_f32 v220, v220\n" \
  "v_accvgpr_read_b32 v214, a177\n" \
  "v_mfma_f32_16x16x32_f16 a[220:223], v[112:115], v[88:91], a[220:223]\n" \
  "ds_read_b128 v[80:83], v240 offset:512\n" \
  "v_med3_f32 v221, v213, s55, v186\n" \
  "v_exp_f32 v221, v221\n" \
  "v_accvgpr_read_b32 v215, a185\n" \
  "v_mfma_f32_16x16x32_f16 a[220:223], v[116:119], v[92:95], a[220:223]\n" \
  "ds_read_b128 v[84:87], v240 offset:8704\n" \
  "v_fma_f32 v220, v220, s80, s80\n" \
  "v_med3_f32 v222, v214, s55, v186\n" \
  "v_mfma_f32_16x16x32_f16 a[252:255], v[124:127], v[92:95], a[252:255]\n" \
  "v_exp_f32 v222, v222\n" \
  "v_accvgpr_read_b32 v216, a165\n" \
  "v_mfma_f32_16x16x32_f16 a[252:255], v[120:123], v[88:91], a[252:255]\n" \
  "v_rcp_f32 v220, v220\n" \
  "v_fma_f32 v221, v221, s80, s80\n" \
  "v_med3_f32 v223, v215, s55, v186\n" \
  "s_waitcnt vmcnt(22) lgkmcnt(2)\n" \
  "v_mfma_f32_16x16x32_f16 a[192:195], v[128:131], v[64:67], a[192:195]\n" \
  "ds_read_b128 v[88:91], v240 offset:768\n" \
  "v_mul_f32 v228, v212, v220\n" \
  "v_exp_f32 v223, v223\n" \
  "v_mfma_f32_16x16x32_f16 a[192:195], v[132:135], v[68:71], a[192:195]\n" \
  "ds_read_b128 v[92:95], v240 offset:8960\n" \
  "v_accvgpr_read_b32 v217, a173\n" \
  "v_rcp_f32 v221, v221\n" \
  "v_mfma_f32_16x16x32_f16 a[224:227], v[140:143], v[68:71], a[224:227]\n" \
  "v_fma_f32 v222, v222, s80, s80\n" \
  "v_med3_f32 v224, v216, s55, v186\n" \
  "v_mul_f32 v229, v213, v221\n" \
  "v_mfma_f32_16x16x32_f16 a[224:227], v[136:139], v[64:67], a[224:227]\n" \
  "v_exp_f32 v224, v224\n" \
  "v_accvgpr_read_b32 v218, a181\n" \
  "v_mfma_f32_16x16x32_f16 a[196:199], v[128:131], v[72:75], a[196:199]\n" \
  "ds_read_b128 v[64:67], v240 offset:1024\n" \
  "v_rcp_f32 v222, v222\n" \
  "v_fma_f32 v223, v223, s80, s80\n" \
  "v_mfma_f32_16x16x32_f16 a[196:199], v[132:135], v[76:79], a[196:199]\n" \
  "ds_read_b128 v[68:71], v240 offset:9216\n" \
  "v_med3_f32 v225, v217, s55, v186\n" \
  "v_mul_f32 v230, v214, v222\n" \
  "v_exp_f32 v225, v225\n" \
  "v_mfma_f32_16x16x32_f16 a[228:231], v[140:143], v[76:79], a[228:231]\n" \
  "v_accvgpr_read_b32 v219, a189\n" \
  "v_rcp_f32 v223, v223\n" \
  "v_mfma_f32_16x16x32_f16 a[228:231], v[136:139], v[72:75], a[228:231]\n" \
  "v_fma_f32 v224, v224, s80, s80\n" \
  "v_med3_f32 v226, v218, s55, v186\n" \
  "s_waitcnt lgkmcnt(2)\n" \
  "v_mfma_f32_16x16x32_f16 a[200:203], v[128:131], v[80:83], a[200:203]\n" \
  "ds_read_b128 v[72:75], v240 offset:1280\n" \
  "v_mul_f32 v231, v215, v223\n" \
  "v_exp_f32 v226, v226\n" \
  "v_mfma_f32_16x16x32_f16 a[200:203], v[132:135], v[84:87], a[200:203]\n" \
  "ds_read_b128 v[76:79], v240 offset:9472\n" \
  "v_rcp_f32 v224, v224\n" \
  "v_fma_f32 v225, v225, s80, s80\n" \
  "v_med3_f32 v227, v219, s55, v186\n" \
  "v_mfma_f32_16x16x32_f16 a[232:235], v[140:143], v[84:87], a[232:235]\n" \
  "v_mul_f32 v232, v216, v224\n" \
  "v_exp_f32 v227, v227\n" \
  "v_mfma_f32_16x16x32_f16 a[232:235], v[136:139], v[80:83], a[232:235]\n" \
  "v_rcp_f32 v225, v225\n" \
  "v_fma_f32 v226, v226, s80, s80\n" \
  "v_mfma_f32_16x16x32_f16 a[204:207], v[128:131], v[88:91], a[204:207]\n" \
  "ds_read_b128 v[80:83], v240 offset:1536\n" \
  "v_mul_f32 v233, v217, v225\n" \
  "v_rcp_f32 v226, v226\n" \
  "v_fma_f32 v227, v227, s80, s80\n" \
  "v_mfma_f32_16x16x32_f16 a[204:207], v[132:135], v[92:95], a[204:207]\n" \
  "ds_read_b128 v[84:87], v240 offset:9728\n" \
  "s_nop 0\n" \
  "v_mul_f32 v234, v218, v226\n" \
  "v_mfma_f32_16x16x32_f16 a[236:239], v[140:143], v[92:95], a[236:239]\n" \
  "v_rcp_f32 v227, v227\n" \
  "s_nop 0\n" \
  "v_mfma_f32_16x16x32_f16 a[236:239], v[136:139], v[88:91], a[236:239]\n" \
  "v_mul_f32 v235, v219, v227\n" \
  "s_nop 0\n" \
  "s_mov_b32 s79, 10616832\n" \
  "s_waitcnt lgkmcnt(2)\n" \
  "v_mfma_f32_16x16x32_f16 a[208:211], v[128:131], v[64:67], a[208:211]\n" \
  "ds_read_b128 v[88:91], v240 offset:1792\n" \
  "s_add_u32 s25, s25, 28\n" \
  "buffer_store_dwordx4 v[228:231], %15, s[72:75], s79 offen nt\n" \
  "buffer_store_dwordx4 v[232:235], %15, s[72:75], s79 offen offset:256 nt\n" \
  "v_mfma_f32_16x16x32_f16 a[208:211], v[132:135], v[68:71], a[208:211]\n" \
  "ds_read_b128 v[92:95], v240 offset:9984\n" \
  "v_accvgpr_read_b32 v188, a162\n" \
  "v_accvgpr_read_b32 v189, a170\n" \
  "v_mfma_f32_16x16x32_f16 a[240:243], v[140:143], v[68:71], a[240:243]\n" \
  "v_med3_f32 v196, v188, s55, v186\n" \
  "v_exp_f32 v196, v196\n" \
  "v_accvgpr_read_b32 v190, a178\n" \
  "v_mfma_f32_16x16x32_f16 a[240:243], v[136:139], v[64:67], a[240:243]\n" \
  "v_med3_f32 v197, v189, s55, v186\n" \
  "v_exp_f32 v197, v197\n" \
  "v_mfma_f32_16x16x32_f16 a[212:215], v[128:131], v[72:75], a[212:215]\n" \
  "ds_read_b128 v[64:67], v241 offset:0\n" \
  "v_accvgpr_read_b32 v191, a186\n" \
  "v_fma_f32 v196, v196, s80, s80\n" \
  "v_mfma_f32_16x16x32_f16 a[212:215], v[132:135], v[76:79], a[212:215]\n" \
  "ds_read_b128 v[68:71], v241 offset:8192\n" \
  "v_med3_f32 v198, v190, s55, v186\n" \
  "v_exp_f32 v198, v198\n" \
  "v_mfma_f32_16x16x32_f16 a[244:247], v[140:143], v[76:79], a[244:247]\n" \
  "v_accvgpr_read_b32 v192, a166\n" \
  "v_rcp_f32 v196, v196\n" \
  "v_fma_f32 v197, v197, s80, s80\n" \
  "v_mfma_f32_16x16x32_f16 a[244:247], v[136:139], v[72:75], a[244:247]\n" \
  "v_med3_f32 v199, v191, s55, v186\n" \
  "v_mul_f32 v204, v188, v196\n" \
  "s_waitcnt lgkmcnt(2)\n" \
  "v_mfma_f32_16x16x32_f16 a[216:219], v[128:131], v[80:83], a[216:219]\n" \
  "ds_read_b128 v[72:75], v241 offset:256\n" \
  "v_exp_f32 v199, v199\n" \
  "v_accvgpr_read_b32 v193, a174\n" \
  "v_mfma_f32_16x16x32_f16 a[216:219], v[132:135], v[84:87], a[216:219]\n" \
  "ds_read_b128 v[76:79], v241 offset:8448\n" \
  "v_rcp_f32 v197, v197\n" \
  "v_fma_f32 v198, v198, s80, s80\n" \
  "v_med3_f32 v200, v192, s55, v186\n" \
  "v_mfma_f32_16x16x32_f16 a[248:251], v[140:143], v[84:87], a[248:251]\n" \
  "v_mul_f32 v205, v189, v197\n" \
  "v_exp_f32 v200, v200\n" \
  "v_mfma_f32_16x16x32_f16 a[248:251], v[136:139], v[80:83], a[248:251]\n" \
  "v_accvgpr_read_b32 v194, a182\n" \
  "v_rcp_f32 v198, v198\n" \
  "v_mfma_f32_16x16x32_f16 a[220:223], v[128:131], v[88:91], a[220:223]\n" \
  "ds_read_b128 v[80:83], v241 offset:512\n" \
  "v_fma_f32 v199, v199, s80, s80\n" \
  "v_med3_f32 v201, v193, s55, v186\n" \
  "v_mul_f32 v206, v190, v198\n" \
  "v_mfma_f32_16x16x32_f16 a[220:223], v[132:135], v[92:95], a[220:223]\n" \
  "ds_read_b128 v[84:87], v241 offset:8704\n" \
  "v_exp_f32 v201, v201\n" \
  "v_accvgpr_read_b32 v195, a190\n" \
  "v_mfma_f32_16x16x32_f16 a[252:255], v[140:143], v[92:95], a[252:255]\n" \
  "v_rcp_f32 v199, v199\n" \
  "v_fma_f32 v200, v200, s80, s80\n" \
  "v_mfma_f32_16x16x32_f16 a[252:255], v[136:139], v[88:91], a[252:255]\n" \
  "v_med3_f32 v202, v194, s55, v186\n" \
  "v_mul_f32 v207, v191, v199\n" \
  "v_exp_f32 v202, v202\n" \
  "s_waitcnt vmcnt(18) lgkmcnt(2)\n" \
  "v_mfma_f32_16x16x32_f16 a[192:195], v[144:147], v[64:67], a[192:195]\n" \
  "ds_read_b128 v[88:91], v241 offset:768\n" \
  "v_rcp_f32 v200, v200\n" \
  "v_fma_f32 v201, v201, s80, s80\n" \
  "v_mfma_f32_16x16x32_f16 a[192:195], v[148:151], v[68:71], a[192:195]\n" \
  "ds_read_b128 v[92:95], v241 offset:8960\n" \
  "v_med3_f32 v203, v195, s55, v186\n" \
  "v_mul_f32 v208, v192, v200\n" \
  "v_mfma_f32_16x16x32_f16 a[224:227], v[156:159], v[68:71], a[224:227]\n" \
  "v_exp_f32 v203, v203\n" \
  "v_rcp_f32 v201, v201\n" \
  "v_fma_f32 v202, v202, s80, s80\n" \
  "v_mfma_f32_16x16x32_f16 a[224:227], v[152:155], v[64:67], a[224:227]\n" \
  "v_mul_f32 v209, v193, v201\n" \
  "v_rcp_f32 v202, v202\n" \
  "v_mfma_f32_16x16x32_f16 a[196:199], v[144:147], v[72:75], a[196:199]\n" \
  "ds_read_b128 v[64:67], v241 offset:1024\n" \
  "v_fma_f32 v203, v203, s80, s80\n" \
  "s_nop 0\n" \
  "v_mfma_f32_16x16x32_f16 a[196:199], v[148:151], v[76:79], a[196:199]\n" \
  "ds_read_b128 v[68:71], v241 offset:9216\n" \
  "v_mul_f32 v210, v194, v202\n" \
  "v_rcp_f32 v203, v203\n" \
  "v_mfma_f32_16x16x32_f16 a[228:231], v[156:159], v[76:79], a[228:231]\n" \
  "s_nop 0\n" \
  "v_mul_f32 v211, v195, v203\n" \
  "s_nop 0\n" \
  "v_mfma_f32_16x16x32_f16 a[228:231], v[152:155], v[72:75], a[228:231]\n" \
  "s_mov_b32 s79, 10747904\n" \
  "s_add_u32 s25, s25, 28\n" \
  "buffer_store_dwordx4 v[204:207], %15, s[72:75], s79 offen nt\n" \
  "s_waitcnt lgkmcnt(2)\n" \
  "v_mfma_f32_16x16x32_f16 a[200:203], v[144:147], v[80:83], a[200:203]\n" \
  "ds_read_b128 v[72:75], v241 offset:1280\n" \
  "buffer_store_dwordx4 v[208:211], %15, s[72:75], s79 offen offset:256 nt\n" \
  "v_accvgpr_read_b32 v212, a163\n" \
  "v_mfma_f32_16x16x32_f16 a[200:203], v[148:151], v[84:87], a[200:203]\n" \
  "ds_read_b128 v[76:79], v241 offset:9472\n" \
  "v_accvgpr_read_b32 v213, a171\n" \
  "v_med3_f32 v220, v212, s55, v186\n" \
  "v_exp_f32 v220, v220\n" \
  "v_mfma_f32_16x16x32_f16 a[232:235], v[156:159], v[84:87], a[232:235]\n" \
  "v_accvgpr_read_b32 v214, a179\n" \
  "v_med3_f32 v221, v213, s55, v186\n" \
  "v_mfma_f32_16x16x32_f16 a[232:235], v[152:155], v[80:83], a[232:235]\n" \
  "v_exp_f32 v221, v221\n" \
  "v_accvgpr_read_b32 v215, a187\n" \
  "v_mfma_f32_16x16x32_f16 a[204:207], v[144:147], v[88:91], a[204:207]\n" \
  "ds_read_b128 v[80:83], v241 offset:1536\n" \
  "v_fma_f32 v220, v220, s80, s80\n" \
  "v_med3_f32 v222, v214, s55, v186\n" \
  "v_exp_f32 v222, v222\n" \
  "v_mfma_f32_16x16x32_f16 a[204:207], v[148:151], v[92:95], a[204:207]\n" \
  "ds_read_b128 v[84:87], v241 offset:9728\n" \
  "v_accvgpr_read_b32 v216, a167\n" \
  "v_rcp_f32 v220, v220\n" \
  "v_mfma_f32_16x16x32_f16 a[236:239], v[156:159], v[92:95], a[236:239]\n" \
  "v_fma_f32 v221, v221, s80, s80\n" \
  "v_med3_f32 v223, v215, s55, v186\n" \
  "v_mfma_f32_16x16x32_f16 a[236:239], v[152:155], v[88:91], a[236:239]\n" \
  "v_mul_f32 v228, v212, v220\n" \
  "v_exp_f32 v223, v223\n" \
  "v_accvgpr_read_b32 v217, a175\n" \
  "s_waitcnt lgkmcnt(2)\n" \
  "v_mfma_f32_16x16x32_f16 a[208:211], v[144:147], v[64:67], a[208:211]\n" \
  "ds_read_b128 v[88:91], v241 offset:1792\n" \
  "v_rcp_f32 v221, v221\n" \
  "v_fma_f32 v222, v222, s80, s80\n" \
  "v_mfma_f32_16x16x32_f16 a[208:211], v[148:151], v[68:71], a[208:211]\n" \
  "ds_read_b128 v[92:95], v241 offset:9984\n" \
  "v_med3_f32 v224, v216, s55, v186\n" \
  "v_mul_f32 v229, v213, v221\n" \
  "v_mfma_f32_16x16x32_f16 a[240:243], v[156:159], v[68:71], a[240:243]\n" \
  "v_exp_f32 v224, v224\n" \
  "v_accvgpr_read_b32 v218, a183\n" \
  "v_mfma_f32_16x16x32_f16 a[240:243], v[152:155], v[64:67], a[240:243]\n" \
  "v_rcp_f32 v222, v222\n" \
  "v_fma_f32 v223, v223, s80, s80\n" \
  "v_med3_f32 v225, v217, s55, v186\n" \
  "v_mfma_f32_16x16x32_f16 a[212:215], v[144:147], v[72:75], a[212:215]\n" \
  "v_mul_f32 v230, v214, v222\n" \
  "v_exp_f32 v225, v225\n" \
  "v_mfma_f32_16x16x32_f16 a[212:215], v[148:151], v[76:79], a[212:215]\n" \
  "v_accvgpr_read_b32 v219, a191\n" \
  "v_rcp_f32 v223, v223\n" \
  "v_mfma_f32_16x16x32_f16 a[244:247], v[156:159], v[76:79], a[244:247]\n" \
  "v_fma_f32 v224, v224, s80, s80\n" \
  "v_med3_f32 v226, v218, s55, v186\n" \
  "v_mul_f32 v231, v215, v223\n" \
  "v_mfma_f32_16x16x32_f16 a[244:247], v[152:155], v[72:75], a[244:247]\n" \
  "v_exp_f32 v226, v226\n" \
  "v_rcp_f32 v224, v224\n" \
  "s_waitcnt lgkmcnt(0)\n" \
  "v_mfma_f32_16x16x32_f16 a[216:219], v[144:147], v[80:83], a[216:219]\n" \
  "v_fma_f32 v225, v225, s80, s80\n" \
  "v_med3_f32 v227, v219, s55, v186\n" \
  "v_mfma_f32_16x16x32_f16 a[216:219], v[148:151], v[84:87], a[216:219]\n" \
  "v_mul_f32 v232, v216, v224\n" \
  "v_exp_f32 v227, v227\n" \
  "v_rcp_f32 v225, v225\n" \
  "v_mfma_f32_16x16x32_f16 a[248:251], v[156:159], v[84:87], a[248:251]\n" \
  "v_fma_f32 v226, v226, s80, s80\n" \
  "v_mul_f32 v233, v217, v225\n" \
  "v_mfma_f32_16x16x32_f16 a[248:251], v[152:155], v[80:83], a[248:251]\n" \
  "v_rcp_f32 v226, v226\n" \
  "v_fma_f32 v227, v227, s80, s80\n" \
  "v_mfma_f32_16x16x32_f16 a[220:223], v[144:147], v[88:91], a[220:223]\n" \
  "s_nop 0\n" \
  "v_mul_f32 v234, v218, v226\n" \
  "v_rcp_f32 v227, v227\n" \
  "v_mfma_f32_16x16x32_f16 a[220:223], v[148:151], v[92:95], a[220:223]\n" \
  "s_nop 0\n" \
  "v_mul_f32 v235, v219, v227\n" \
  "v_mfma_f32_16x16x32_f16 a[252:255], v[156:159], v[92:95], a[252:255]\n" \
  "s_nop 0\n" \
  "s_mov_b32 s79, 10878976\n" \
  "v_mfma_f32_16x16x32_f16 a[252:255], v[152:155], v[88:91], a[252:255]\n" \
  "s_add_u32 s25, s25, 28\n" \
  "buffer_store_dwordx4 v[228:231], %15, s[72:75], s79 offen nt\n" \
  "buffer_store_dwordx4 v[232:235], %15, s[72:75], s79 offen offset:256 nt\n" \
  "s_memrealtime s[62:63]\n" \
  "s_nop 7\n" \
  "s_nop 1\n" \
  "v_accvgpr_read_b32 v188, a192\n" \
  "v_accvgpr_read_b32 v189, a200\n" \
  "v_med3_f32 v196, v188, s55, v186\n" \
  "v_exp_f32 v196, v196\n" \
  "v_accvgpr_read_b32 v190, a208\n" \
  "v_med3_f32 v197, v189, s55, v186\n" \
  "v_exp_f32 v197, v197\n" \
  "v_accvgpr_read_b32 v191, a216\n" \
  "v_fma_f32 v196, v196, s80, s80\n" \
  "v_med3_f32 v198, v190, s55, v186\n" \
  "v_exp_f32 v198, v198\n" \
  "v_accvgpr_read_b32 v192, a196\n" \
  "v_rcp_f32 v196, v196\n" \
  "v_fma_f32 v197, v197, s80, s80\n" \
  "v_med3_f32 v199, v191, s55, v186\n" \
  "v_mul_f32 v204, v188, v196\n" \
  "v_exp_f32 v199, v199\n" \
  "v_accvgpr_read_b32 v193, a204\n" \
  "v_rcp_f32 v197, v197\n" \
  "v_fma_f32 v198, v198, s80, s80\n" \
  "v_med3_f32 v200, v192, s55, v186\n" \
  "v_mul_f32 v205, v189, v197\n" \
  "v_exp_f32 v200, v200\n" \
  "v_accvgpr_read_b32 v194, a212\n" \
  "v_rcp_f32 v198, v198\n" \
  "v_fma_f32 v199, v199, s80, s80\n" \
  "v_med3_f32 v201, v193, s55, v186\n" \
  "s_memrealtime s[38:39]\n" \
  "v_mul_f32 v206, v190, v198\n" \
  "v_exp_f32 v201, v201\n" \
  "v_accvgpr_read_b32 v195, a220\n" \
  "v_rcp_f32 v199, v199\n" \
  "v_fma_f32 v200, v200, s80, s80\n" \
  "v_med3_f32 v202, v194, s55, v186\n" \
  "v_mul_f32 v207, v191, v199\n" \
  "v_exp_f32 v202, v202\n" \
  "v_rcp_f32 v200, v200\n" \
  "v_fma_f32 v201, v201, s80, s80\n" \
  "v_med3_f32 v203, v195, s55, v186\n" \
  "v_mul_f32 v208, v192, v200\n" \
  "v_exp_f32 v203, v203\n" \
  "v_rcp_f32 v201, v201\n" \
  "v_fma_f32 v202, v202, s80, s80\n" \
  "v_mul_f32 v209, v193, v201\n" \
  "v_rcp_f32 v202, v202\n" \
  "v_fma_f32 v203, v203, s80, s80\n" \
  "s_nop 0\n" \
  "v_mul_f32 v210, v194, v202\n" \
  "v_rcp_f32 v203, v203\n" \
  "s_nop 0\n" \
  "v_mul_f32 v211, v195, v203\n" \
  "s_nop 0\n" \
  "s_mov_b32 s79, 12582912\n" \
  "EW24_%=:\n" \
  "s_waitcnt lgkmcnt(0)\n" \
  "s_sub_u32 s24, s38, s25\n" \
  "s_cmp_lt_i32 s24, 0\n" \
  "s_cbranch_scc0 EG24_%=\n" \
  "s_memrealtime s[38:39]\n" \
  "s_branch EW24_%=\n" \
  "EG24_%=:\n" \
  "s_add_u32 s25, s25, 28\n" \
  "buffer_store_dwordx4 v[204:207], %15, s[72:75], s79 offen nt\n" \
  "buffer_store_dwordx4 v[208:211], %15, s[72:75], s79 offen offset:256 nt\n" \
  "v_accvgpr_read_b32 v212, a193\n" \
  "v_accvgpr_read_b32 v213, a201\n" \
  "v_med3_f32 v220, v212, s55, v186\n" \
  "v_exp_f32 v220, v220\n" \
  "v_accvgpr_read_b32 v214, a209\n" \
  "v_med3_f32 v221, v213, s55, v186\n" \
  "v_exp_f32 v221, v221\n" \
  "v_accvgpr_read_b32 v215, a217\n" \
  "v_fma_f32 v220, v220, s80, s80\n" \
  "v_med3_f32 v222, v214, s55, v186\n" \
  "v_exp_f32 v222, v222\n" \
  "v_accvgpr_read_b32 v216, a197\n" \
  "v_rcp_f32 v220, v220\n" \
  "v_fma_f32 v221, v221, s80, s80\n" \
  "v_med3_f32 v223, v215, s55, v186\n" \
  "v_mul_f32 v228, v212, v220\n" \
  "v_exp_f32 v223, v223\n" \
  "v_accvgpr_read_b32 v217, a205\n" \
  "v_rcp_f32 v221, v221\n" \
  "v_fma_f32 v222, v222, s80, s80\n" \
  "v_med3_f32 v224, v216, s55, v186\n" \
  "v_mul_f32 v229, v213, v221\n" \
  "v_exp_f32 v224, v224\n" \
  "v_accvgpr_read_b32 v218, a213\n" \
  "v_rcp_f32 v222, v222\n" \
  "v_fma_f32 v223, v223, s80, s80\n" \
  "v_med3_f32 v225, v217, s55, v186\n" \
  "s_memrealtime s[38:39]\n" \
  "v_mul_f32 v230, v214, v222\n" \
  "v_exp_f32 v225, v225\n" \
  "v_accvgpr_read_b32 v219, a221\n" \
  "v_rcp_f32 v223, v223\n" \
  "v_fma_f32 v224, v224, s80, s80\n" \
  "v_med3_f32 v226, v218, s55, v186\n" \
  "v_mul_f32 v231, v215, v223\n" \
  "v_exp_f32 v226, v226\n" \
  "v_rcp_f32 v224, v224\n" \
  "v_fma_f32 v225, v225, s80, s80\n" \
  "v_med3_f32 v227, v219, s55, v186\n" \
  "v_mul_f32 v232, v216, v224\n" \
  "v_exp_f32 v227, v227\n" \
  "v_rcp_f32 v225, v225\n" \
  "v_fma_f32 v226, v226, s80, s80\n" \
  "v_mul_f32 v233, v217, v225\n" \
  "v_rcp_f32 v226, v226\n" \
  "v_fma_f32 v227, v227, s80, s80\n" \
  "s_nop 0\n" \
  "v_mul_f32 v234, v218, v226\n" \
  "v_rcp_f32 v227, v227\n" \
  "s_nop 0\n" \
  "v_mul_f32 v235, v219, v227\n" \
  "s_nop 0\n" \
  "s_mov_b32 s79, 12713984\n" \
  "EW25_%=:\n" \
  "s_waitcnt lgkmcnt(0)\n" \
  "s_sub_u32 s24, s38, s25\n" \
  "s_cmp_lt_i32 s24, 0\n" \
  "s_cbranch_scc0 EG25_%=\n" \
  "s_memrealtime s[38:39]\n" \
  "s_branch EW25_%=\n" \
  "EG25_%=:\n" \
  "s_add_u32 s25, s25, 28\n" \
  "buffer_store_dwordx4 v[228:231], %15, s[72:75], s79 offen nt\n" \
  "buffer_store_dwordx4 v[232:235], %15, s[72:75], s79 offen offset:256 nt\n" \
  "v_accvgpr_read_b32 v188, a194\n" \
  "v_accvgpr_read_b32 v189, a202\n" \
  "v_med3_f32 v196, v188, s55, v186\n" \
  "v_exp_f32 v196, v196\n" \
  "v_accvgpr_read_b32 v190, a210\n" \
  "v_med3_f32 v197, v189, s55, v186\n" \
  "v_exp_f32 v197, v197\n" \
  "v_accvgpr_read_b32 v191, a218\n" \
  "v_fma_f32 v196, v196, s80, s80\n" \
  "v_med3_f32 v198, v190, s55, v186\n" \
  "v_exp_f32 v198, v198\n" \
  "v_accvgpr_read_b32 v192, a198\n" \
  "v_rcp_f32 v196, v196\n" \
  "v_fma_f32 v197, v197, s80, s80\n" \
  "v_med3_f32 v199, v191, s55, v186\n" \
  "v_mul_f32 v204, v188, v196\n" \
  "v_exp_f32 v199, v199\n" \
  "v_accvgpr_read_b32 v193, a206\n" \
  "v_rcp_f32 v197, v197\n" \
  "v_fma_f32 v198, v198, s80, s80\n" \
  "v_med3_f32 v200, v192, s55, v186\n" \
  "v_mul_f32 v205, v189, v197\n" \
  "v_exp_f32 v200, v200\n" \
  "v_accvgpr_read_b32 v194, a214\n" \
  "v_rcp_f32 v198, v198\n" \
  "v_fma_f32 v199, v199, s80, s80\n" \
  "v_med3_f32 v201, v193, s55, v186\n" \
  "s_memrealtime s[38:39]\n" \
  "v_mul_f32 v206, v190, v198\n" \
  "v_exp_f32 v201, v201\n" \
  "v_accvgpr_read_b32 v195, a222\n" \
  "v_rcp_f32 v199, v199\n" \
  "v_fma_f32 v200, v200, s80, s80\n" \
  "v_med3_f32 v202, v194, s55, v186\n" \
  "v_mul_f32 v207, v191, v199\n" \
  "v_exp_f32 v202, v202\n" \
  "v_rcp_f32 v200, v200\n" \
  "v_fma_f32 v201, v201, s80, s80\n" \
  "v_med3_f32 v203, v195, s55, v186\n" \
  "v_mul_f32 v208, v192, v200\n" \
  "v_exp_f32 v203, v203\n" \
  "v_rcp_f32 v201, v201\n" \
  "v_fma_f32 v202, v202, s80, s80\n" \
  "v_mul_f32 v209, v193, v201\n" \
  "v_rcp_f32 v202, v202\n" \
  "v_fma_f32 v203, v203, s80, s80\n" \
  "s_nop 0\n" \
  "v_mul_f32 v210, v194, v202\n" \
  "v_rcp_f32 v203, v203\n" \
  "s_nop 0\n" \
  "v_mul_f32 v211, v195, v203\n" \
  "s_nop 0\n" \
  "s_mov_b32 s79, 12845056\n" \
  "EW26_%=:\n" \
  "s_waitcnt lgkmcnt(0)\n" \
  "s_sub_u32 s24, s38, s25\n" \
  "s_cmp_lt_i32 s24, 0\n" \
  "s_cbranch_scc0 EG26_%=\n" \
  "s_memrealtime s[38:39]\n" \
  "s_branch EW26_%=\n" \
  "EG26_%=:\n" \
  "s_add_u32 s25, s25, 28\n" \
  "buffer_store_dwordx4 v[204:207], %15, s[72:75], s79 offen nt\n" \
  "buffer_store_dwordx4 v[208:211], %15, s[72:75], s79 offen offset:256 nt\n" \
  "v_accvgpr_read_b32 v212, a195\n" \
  "v_accvgpr_read_b32 v213, a203\n" \
  "v_med3_f32 v220, v212, s55, v186\n" \
  "v_exp_f32 v220, v220\n" \
  "v_accvgpr_read_b32 v214, a211\n" \
  "v_med3_f32 v221, v213, s55, v186\n" \
  "v_exp_f32 v221, v221\n" \
  "v_accvgpr_read_b32 v215, a219\n" \
  "v_fma_f32 v220, v220, s80, s80\n" \
  "v_med3_f32 v222, v214, s55, v186\n" \
  "v_exp_f32 v222, v222\n" \
  "v_accvgpr_read_b32 v216, a199\n" \
  "v_rcp_f32 v220, v220\n" \
  "v_fma_f32 v221, v221, s80, s80\n" \
  "v_med3_f32 v223, v215, s55, v186\n" \
  "v_mul_f32 v228, v212, v220\n" \
  "v_exp_f32 v223, v223\n" \
  "v_accvgpr_read_b32 v217, a207\n" \
  "v_rcp_f32 v221, v221\n" \
  "v_fma_f32 v222, v222, s80, s80\n" \
  "v_med3_f32 v224, v216, s55, v186\n" \
  "v_mul_f32 v229, v213, v221\n" \
  "v_exp_f32 v224, v224\n" \
  "v_accvgpr_read_b32 v218, a215\n" \
  "v_rcp_f32 v222, v222\n" \
  "v_fma_f32 v223, v223, s80, s80\n" \
  "v_med3_f32 v225, v217, s55, v186\n" \
  "s_memrealtime s[38:39]\n" \
  "v_mul_f32 v230, v214, v222\n" \
  "v_exp_f32 v225, v225\n" \
  "v_accvgpr_read_b32 v219, a223\n" \
  "v_rcp_f32 v223, v223\n" \
  "v_fma_f32 v224, v224, s80, s80\n" \
  "v_med3_f32 v226, v218, s55, v186\n" \
  "v_mul_f32 v231, v215, v223\n" \
  "v_exp_f32 v226, v226\n" \
  "v_rcp_f32 v224, v224\n" \
  "v_fma_f32 v225, v225, s80, s80\n" \
  "v_med3_f32 v227, v219, s55, v186\n" \
  "v_mul_f32 v232, v216, v224\n" \
  "v_exp_f32 v227, v227\n" \
  "v_rcp_f32 v225, v225\n" \
  "v_fma_f32 v226, v226, s80, s80\n" \
  "v_mul_f32 v233, v217, v225\n" \
  "v_rcp_f32 v226, v226\n" \
  "v_fma_f32 v227, v227, s80, s80\n" \
  "s_nop 0\n" \
  "v_mul_f32 v234, v218, v226\n" \
  "v_rcp_f32 v227, v227\n" \
  "s_nop 0\n" \
  "v_mul_f32 v235, v219, v227\n" \
  "s_nop 0\n" \
  "s_mov_b32 s79, 12976128\n" \
  "EW27_%=:\n" \
  "s_waitcnt lgkmcnt(0)\n" \
  "s_sub_u32 s24, s38, s25\n" \
  "s_cmp_lt_i32 s24, 0\n" \
  "s_cbranch_scc0 EG27_%=\n" \
  "s_memrealtime s[38:39]\n" \
  "s_branch EW27_%=\n" \
  "EG27_%=:\n" \
  "s_add_u32 s25, s25, 28\n" \
  "buffer_store_dwordx4 v[228:231], %15, s[72:75], s79 offen nt\n" \
  "buffer_store_dwordx4 v[232:235], %15, s[72:75], s79 offen offset:256 nt\n" \
  "v_accvgpr_read_b32 v188, a224\n" \
  "v_accvgpr_read_b32 v189, a232\n" \
  "v_med3_f32 v196, v188, s55, v186\n" \
  "v_exp_f32 v196, v196\n" \
  "v_accvgpr_read_b32 v190, a240\n" \
  "v_med3_f32 v197, v189, s55, v186\n" \
  "v_exp_f32 v197, v197\n" \
  "v_accvgpr_read_b32 v191, a248\n" \
  "v_fma_f32 v196, v196, s80, s80\n" \
  "v_med3_f32 v198, v190, s55, v186\n" \
  "v_exp_f32 v198, v198\n" \
  "v_accvgpr_read_b32 v192, a228\n" \
  "v_rcp_f32 v196, v196\n" \
  "v_fma_f32 v197, v197, s80, s80\n" \
  "v_med3_f32 v199, v191, s55, v186\n" \
  "v_mul_f32 v204, v188, v196\n" \
  "v_exp_f32 v199, v199\n" \
  "v_accvgpr_read_b32 v193, a236\n" \
  "v_rcp_f32 v197, v197\n" \
  "v_fma_f32 v198, v198, s80, s80\n" \
  "v_med3_f32 v200, v192, s55, v186\n" \
  "v_mul_f32 v205, v189, v197\n" \
  "v_exp_f32 v200, v200\n" \
  "v_accvgpr_read_b32 v194, a244\n" \
  "v_rcp_f32 v198, v198\n" \
  "v_fma_f32 v199, v199, s80, s80\n" \
  "v_med3_f32 v201, v193, s55, v186\n" \
  "s_memrealtime s[38:39]\n" \
  "v_mul_f32 v206, v190, v198\n" \
  "v_exp_f32 v201, v201\n" \
  "v_accvgpr_read_b32 v195, a252\n" \
  "v_rcp_f32 v199, v199\n" \
  "v_fma_f32 v200, v200, s80, s80\n" \
  "v_med3_f32 v202, v194, s55, v186\n" \
  "v_mul_f32 v207, v191, v199\n" \
  "v_exp_f32 v202, v202\n" \
  "v_rcp_f32 v200, v200\n" \
  "v_fma_f32 v201, v201, s80, s80\n" \
  "v_med3_f32 v203, v195, s55, v186\n" \
  "v_mul_f32 v208, v192, v200\n" \
  "v_exp_f32 v203, v203\n" \
  "v_rcp_f32 v201, v201\n" \
  "v_fma_f32 v202, v202, s80, s80\n" \
  "v_mul_f32 v209, v193, v201\n" \
  "v_rcp_f32 v202, v202\n" \
  "v_fma_f32 v203, v203, s80, s80\n" \
  "s_nop 0\n" \
  "v_mul_f32 v210, v194, v202\n" \
  "v_rcp_f32 v203, v203\n" \
  "s_nop 0\n" \
  "v_mul_f32 v211, v195, v203\n" \
  "s_nop 0\n" \
  "s_mov_b32 s79, 14680064\n" \
  "EW28_%=:\n" \
  "s_waitcnt lgkmcnt(0)\n" \
  "s_sub_u32 s24, s38, s25\n" \
  "s_cmp_lt_i32 s24, 0\n" \
  "s_cbranch_scc0 EG28_%=\n" \
  "s_memrealtime s[38:39]\n" \
  "s_branch EW28_%=\n" \
  "EG28_%=:\n" \
  "s_add_u32 s25, s25, 28\n" \
  "buffer_store_dwordx4 v[204:207], %15, s[72:75], s79 offen nt\n" \
  "buffer_store_dwordx4 v[208:211], %15, s[72:75], s79 offen offset:256 nt\n" \
  "v_accvgpr_read_b32 v212, a225\n" \
  "v_accvgpr_read_b32 v213, a233\n" \
  "v_med3_f32 v220, v212, s55, v186\n" \
  "v_exp_f32 v220, v220\n" \
  "v_accvgpr_read_b32 v214, a241\n" \
  "v_med3_f32 v221, v213, s55, v186\n" \
  "v_exp_f32 v221, v221\n" \
  "v_accvgpr_read_b32 v215, a249\n" \
  "v_fma_f32 v220, v220, s80, s80\n" \
  "v_med3_f32 v222, v214, s55, v186\n" \
  "v_exp_f32 v222, v222\n" \
  "v_accvgpr_read_b32 v216, a229\n" \
  "v_rcp_f32 v220, v220\n" \
  "v_fma_f32 v221, v221, s80, s80\n" \
  "v_med3_f32 v223, v215, s55, v186\n" \
  "v_mul_f32 v228, v212, v220\n" \
  "v_exp_f32 v223, v223\n" \
  "v_accvgpr_read_b32 v217, a237\n" \
  "v_rcp_f32 v221, v221\n" \
  "v_fma_f32 v222, v222, s80, s80\n" \
  "v_med3_f32 v224, v216, s55, v186\n" \
  "v_mul_f32 v229, v213, v221\n" \
  "v_exp_f32 v224, v224\n" \
  "v_accvgpr_read_b32 v218, a245\n" \
  "v_rcp_f32 v222, v222\n" \
  "v_fma_f32 v223, v223, s80, s80\n" \
  "v_med3_f32 v225, v217, s55, v186\n" \
  "s_memrealtime s[38:39]\n" \
  "v_mul_f32 v230, v214, v222\n" \
  "v_exp_f32 v225, v225\n" \
  "v_accvgpr_read_b32 v219, a253\n" \
  "v_rcp_f32 v223, v223\n" \
  "v_fma_f32 v224, v224, s80, s80\n" \
  "v_med3_f32 v226, v218, s55, v186\n" \
  "v_mul_f32 v231, v215, v223\n" \
  "v_exp_f32 v226, v226\n" \
  "v_rcp_f32 v224, v224\n" \
  "v_fma_f32 v225, v225, s80, s80\n" \
  "v_med3_f32 v227, v219, s55, v186\n" \
  "v_mul_f32 v232, v216, v224\n" \
  "v_exp_f32 v227, v227\n" \
  "v_rcp_f32 v225, v225\n" \
  "v_fma_f32 v226, v226, s80, s80\n" \
  "v_mul_f32 v233, v217, v225\n" \
  "v_rcp_f32 v226, v226\n" \
  "v_fma_f32 v227, v227, s80, s80\n" \
  "s_nop 0\n" \
  "v_mul_f32 v234, v218, v226\n" \
  "v_rcp_f32 v227, v227\n" \
  "s_nop 0\n" \
  "v_mul_f32 v235, v219, v227\n" \
  "s_nop 0\n" \
  "s_mov_b32 s79, 14811136\n" \
  "EW29_%=:\n" \
  "s_waitcnt lgkmcnt(0)\n" \
  "s_sub_u32 s24, s38, s25\n" \
  "s_cmp_lt_i32 s24, 0\n" \
  "s_cbranch_scc0 EG29_%=\n" \
  "s_memrealtime s[38:39]\n" \
  "s_branch EW29_%=\n" \
  "EG29_%=:\n" \
  "s_add_u32 s25, s25, 28\n" \
  "buffer_store_dwordx4 v[228:231], %15, s[72:75], s79 offen nt\n" \
  "buffer_store_dwordx4 v[232:235], %15, s[72:75], s79 offen offset:256 nt\n" \
  "v_accvgpr_read_b32 v188, a226\n" \
  "v_accvgpr_read_b32 v189, a234\n" \
  "v_med3_f32 v196, v188, s55, v186\n" \
  "v_exp_f32 v196, v196\n" \
  "v_accvgpr_read_b32 v190, a242\n" \
  "v_med3_f32 v197, v189, s55, v186\n" \
  "v_exp_f32 v197, v197\n" \
  "v_accvgpr_read_b32 v191, a250\n" \
  "v_fma_f32 v196, v196, s80, s80\n" \
  "v_med3_f32 v198, v190, s55, v186\n" \
  "v_exp_f32 v198, v198\n" \
  "v_accvgpr_read_b32 v192, a230\n" \
  "v_rcp_f32 v196, v196\n" \
  "v_fma_f32 v197, v197, s80, s80\n" \
  "v_med3_f32 v199, v191, s55, v186\n" \
  "v_mul_f32 v204, v188, v196\n" \
  "v_exp_f32 v199, v199\n" \
  "v_accvgpr_read_b32 v193, a238\n" \
  "v_rcp_f32 v197, v197\n" \
  "v_fma_f32 v198, v198, s80, s80\n" \
  "v_med3_f32 v200, v192, s55, v186\n" \
  "v_mul_f32 v205, v189, v197\n" \
  "v_exp_f32 v200, v200\n" \
  "v_accvgpr_read_b32 v194, a246\n" \
  "v_rcp_f32 v198, v198\n" \
  "v_fma_f32 v199, v199, s80, s80\n" \
  "v_med3_f32 v201, v193, s55, v186\n" \
  "s_memrealtime s[38:39]\n" \
  "v_mul_f32 v206, v190, v198\n" \
  "v_exp_f32 v201, v201\n" \
  "v_accvgpr_read_b32 v195, a254\n" \
  "v_rcp_f32 v199, v199\n" \
  "v_fma_f32 v200, v200, s80, s80\n" \
  "v_med3_f32 v202, v194, s55, v186\n" \
  "v_mul_f32 v207, v191, v199\n" \
  "v_exp_f32 v202, v202\n" \
  "v_rcp_f32 v200, v200\n" \
  "v_fma_f32 v201, v201, s80, s80\n" \
  "v_med3_f32 v203, v195, s55, v186\n" \
  "v_mul_f32 v208, v192, v200\n" \
  "v_exp_f32 v203, v203\n" \
  "v_rcp_f32 v201, v201\n" \
  "v_fma_f32 v202, v202, s80, s80\n" \
  "v_mul_f32 v209, v193, v201\n" \
  "v_rcp_f32 v202, v202\n" \
  "v_fma_f32 v203, v203, s80, s80\n" \
  "s_nop 0\n" \
  "v_mul_f32 v210, v194, v202\n" \
  "v_rcp_f32 v203, v203\n" \
  "s_nop 0\n" \
  "v_mul_f32 v211, v195, v203\n" \
  "s_nop 0\n" \
  "s_mov_b32 s79, 14942208\n" \
  "EW30_%=:\n" \
  "s_waitcnt lgkmcnt(0)\n" \
  "s_sub_u32 s24, s38, s25\n" \
  "s_cmp_lt_i32 s24, 0\n" \
  "s_cbranch_scc0 EG30_%=\n" \
  "s_memrealtime s[38:39]\n" \
  "s_branch EW30_%=\n" \
  "EG30_%=:\n" \
  "s_add_u32 s25, s25, 28\n" \
  "buffer_store_dwordx4 v[204:207], %15, s[72:75], s79 offen nt\n" \
  "buffer_store_dwordx4 v[208:211], %15, s[72:75], s79 offen offset:256 nt\n" \
  "v_accvgpr_read_b32 v212, a227\n" \
  "v_accvgpr_read_b32 v213, a235\n" \
  "v_med3_f32 v220, v212, s55, v186\n" \
  "v_exp_f32 v220, v220\n" \
  "v_accvgpr_read_b32 v214, a243\n" \
  "v_med3_f32 v221, v213, s55, v186\n" \
  "v_exp_f32 v221, v221\n" \
  "v_accvgpr_read_b32 v215, a251\n" \
  "v_fma_f32 v220, v220, s80, s80\n" \
  "v_med3_f32 v222, v214, s55, v186\n" \
  "v_exp_f32 v222, v222\n" \
  "v_accvgpr_read_b32 v216, a231\n" \
  "v_rcp_f32 v220, v220\n" \
  "v_fma_f32 v221, v221, s80, s80\n" \
  "v_med3_f32 v223, v215, s55, v186\n" \
  "v_mul_f32 v228, v212, v220\n" \
  "v_exp_f32 v223, v223\n" \
  "v_accvgpr_read_b32 v217, a239\n" \
  "v_rcp_f32 v221, v221\n" \
  "v_fma_f32 v222, v222, s80, s80\n" \
  "v_med3_f32 v224, v216, s55, v186\n" \
  "v_mul_f32 v229, v213, v221\n" \
  "v_exp_f32 v224, v224\n" \
  "v_accvgpr_read_b32 v218, a247\n" \
  "v_rcp_f32 v222, v222\n" \
  "v_fma_f32 v223, v223, s80, s80\n" \
  "v_med3_f32 v225, v217, s55, v186\n" \
  "s_memrealtime s[38:39]\n" \
  "v_mul_f32 v230, v214, v222\n" \
  "v_exp_f32 v225, v225\n" \
  "v_accvgpr_read_b32 v219, a255\n" \
  "v_rcp_f32 v223, v223\n" \
  "v_fma_f32 v224, v224, s80, s80\n" \
  "v_med3_f32 v226, v218, s55, v186\n" \
  "v_mul_f32 v231, v215, v223\n" \
  "v_exp_f32 v226, v226\n" \
  "v_rcp_f32 v224, v224\n" \
  "v_fma_f32 v225, v225, s80, s80\n" \
  "v_med3_f32 v227, v219, s55, v186\n" \
  "v_mul_f32 v232, v216, v224\n" \
  "v_exp_f32 v227, v227\n" \
  "v_rcp_f32 v225, v225\n" \
  "v_fma_f32 v226, v226, s80, s80\n" \
  "v_mul_f32 v233, v217, v225\n" \
  "v_rcp_f32 v226, v226\n" \
  "v_fma_f32 v227, v227, s80, s80\n" \
  "s_nop 0\n" \
  "v_mul_f32 v234, v218, v226\n" \
  "v_rcp_f32 v227, v227\n" \
  "s_nop 0\n" \
  "v_mul_f32 v235, v219, v227\n" \
  "s_nop 0\n" \
  "s_mov_b32 s79, 15073280\n" \
  "EW31_%=:\n" \
  "s_waitcnt lgkmcnt(0)\n" \
  "s_sub_u32 s24, s38, s25\n" \
  "s_cmp_lt_i32 s24, 0\n" \
  "s_cbranch_scc0 EG31_%=\n" \
  "s_memrealtime s[38:39]\n" \
  "s_branch EW31_%=\n" \
  "EG31_%=:\n" \
  "s_add_u32 s25, s25, 28\n" \
  "buffer_store_dwordx4 v[228:231], %15, s[72:75], s79 offen nt\n" \
  "buffer_store_dwordx4 v[232:235], %15, s[72:75], s79 offen offset:256 nt\n" \
  "s_waitcnt lgkmcnt(0)\n" \
  "v_mov_b32 v168, s96\n" \
  "v_mov_b32 v169, s62\n" \
  "buffer_store_dwordx2 v[168:169], v176, s[92:95], s59 offen sc0 sc1\n" \
  "s_waitcnt vmcnt(0)\n" \
  "CPFE%=:\n" \
  ""
#define FUSED_CLOBBERS "v0","v1","v2","v3","v4","v5","v6","v7","v8","v9","v10","v11","v12","v13","v14","v15","v16","v17","v18","v19","v20","v21","v22","v23","v24","v25","v26","v27","v28","v29","v30","v31","v32","v33","v34","v35","v36","v37","v38","v39","v40","v41","v42","v43","v44","v45","v46","v47","v48","v49","v50","v51","v52","v53","v54","v55","v56","v57","v58","v59","v60","v61","v62","v63","v64","v65","v66","v67","v68","v69","v70","v71","v72","v73","v74","v75","v76","v77","v78","v79","v80","v81","v82","v83","v84","v85","v86","v87","v88","v89","v90","v91","v92","v93","v94","v95","v96","v97","v98","v99","v100","v101","v102","v103","v104","v105","v106","v107","v108","v109","v110","v111","v112","v113","v114","v115","v116","v117","v118","v119","v120","v121","v122","v123","v124","v125","v126","v127","v128","v129","v130","v131","v132","v133","v134","v135","v136","v137","v138","v139","v140","v141","v142","v143","v144","v145","v146","v147","v148","v149","v150","v151","v152","v153","v154","v155","v156","v157","v158","v159","v160","v161","v162","v163","v164","v165","v166","v167","v168","v169","v170","v171","v172","v173","v174","v175","v176","v177","v178","v179","v180","v181","v182","v183","v184","v185","v186","v187","v188","v189","v190","v191","v192","v193","v194","v195","v196","v197","v198","v199","v200","v201","v202","v203","v204","v205","v206","v207","v208","v209","v210","v211","v212","v213","v214","v215","v216","v217","v218","v219","v220","v221","v222","v223","v224","v225","v226","v227","v228","v229","v230","v231","v232","v233","v234","v235","v236","v237","v238","v239","v240","v241","a0","a1","a2","a3","a4","a5","a6","a7","a8","a9","a10","a11","a12","a13","a14","a15","a16","a17","a18","a19","a20","a21","a22","a23","a24","a25","a26","a27","a28","a29","a30","a31","a32","a33","a34","a35","a36","a37","a38","a39","a40","a41","a42","a43","a44","a45","a46","a47","a48","a49","a50","a51","a52","a53","a54","a55","a56","a57","a58","a59","a60","a61","a62","a63","a64","a65","a66","a67","a68","a69","a70","a71","a72","a73","a74","a75","a76","a77","a78","a79","a80","a81","a82","a83","a84","a85","a86","a87","a88","a89","a90","a91","a92","a93","a94","a95","a96","a97","a98","a99","a100","a101","a102","a103","a104","a105","a106","a107","a108","a109","a110","a111","a112","a113","a114","a115","a116","a117","a118","a119","a120","a121","a122","a123","a124","a125","a126","a127","a128","a129","a130","a131","a132","a133","a134","a135","a136","a137","a138","a139","a140","a141","a142","a143","a144","a145","a146","a147","a148","a149","a150","a151","a152","a153","a154","a155","a156","a157","a158","a159","a160","a161","a162","a163","a164","a165","a166","a167","a168","a169","a170","a171","a172","a173","a174","a175","a176","a177","a178","a179","a180","a181","a182","a183","a184","a185","a186","a187","a188","a189","a190","a191","a192","a193","a194","a195","a196","a197","a198","a199","a200","a201","a202","a203","a204","a205","a206","a207","a208","a209","a210","a211","a212","a213","a214","a215","a216","a217","a218","a219","a220","a221","a222","a223","a224","a225","a226","a227","a228","a229","a230","a231","a232","a233","a234","a235","a236","a237","a238","a239","a240","a241","a242","a243","a244","a245","a246","a247","a248","a249","a250","a251","a252","a253","a254","a255","s22","s23","s24","s25","s26","s27","s28","s29","s30","s31","s34","s35","s36","s37","s38","s39","s40","s41","s42","s43","s44","s45","s46","s47","s48","s49","s50","s51","s52","s53","s54","s55","s56","s57","s58","s59","s60","s61","s62","s63","s64","s65","s66","s67","s68","s69","s70","s71","s72","s73","s74","s75","s76","s77","s78","s79","s80","s81","s82","s83","s84","s85","s86","s87","s88","s89","s90","s91","s92","s93","s94","s95","s96","s97","s98","s99","scc","vcc","memory"

namespace {
constexpr int kM = 512, kN = 32768, kK = 4096, kRanks = 8;
constexpr unsigned kRow = kN * 4;
constexpr unsigned kCtl = 0x800000;
constexpr size_t kWsNeed = kCtl + 0x10000;

struct Params {
  const float* x;
  const float* w;
  float* out;
  char* ws[kRanks];
  int rank;
};

__global__ __launch_bounds__(256, 1) void fused_a2a_gemm_silu(Params p) {
  __shared__ __attribute__((aligned(16))) char lds[LDS_BYTES];
  const int tid = threadIdx.x;
  const int w = __builtin_amdgcn_readfirstlane(tid >> 6);
  const unsigned lane = tid & 63;
  const unsigned nb = blockIdx.x;
  const unsigned cb = nb;
  const unsigned rank = (unsigned)p.rank & (kRanks - 1);
  const unsigned ldsb = (unsigned)(size_t)(__attribute__((address_space(3))) char*)lds;
  const size_t wfull = (size_t)p.w + cb * 512;
  const unsigned wlow = (unsigned)(wfull & 0xffff);
  const unsigned flip = (__builtin_popcount(wlow & 0x7e00) & 1) ^ 1;
  const char* wbase = (const char*)(wfull - wlow) + (size_t)w * 8 * kRow - (size_t)flip * 0x80000;
  const unsigned hrow = ((lane >> 5) ^ (lane >> 3)) & 1;
  const unsigned wvoff = hrow * kRow + wlow + (lane & 31) * 16 + flip * 0x80000;
  const unsigned avoff = w * 8192 + lane * 16;
  const unsigned lw = ldsb + w * 2048 + (lane & 31) * 16 + hrow * 8;
  const unsigned lr = ldsb + (lane >> 4) * 2048 + (lane & 15) * 16;
  const unsigned ovoff = (128 * w + 4 * (lane >> 4)) * kRow + cb * 512 + (lane & 15) * 16;
  const unsigned pfoff = (w == 0 && lane < 16) ? ((nb >> 3) * 16 + lane) * 128 : 0x7ff00000u;
  const unsigned dest = (rank + (nb >> 5)) & (kRanks - 1);
  const unsigned mt = nb & 31;
  const char* xbase = (const char*)p.x + (size_t)(dest * 512 + mt * 16) * 2048;
  const unsigned xvoff = (lane & 15) * 2048 + w * 128 + (lane >> 4) * 32;
  char* own = p.ws[0];
  char* dws = p.ws[(nb >> 5) & (kRanks - 1)];
  const unsigned dsoff = rank * 0x80000 + w * 0x8000 + mt * 1024;
  const unsigned smask = dest == rank ? 0u : ((mt & 1) ? 0x55u : 0x2bu);
  const unsigned fvoff = lane < 32 ? lane * 128 : 0x7ff00000u;
  const unsigned cnt_off = 0x1000 + nb * 8;
  const char* cntp = own + kCtl + cnt_off;
  asm volatile(FUSED_ASM
               :
               : "s"(wbase), "s"(own), "s"(p.out), "s"(rank), "s"(xbase), "s"(dws), "s"(cntp), "s"(dsoff), "s"(smask),
                 "s"(cnt_off), "s"(w), "v"(wvoff), "v"(avoff), "v"(lw), "v"(lr), "v"(ovoff), "v"(pfoff), "v"(xvoff),
                 "v"(fvoff), "v"(lane)
               : FUSED_CLOBBERS);
}
}

extern "C" void kernel_launch(int rank, int nranks, ncclComm_t comm, hipStream_t stream, void* const* d_in,
                              const int* in_sizes, int n_in, void* d_out, int out_size, const int* in_dtypes,
                              int out_dtype, void* const* d_out_all, void* const* d_ws_all, size_t ws_bytes) {
  if (nranks != kRanks || (unsigned)rank >= (unsigned)kRanks || n_in < 2 || ws_bytes < kWsNeed ||
      in_sizes[0] != kK * kM || in_sizes[1] != kK * kN || out_size != kM * kN) {
    fprintf(stderr, "kernel_launch: unexpected problem shape\n");
    return;
  }
  Params p;
  p.x = (const float*)d_in[0];
  p.w = (const float*)d_in[1];
  p.out = (float*)d_out;
  for (int i = 0; i < kRanks; i++) p.ws[i] = (char*)d_ws_all[(rank + i) % kRanks];
  p.rank = rank;
  fused_a2a_gemm_silu<<<256, 256, 0, stream>>>(p);
  hipError_t e = hipGetLastError();
  if (e != hipSuccess) fprintf(stderr, "HIP error %s\n", hipGetErrorString(e));
}
```
